```python
import numpy as np

kernel_name = "gpu_gemm_a2a_m4096_k4096_n16384_f32_reluq8"


ABSMAX_RTOL = 0.02

COMPUTE_DTYPE = "float32"
EPILOGUE = "relu_quant_int8"
IN_QUANT = None
OUT_QUANT = "int8"
IN_DTYPES = {"x": "float32", "w_mat": "float32"}
WORLD_SIZE = 8

def setup_inputs():

    x = np.random.randn(4096, 4096).astype(np.float32)
    w_mat = (np.random.randn(4096, 16384) / np.sqrt(4096)).astype(np.float32)
    return {"x": x, "w_mat": w_mat}

def _quant_dequant(y, amax):
    scale = amax / 127.0
    q = np.clip(np.round(y / scale), -127.0, 127.0)
    return (q * scale).astype(np.float32)
def _epilogue(y):
    y = np.maximum(y, 0.0)
    return _quant_dequant(y, np.abs(y).max())

def reference(x, w_mat):
    y = (x @ w_mat).astype(np.float32)
    return _epilogue(y)
```

```cpp
#include <hip/hip_runtime.h>
#include <rccl/rccl.h>
#include <cstdio>

typedef unsigned u32;
typedef unsigned long long u64;
typedef u32 u32x2 __attribute__((ext_vector_type(2)));
typedef u32 u32x4 __attribute__((ext_vector_type(4)));
typedef float f32x4 __attribute__((ext_vector_type(4)));
typedef float f32x32 __attribute__((ext_vector_type(32)));

#define MAIN_ASM \
  "s_load_dwordx16 s[36:51], %[ka], 0\n\t" \
  "v_mov_b32 v4, %[vW0]\n\t" \
  "v_add_u32 v5, 65536, %[vW0]\n\t" \
  "v_add_u32 v6, 131072, %[vW0]\n\t" \
  "v_add_u32 v7, 196608, %[vW0]\n\t" \
  "v_add_u32 v8, 262144, %[vW0]\n\t" \
  "v_add_u32 v9, 327680, %[vW0]\n\t" \
  "v_add_u32 v10, 393216, %[vW0]\n\t" \
  "v_add_u32 v11, 458752, %[vW0]\n\t" \
  "v_mov_b32 v12, %[vL0]\n\t" \
  "v_add_u32 v13, %[sM0], %[vL0]\n\t" \
  "v_mov_b32 v14, %[vPA]\n\t" \
  "v_mov_b32 v15, %[vOff]\n\t" \
  "v_mov_b32 v224, %[sPM]\n\t" \
  "s_mov_b32 s84, 0x05010400\n\t" \
  "s_mov_b32 s85, 0x07030602\n\t" \
  "s_mov_b32 s63, 0x83838383\n\t" \
  "s_mov_b32 s64, 0x80808080\n\t" \
  "s_lshr_b32 s77, %[sig], 4\n\t" \
  "s_lshl_b32 s77, s77, 1\n\t" \
  "s_mov_b32 s68, 0\n\t" \
  "s_mov_b32 s69, 1\n\t" \
  "s_mov_b32 s70, 12\n\t" \
  "s_cmp_lt_u32 %[sig], 5\n\t" \
  "s_cselect_b32 s71, 1, 0\n\t" \
  "s_and_b32 s81, %[sig], 15\n\t" \
  "s_cmp_ge_u32 s81, 16\n\t" \
  "s_cselect_b32 s81, 64, 0\n\t" \
  "s_or_b32 s71, s71, s81\n\t" \
  "s_sub_u32 s80, %[due], 40\n\t" \
  "s_cmp_lt_u32 %[sig], 5\n\t" \
  "s_cselect_b32 s76, %[due], s80\n\t" \
  "s_waitcnt lgkmcnt(0)\n\t" \
  "v_writelane_b32 v232, s36, 0\n\t" \
  "v_writelane_b32 v233, s37, 0\n\t" \
  "v_writelane_b32 v232, s38, 1\n\t" \
  "v_writelane_b32 v233, s39, 1\n\t" \
  "v_writelane_b32 v232, s40, 2\n\t" \
  "v_writelane_b32 v233, s41, 2\n\t" \
  "v_writelane_b32 v232, s42, 3\n\t" \
  "v_writelane_b32 v233, s43, 3\n\t" \
  "v_writelane_b32 v232, s44, 4\n\t" \
  "v_writelane_b32 v233, s45, 4\n\t" \
  "v_writelane_b32 v232, s46, 5\n\t" \
  "v_writelane_b32 v233, s47, 5\n\t" \
  "v_writelane_b32 v232, s48, 6\n\t" \
  "v_writelane_b32 v233, s49, 6\n\t" \
  "v_writelane_b32 v232, s50, 7\n\t" \
  "v_writelane_b32 v233, s51, 7\n\t" \
  "v_add_co_u32 v232, vcc, %[fo], v232\n\t" \
  "s_nop 1\n\t" \
  "v_addc_co_u32 v233, vcc, 0, v233, vcc\n\t" \
  "s_nop 4\n\t" \
  "s_mov_b64 vcc, 0\n\t" \
  "s_cmp_lt_u32 %[sig], 5\n\t" \
  "s_cbranch_scc0 Lea1_%=\n\t" \
  "Lea0_%=:\n\t" \
  "s_memrealtime s[72:73]\n\t" \
  "s_waitcnt lgkmcnt(0)\n\t" \
  "s_sub_u32 s81, s72, s76\n\t" \
  "s_cmp_ge_i32 s81, 0\n\t" \
  "s_cbranch_scc0 Lea0_%=\n\t" \
  "s_getpc_b64 s[74:75]\n\t" \
  "s_branch Lpush_%=\n\t" \
  "v_accvgpr_write_b32 a0, 0\n\t" \
  "v_accvgpr_write_b32 a1, 0\n\t" \
  "v_accvgpr_write_b32 a2, 0\n\t" \
  "v_accvgpr_write_b32 a3, 0\n\t" \
  "v_accvgpr_write_b32 a4, 0\n\t" \
  "v_accvgpr_write_b32 a5, 0\n\t" \
  "v_accvgpr_write_b32 a6, 0\n\t" \
  "v_accvgpr_write_b32 a7, 0\n\t" \
  "v_accvgpr_write_b32 a8, 0\n\t" \
  "v_accvgpr_write_b32 a9, 0\n\t" \
  "v_accvgpr_write_b32 a10, 0\n\t" \
  "v_accvgpr_write_b32 a11, 0\n\t" \
  "v_accvgpr_write_b32 a12, 0\n\t" \
  "v_accvgpr_write_b32 a13, 0\n\t" \
  "v_accvgpr_write_b32 a14, 0\n\t" \
  "v_accvgpr_write_b32 a15, 0\n\t" \
  "v_accvgpr_write_b32 a16, 0\n\t" \
  "v_accvgpr_write_b32 a17, 0\n\t" \
  "v_accvgpr_write_b32 a18, 0\n\t" \
  "v_accvgpr_write_b32 a19, 0\n\t" \
  "v_accvgpr_write_b32 a20, 0\n\t" \
  "v_accvgpr_write_b32 a21, 0\n\t" \
  "v_accvgpr_write_b32 a22, 0\n\t" \
  "v_accvgpr_write_b32 a23, 0\n\t" \
  "v_accvgpr_write_b32 a24, 0\n\t" \
  "v_accvgpr_write_b32 a25, 0\n\t" \
  "v_accvgpr_write_b32 a26, 0\n\t" \
  "v_accvgpr_write_b32 a27, 0\n\t" \
  "v_accvgpr_write_b32 a28, 0\n\t" \
  "v_accvgpr_write_b32 a29, 0\n\t" \
  "v_accvgpr_write_b32 a30, 0\n\t" \
  "v_accvgpr_write_b32 a31, 0\n\t" \
  "v_accvgpr_write_b32 a32, 0\n\t" \
  "v_accvgpr_write_b32 a33, 0\n\t" \
  "v_accvgpr_write_b32 a34, 0\n\t" \
  "v_accvgpr_write_b32 a35, 0\n\t" \
  "v_accvgpr_write_b32 a36, 0\n\t" \
  "v_accvgpr_write_b32 a37, 0\n\t" \
  "v_accvgpr_write_b32 a38, 0\n\t" \
  "v_accvgpr_write_b32 a39, 0\n\t" \
  "v_accvgpr_write_b32 a40, 0\n\t" \
  "v_accvgpr_write_b32 a41, 0\n\t" \
  "v_accvgpr_write_b32 a42, 0\n\t" \
  "v_accvgpr_write_b32 a43, 0\n\t" \
  "v_accvgpr_write_b32 a44, 0\n\t" \
  "v_accvgpr_write_b32 a45, 0\n\t" \
  "v_accvgpr_write_b32 a46, 0\n\t" \
  "v_accvgpr_write_b32 a47, 0\n\t" \
  "v_accvgpr_write_b32 a48, 0\n\t" \
  "v_accvgpr_write_b32 a49, 0\n\t" \
  "v_accvgpr_write_b32 a50, 0\n\t" \
  "v_accvgpr_write_b32 a51, 0\n\t" \
  "v_accvgpr_write_b32 a52, 0\n\t" \
  "v_accvgpr_write_b32 a53, 0\n\t" \
  "v_accvgpr_write_b32 a54, 0\n\t" \
  "v_accvgpr_write_b32 a55, 0\n\t" \
  "v_accvgpr_write_b32 a56, 0\n\t" \
  "v_accvgpr_write_b32 a57, 0\n\t" \
  "v_accvgpr_write_b32 a58, 0\n\t" \
  "v_accvgpr_write_b32 a59, 0\n\t" \
  "v_accvgpr_write_b32 a60, 0\n\t" \
  "v_accvgpr_write_b32 a61, 0\n\t" \
  "v_accvgpr_write_b32 a62, 0\n\t" \
  "v_accvgpr_write_b32 a63, 0\n\t" \
  "v_accvgpr_write_b32 a64, 0\n\t" \
  "v_accvgpr_write_b32 a65, 0\n\t" \
  "v_accvgpr_write_b32 a66, 0\n\t" \
  "v_accvgpr_write_b32 a67, 0\n\t" \
  "v_accvgpr_write_b32 a68, 0\n\t" \
  "v_accvgpr_write_b32 a69, 0\n\t" \
  "v_accvgpr_write_b32 a70, 0\n\t" \
  "v_accvgpr_write_b32 a71, 0\n\t" \
  "v_accvgpr_write_b32 a72, 0\n\t" \
  "v_accvgpr_write_b32 a73, 0\n\t" \
  "v_accvgpr_write_b32 a74, 0\n\t" \
  "v_accvgpr_write_b32 a75, 0\n\t" \
  "v_accvgpr_write_b32 a76, 0\n\t" \
  "v_accvgpr_write_b32 a77, 0\n\t" \
  "v_accvgpr_write_b32 a78, 0\n\t" \
  "v_accvgpr_write_b32 a79, 0\n\t" \
  "v_accvgpr_write_b32 a80, 0\n\t" \
  "v_accvgpr_write_b32 a81, 0\n\t" \
  "v_accvgpr_write_b32 a82, 0\n\t" \
  "v_accvgpr_write_b32 a83, 0\n\t" \
  "v_accvgpr_write_b32 a84, 0\n\t" \
  "v_accvgpr_write_b32 a85, 0\n\t" \
  "v_accvgpr_write_b32 a86, 0\n\t" \
  "v_accvgpr_write_b32 a87, 0\n\t" \
  "v_accvgpr_write_b32 a88, 0\n\t" \
  "v_accvgpr_write_b32 a89, 0\n\t" \
  "v_accvgpr_write_b32 a90, 0\n\t" \
  "v_accvgpr_write_b32 a91, 0\n\t" \
  "v_accvgpr_write_b32 a92, 0\n\t" \
  "v_accvgpr_write_b32 a93, 0\n\t" \
  "v_accvgpr_write_b32 a94, 0\n\t" \
  "v_accvgpr_write_b32 a95, 0\n\t" \
  "v_accvgpr_write_b32 a96, 0\n\t" \
  "v_accvgpr_write_b32 a97, 0\n\t" \
  "v_accvgpr_write_b32 a98, 0\n\t" \
  "v_accvgpr_write_b32 a99, 0\n\t" \
  "v_accvgpr_write_b32 a100, 0\n\t" \
  "v_accvgpr_write_b32 a101, 0\n\t" \
  "v_accvgpr_write_b32 a102, 0\n\t" \
  "v_accvgpr_write_b32 a103, 0\n\t" \
  "v_accvgpr_write_b32 a104, 0\n\t" \
  "v_accvgpr_write_b32 a105, 0\n\t" \
  "v_accvgpr_write_b32 a106, 0\n\t" \
  "v_accvgpr_write_b32 a107, 0\n\t" \
  "v_accvgpr_write_b32 a108, 0\n\t" \
  "v_accvgpr_write_b32 a109, 0\n\t" \
  "v_accvgpr_write_b32 a110, 0\n\t" \
  "v_accvgpr_write_b32 a111, 0\n\t" \
  "v_accvgpr_write_b32 a112, 0\n\t" \
  "v_accvgpr_write_b32 a113, 0\n\t" \
  "v_accvgpr_write_b32 a114, 0\n\t" \
  "v_accvgpr_write_b32 a115, 0\n\t" \
  "v_accvgpr_write_b32 a116, 0\n\t" \
  "v_accvgpr_write_b32 a117, 0\n\t" \
  "v_accvgpr_write_b32 a118, 0\n\t" \
  "v_accvgpr_write_b32 a119, 0\n\t" \
  "v_accvgpr_write_b32 a120, 0\n\t" \
  "v_accvgpr_write_b32 a121, 0\n\t" \
  "v_accvgpr_write_b32 a122, 0\n\t" \
  "v_accvgpr_write_b32 a123, 0\n\t" \
  "v_accvgpr_write_b32 a124, 0\n\t" \
  "v_accvgpr_write_b32 a125, 0\n\t" \
  "v_accvgpr_write_b32 a126, 0\n\t" \
  "v_accvgpr_write_b32 a127, 0\n\t" \
  "v_accvgpr_write_b32 a128, 0\n\t" \
  "v_accvgpr_write_b32 a129, 0\n\t" \
  "v_accvgpr_write_b32 a130, 0\n\t" \
  "v_accvgpr_write_b32 a131, 0\n\t" \
  "v_accvgpr_write_b32 a132, 0\n\t" \
  "v_accvgpr_write_b32 a133, 0\n\t" \
  "v_accvgpr_write_b32 a134, 0\n\t" \
  "v_accvgpr_write_b32 a135, 0\n\t" \
  "v_accvgpr_write_b32 a136, 0\n\t" \
  "v_accvgpr_write_b32 a137, 0\n\t" \
  "v_accvgpr_write_b32 a138, 0\n\t" \
  "v_accvgpr_write_b32 a139, 0\n\t" \
  "v_accvgpr_write_b32 a140, 0\n\t" \
  "v_accvgpr_write_b32 a141, 0\n\t" \
  "v_accvgpr_write_b32 a142, 0\n\t" \
  "v_accvgpr_write_b32 a143, 0\n\t" \
  "v_accvgpr_write_b32 a144, 0\n\t" \
  "v_accvgpr_write_b32 a145, 0\n\t" \
  "v_accvgpr_write_b32 a146, 0\n\t" \
  "v_accvgpr_write_b32 a147, 0\n\t" \
  "v_accvgpr_write_b32 a148, 0\n\t" \
  "v_accvgpr_write_b32 a149, 0\n\t" \
  "v_accvgpr_write_b32 a150, 0\n\t" \
  "v_accvgpr_write_b32 a151, 0\n\t" \
  "v_accvgpr_write_b32 a152, 0\n\t" \
  "v_accvgpr_write_b32 a153, 0\n\t" \
  "v_accvgpr_write_b32 a154, 0\n\t" \
  "v_accvgpr_write_b32 a155, 0\n\t" \
  "v_accvgpr_write_b32 a156, 0\n\t" \
  "v_accvgpr_write_b32 a157, 0\n\t" \
  "v_accvgpr_write_b32 a158, 0\n\t" \
  "v_accvgpr_write_b32 a159, 0\n\t" \
  "v_accvgpr_write_b32 a160, 0\n\t" \
  "v_accvgpr_write_b32 a161, 0\n\t" \
  "v_accvgpr_write_b32 a162, 0\n\t" \
  "v_accvgpr_write_b32 a163, 0\n\t" \
  "v_accvgpr_write_b32 a164, 0\n\t" \
  "v_accvgpr_write_b32 a165, 0\n\t" \
  "v_accvgpr_write_b32 a166, 0\n\t" \
  "v_accvgpr_write_b32 a167, 0\n\t" \
  "v_accvgpr_write_b32 a168, 0\n\t" \
  "v_accvgpr_write_b32 a169, 0\n\t" \
  "v_accvgpr_write_b32 a170, 0\n\t" \
  "v_accvgpr_write_b32 a171, 0\n\t" \
  "v_accvgpr_write_b32 a172, 0\n\t" \
  "v_accvgpr_write_b32 a173, 0\n\t" \
  "v_accvgpr_write_b32 a174, 0\n\t" \
  "v_accvgpr_write_b32 a175, 0\n\t" \
  "v_accvgpr_write_b32 a176, 0\n\t" \
  "v_accvgpr_write_b32 a177, 0\n\t" \
  "v_accvgpr_write_b32 a178, 0\n\t" \
  "v_accvgpr_write_b32 a179, 0\n\t" \
  "v_accvgpr_write_b32 a180, 0\n\t" \
  "v_accvgpr_write_b32 a181, 0\n\t" \
  "v_accvgpr_write_b32 a182, 0\n\t" \
  "v_accvgpr_write_b32 a183, 0\n\t" \
  "v_accvgpr_write_b32 a184, 0\n\t" \
  "v_accvgpr_write_b32 a185, 0\n\t" \
  "v_accvgpr_write_b32 a186, 0\n\t" \
  "v_accvgpr_write_b32 a187, 0\n\t" \
  "v_accvgpr_write_b32 a188, 0\n\t" \
  "v_accvgpr_write_b32 a189, 0\n\t" \
  "v_accvgpr_write_b32 a190, 0\n\t" \
  "v_accvgpr_write_b32 a191, 0\n\t" \
  "v_accvgpr_write_b32 a192, 0\n\t" \
  "v_accvgpr_write_b32 a193, 0\n\t" \
  "v_accvgpr_write_b32 a194, 0\n\t" \
  "v_accvgpr_write_b32 a195, 0\n\t" \
  "v_accvgpr_write_b32 a196, 0\n\t" \
  "v_accvgpr_write_b32 a197, 0\n\t" \
  "v_accvgpr_write_b32 a198, 0\n\t" \
  "v_accvgpr_write_b32 a199, 0\n\t" \
  "v_accvgpr_write_b32 a200, 0\n\t" \
  "v_accvgpr_write_b32 a201, 0\n\t" \
  "v_accvgpr_write_b32 a202, 0\n\t" \
  "v_accvgpr_write_b32 a203, 0\n\t" \
  "v_accvgpr_write_b32 a204, 0\n\t" \
  "v_accvgpr_write_b32 a205, 0\n\t" \
  "v_accvgpr_write_b32 a206, 0\n\t" \
  "v_accvgpr_write_b32 a207, 0\n\t" \
  "v_accvgpr_write_b32 a208, 0\n\t" \
  "v_accvgpr_write_b32 a209, 0\n\t" \
  "v_accvgpr_write_b32 a210, 0\n\t" \
  "v_accvgpr_write_b32 a211, 0\n\t" \
  "v_accvgpr_write_b32 a212, 0\n\t" \
  "v_accvgpr_write_b32 a213, 0\n\t" \
  "v_accvgpr_write_b32 a214, 0\n\t" \
  "v_accvgpr_write_b32 a215, 0\n\t" \
  "v_accvgpr_write_b32 a216, 0\n\t" \
  "v_accvgpr_write_b32 a217, 0\n\t" \
  "v_accvgpr_write_b32 a218, 0\n\t" \
  "v_accvgpr_write_b32 a219, 0\n\t" \
  "v_accvgpr_write_b32 a220, 0\n\t" \
  "v_accvgpr_write_b32 a221, 0\n\t" \
  "v_accvgpr_write_b32 a222, 0\n\t" \
  "v_accvgpr_write_b32 a223, 0\n\t" \
  "v_accvgpr_write_b32 a224, 0\n\t" \
  "v_accvgpr_write_b32 a225, 0\n\t" \
  "v_accvgpr_write_b32 a226, 0\n\t" \
  "v_accvgpr_write_b32 a227, 0\n\t" \
  "v_accvgpr_write_b32 a228, 0\n\t" \
  "v_accvgpr_write_b32 a229, 0\n\t" \
  "v_accvgpr_write_b32 a230, 0\n\t" \
  "v_accvgpr_write_b32 a231, 0\n\t" \
  "v_accvgpr_write_b32 a232, 0\n\t" \
  "v_accvgpr_write_b32 a233, 0\n\t" \
  "v_accvgpr_write_b32 a234, 0\n\t" \
  "v_accvgpr_write_b32 a235, 0\n\t" \
  "v_accvgpr_write_b32 a236, 0\n\t" \
  "v_accvgpr_write_b32 a237, 0\n\t" \
  "v_accvgpr_write_b32 a238, 0\n\t" \
  "v_accvgpr_write_b32 a239, 0\n\t" \
  "v_accvgpr_write_b32 a240, 0\n\t" \
  "v_accvgpr_write_b32 a241, 0\n\t" \
  "v_accvgpr_write_b32 a242, 0\n\t" \
  "v_accvgpr_write_b32 a243, 0\n\t" \
  "v_accvgpr_write_b32 a244, 0\n\t" \
  "v_accvgpr_write_b32 a245, 0\n\t" \
  "v_accvgpr_write_b32 a246, 0\n\t" \
  "v_accvgpr_write_b32 a247, 0\n\t" \
  "v_accvgpr_write_b32 a248, 0\n\t" \
  "v_accvgpr_write_b32 a249, 0\n\t" \
  "v_accvgpr_write_b32 a250, 0\n\t" \
  "v_accvgpr_write_b32 a251, 0\n\t" \
  "v_accvgpr_write_b32 a252, 0\n\t" \
  "v_accvgpr_write_b32 a253, 0\n\t" \
  "v_accvgpr_write_b32 a254, 0\n\t" \
  "v_accvgpr_write_b32 a255, 0\n\t" \
  "s_bitcmp1_b32 s77, 0\n\t" \
  "s_cbranch_scc0 Lea1_%=\n\t" \
  "s_waitcnt vmcnt(0)\n\t" \
  "s_getpc_b64 s[74:75]\n\t" \
  "s_branch Lflag_%=\n\t" \
  "Lea1_%=:\n\t" \
  "buffer_load_dwordx4 v[80:83], v4, %[srdW], %[sW] offen\n\t" \
  "buffer_load_dwordx4 v[84:87], v5, %[srdW], %[sW] offen\n\t" \
  "buffer_load_dwordx4 v[88:91], v6, %[srdW], %[sW] offen\n\t" \
  "buffer_load_dwordx4 v[92:95], v7, %[srdW], %[sW] offen\n\t" \
  "buffer_load_dwordx4 v[96:99], v8, %[srdW], %[sW] offen\n\t" \
  "buffer_load_dwordx4 v[100:103], v9, %[srdW], %[sW] offen\n\t" \
  "buffer_load_dwordx4 v[104:107], v10, %[srdW], %[sW] offen\n\t" \
  "buffer_load_dwordx4 v[108:111], v11, %[srdW], %[sW] offen\n\t" \
  "s_add_u32 %[sW], %[sW], 2097152\n\t" \
  "buffer_load_dwordx4 v[112:115], v4, %[srdW], %[sW] offen\n\t" \
  "buffer_load_dwordx4 v[116:119], v5, %[srdW], %[sW] offen\n\t" \
  "buffer_load_dwordx4 v[120:123], v6, %[srdW], %[sW] offen\n\t" \
  "buffer_load_dwordx4 v[124:127], v7, %[srdW], %[sW] offen\n\t" \
  "buffer_load_dwordx4 v[128:131], v8, %[srdW], %[sW] offen\n\t" \
  "buffer_load_dwordx4 v[132:135], v9, %[srdW], %[sW] offen\n\t" \
  "buffer_load_dwordx4 v[136:139], v10, %[srdW], %[sW] offen\n\t" \
  "buffer_load_dwordx4 v[140:143], v11, %[srdW], %[sW] offen\n\t" \
  "s_add_u32 %[sW], %[sW], 2097152\n\t" \
  "buffer_load_dwordx4 v[144:147], v4, %[srdW], %[sW] offen\n\t" \
  "buffer_load_dwordx4 v[148:151], v5, %[srdW], %[sW] offen\n\t" \
  "buffer_load_dwordx4 v[152:155], v6, %[srdW], %[sW] offen\n\t" \
  "buffer_load_dwordx4 v[156:159], v7, %[srdW], %[sW] offen\n\t" \
  "buffer_load_dwordx4 v[160:163], v8, %[srdW], %[sW] offen\n\t" \
  "buffer_load_dwordx4 v[164:167], v9, %[srdW], %[sW] offen\n\t" \
  "buffer_load_dwordx4 v[168:171], v10, %[srdW], %[sW] offen\n\t" \
  "buffer_load_dwordx4 v[172:175], v11, %[srdW], %[sW] offen\n\t" \
  "s_add_u32 %[sW], %[sW], 2097152\n\t" \
  "v_add_u32 v225, 2097152, %[vPL]\n\t" \
  "buffer_load_dword v226, v225, %[srdW], 0 offen sc1\n\t" \
  "v_add_u32 v225, 14680064, %[vPL]\n\t" \
  "s_cmp_lt_u32 %[sig], 5\n\t" \
  "s_cbranch_scc1 Lea2_%=\n\t" \
  "v_accvgpr_write_b32 a0, 0\n\t" \
  "v_accvgpr_write_b32 a1, 0\n\t" \
  "v_accvgpr_write_b32 a2, 0\n\t" \
  "v_accvgpr_write_b32 a3, 0\n\t" \
  "v_accvgpr_write_b32 a4, 0\n\t" \
  "v_accvgpr_write_b32 a5, 0\n\t" \
  "v_accvgpr_write_b32 a6, 0\n\t" \
  "v_accvgpr_write_b32 a7, 0\n\t" \
  "v_accvgpr_write_b32 a8, 0\n\t" \
  "v_accvgpr_write_b32 a9, 0\n\t" \
  "v_accvgpr_write_b32 a10, 0\n\t" \
  "v_accvgpr_write_b32 a11, 0\n\t" \
  "v_accvgpr_write_b32 a12, 0\n\t" \
  "v_accvgpr_write_b32 a13, 0\n\t" \
  "v_accvgpr_write_b32 a14, 0\n\t" \
  "v_accvgpr_write_b32 a15, 0\n\t" \
  "v_accvgpr_write_b32 a16, 0\n\t" \
  "v_accvgpr_write_b32 a17, 0\n\t" \
  "v_accvgpr_write_b32 a18, 0\n\t" \
  "v_accvgpr_write_b32 a19, 0\n\t" \
  "v_accvgpr_write_b32 a20, 0\n\t" \
  "v_accvgpr_write_b32 a21, 0\n\t" \
  "v_accvgpr_write_b32 a22, 0\n\t" \
  "v_accvgpr_write_b32 a23, 0\n\t" \
  "v_accvgpr_write_b32 a24, 0\n\t" \
  "v_accvgpr_write_b32 a25, 0\n\t" \
  "v_accvgpr_write_b32 a26, 0\n\t" \
  "v_accvgpr_write_b32 a27, 0\n\t" \
  "v_accvgpr_write_b32 a28, 0\n\t" \
  "v_accvgpr_write_b32 a29, 0\n\t" \
  "v_accvgpr_write_b32 a30, 0\n\t" \
  "v_accvgpr_write_b32 a31, 0\n\t" \
  "v_accvgpr_write_b32 a32, 0\n\t" \
  "v_accvgpr_write_b32 a33, 0\n\t" \
  "v_accvgpr_write_b32 a34, 0\n\t" \
  "v_accvgpr_write_b32 a35, 0\n\t" \
  "v_accvgpr_write_b32 a36, 0\n\t" \
  "v_accvgpr_write_b32 a37, 0\n\t" \
  "v_accvgpr_write_b32 a38, 0\n\t" \
  "v_accvgpr_write_b32 a39, 0\n\t" \
  "v_accvgpr_write_b32 a40, 0\n\t" \
  "v_accvgpr_write_b32 a41, 0\n\t" \
  "v_accvgpr_write_b32 a42, 0\n\t" \
  "v_accvgpr_write_b32 a43, 0\n\t" \
  "v_accvgpr_write_b32 a44, 0\n\t" \
  "v_accvgpr_write_b32 a45, 0\n\t" \
  "v_accvgpr_write_b32 a46, 0\n\t" \
  "v_accvgpr_write_b32 a47, 0\n\t" \
  "v_accvgpr_write_b32 a48, 0\n\t" \
  "v_accvgpr_write_b32 a49, 0\n\t" \
  "v_accvgpr_write_b32 a50, 0\n\t" \
  "v_accvgpr_write_b32 a51, 0\n\t" \
  "v_accvgpr_write_b32 a52, 0\n\t" \
  "v_accvgpr_write_b32 a53, 0\n\t" \
  "v_accvgpr_write_b32 a54, 0\n\t" \
  "v_accvgpr_write_b32 a55, 0\n\t" \
  "v_accvgpr_write_b32 a56, 0\n\t" \
  "v_accvgpr_write_b32 a57, 0\n\t" \
  "v_accvgpr_write_b32 a58, 0\n\t" \
  "v_accvgpr_write_b32 a59, 0\n\t" \
  "v_accvgpr_write_b32 a60, 0\n\t" \
  "v_accvgpr_write_b32 a61, 0\n\t" \
  "v_accvgpr_write_b32 a62, 0\n\t" \
  "v_accvgpr_write_b32 a63, 0\n\t" \
  "v_accvgpr_write_b32 a64, 0\n\t" \
  "v_accvgpr_write_b32 a65, 0\n\t" \
  "v_accvgpr_write_b32 a66, 0\n\t" \
  "v_accvgpr_write_b32 a67, 0\n\t" \
  "v_accvgpr_write_b32 a68, 0\n\t" \
  "v_accvgpr_write_b32 a69, 0\n\t" \
  "v_accvgpr_write_b32 a70, 0\n\t" \
  "v_accvgpr_write_b32 a71, 0\n\t" \
  "v_accvgpr_write_b32 a72, 0\n\t" \
  "v_accvgpr_write_b32 a73, 0\n\t" \
  "v_accvgpr_write_b32 a74, 0\n\t" \
  "v_accvgpr_write_b32 a75, 0\n\t" \
  "v_accvgpr_write_b32 a76, 0\n\t" \
  "v_accvgpr_write_b32 a77, 0\n\t" \
  "v_accvgpr_write_b32 a78, 0\n\t" \
  "v_accvgpr_write_b32 a79, 0\n\t" \
  "v_accvgpr_write_b32 a80, 0\n\t" \
  "v_accvgpr_write_b32 a81, 0\n\t" \
  "v_accvgpr_write_b32 a82, 0\n\t" \
  "v_accvgpr_write_b32 a83, 0\n\t" \
  "v_accvgpr_write_b32 a84, 0\n\t" \
  "v_accvgpr_write_b32 a85, 0\n\t" \
  "v_accvgpr_write_b32 a86, 0\n\t" \
  "v_accvgpr_write_b32 a87, 0\n\t" \
  "v_accvgpr_write_b32 a88, 0\n\t" \
  "v_accvgpr_write_b32 a89, 0\n\t" \
  "v_accvgpr_write_b32 a90, 0\n\t" \
  "v_accvgpr_write_b32 a91, 0\n\t" \
  "v_accvgpr_write_b32 a92, 0\n\t" \
  "v_accvgpr_write_b32 a93, 0\n\t" \
  "v_accvgpr_write_b32 a94, 0\n\t" \
  "v_accvgpr_write_b32 a95, 0\n\t" \
  "v_accvgpr_write_b32 a96, 0\n\t" \
  "v_accvgpr_write_b32 a97, 0\n\t" \
  "v_accvgpr_write_b32 a98, 0\n\t" \
  "v_accvgpr_write_b32 a99, 0\n\t" \
  "v_accvgpr_write_b32 a100, 0\n\t" \
  "v_accvgpr_write_b32 a101, 0\n\t" \
  "v_accvgpr_write_b32 a102, 0\n\t" \
  "v_accvgpr_write_b32 a103, 0\n\t" \
  "v_accvgpr_write_b32 a104, 0\n\t" \
  "v_accvgpr_write_b32 a105, 0\n\t" \
  "v_accvgpr_write_b32 a106, 0\n\t" \
  "v_accvgpr_write_b32 a107, 0\n\t" \
  "v_accvgpr_write_b32 a108, 0\n\t" \
  "v_accvgpr_write_b32 a109, 0\n\t" \
  "v_accvgpr_write_b32 a110, 0\n\t" \
  "v_accvgpr_write_b32 a111, 0\n\t" \
  "v_accvgpr_write_b32 a112, 0\n\t" \
  "v_accvgpr_write_b32 a113, 0\n\t" \
  "v_accvgpr_write_b32 a114, 0\n\t" \
  "v_accvgpr_write_b32 a115, 0\n\t" \
  "v_accvgpr_write_b32 a116, 0\n\t" \
  "v_accvgpr_write_b32 a117, 0\n\t" \
  "v_accvgpr_write_b32 a118, 0\n\t" \
  "v_accvgpr_write_b32 a119, 0\n\t" \
  "v_accvgpr_write_b32 a120, 0\n\t" \
  "v_accvgpr_write_b32 a121, 0\n\t" \
  "v_accvgpr_write_b32 a122, 0\n\t" \
  "v_accvgpr_write_b32 a123, 0\n\t" \
  "v_accvgpr_write_b32 a124, 0\n\t" \
  "v_accvgpr_write_b32 a125, 0\n\t" \
  "v_accvgpr_write_b32 a126, 0\n\t" \
  "v_accvgpr_write_b32 a127, 0\n\t" \
  "v_accvgpr_write_b32 a128, 0\n\t" \
  "v_accvgpr_write_b32 a129, 0\n\t" \
  "v_accvgpr_write_b32 a130, 0\n\t" \
  "v_accvgpr_write_b32 a131, 0\n\t" \
  "v_accvgpr_write_b32 a132, 0\n\t" \
  "v_accvgpr_write_b32 a133, 0\n\t" \
  "v_accvgpr_write_b32 a134, 0\n\t" \
  "v_accvgpr_write_b32 a135, 0\n\t" \
  "v_accvgpr_write_b32 a136, 0\n\t" \
  "v_accvgpr_write_b32 a137, 0\n\t" \
  "v_accvgpr_write_b32 a138, 0\n\t" \
  "v_accvgpr_write_b32 a139, 0\n\t" \
  "v_accvgpr_write_b32 a140, 0\n\t" \
  "v_accvgpr_write_b32 a141, 0\n\t" \
  "v_accvgpr_write_b32 a142, 0\n\t" \
  "v_accvgpr_write_b32 a143, 0\n\t" \
  "v_accvgpr_write_b32 a144, 0\n\t" \
  "v_accvgpr_write_b32 a145, 0\n\t" \
  "v_accvgpr_write_b32 a146, 0\n\t" \
  "v_accvgpr_write_b32 a147, 0\n\t" \
  "v_accvgpr_write_b32 a148, 0\n\t" \
  "v_accvgpr_write_b32 a149, 0\n\t" \
  "v_accvgpr_write_b32 a150, 0\n\t" \
  "v_accvgpr_write_b32 a151, 0\n\t" \
  "v_accvgpr_write_b32 a152, 0\n\t" \
  "v_accvgpr_write_b32 a153, 0\n\t" \
  "v_accvgpr_write_b32 a154, 0\n\t" \
  "v_accvgpr_write_b32 a155, 0\n\t" \
  "v_accvgpr_write_b32 a156, 0\n\t" \
  "v_accvgpr_write_b32 a157, 0\n\t" \
  "v_accvgpr_write_b32 a158, 0\n\t" \
  "v_accvgpr_write_b32 a159, 0\n\t" \
  "v_accvgpr_write_b32 a160, 0\n\t" \
  "v_accvgpr_write_b32 a161, 0\n\t" \
  "v_accvgpr_write_b32 a162, 0\n\t" \
  "v_accvgpr_write_b32 a163, 0\n\t" \
  "v_accvgpr_write_b32 a164, 0\n\t" \
  "v_accvgpr_write_b32 a165, 0\n\t" \
  "v_accvgpr_write_b32 a166, 0\n\t" \
  "v_accvgpr_write_b32 a167, 0\n\t" \
  "v_accvgpr_write_b32 a168, 0\n\t" \
  "v_accvgpr_write_b32 a169, 0\n\t" \
  "v_accvgpr_write_b32 a170, 0\n\t" \
  "v_accvgpr_write_b32 a171, 0\n\t" \
  "v_accvgpr_write_b32 a172, 0\n\t" \
  "v_accvgpr_write_b32 a173, 0\n\t" \
  "v_accvgpr_write_b32 a174, 0\n\t" \
  "v_accvgpr_write_b32 a175, 0\n\t" \
  "v_accvgpr_write_b32 a176, 0\n\t" \
  "v_accvgpr_write_b32 a177, 0\n\t" \
  "v_accvgpr_write_b32 a178, 0\n\t" \
  "v_accvgpr_write_b32 a179, 0\n\t" \
  "v_accvgpr_write_b32 a180, 0\n\t" \
  "v_accvgpr_write_b32 a181, 0\n\t" \
  "v_accvgpr_write_b32 a182, 0\n\t" \
  "v_accvgpr_write_b32 a183, 0\n\t" \
  "v_accvgpr_write_b32 a184, 0\n\t" \
  "v_accvgpr_write_b32 a185, 0\n\t" \
  "v_accvgpr_write_b32 a186, 0\n\t" \
  "v_accvgpr_write_b32 a187, 0\n\t" \
  "v_accvgpr_write_b32 a188, 0\n\t" \
  "v_accvgpr_write_b32 a189, 0\n\t" \
  "v_accvgpr_write_b32 a190, 0\n\t" \
  "v_accvgpr_write_b32 a191, 0\n\t" \
  "v_accvgpr_write_b32 a192, 0\n\t" \
  "v_accvgpr_write_b32 a193, 0\n\t" \
  "v_accvgpr_write_b32 a194, 0\n\t" \
  "v_accvgpr_write_b32 a195, 0\n\t" \
  "v_accvgpr_write_b32 a196, 0\n\t" \
  "v_accvgpr_write_b32 a197, 0\n\t" \
  "v_accvgpr_write_b32 a198, 0\n\t" \
  "v_accvgpr_write_b32 a199, 0\n\t" \
  "v_accvgpr_write_b32 a200, 0\n\t" \
  "v_accvgpr_write_b32 a201, 0\n\t" \
  "v_accvgpr_write_b32 a202, 0\n\t" \
  "v_accvgpr_write_b32 a203, 0\n\t" \
  "v_accvgpr_write_b32 a204, 0\n\t" \
  "v_accvgpr_write_b32 a205, 0\n\t" \
  "v_accvgpr_write_b32 a206, 0\n\t" \
  "v_accvgpr_write_b32 a207, 0\n\t" \
  "v_accvgpr_write_b32 a208, 0\n\t" \
  "v_accvgpr_write_b32 a209, 0\n\t" \
  "v_accvgpr_write_b32 a210, 0\n\t" \
  "v_accvgpr_write_b32 a211, 0\n\t" \
  "v_accvgpr_write_b32 a212, 0\n\t" \
  "v_accvgpr_write_b32 a213, 0\n\t" \
  "v_accvgpr_write_b32 a214, 0\n\t" \
  "v_accvgpr_write_b32 a215, 0\n\t" \
  "v_accvgpr_write_b32 a216, 0\n\t" \
  "v_accvgpr_write_b32 a217, 0\n\t" \
  "v_accvgpr_write_b32 a218, 0\n\t" \
  "v_accvgpr_write_b32 a219, 0\n\t" \
  "v_accvgpr_write_b32 a220, 0\n\t" \
  "v_accvgpr_write_b32 a221, 0\n\t" \
  "v_accvgpr_write_b32 a222, 0\n\t" \
  "v_accvgpr_write_b32 a223, 0\n\t" \
  "v_accvgpr_write_b32 a224, 0\n\t" \
  "v_accvgpr_write_b32 a225, 0\n\t" \
  "v_accvgpr_write_b32 a226, 0\n\t" \
  "v_accvgpr_write_b32 a227, 0\n\t" \
  "v_accvgpr_write_b32 a228, 0\n\t" \
  "v_accvgpr_write_b32 a229, 0\n\t" \
  "v_accvgpr_write_b32 a230, 0\n\t" \
  "v_accvgpr_write_b32 a231, 0\n\t" \
  "v_accvgpr_write_b32 a232, 0\n\t" \
  "v_accvgpr_write_b32 a233, 0\n\t" \
  "v_accvgpr_write_b32 a234, 0\n\t" \
  "v_accvgpr_write_b32 a235, 0\n\t" \
  "v_accvgpr_write_b32 a236, 0\n\t" \
  "v_accvgpr_write_b32 a237, 0\n\t" \
  "v_accvgpr_write_b32 a238, 0\n\t" \
  "v_accvgpr_write_b32 a239, 0\n\t" \
  "v_accvgpr_write_b32 a240, 0\n\t" \
  "v_accvgpr_write_b32 a241, 0\n\t" \
  "v_accvgpr_write_b32 a242, 0\n\t" \
  "v_accvgpr_write_b32 a243, 0\n\t" \
  "v_accvgpr_write_b32 a244, 0\n\t" \
  "v_accvgpr_write_b32 a245, 0\n\t" \
  "v_accvgpr_write_b32 a246, 0\n\t" \
  "v_accvgpr_write_b32 a247, 0\n\t" \
  "v_accvgpr_write_b32 a248, 0\n\t" \
  "v_accvgpr_write_b32 a249, 0\n\t" \
  "v_accvgpr_write_b32 a250, 0\n\t" \
  "v_accvgpr_write_b32 a251, 0\n\t" \
  "v_accvgpr_write_b32 a252, 0\n\t" \
  "v_accvgpr_write_b32 a253, 0\n\t" \
  "v_accvgpr_write_b32 a254, 0\n\t" \
  "v_accvgpr_write_b32 a255, 0\n\t" \
  "Lea2_%=:\n\t" \
  "s_getpc_b64 s[86:87]\n\t" \
  "s_branch Lspin_%=\n\t" \
  "s_mov_b32 s53, s59\n\t" \
  "buffer_load_dwordx4 v[176:179], v12, %[srdX], %[sX] offen sc0 sc1\n\t" \
  "buffer_load_dwordx4 v[180:183], v12, %[srdX], %[sX] offen offset:1024 sc0 sc1\n\t" \
  "buffer_load_dwordx3 v[184:186], %[vL12], %[srdX], %[sX] offen offset:2048 sc0 sc1\n\t" \
  "buffer_load_dword v187, %[vL4], %[srdX], %[sX] offen offset:2816 sc0 sc1\n\t" \
  "s_add_u32 %[sX], %[sX], 12288\n\t" \
  "s_add_u32 %[sM], %[sM], 512\n\t" \
  "s_getpc_b64 s[86:87]\n\t" \
  "s_branch Lspin_%=\n\t" \
  "s_mov_b32 s55, s59\n\t" \
  "buffer_load_dwordx4 v[188:191], v12, %[srdX], %[sX] offen sc0 sc1\n\t" \
  "buffer_load_dwordx4 v[192:195], v12, %[srdX], %[sX] offen offset:1024 sc0 sc1\n\t" \
  "buffer_load_dwordx3 v[196:198], %[vL12], %[srdX], %[sX] offen offset:2048 sc0 sc1\n\t" \
  "buffer_load_dword v199, %[vL4], %[srdX], %[sX] offen offset:2816 sc0 sc1\n\t" \
  "s_add_u32 %[sX], %[sX], 12288\n\t" \
  "s_add_u32 %[sM], %[sM], 512\n\t" \
  "s_waitcnt vmcnt(4)\n\t" \
  "s_and_b32 s62, s53, 0x7f807f80\n\t" \
  "v_and_b32 v200, s63, v184\n\t" \
  "v_alignbit_b32 v201, v184, v184, 3\n\t" \
  "v_perm_b32 v204, v200, v176, s84\n\t" \
  "v_and_b32 v201, s63, v201\n\t" \
  "v_perm_b32 v205, v200, v176, s85\n\t" \
  "v_add_u32 v204, s62, v204\n\t" \
  "v_perm_b32 v206, v201, v177, s84\n\t" \
  "v_add_u32 v205, s62, v205\n\t" \
  "v_perm_b32 v207, v201, v177, s85\n\t" \
  "v_add_u32 v206, s62, v206\n\t" \
  "v_add_u32 v207, s62, v207\n\t" \
  "ds_write_b128 v13, v[204:207] offset:0\n\t" \
  "v_and_b32 v202, s63, v185\n\t" \
  "v_alignbit_b32 v203, v185, v185, 3\n\t" \
  "v_perm_b32 v208, v202, v178, s84\n\t" \
  "v_and_b32 v203, s63, v203\n\t" \
  "v_perm_b32 v209, v202, v178, s85\n\t" \
  "v_add_u32 v208, s62, v208\n\t" \
  "v_perm_b32 v210, v203, v179, s84\n\t" \
  "v_add_u32 v209, s62, v209\n\t" \
  "v_perm_b32 v211, v203, v179, s85\n\t" \
  "v_add_u32 v210, s62, v210\n\t" \
  "v_add_u32 v211, s62, v211\n\t" \
  "ds_write_b128 v13, v[208:211] offset:1024\n\t" \
  "v_and_b32 v200, s63, v186\n\t" \
  "v_alignbit_b32 v201, v186, v186, 3\n\t" \
  "v_perm_b32 v212, v200, v180, s84\n\t" \
  "v_and_b32 v201, s63, v201\n\t" \
  "v_perm_b32 v213, v200, v180, s85\n\t" \
  "v_add_u32 v212, s62, v212\n\t" \
  "v_perm_b32 v214, v201, v181, s84\n\t" \
  "v_add_u32 v213, s62, v213\n\t" \
  "v_perm_b32 v215, v201, v181, s85\n\t" \
  "v_add_u32 v214, s62, v214\n\t" \
  "v_add_u32 v215, s62, v215\n\t" \
  "ds_write_b128 v13, v[212:215] offset:2048\n\t" \
  "v_lshrrev_b32 v202, 5, v184\n\t" \
  "v_lshlrev_b32 v204, 2, v186\n\t" \
  "v_lshrrev_b32 v203, 5, v185\n\t" \
  "v_lshlrev_b32 v205, 1, v186\n\t" \
  "v_bfi_b32 v202, s64, v204, v202\n\t" \
  "v_bfi_b32 v203, s64, v205, v203\n\t" \
  "v_and_b32 v202, s63, v202\n\t" \
  "v_perm_b32 v204, v202, v182, s84\n\t" \
  "v_and_b32 v203, s63, v203\n\t" \
  "v_perm_b32 v205, v202, v182, s85\n\t" \
  "v_add_u32 v204, s62, v204\n\t" \
  "v_perm_b32 v206, v203, v183, s84\n\t" \
  "v_add_u32 v205, s62, v205\n\t" \
  "v_perm_b32 v207, v203, v183, s85\n\t" \
  "v_add_u32 v206, s62, v206\n\t" \
  "v_add_u32 v207, s62, v207\n\t" \
  "v_lshrrev_b32 v200, 16, v187\n\t" \
  "ds_write_b128 v13, v[204:207] offset:3072\n\t" \
  "s_bfm_b64 exec, s53, 0\n\t" \
  "s_cbranch_execz Lst0_%=\n\t" \
  "ds_write_b16 v200, v187 offset:0\n\t" \
  "Lbk0_%=:\n\t" \
  "s_mov_b64 exec, -1\n\t" \
  "v_cvt_pk_bf16_f32 v48, v80, v84\n\t" \
  "v_cvt_pk_bf16_f32 v52, v81, v85\n\t" \
  "v_cvt_pk_bf16_f32 v56, v82, v86\n\t" \
  "v_cvt_pk_bf16_f32 v60, v83, v87\n\t" \
  "v_cvt_pk_bf16_f32 v49, v88, v92\n\t" \
  "v_cvt_pk_bf16_f32 v53, v89, v93\n\t" \
  "v_cvt_pk_bf16_f32 v57, v90, v94\n\t" \
  "v_cvt_pk_bf16_f32 v61, v91, v95\n\t" \
  "v_cvt_pk_bf16_f32 v50, v96, v100\n\t" \
  "v_cvt_pk_bf16_f32 v54, v97, v101\n\t" \
  "v_cvt_pk_bf16_f32 v58, v98, v102\n\t" \
  "v_cvt_pk_bf16_f32 v62, v99, v103\n\t" \
  "v_cvt_pk_bf16_f32 v51, v104, v108\n\t" \
  "v_cvt_pk_bf16_f32 v55, v105, v109\n\t" \
  "v_cvt_pk_bf16_f32 v59, v106, v110\n\t" \
  "v_cvt_pk_bf16_f32 v63, v107, v111\n\t" \
  "buffer_load_dwordx4 v[80:83], v4, %[srdW], %[sW] offen\n\t" \
  "buffer_load_dwordx4 v[84:87], v5, %[srdW], %[sW] offen\n\t" \
  "buffer_load_dwordx4 v[88:91], v6, %[srdW], %[sW] offen\n\t" \
  "buffer_load_dwordx4 v[92:95], v7, %[srdW], %[sW] offen\n\t" \
  "buffer_load_dwordx4 v[96:99], v8, %[srdW], %[sW] offen\n\t" \
  "buffer_load_dwordx4 v[100:103], v9, %[srdW], %[sW] offen\n\t" \
  "buffer_load_dwordx4 v[104:107], v10, %[srdW], %[sW] offen\n\t" \
  "buffer_load_dwordx4 v[108:111], v11, %[srdW], %[sW] offen\n\t" \
  "s_add_u32 %[sW], %[sW], 2097152\n\t" \
  "s_mov_b32 s68, 0\n\t" \
  "s_mov_b64 vcc, 0\n\t" \
  "s_getpc_b64 s[86:87]\n\t" \
  "s_branch Lspin_%=\n\t" \
  "s_mov_b32 s57, s59\n\t" \
  "buffer_load_dwordx4 v[176:179], v12, %[srdX], %[sX] offen sc0 sc1\n\t" \
  "buffer_load_dwordx4 v[180:183], v12, %[srdX], %[sX] offen offset:1024 sc0 sc1\n\t" \
  "buffer_load_dwordx3 v[184:186], %[vL12], %[srdX], %[sX] offen offset:2048 sc0 sc1\n\t" \
  "buffer_load_dword v187, %[vL4], %[srdX], %[sX] offen offset:2816 sc0 sc1\n\t" \
  "s_add_u32 %[sX], %[sX], 12288\n\t" \
  "s_add_u32 %[sM], %[sM], 512\n\t" \
  "s_load_dwordx2 s[52:53], %[ws], %[sM] glc\n\t" \
  "s_waitcnt lgkmcnt(0)\n\t" \
  "s_barrier\n\t" \
  "ds_read_b128 v[16:19], v12 offset:0\n\t" \
  "ds_read_b128 v[20:23], v12 offset:1024\n\t" \
  "ds_read_b128 v[24:27], v12 offset:2048\n\t" \
  "ds_read_b128 v[28:31], v12 offset:3072\n\t" \
  "ds_read_b128 v[32:35], v12 offset:4096\n\t" \
  "ds_read_b128 v[36:39], v12 offset:5120\n\t" \
  "ds_read_b128 v[40:43], v12 offset:6144\n\t" \
  "s_waitcnt lgkmcnt(5)\n\t" \
  ".p2align 6\n\t" \
  "Lloop_%=:\n\t" \
  "v_mfma_f32_16x16x32_bf16 a[0:3], v[48:51], v[16:19], a[0:3]\n\t" \
  "ds_read_b128 v[44:47], v12 offset:7168\n\t" \
  "v_cvt_pk_bf16_f32 v64, v112, v116\n\t" \
  "v_mfma_f32_16x16x32_bf16 a[4:7], v[52:55], v[16:19], a[4:7]\n\t" \
  "v_cvt_pk_bf16_f32 v68, v113, v117\n\t" \
  "v_cvt_pk_bf16_f32 v72, v114, v118\n\t" \
  "buffer_load_dword v226, v225, %[srdW], 0 offen sc1\n\t" \
  "v_mfma_f32_16x16x32_bf16 a[8:11], v[56:59], v[16:19], a[8:11]\n\t" \
  "v_cvt_pk_bf16_f32 v76, v115, v119\n\t" \
  "v_cvt_pk_bf16_f32 v65, v120, v124\n\t" \
  "v_add_u32 v225, 12582912, v225\n\t" \
  "v_mfma_f32_16x16x32_bf16 a[12:15], v[60:63], v[16:19], a[12:15]\n\t" \
  "v_cvt_pk_bf16_f32 v69, v121, v125\n\t" \
  "v_cvt_pk_bf16_f32 v73, v122, v126\n\t" \
  "v_mfma_f32_16x16x32_bf16 a[16:19], v[48:51], v[20:23], a[16:19]\n\t" \
  "ds_read_b128 v[16:19], v12 offset:8192\n\t" \
  "v_cvt_pk_bf16_f32 v77, v123, v127\n\t" \
  "v_mfma_f32_16x16x32_bf16 a[20:23], v[52:55], v[20:23], a[20:23]\n\t" \
  "v_cvt_pk_bf16_f32 v66, v128, v132\n\t" \
  "v_cvt_pk_bf16_f32 v70, v129, v133\n\t" \
  "v_mfma_f32_16x16x32_bf16 a[24:27], v[56:59], v[20:23], a[24:27]\n\t" \
  "v_cvt_pk_bf16_f32 v74, v130, v134\n\t" \
  "v_cvt_pk_bf16_f32 v78, v131, v135\n\t" \
  "v_mfma_f32_16x16x32_bf16 a[28:31], v[60:63], v[20:23], a[28:31]\n\t" \
  "s_waitcnt lgkmcnt(5)\n\t" \
  "v_cvt_pk_bf16_f32 v67, v136, v140\n\t" \
  "v_cvt_pk_bf16_f32 v71, v137, v141\n\t" \
  "v_mfma_f32_16x16x32_bf16 a[32:35], v[48:51], v[24:27], a[32:35]\n\t" \
  "ds_read_b128 v[20:23], v12 offset:9216\n\t" \
  "v_cvt_pk_bf16_f32 v75, v138, v142\n\t" \
  "v_mfma_f32_16x16x32_bf16 a[36:39], v[52:55], v[24:27], a[36:39]\n\t" \
  "v_cvt_pk_bf16_f32 v79, v139, v143\n\t" \
  "v_mfma_f32_16x16x32_bf16 a[40:43], v[56:59], v[24:27], a[40:43]\n\t" \
  "s_cbranch_vccnz Lst1_%=\n\t" \
  "s_waitcnt vmcnt(13)\n\t" \
  "Lbk1_%=:\n\t" \
  "buffer_load_dwordx4 v[112:115], v4, %[srdW], %[sW] offen\n\t" \
  "v_mfma_f32_16x16x32_bf16 a[44:47], v[60:63], v[24:27], a[44:47]\n\t" \
  "s_and_b32 s62, s55, 0x7f807f80\n\t" \
  "v_and_b32 v200, s63, v196\n\t" \
  "v_alignbit_b32 v201, v196, v196, 3\n\t" \
  "v_mfma_f32_16x16x32_bf16 a[48:51], v[48:51], v[28:31], a[48:51]\n\t" \
  "ds_read_b128 v[24:27], v12 offset:10240\n\t" \
  "v_perm_b32 v204, v200, v188, s84\n\t" \
  "v_mfma_f32_16x16x32_bf16 a[52:55], v[52:55], v[28:31], a[52:55]\n\t" \
  "v_and_b32 v201, s63, v201\n\t" \
  "v_perm_b32 v205, v200, v188, s85\n\t" \
  "v_mfma_f32_16x16x32_bf16 a[56:59], v[56:59], v[28:31], a[56:59]\n\t" \
  "v_add_u32 v204, s62, v204\n\t" \
  "v_perm_b32 v206, v201, v189, s84\n\t" \
  "v_mfma_f32_16x16x32_bf16 a[60:63], v[60:63], v[28:31], a[60:63]\n\t" \
  "s_waitcnt lgkmcnt(5)\n\t" \
  "buffer_load_dwordx4 v[116:119], v5, %[srdW], %[sW] offen\n\t" \
  "v_mfma_f32_16x16x32_bf16 a[64:67], v[48:51], v[32:35], a[64:67]\n\t" \
  "ds_read_b128 v[28:31], v12 offset:11264\n\t" \
  "v_add_u32 v205, s62, v205\n\t" \
  "v_mfma_f32_16x16x32_bf16 a[68:71], v[52:55], v[32:35], a[68:71]\n\t" \
  "v_perm_b32 v207, v201, v189, s85\n\t" \
  "v_add_u32 v206, s62, v206\n\t" \
  "v_mfma_f32_16x16x32_bf16 a[72:75], v[56:59], v[32:35], a[72:75]\n\t" \
  "v_add_u32 v207, s62, v207\n\t" \
  "ds_write_b128 v13, v[204:207] offset:16384\n\t" \
  "v_mfma_f32_16x16x32_bf16 a[76:79], v[60:63], v[32:35], a[76:79]\n\t" \
  "v_and_b32 v202, s63, v197\n\t" \
  "v_alignbit_b32 v203, v197, v197, 3\n\t" \
  "v_mfma_f32_16x16x32_bf16 a[80:83], v[48:51], v[36:39], a[80:83]\n\t" \
  "s_memrealtime s[72:73]\n\t" \
  "ds_read_b128 v[32:35], v12 offset:12288\n\t" \
  "v_perm_b32 v208, v202, v190, s84\n\t" \
  "v_mfma_f32_16x16x32_bf16 a[84:87], v[52:55], v[36:39], a[84:87]\n\t" \
  "buffer_load_dwordx4 v[120:123], v6, %[srdW], %[sW] offen\n\t" \
  "v_mfma_f32_16x16x32_bf16 a[88:91], v[56:59], v[36:39], a[88:91]\n\t" \
  "v_and_b32 v203, s63, v203\n\t" \
  "v_perm_b32 v209, v202, v190, s85\n\t" \
  "v_mfma_f32_16x16x32_bf16 a[92:95], v[60:63], v[36:39], a[92:95]\n\t" \
  "s_waitcnt lgkmcnt(6)\n\t" \
  "v_add_u32 v208, s62, v208\n\t" \
  "v_perm_b32 v210, v203, v191, s84\n\t" \
  "v_mfma_f32_16x16x32_bf16 a[96:99], v[48:51], v[40:43], a[96:99]\n\t" \
  "ds_read_b128 v[36:39], v12 offset:13312\n\t" \
  "v_add_u32 v209, s62, v209\n\t" \
  "v_mfma_f32_16x16x32_bf16 a[100:103], v[52:55], v[40:43], a[100:103]\n\t" \
  "v_perm_b32 v211, v203, v191, s85\n\t" \
  "v_add_u32 v210, s62, v210\n\t" \
  "v_mfma_f32_16x16x32_bf16 a[104:107], v[56:59], v[40:43], a[104:107]\n\t" \
  "buffer_load_dwordx4 v[124:127], v7, %[srdW], %[sW] offen\n\t" \
  "v_mfma_f32_16x16x32_bf16 a[108:111], v[60:63], v[40:43], a[108:111]\n\t" \
  "v_add_u32 v211, s62, v211\n\t" \
  "ds_write_b128 v13, v[208:211] offset:17408\n\t" \
  "v_mfma_f32_16x16x32_bf16 a[112:115], v[48:51], v[44:47], a[112:115]\n\t" \
  "ds_read_b128 v[40:43], v12 offset:14336\n\t" \
  "v_and_b32 v200, s63, v198\n\t" \
  "v_mfma_f32_16x16x32_bf16 a[116:119], v[52:55], v[44:47], a[116:119]\n\t" \
  "v_alignbit_b32 v201, v198, v198, 3\n\t" \
  "v_perm_b32 v212, v200, v192, s84\n\t" \
  "v_mfma_f32_16x16x32_bf16 a[120:123], v[56:59], v[44:47], a[120:123]\n\t" \
  "v_and_b32 v201, s63, v201\n\t" \
  "v_perm_b32 v213, v200, v192, s85\n\t" \
  "v_mfma_f32_16x16x32_bf16 a[124:127], v[60:63], v[44:47], a[124:127]\n\t" \
  "s_waitcnt lgkmcnt(7)\n\t" \
  "buffer_load_dwordx4 v[128:131], v8, %[srdW], %[sW] offen\n\t" \
  "v_mfma_f32_16x16x32_bf16 a[128:131], v[48:51], v[16:19], a[128:131]\n\t" \
  "ds_read_b128 v[44:47], v12 offset:15360\n\t" \
  "v_add_u32 v212, s62, v212\n\t" \
  "v_mfma_f32_16x16x32_bf16 a[132:135], v[52:55], v[16:19], a[132:135]\n\t" \
  "v_perm_b32 v214, v201, v193, s84\n\t" \
  "v_add_u32 v213, s62, v213\n\t" \
  "v_mfma_f32_16x16x32_bf16 a[136:139], v[56:59], v[16:19], a[136:139]\n\t" \
  "v_perm_b32 v215, v201, v193, s85\n\t" \
  "v_add_u32 v214, s62, v214\n\t" \
  "v_mfma_f32_16x16x32_bf16 a[140:143], v[60:63], v[16:19], a[140:143]\n\t" \
  "v_add_u32 v215, s62, v215\n\t" \
  "ds_write_b128 v13, v[212:215] offset:18432\n\t" \
  "v_mfma_f32_16x16x32_bf16 a[144:147], v[48:51], v[20:23], a[144:147]\n\t" \
  "buffer_load_dwordx4 v[132:135], v9, %[srdW], %[sW] offen\n\t" \
  "v_mfma_f32_16x16x32_bf16 a[148:151], v[52:55], v[20:23], a[148:151]\n\t" \
  "v_lshrrev_b32 v202, 5, v196\n\t" \
  "v_lshlrev_b32 v204, 2, v198\n\t" \
  "v_mfma_f32_16x16x32_bf16 a[152:155], v[56:59], v[20:23], a[152:155]\n\t" \
  "v_lshrrev_b32 v203, 5, v197\n\t" \
  "v_lshlrev_b32 v205, 1, v198\n\t" \
  "v_mfma_f32_16x16x32_bf16 a[156:159], v[60:63], v[20:23], a[156:159]\n\t" \
  "s_waitcnt lgkmcnt(7)\n\t" \
  "v_bfi_b32 v202, s64, v204, v202\n\t" \
  "v_bfi_b32 v203, s64, v205, v203\n\t" \
  "v_mfma_f32_16x16x32_bf16 a[160:163], v[48:51], v[24:27], a[160:163]\n\t" \
  "v_and_b32 v202, s63, v202\n\t" \
  "v_perm_b32 v204, v202, v194, s84\n\t" \
  "v_mfma_f32_16x16x32_bf16 a[164:167], v[52:55], v[24:27], a[164:167]\n\t" \
  "buffer_load_dwordx4 v[136:139], v10, %[srdW], %[sW] offen\n\t" \
  "v_mfma_f32_16x16x32_bf16 a[168:171], v[56:59], v[24:27], a[168:171]\n\t" \
  "v_and_b32 v203, s63, v203\n\t" \
  "v_perm_b32 v205, v202, v194, s85\n\t" \
  "v_mfma_f32_16x16x32_bf16 a[172:175], v[60:63], v[24:27], a[172:175]\n\t" \
  "v_add_u32 v204, s62, v204\n\t" \
  "v_perm_b32 v206, v203, v195, s84\n\t" \
  "v_mfma_f32_16x16x32_bf16 a[176:179], v[48:51], v[28:31], a[176:179]\n\t" \
  "v_add_u32 v205, s62, v205\n\t" \
  "v_perm_b32 v207, v203, v195, s85\n\t" \
  "v_mfma_f32_16x16x32_bf16 a[180:183], v[52:55], v[28:31], a[180:183]\n\t" \
  "v_add_u32 v206, s62, v206\n\t" \
  "v_add_u32 v207, s62, v207\n\t" \
  "v_mfma_f32_16x16x32_bf16 a[184:187], v[56:59], v[28:31], a[184:187]\n\t" \
  "buffer_load_dwordx4 v[140:143], v11, %[srdW], %[sW] offen\n\t" \
  "v_mfma_f32_16x16x32_bf16 a[188:191], v[60:63], v[28:31], a[188:191]\n\t" \
  "s_waitcnt lgkmcnt(4)\n\t" \
  "s_add_u32 %[sW], %[sW], 2097152\n\t" \
  "v_lshrrev_b32 v200, 16, v199\n\t" \
  "ds_write_b128 v13, v[204:207] offset:19456\n\t" \
  "v_mfma_f32_16x16x32_bf16 a[192:195], v[48:51], v[32:35], a[192:195]\n\t" \
  "s_bfm_b64 exec, s55, 0\n\t" \
  "s_cbranch_execz Lst2_%=\n\t" \
  "ds_write_b16 v200, v199 offset:16384\n\t" \
  "Lbk2_%=:\n\t" \
  "s_mov_b64 exec, -1\n\t" \
  "v_mfma_f32_16x16x32_bf16 a[196:199], v[52:55], v[32:35], a[196:199]\n\t" \
  "v_mfma_f32_16x16x32_bf16 a[200:203], v[56:59], v[32:35], a[200:203]\n\t" \
  "v_mfma_f32_16x16x32_bf16 a[204:207], v[60:63], v[32:35], a[204:207]\n\t" \
  "s_waitcnt lgkmcnt(0)\n\t" \
  "s_barrier\n\t" \
  "v_mfma_f32_16x16x32_bf16 a[208:211], v[48:51], v[36:39], a[208:211]\n\t" \
  "s_cmp_lt_u32 s52, %[cmp]\n\t" \
  "s_cbranch_scc1 Lst3_%=\n\t" \
  "Lbk3_%=:\n\t" \
  "s_add_u32 %[sM], %[sM], 512\n\t" \
  "s_load_dwordx2 s[54:55], %[ws], %[sM] glc\n\t" \
  "ds_read_b128 v[16:19], v12 offset:16384\n\t" \
  "v_mfma_f32_16x16x32_bf16 a[212:215], v[52:55], v[36:39], a[212:215]\n\t" \
  "ds_read_b128 v[20:23], v12 offset:17408\n\t" \
  "buffer_load_dwordx4 v[188:191], v12, %[srdX], %[sX] offen sc0 sc1\n\t" \
  "v_mfma_f32_16x16x32_bf16 a[216:219], v[56:59], v[36:39], a[216:219]\n\t" \
  "ds_read_b128 v[24:27], v12 offset:18432\n\t" \
  "v_mfma_f32_16x16x32_bf16 a[220:223], v[60:63], v[36:39], a[220:223]\n\t" \
  "ds_read_b128 v[28:31], v12 offset:19456\n\t" \
  "buffer_load_dwordx4 v[192:195], v12, %[srdX], %[sX] offen offset:1024 sc0 sc1\n\t" \
  "v_mfma_f32_16x16x32_bf16 a[224:227], v[48:51], v[40:43], a[224:227]\n\t" \
  "ds_read_b128 v[32:35], v12 offset:20480\n\t" \
  "v_mfma_f32_16x16x32_bf16 a[228:231], v[52:55], v[40:43], a[228:231]\n\t" \
  "ds_read_b128 v[36:39], v12 offset:21504\n\t" \
  "buffer_load_dwordx3 v[196:198], %[vL12], %[srdX], %[sX] offen offset:2048 sc0 sc1\n\t" \
  "v_mfma_f32_16x16x32_bf16 a[232:235], v[56:59], v[40:43], a[232:235]\n\t" \
  "v_mfma_f32_16x16x32_bf16 a[236:239], v[60:63], v[40:43], a[236:239]\n\t" \
  "buffer_load_dword v199, %[vL4], %[srdX], %[sX] offen offset:2816 sc0 sc1\n\t" \
  "v_mfma_f32_16x16x32_bf16 a[240:243], v[48:51], v[44:47], a[240:243]\n\t" \
  "s_add_u32 %[sX], %[sX], 12288\n\t" \
  "s_sub_u32 s81, s72, s76\n\t" \
  "s_cmp_ge_i32 s81, 0\n\t" \
  "s_cbranch_scc1 Lst4_%=\n\t" \
  "Lbk4_%=:\n\t" \
  "ds_read_b128 v[40:43], v12 offset:22528\n\t" \
  "v_mfma_f32_16x16x32_bf16 a[244:247], v[52:55], v[44:47], a[244:247]\n\t" \
  "v_mfma_f32_16x16x32_bf16 a[248:251], v[56:59], v[44:47], a[248:251]\n\t" \
  "v_mfma_f32_16x16x32_bf16 a[252:255], v[60:63], v[44:47], a[252:255]\n\t" \
  "s_waitcnt lgkmcnt(5)\n\t" \
  "v_mfma_f32_16x16x32_bf16 a[0:3], v[64:67], v[16:19], a[0:3]\n\t" \
  "ds_read_b128 v[44:47], v12 offset:23552\n\t" \
  "v_cvt_pk_bf16_f32 v48, v144, v148\n\t" \
  "v_mfma_f32_16x16x32_bf16 a[4:7], v[68:71], v[16:19], a[4:7]\n\t" \
  "v_cvt_pk_bf16_f32 v52, v145, v149\n\t" \
  "v_cvt_pk_bf16_f32 v56, v146, v150\n\t" \
  "v_mfma_f32_16x16x32_bf16 a[8:11], v[72:75], v[16:19], a[8:11]\n\t" \
  "v_cvt_pk_bf16_f32 v60, v147, v151\n\t" \
  "v_cvt_pk_bf16_f32 v49, v152, v156\n\t" \
  "v_mfma_f32_16x16x32_bf16 a[12:15], v[76:79], v[16:19], a[12:15]\n\t" \
  "v_cvt_pk_bf16_f32 v53, v153, v157\n\t" \
  "v_cvt_pk_bf16_f32 v57, v154, v158\n\t" \
  "v_mfma_f32_16x16x32_bf16 a[16:19], v[64:67], v[20:23], a[16:19]\n\t" \
  "ds_read_b128 v[16:19], v12 offset:24576\n\t" \
  "v_cvt_pk_bf16_f32 v61, v155, v159\n\t" \
  "v_mfma_f32_16x16x32_bf16 a[20:23], v[68:71], v[20:23], a[20:23]\n\t" \
  "v_cvt_pk_bf16_f32 v50, v160, v164\n\t" \
  "v_cvt_pk_bf16_f32 v54, v161, v165\n\t" \
  "v_mfma_f32_16x16x32_bf16 a[24:27], v[72:75], v[20:23], a[24:27]\n\t" \
  "v_cvt_pk_bf16_f32 v58, v162, v166\n\t" \
  "v_cvt_pk_bf16_f32 v62, v163, v167\n\t" \
  "v_mfma_f32_16x16x32_bf16 a[28:31], v[76:79], v[20:23], a[28:31]\n\t" \
  "s_waitcnt lgkmcnt(5)\n\t" \
  "v_cvt_pk_bf16_f32 v51, v168, v172\n\t" \
  "v_cvt_pk_bf16_f32 v55, v169, v173\n\t" \
  "v_mfma_f32_16x16x32_bf16 a[32:35], v[64:67], v[24:27], a[32:35]\n\t" \
  "ds_read_b128 v[20:23], v12 offset:25600\n\t" \
  "v_cvt_pk_bf16_f32 v59, v170, v174\n\t" \
  "v_mfma_f32_16x16x32_bf16 a[36:39], v[68:71], v[24:27], a[36:39]\n\t" \
  "v_cvt_pk_bf16_f32 v63, v171, v175\n\t" \
  "v_mfma_f32_16x16x32_bf16 a[40:43], v[72:75], v[24:27], a[40:43]\n\t" \
  "s_cbranch_vccnz Lst5_%=\n\t" \
  "s_waitcnt vmcnt(13)\n\t" \
  "Lbk5_%=:\n\t" \
  "buffer_load_dwordx4 v[144:147], v4, %[srdW], %[sW] offen\n\t" \
  "v_mfma_f32_16x16x32_bf16 a[44:47], v[76:79], v[24:27], a[44:47]\n\t" \
  "s_and_b32 s62, s57, 0x7f807f80\n\t" \
  "v_and_b32 v200, s63, v184\n\t" \
  "v_alignbit_b32 v201, v184, v184, 3\n\t" \
  "v_mfma_f32_16x16x32_bf16 a[48:51], v[64:67], v[28:31], a[48:51]\n\t" \
  "ds_read_b128 v[24:27], v12 offset:26624\n\t" \
  "v_perm_b32 v204, v200, v176, s84\n\t" \
  "v_mfma_f32_16x16x32_bf16 a[52:55], v[68:71], v[28:31], a[52:55]\n\t" \
  "v_and_b32 v201, s63, v201\n\t" \
  "v_perm_b32 v205, v200, v176, s85\n\t" \
  "v_mfma_f32_16x16x32_bf16 a[56:59], v[72:75], v[28:31], a[56:59]\n\t" \
  "v_add_u32 v204, s62, v204\n\t" \
  "v_perm_b32 v206, v201, v177, s84\n\t" \
  "v_mfma_f32_16x16x32_bf16 a[60:63], v[76:79], v[28:31], a[60:63]\n\t" \
  "s_waitcnt lgkmcnt(5)\n\t" \
  "buffer_load_dwordx4 v[148:151], v5, %[srdW], %[sW] offen\n\t" \
  "v_mfma_f32_16x16x32_bf16 a[64:67], v[64:67], v[32:35], a[64:67]\n\t" \
  "ds_read_b128 v[28:31], v12 offset:27648\n\t" \
  "v_add_u32 v205, s62, v205\n\t" \
  "v_mfma_f32_16x16x32_bf16 a[68:71], v[68:71], v[32:35], a[68:71]\n\t" \
  "v_perm_b32 v207, v201, v177, s85\n\t" \
  "v_add_u32 v206, s62, v206\n\t" \
  "v_mfma_f32_16x16x32_bf16 a[72:75], v[72:75], v[32:35], a[72:75]\n\t" \
  "v_add_u32 v207, s62, v207\n\t" \
  "ds_write_b128 v13, v[204:207] offset:0\n\t" \
  "v_mfma_f32_16x16x32_bf16 a[76:79], v[76:79], v[32:35], a[76:79]\n\t" \
  "v_and_b32 v202, s63, v185\n\t" \
  "v_alignbit_b32 v203, v185, v185, 3\n\t" \
  "v_mfma_f32_16x16x32_bf16 a[80:83], v[64:67], v[36:39], a[80:83]\n\t" \
  "s_memrealtime s[72:73]\n\t" \
  "ds_read_b128 v[32:35], v12 offset:28672\n\t" \
  "v_perm_b32 v208, v202, v178, s84\n\t" \
  "v_mfma_f32_16x16x32_bf16 a[84:87], v[68:71], v[36:39], a[84:87]\n\t" \
  "buffer_load_dwordx4 v[152:155], v6, %[srdW], %[sW] offen\n\t" \
  "v_mfma_f32_16x16x32_bf16 a[88:91], v[72:75], v[36:39], a[88:91]\n\t" \
  "v_and_b32 v203, s63, v203\n\t" \
  "v_perm_b32 v209, v202, v178, s85\n\t" \
  "v_mfma_f32_16x16x32_bf16 a[92:95], v[76:79], v[36:39], a[92:95]\n\t" \
  "s_waitcnt lgkmcnt(6)\n\t" \
  "v_add_u32 v208, s62, v208\n\t" \
  "v_perm_b32 v210, v203, v179, s84\n\t" \
  "v_mfma_f32_16x16x32_bf16 a[96:99], v[64:67], v[40:43], a[96:99]\n\t" \
  "ds_read_b128 v[36:39], v12 offset:29696\n\t" \
  "v_add_u32 v209, s62, v209\n\t" \
  "v_mfma_f32_16x16x32_bf16 a[100:103], v[68:71], v[40:43], a[100:103]\n\t" \
  "v_perm_b32 v211, v203, v179, s85\n\t" \
  "v_add_u32 v210, s62, v210\n\t" \
  "v_mfma_f32_16x16x32_bf16 a[104:107], v[72:75], v[40:43], a[104:107]\n\t" \
  "buffer_load_dwordx4 v[156:159], v7, %[srdW], %[sW] offen\n\t" \
  "v_mfma_f32_16x16x32_bf16 a[108:111], v[76:79], v[40:43], a[108:111]\n\t" \
  "v_add_u32 v211, s62, v211\n\t" \
  "ds_write_b128 v13, v[208:211] offset:1024\n\t" \
  "v_mfma_f32_16x16x32_bf16 a[112:115], v[64:67], v[44:47], a[112:115]\n\t" \
  "ds_read_b128 v[40:43], v12 offset:30720\n\t" \
  "v_and_b32 v200, s63, v186\n\t" \
  "v_mfma_f32_16x16x32_bf16 a[116:119], v[68:71], v[44:47], a[116:119]\n\t" \
  "v_alignbit_b32 v201, v186, v186, 3\n\t" \
  "v_perm_b32 v212, v200, v180, s84\n\t" \
  "v_mfma_f32_16x16x32_bf16 a[120:123], v[72:75], v[44:47], a[120:123]\n\t" \
  "v_and_b32 v201, s63, v201\n\t" \
  "v_perm_b32 v213, v200, v180, s85\n\t" \
  "v_mfma_f32_16x16x32_bf16 a[124:127], v[76:79], v[44:47], a[124:127]\n\t" \
  "s_waitcnt lgkmcnt(7)\n\t" \
  "buffer_load_dwordx4 v[160:163], v8, %[srdW], %[sW] offen\n\t" \
  "v_mfma_f32_16x16x32_bf16 a[128:131], v[64:67], v[16:19], a[128:131]\n\t" \
  "ds_read_b128 v[44:47], v12 offset:31744\n\t" \
  "v_add_u32 v212, s62, v212\n\t" \
  "v_mfma_f32_16x16x32_bf16 a[132:135], v[68:71], v[16:19], a[132:135]\n\t" \
  "v_perm_b32 v214, v201, v181, s84\n\t" \
  "v_add_u32 v213, s62, v213\n\t" \
  "v_mfma_f32_16x16x32_bf16 a[136:139], v[72:75], v[16:19], a[136:139]\n\t" \
  "v_perm_b32 v215, v201, v181, s85\n\t" \
  "v_add_u32 v214, s62, v214\n\t" \
  "v_mfma_f32_16x16x32_bf16 a[140:143], v[76:79], v[16:19], a[140:143]\n\t" \
  "v_add_u32 v215, s62, v215\n\t" \
  "ds_write_b128 v13, v[212:215] offset:2048\n\t" \
  "v_mfma_f32_16x16x32_bf16 a[144:147], v[64:67], v[20:23], a[144:147]\n\t" \
  "buffer_load_dwordx4 v[164:167], v9, %[srdW], %[sW] offen\n\t" \
  "v_mfma_f32_16x16x32_bf16 a[148:151], v[68:71], v[20:23], a[148:151]\n\t" \
  "v_lshrrev_b32 v202, 5, v184\n\t" \
  "v_lshlrev_b32 v204, 2, v186\n\t" \
  "v_mfma_f32_16x16x32_bf16 a[152:155], v[72:75], v[20:23], a[152:155]\n\t" \
  "v_lshrrev_b32 v203, 5, v185\n\t" \
  "v_lshlrev_b32 v205, 1, v186\n\t" \
  "v_mfma_f32_16x16x32_bf16 a[156:159], v[76:79], v[20:23], a[156:159]\n\t" \
  "s_waitcnt lgkmcnt(7)\n\t" \
  "v_bfi_b32 v202, s64, v204, v202\n\t" \
  "v_bfi_b32 v203, s64, v205, v203\n\t" \
  "v_mfma_f32_16x16x32_bf16 a[160:163], v[64:67], v[24:27], a[160:163]\n\t" \
  "v_and_b32 v202, s63, v202\n\t" \
  "v_perm_b32 v204, v202, v182, s84\n\t" \
  "v_mfma_f32_16x16x32_bf16 a[164:167], v[68:71], v[24:27], a[164:167]\n\t" \
  "buffer_load_dwordx4 v[168:171], v10, %[srdW], %[sW] offen\n\t" \
  "v_mfma_f32_16x16x32_bf16 a[168:171], v[72:75], v[24:27], a[168:171]\n\t" \
  "v_and_b32 v203, s63, v203\n\t" \
  "v_perm_b32 v205, v202, v182, s85\n\t" \
  "v_mfma_f32_16x16x32_bf16 a[172:175], v[76:79], v[24:27], a[172:175]\n\t" \
  "v_add_u32 v204, s62, v204\n\t" \
  "v_perm_b32 v206, v203, v183, s84\n\t" \
  "v_mfma_f32_16x16x32_bf16 a[176:179], v[64:67], v[28:31], a[176:179]\n\t" \
  "v_add_u32 v205, s62, v205\n\t" \
  "v_perm_b32 v207, v203, v183, s85\n\t" \
  "v_mfma_f32_16x16x32_bf16 a[180:183], v[68:71], v[28:31], a[180:183]\n\t" \
  "v_add_u32 v206, s62, v206\n\t" \
  "v_add_u32 v207, s62, v207\n\t" \
  "v_mfma_f32_16x16x32_bf16 a[184:187], v[72:75], v[28:31], a[184:187]\n\t" \
  "buffer_load_dwordx4 v[172:175], v11, %[srdW], %[sW] offen\n\t" \
  "v_mfma_f32_16x16x32_bf16 a[188:191], v[76:79], v[28:31], a[188:191]\n\t" \
  "s_waitcnt lgkmcnt(4)\n\t" \
  "s_add_u32 %[sW], %[sW], 2097152\n\t" \
  "v_lshrrev_b32 v200, 16, v187\n\t" \
  "ds_write_b128 v13, v[204:207] offset:3072\n\t" \
  "v_mfma_f32_16x16x32_bf16 a[192:195], v[64:67], v[32:35], a[192:195]\n\t" \
  "s_bfm_b64 exec, s57, 0\n\t" \
  "s_cbranch_execz Lst6_%=\n\t" \
  "ds_write_b16 v200, v187 offset:0\n\t" \
  "Lbk6_%=:\n\t" \
  "s_mov_b64 exec, -1\n\t" \
  "v_mfma_f32_16x16x32_bf16 a[196:199], v[68:71], v[32:35], a[196:199]\n\t" \
  "v_mfma_f32_16x16x32_bf16 a[200:203], v[72:75], v[32:35], a[200:203]\n\t" \
  "v_mfma_f32_16x16x32_bf16 a[204:207], v[76:79], v[32:35], a[204:207]\n\t" \
  "s_waitcnt lgkmcnt(0)\n\t" \
  "s_barrier\n\t" \
  "v_mfma_f32_16x16x32_bf16 a[208:211], v[64:67], v[36:39], a[208:211]\n\t" \
  "s_cmp_lt_u32 s54, %[cmp]\n\t" \
  "s_cbranch_scc1 Lst7_%=\n\t" \
  "Lbk7_%=:\n\t" \
  "s_add_u32 %[sM], %[sM], 512\n\t" \
  "s_load_dwordx2 s[56:57], %[ws], %[sM] glc\n\t" \
  "ds_read_b128 v[16:19], v12 offset:0\n\t" \
  "v_mfma_f32_16x16x32_bf16 a[212:215], v[68:71], v[36:39], a[212:215]\n\t" \
  "ds_read_b128 v[20:23], v12 offset:1024\n\t" \
  "buffer_load_dwordx4 v[176:179], v12, %[srdX], %[sX] offen sc0 sc1\n\t" \
  "v_mfma_f32_16x16x32_bf16 a[216:219], v[72:75], v[36:39], a[216:219]\n\t" \
  "ds_read_b128 v[24:27], v12 offset:2048\n\t" \
  "v_mfma_f32_16x16x32_bf16 a[220:223], v[76:79], v[36:39], a[220:223]\n\t" \
  "ds_read_b128 v[28:31], v12 offset:3072\n\t" \
  "buffer_load_dwordx4 v[180:183], v12, %[srdX], %[sX] offen offset:1024 sc0 sc1\n\t" \
  "v_mfma_f32_16x16x32_bf16 a[224:227], v[64:67], v[40:43], a[224:227]\n\t" \
  "ds_read_b128 v[32:35], v12 offset:4096\n\t" \
  "v_mfma_f32_16x16x32_bf16 a[228:231], v[68:71], v[40:43], a[228:231]\n\t" \
  "ds_read_b128 v[36:39], v12 offset:5120\n\t" \
  "buffer_load_dwordx3 v[184:186], %[vL12], %[srdX], %[sX] offen offset:2048 sc0 sc1\n\t" \
  "v_mfma_f32_16x16x32_bf16 a[232:235], v[72:75], v[40:43], a[232:235]\n\t" \
  "v_mfma_f32_16x16x32_bf16 a[236:239], v[76:79], v[40:43], a[236:239]\n\t" \
  "buffer_load_dword v187, %[vL4], %[srdX], %[sX] offen offset:2816 sc0 sc1\n\t" \
  "v_mfma_f32_16x16x32_bf16 a[240:243], v[64:67], v[44:47], a[240:243]\n\t" \
  "s_add_u32 %[sX], %[sX], 12288\n\t" \
  "s_sub_u32 s81, s72, s76\n\t" \
  "s_cmp_ge_i32 s81, 0\n\t" \
  "s_cbranch_scc1 Lst8_%=\n\t" \
  "Lbk8_%=:\n\t" \
  "ds_read_b128 v[40:43], v12 offset:6144\n\t" \
  "v_mfma_f32_16x16x32_bf16 a[244:247], v[68:71], v[44:47], a[244:247]\n\t" \
  "v_mfma_f32_16x16x32_bf16 a[248:251], v[72:75], v[44:47], a[248:251]\n\t" \
  "v_mfma_f32_16x16x32_bf16 a[252:255], v[76:79], v[44:47], a[252:255]\n\t" \
  "s_waitcnt lgkmcnt(5)\n\t" \
  "v_mfma_f32_16x16x32_bf16 a[0:3], v[48:51], v[16:19], a[0:3]\n\t" \
  "ds_read_b128 v[44:47], v12 offset:7168\n\t" \
  "v_cvt_pk_bf16_f32 v64, v80, v84\n\t" \
  "v_mfma_f32_16x16x32_bf16 a[4:7], v[52:55], v[16:19], a[4:7]\n\t" \
  "v_cvt_pk_bf16_f32 v68, v81, v85\n\t" \
  "v_cvt_pk_bf16_f32 v72, v82, v86\n\t" \
  "v_mfma_f32_16x16x32_bf16 a[8:11], v[56:59], v[16:19], a[8:11]\n\t" \
  "v_cvt_pk_bf16_f32 v76, v83, v87\n\t" \
  "v_cvt_pk_bf16_f32 v65, v88, v92\n\t" \
  "v_mfma_f32_16x16x32_bf16 a[12:15], v[60:63], v[16:19], a[12:15]\n\t" \
  "v_cvt_pk_bf16_f32 v69, v89, v93\n\t" \
  "v_cvt_pk_bf16_f32 v73, v90, v94\n\t" \
  "v_mfma_f32_16x16x32_bf16 a[16:19], v[48:51], v[20:23], a[16:19]\n\t" \
  "ds_read_b128 v[16:19], v12 offset:8192\n\t" \
  "v_cvt_pk_bf16_f32 v77, v91, v95\n\t" \
  "v_mfma_f32_16x16x32_bf16 a[20:23], v[52:55], v[20:23], a[20:23]\n\t" \
  "v_cvt_pk_bf16_f32 v66, v96, v100\n\t" \
  "v_cvt_pk_bf16_f32 v70, v97, v101\n\t" \
  "v_mfma_f32_16x16x32_bf16 a[24:27], v[56:59], v[20:23], a[24:27]\n\t" \
  "v_cvt_pk_bf16_f32 v74, v98, v102\n\t" \
  "v_cvt_pk_bf16_f32 v78, v99, v103\n\t" \
  "v_mfma_f32_16x16x32_bf16 a[28:31], v[60:63], v[20:23], a[28:31]\n\t" \
  "s_waitcnt lgkmcnt(5)\n\t" \
  "v_cvt_pk_bf16_f32 v67, v104, v108\n\t" \
  "v_cvt_pk_bf16_f32 v71, v105, v109\n\t" \
  "v_mfma_f32_16x16x32_bf16 a[32:35], v[48:51], v[24:27], a[32:35]\n\t" \
  "ds_read_b128 v[20:23], v12 offset:9216\n\t" \
  "v_cvt_pk_bf16_f32 v75, v106, v110\n\t" \
  "v_mfma_f32_16x16x32_bf16 a[36:39], v[52:55], v[24:27], a[36:39]\n\t" \
  "v_cvt_pk_bf16_f32 v79, v107, v111\n\t" \
  "v_mfma_f32_16x16x32_bf16 a[40:43], v[56:59], v[24:27], a[40:43]\n\t" \
  "s_cbranch_vccnz Lst9_%=\n\t" \
  "s_waitcnt vmcnt(12)\n\t" \
  "Lbk9_%=:\n\t" \
  "buffer_load_dwordx4 v[80:83], v4, %[srdW], %[sW] offen\n\t" \
  "v_mfma_f32_16x16x32_bf16 a[44:47], v[60:63], v[24:27], a[44:47]\n\t" \
  "s_and_b32 s62, s53, 0x7f807f80\n\t" \
  "v_and_b32 v200, s63, v196\n\t" \
  "v_alignbit_b32 v201, v196, v196, 3\n\t" \
  "v_mfma_f32_16x16x32_bf16 a[48:51], v[48:51], v[28:31], a[48:51]\n\t" \
  "ds_read_b128 v[24:27], v12 offset:10240\n\t" \
  "v_perm_b32 v204, v200, v188, s84\n\t" \
  "v_mfma_f32_16x16x32_bf16 a[52:55], v[52:55], v[28:31], a[52:55]\n\t" \
  "v_and_b32 v201, s63, v201\n\t" \
  "v_perm_b32 v205, v200, v188, s85\n\t" \
  "v_mfma_f32_16x16x32_bf16 a[56:59], v[56:59], v[28:31], a[56:59]\n\t" \
  "v_add_u32 v204, s62, v204\n\t" \
  "v_perm_b32 v206, v201, v189, s84\n\t" \
  "v_mfma_f32_16x16x32_bf16 a[60:63], v[60:63], v[28:31], a[60:63]\n\t" \
  "s_waitcnt lgkmcnt(5)\n\t" \
  "buffer_load_dwordx4 v[84:87], v5, %[srdW], %[sW] offen\n\t" \
  "v_mfma_f32_16x16x32_bf16 a[64:67], v[48:51], v[32:35], a[64:67]\n\t" \
  "ds_read_b128 v[28:31], v12 offset:11264\n\t" \
  "v_add_u32 v205, s62, v205\n\t" \
  "v_mfma_f32_16x16x32_bf16 a[68:71], v[52:55], v[32:35], a[68:71]\n\t" \
  "v_perm_b32 v207, v201, v189, s85\n\t" \
  "v_add_u32 v206, s62, v206\n\t" \
  "v_mfma_f32_16x16x32_bf16 a[72:75], v[56:59], v[32:35], a[72:75]\n\t" \
  "v_add_u32 v207, s62, v207\n\t" \
  "ds_write_b128 v13, v[204:207] offset:16384\n\t" \
  "v_mfma_f32_16x16x32_bf16 a[76:79], v[60:63], v[32:35], a[76:79]\n\t" \
  "v_and_b32 v202, s63, v197\n\t" \
  "v_alignbit_b32 v203, v197, v197, 3\n\t" \
  "v_mfma_f32_16x16x32_bf16 a[80:83], v[48:51], v[36:39], a[80:83]\n\t" \
  "s_memrealtime s[72:73]\n\t" \
  "ds_read_b128 v[32:35], v12 offset:12288\n\t" \
  "v_perm_b32 v208, v202, v190, s84\n\t" \
  "v_mfma_f32_16x16x32_bf16 a[84:87], v[52:55], v[36:39], a[84:87]\n\t" \
  "buffer_load_dwordx4 v[88:91], v6, %[srdW], %[sW] offen\n\t" \
  "v_mfma_f32_16x16x32_bf16 a[88:91], v[56:59], v[36:39], a[88:91]\n\t" \
  "v_and_b32 v203, s63, v203\n\t" \
  "v_perm_b32 v209, v202, v190, s85\n\t" \
  "v_mfma_f32_16x16x32_bf16 a[92:95], v[60:63], v[36:39], a[92:95]\n\t" \
  "s_waitcnt lgkmcnt(6)\n\t" \
  "v_add_u32 v208, s62, v208\n\t" \
  "v_perm_b32 v210, v203, v191, s84\n\t" \
  "v_mfma_f32_16x16x32_bf16 a[96:99], v[48:51], v[40:43], a[96:99]\n\t" \
  "ds_read_b128 v[36:39], v12 offset:13312\n\t" \
  "v_add_u32 v209, s62, v209\n\t" \
  "v_mfma_f32_16x16x32_bf16 a[100:103], v[52:55], v[40:43], a[100:103]\n\t" \
  "v_perm_b32 v211, v203, v191, s85\n\t" \
  "v_add_u32 v210, s62, v210\n\t" \
  "v_mfma_f32_16x16x32_bf16 a[104:107], v[56:59], v[40:43], a[104:107]\n\t" \
  "buffer_load_dwordx4 v[92:95], v7, %[srdW], %[sW] offen\n\t" \
  "v_mfma_f32_16x16x32_bf16 a[108:111], v[60:63], v[40:43], a[108:111]\n\t" \
  "v_add_u32 v211, s62, v211\n\t" \
  "ds_write_b128 v13, v[208:211] offset:17408\n\t" \
  "v_mfma_f32_16x16x32_bf16 a[112:115], v[48:51], v[44:47], a[112:115]\n\t" \
  "ds_read_b128 v[40:43], v12 offset:14336\n\t" \
  "v_and_b32 v200, s63, v198\n\t" \
  "v_mfma_f32_16x16x32_bf16 a[116:119], v[52:55], v[44:47], a[116:119]\n\t" \
  "v_alignbit_b32 v201, v198, v198, 3\n\t" \
  "v_perm_b32 v212, v200, v192, s84\n\t" \
  "v_mfma_f32_16x16x32_bf16 a[120:123], v[56:59], v[44:47], a[120:123]\n\t" \
  "v_and_b32 v201, s63, v201\n\t" \
  "v_perm_b32 v213, v200, v192, s85\n\t" \
  "v_mfma_f32_16x16x32_bf16 a[124:127], v[60:63], v[44:47], a[124:127]\n\t" \
  "s_waitcnt lgkmcnt(7)\n\t" \
  "buffer_load_dwordx4 v[96:99], v8, %[srdW], %[sW] offen\n\t" \
  "v_mfma_f32_16x16x32_bf16 a[128:131], v[48:51], v[16:19], a[128:131]\n\t" \
  "ds_read_b128 v[44:47], v12 offset:15360\n\t" \
  "v_add_u32 v212, s62, v212\n\t" \
  "v_mfma_f32_16x16x32_bf16 a[132:135], v[52:55], v[16:19], a[132:135]\n\t" \
  "v_perm_b32 v214, v201, v193, s84\n\t" \
  "v_add_u32 v213, s62, v213\n\t" \
  "v_mfma_f32_16x16x32_bf16 a[136:139], v[56:59], v[16:19], a[136:139]\n\t" \
  "v_perm_b32 v215, v201, v193, s85\n\t" \
  "v_add_u32 v214, s62, v214\n\t" \
  "v_mfma_f32_16x16x32_bf16 a[140:143], v[60:63], v[16:19], a[140:143]\n\t" \
  "v_add_u32 v215, s62, v215\n\t" \
  "ds_write_b128 v13, v[212:215] offset:18432\n\t" \
  "v_mfma_f32_16x16x32_bf16 a[144:147], v[48:51], v[20:23], a[144:147]\n\t" \
  "buffer_load_dwordx4 v[100:103], v9, %[srdW], %[sW] offen\n\t" \
  "v_mfma_f32_16x16x32_bf16 a[148:151], v[52:55], v[20:23], a[148:151]\n\t" \
  "v_lshrrev_b32 v202, 5, v196\n\t" \
  "v_lshlrev_b32 v204, 2, v198\n\t" \
  "v_mfma_f32_16x16x32_bf16 a[152:155], v[56:59], v[20:23], a[152:155]\n\t" \
  "v_lshrrev_b32 v203, 5, v197\n\t" \
  "v_lshlrev_b32 v205, 1, v198\n\t" \
  "v_mfma_f32_16x16x32_bf16 a[156:159], v[60:63], v[20:23], a[156:159]\n\t" \
  "s_waitcnt lgkmcnt(7)\n\t" \
  "v_bfi_b32 v202, s64, v204, v202\n\t" \
  "v_bfi_b32 v203, s64, v205, v203\n\t" \
  "v_mfma_f32_16x16x32_bf16 a[160:163], v[48:51], v[24:27], a[160:163]\n\t" \
  "v_and_b32 v202, s63, v202\n\t" \
  "v_perm_b32 v204, v202, v194, s84\n\t" \
  "v_mfma_f32_16x16x32_bf16 a[164:167], v[52:55], v[24:27], a[164:167]\n\t" \
  "buffer_load_dwordx4 v[104:107], v10, %[srdW], %[sW] offen\n\t" \
  "v_mfma_f32_16x16x32_bf16 a[168:171], v[56:59], v[24:27], a[168:171]\n\t" \
  "v_and_b32 v203, s63, v203\n\t" \
  "v_perm_b32 v205, v202, v194, s85\n\t" \
  "v_mfma_f32_16x16x32_bf16 a[172:175], v[60:63], v[24:27], a[172:175]\n\t" \
  "v_add_u32 v204, s62, v204\n\t" \
  "v_perm_b32 v206, v203, v195, s84\n\t" \
  "v_mfma_f32_16x16x32_bf16 a[176:179], v[48:51], v[28:31], a[176:179]\n\t" \
  "v_add_u32 v205, s62, v205\n\t" \
  "v_perm_b32 v207, v203, v195, s85\n\t" \
  "v_mfma_f32_16x16x32_bf16 a[180:183], v[52:55], v[28:31], a[180:183]\n\t" \
  "v_add_u32 v206, s62, v206\n\t" \
  "v_add_u32 v207, s62, v207\n\t" \
  "v_mfma_f32_16x16x32_bf16 a[184:187], v[56:59], v[28:31], a[184:187]\n\t" \
  "buffer_load_dwordx4 v[108:111], v11, %[srdW], %[sW] offen\n\t" \
  "v_mfma_f32_16x16x32_bf16 a[188:191], v[60:63], v[28:31], a[188:191]\n\t" \
  "s_waitcnt lgkmcnt(4)\n\t" \
  "s_add_u32 %[sW], %[sW], 2097152\n\t" \
  "v_lshrrev_b32 v200, 16, v199\n\t" \
  "ds_write_b128 v13, v[204:207] offset:19456\n\t" \
  "v_mfma_f32_16x16x32_bf16 a[192:195], v[48:51], v[32:35], a[192:195]\n\t" \
  "s_bfm_b64 exec, s53, 0\n\t" \
  "s_cbranch_execz Lst10_%=\n\t" \
  "ds_write_b16 v200, v199 offset:16384\n\t" \
  "Lbk10_%=:\n\t" \
  "s_mov_b64 exec, -1\n\t" \
  "v_mfma_f32_16x16x32_bf16 a[196:199], v[52:55], v[32:35], a[196:199]\n\t" \
  "v_mfma_f32_16x16x32_bf16 a[200:203], v[56:59], v[32:35], a[200:203]\n\t" \
  "v_mfma_f32_16x16x32_bf16 a[204:207], v[60:63], v[32:35], a[204:207]\n\t" \
  "s_waitcnt lgkmcnt(0)\n\t" \
  "s_barrier\n\t" \
  "v_mfma_f32_16x16x32_bf16 a[208:211], v[48:51], v[36:39], a[208:211]\n\t" \
  "s_cmp_lt_u32 s56, %[cmp]\n\t" \
  "s_cbranch_scc1 Lst11_%=\n\t" \
  "Lbk11_%=:\n\t" \
  "s_add_u32 %[sM], %[sM], 512\n\t" \
  "s_load_dwordx2 s[52:53], %[ws], %[sM] glc\n\t" \
  "ds_read_b128 v[16:19], v12 offset:16384\n\t" \
  "v_mfma_f32_16x16x32_bf16 a[212:215], v[52:55], v[36:39], a[212:215]\n\t" \
  "ds_read_b128 v[20:23], v12 offset:17408\n\t" \
  "buffer_load_dwordx4 v[188:191], v12, %[srdX], %[sX] offen sc0 sc1\n\t" \
  "v_mfma_f32_16x16x32_bf16 a[216:219], v[56:59], v[36:39], a[216:219]\n\t" \
  "ds_read_b128 v[24:27], v12 offset:18432\n\t" \
  "v_mfma_f32_16x16x32_bf16 a[220:223], v[60:63], v[36:39], a[220:223]\n\t" \
  "ds_read_b128 v[28:31], v12 offset:19456\n\t" \
  "buffer_load_dwordx4 v[192:195], v12, %[srdX], %[sX] offen offset:1024 sc0 sc1\n\t" \
  "v_mfma_f32_16x16x32_bf16 a[224:227], v[48:51], v[40:43], a[224:227]\n\t" \
  "ds_read_b128 v[32:35], v12 offset:20480\n\t" \
  "v_mfma_f32_16x16x32_bf16 a[228:231], v[52:55], v[40:43], a[228:231]\n\t" \
  "ds_read_b128 v[36:39], v12 offset:21504\n\t" \
  "buffer_load_dwordx3 v[196:198], %[vL12], %[srdX], %[sX] offen offset:2048 sc0 sc1\n\t" \
  "v_mfma_f32_16x16x32_bf16 a[232:235], v[56:59], v[40:43], a[232:235]\n\t" \
  "v_mfma_f32_16x16x32_bf16 a[236:239], v[60:63], v[40:43], a[236:239]\n\t" \
  "buffer_load_dword v199, %[vL4], %[srdX], %[sX] offen offset:2816 sc0 sc1\n\t" \
  "v_mfma_f32_16x16x32_bf16 a[240:243], v[48:51], v[44:47], a[240:243]\n\t" \
  "s_add_u32 %[sX], %[sX], 12288\n\t" \
  "s_sub_u32 s81, s72, s76\n\t" \
  "s_cmp_ge_i32 s81, 0\n\t" \
  "s_cbranch_scc1 Lst12_%=\n\t" \
  "Lbk12_%=:\n\t" \
  "ds_read_b128 v[40:43], v12 offset:22528\n\t" \
  "v_mfma_f32_16x16x32_bf16 a[244:247], v[52:55], v[44:47], a[244:247]\n\t" \
  "v_mfma_f32_16x16x32_bf16 a[248:251], v[56:59], v[44:47], a[248:251]\n\t" \
  "v_mfma_f32_16x16x32_bf16 a[252:255], v[60:63], v[44:47], a[252:255]\n\t" \
  "s_waitcnt lgkmcnt(5)\n\t" \
  "v_mfma_f32_16x16x32_bf16 a[0:3], v[64:67], v[16:19], a[0:3]\n\t" \
  "ds_read_b128 v[44:47], v12 offset:23552\n\t" \
  "v_cvt_pk_bf16_f32 v48, v112, v116\n\t" \
  "v_mfma_f32_16x16x32_bf16 a[4:7], v[68:71], v[16:19], a[4:7]\n\t" \
  "v_cvt_pk_bf16_f32 v52, v113, v117\n\t" \
  "v_cvt_pk_bf16_f32 v56, v114, v118\n\t" \
  "v_mfma_f32_16x16x32_bf16 a[8:11], v[72:75], v[16:19], a[8:11]\n\t" \
  "v_cvt_pk_bf16_f32 v60, v115, v119\n\t" \
  "v_cvt_pk_bf16_f32 v49, v120, v124\n\t" \
  "v_mfma_f32_16x16x32_bf16 a[12:15], v[76:79], v[16:19], a[12:15]\n\t" \
  "v_cvt_pk_bf16_f32 v53, v121, v125\n\t" \
  "v_cvt_pk_bf16_f32 v57, v122, v126\n\t" \
  "v_mfma_f32_16x16x32_bf16 a[16:19], v[64:67], v[20:23], a[16:19]\n\t" \
  "ds_read_b128 v[16:19], v12 offset:24576\n\t" \
  "v_cvt_pk_bf16_f32 v61, v123, v127\n\t" \
  "v_mfma_f32_16x16x32_bf16 a[20:23], v[68:71], v[20:23], a[20:23]\n\t" \
  "v_cvt_pk_bf16_f32 v50, v128, v132\n\t" \
  "v_cvt_pk_bf16_f32 v54, v129, v133\n\t" \
  "v_mfma_f32_16x16x32_bf16 a[24:27], v[72:75], v[20:23], a[24:27]\n\t" \
  "v_cvt_pk_bf16_f32 v58, v130, v134\n\t" \
  "v_cvt_pk_bf16_f32 v62, v131, v135\n\t" \
  "v_mfma_f32_16x16x32_bf16 a[28:31], v[76:79], v[20:23], a[28:31]\n\t" \
  "s_waitcnt lgkmcnt(5)\n\t" \
  "v_cvt_pk_bf16_f32 v51, v136, v140\n\t" \
  "v_cvt_pk_bf16_f32 v55, v137, v141\n\t" \
  "v_mfma_f32_16x16x32_bf16 a[32:35], v[64:67], v[24:27], a[32:35]\n\t" \
  "ds_read_b128 v[20:23], v12 offset:25600\n\t" \
  "v_cvt_pk_bf16_f32 v59, v138, v142\n\t" \
  "v_mfma_f32_16x16x32_bf16 a[36:39], v[68:71], v[24:27], a[36:39]\n\t" \
  "v_cvt_pk_bf16_f32 v63, v139, v143\n\t" \
  "v_mfma_f32_16x16x32_bf16 a[40:43], v[72:75], v[24:27], a[40:43]\n\t" \
  "s_cbranch_vccnz Lst13_%=\n\t" \
  "s_waitcnt vmcnt(12)\n\t" \
  "Lbk13_%=:\n\t" \
  "buffer_load_dwordx4 v[112:115], v4, %[srdW], %[sW] offen\n\t" \
  "v_mfma_f32_16x16x32_bf16 a[44:47], v[76:79], v[24:27], a[44:47]\n\t" \
  "s_and_b32 s62, s55, 0x7f807f80\n\t" \
  "v_and_b32 v200, s63, v184\n\t" \
  "v_alignbit_b32 v201, v184, v184, 3\n\t" \
  "v_mfma_f32_16x16x32_bf16 a[48:51], v[64:67], v[28:31], a[48:51]\n\t" \
  "ds_read_b128 v[24:27], v12 offset:26624\n\t" \
  "v_perm_b32 v204, v200, v176, s84\n\t" \
  "v_mfma_f32_16x16x32_bf16 a[52:55], v[68:71], v[28:31], a[52:55]\n\t" \
  "v_and_b32 v201, s63, v201\n\t" \
  "v_perm_b32 v205, v200, v176, s85\n\t" \
  "v_mfma_f32_16x16x32_bf16 a[56:59], v[72:75], v[28:31], a[56:59]\n\t" \
  "v_add_u32 v204, s62, v204\n\t" \
  "v_perm_b32 v206, v201, v177, s84\n\t" \
  "v_mfma_f32_16x16x32_bf16 a[60:63], v[76:79], v[28:31], a[60:63]\n\t" \
  "s_waitcnt lgkmcnt(5)\n\t" \
  "buffer_load_dwordx4 v[116:119], v5, %[srdW], %[sW] offen\n\t" \
  "v_mfma_f32_16x16x32_bf16 a[64:67], v[64:67], v[32:35], a[64:67]\n\t" \
  "ds_read_b128 v[28:31], v12 offset:27648\n\t" \
  "v_add_u32 v205, s62, v205\n\t" \
  "v_mfma_f32_16x16x32_bf16 a[68:71], v[68:71], v[32:35], a[68:71]\n\t" \
  "v_perm_b32 v207, v201, v177, s85\n\t" \
  "v_add_u32 v206, s62, v206\n\t" \
  "v_mfma_f32_16x16x32_bf16 a[72:75], v[72:75], v[32:35], a[72:75]\n\t" \
  "v_add_u32 v207, s62, v207\n\t" \
  "ds_write_b128 v13, v[204:207] offset:0\n\t" \
  "v_mfma_f32_16x16x32_bf16 a[76:79], v[76:79], v[32:35], a[76:79]\n\t" \
  "v_and_b32 v202, s63, v185\n\t" \
  "v_alignbit_b32 v203, v185, v185, 3\n\t" \
  "v_mfma_f32_16x16x32_bf16 a[80:83], v[64:67], v[36:39], a[80:83]\n\t" \
  "s_memrealtime s[72:73]\n\t" \
  "ds_read_b128 v[32:35], v12 offset:28672\n\t" \
  "v_perm_b32 v208, v202, v178, s84\n\t" \
  "v_mfma_f32_16x16x32_bf16 a[84:87], v[68:71], v[36:39], a[84:87]\n\t" \
  "buffer_load_dwordx4 v[120:123], v6, %[srdW], %[sW] offen\n\t" \
  "v_mfma_f32_16x16x32_bf16 a[88:91], v[72:75], v[36:39], a[88:91]\n\t" \
  "v_and_b32 v203, s63, v203\n\t" \
  "v_perm_b32 v209, v202, v178, s85\n\t" \
  "v_mfma_f32_16x16x32_bf16 a[92:95], v[76:79], v[36:39], a[92:95]\n\t" \
  "s_waitcnt lgkmcnt(6)\n\t" \
  "v_add_u32 v208, s62, v208\n\t" \
  "v_perm_b32 v210, v203, v179, s84\n\t" \
  "v_mfma_f32_16x16x32_bf16 a[96:99], v[64:67], v[40:43], a[96:99]\n\t" \
  "ds_read_b128 v[36:39], v12 offset:29696\n\t" \
  "v_add_u32 v209, s62, v209\n\t" \
  "v_mfma_f32_16x16x32_bf16 a[100:103], v[68:71], v[40:43], a[100:103]\n\t" \
  "v_perm_b32 v211, v203, v179, s85\n\t" \
  "v_add_u32 v210, s62, v210\n\t" \
  "v_mfma_f32_16x16x32_bf16 a[104:107], v[72:75], v[40:43], a[104:107]\n\t" \
  "buffer_load_dwordx4 v[124:127], v7, %[srdW], %[sW] offen\n\t" \
  "v_mfma_f32_16x16x32_bf16 a[108:111], v[76:79], v[40:43], a[108:111]\n\t" \
  "v_add_u32 v211, s62, v211\n\t" \
  "ds_write_b128 v13, v[208:211] offset:1024\n\t" \
  "v_mfma_f32_16x16x32_bf16 a[112:115], v[64:67], v[44:47], a[112:115]\n\t" \
  "ds_read_b128 v[40:43], v12 offset:30720\n\t" \
  "v_and_b32 v200, s63, v186\n\t" \
  "v_mfma_f32_16x16x32_bf16 a[116:119], v[68:71], v[44:47], a[116:119]\n\t" \
  "v_alignbit_b32 v201, v186, v186, 3\n\t" \
  "v_perm_b32 v212, v200, v180, s84\n\t" \
  "v_mfma_f32_16x16x32_bf16 a[120:123], v[72:75], v[44:47], a[120:123]\n\t" \
  "v_and_b32 v201, s63, v201\n\t" \
  "v_perm_b32 v213, v200, v180, s85\n\t" \
  "v_mfma_f32_16x16x32_bf16 a[124:127], v[76:79], v[44:47], a[124:127]\n\t" \
  "s_waitcnt lgkmcnt(7)\n\t" \
  "buffer_load_dwordx4 v[128:131], v8, %[srdW], %[sW] offen\n\t" \
  "v_mfma_f32_16x16x32_bf16 a[128:131], v[64:67], v[16:19], a[128:131]\n\t" \
  "ds_read_b128 v[44:47], v12 offset:31744\n\t" \
  "v_add_u32 v212, s62, v212\n\t" \
  "v_mfma_f32_16x16x32_bf16 a[132:135], v[68:71], v[16:19], a[132:135]\n\t" \
  "v_perm_b32 v214, v201, v181, s84\n\t" \
  "v_add_u32 v213, s62, v213\n\t" \
  "v_mfma_f32_16x16x32_bf16 a[136:139], v[72:75], v[16:19], a[136:139]\n\t" \
  "v_perm_b32 v215, v201, v181, s85\n\t" \
  "v_add_u32 v214, s62, v214\n\t" \
  "v_mfma_f32_16x16x32_bf16 a[140:143], v[76:79], v[16:19], a[140:143]\n\t" \
  "v_add_u32 v215, s62, v215\n\t" \
  "ds_write_b128 v13, v[212:215] offset:2048\n\t" \
  "v_mfma_f32_16x16x32_bf16 a[144:147], v[64:67], v[20:23], a[144:147]\n\t" \
  "buffer_load_dwordx4 v[132:135], v9, %[srdW], %[sW] offen\n\t" \
  "v_mfma_f32_16x16x32_bf16 a[148:151], v[68:71], v[20:23], a[148:151]\n\t" \
  "v_lshrrev_b32 v202, 5, v184\n\t" \
  "v_lshlrev_b32 v204, 2, v186\n\t" \
  "v_mfma_f32_16x16x32_bf16 a[152:155], v[72:75], v[20:23], a[152:155]\n\t" \
  "v_lshrrev_b32 v203, 5, v185\n\t" \
  "v_lshlrev_b32 v205, 1, v186\n\t" \
  "v_mfma_f32_16x16x32_bf16 a[156:159], v[76:79], v[20:23], a[156:159]\n\t" \
  "s_waitcnt lgkmcnt(7)\n\t" \
  "v_bfi_b32 v202, s64, v204, v202\n\t" \
  "v_bfi_b32 v203, s64, v205, v203\n\t" \
  "v_mfma_f32_16x16x32_bf16 a[160:163], v[64:67], v[24:27], a[160:163]\n\t" \
  "v_and_b32 v202, s63, v202\n\t" \
  "v_perm_b32 v204, v202, v182, s84\n\t" \
  "v_mfma_f32_16x16x32_bf16 a[164:167], v[68:71], v[24:27], a[164:167]\n\t" \
  "buffer_load_dwordx4 v[136:139], v10, %[srdW], %[sW] offen\n\t" \
  "v_mfma_f32_16x16x32_bf16 a[168:171], v[72:75], v[24:27], a[168:171]\n\t" \
  "v_and_b32 v203, s63, v203\n\t" \
  "v_perm_b32 v205, v202, v182, s85\n\t" \
  "v_mfma_f32_16x16x32_bf16 a[172:175], v[76:79], v[24:27], a[172:175]\n\t" \
  "v_add_u32 v204, s62, v204\n\t" \
  "v_perm_b32 v206, v203, v183, s84\n\t" \
  "v_mfma_f32_16x16x32_bf16 a[176:179], v[64:67], v[28:31], a[176:179]\n\t" \
  "v_add_u32 v205, s62, v205\n\t" \
  "v_perm_b32 v207, v203, v183, s85\n\t" \
  "v_mfma_f32_16x16x32_bf16 a[180:183], v[68:71], v[28:31], a[180:183]\n\t" \
  "v_add_u32 v206, s62, v206\n\t" \
  "v_add_u32 v207, s62, v207\n\t" \
  "v_mfma_f32_16x16x32_bf16 a[184:187], v[72:75], v[28:31], a[184:187]\n\t" \
  "buffer_load_dwordx4 v[140:143], v11, %[srdW], %[sW] offen\n\t" \
  "v_mfma_f32_16x16x32_bf16 a[188:191], v[76:79], v[28:31], a[188:191]\n\t" \
  "s_waitcnt lgkmcnt(4)\n\t" \
  "s_add_u32 %[sW], %[sW], 2097152\n\t" \
  "v_lshrrev_b32 v200, 16, v187\n\t" \
  "ds_write_b128 v13, v[204:207] offset:3072\n\t" \
  "v_mfma_f32_16x16x32_bf16 a[192:195], v[64:67], v[32:35], a[192:195]\n\t" \
  "s_bfm_b64 exec, s55, 0\n\t" \
  "s_cbranch_execz Lst14_%=\n\t" \
  "ds_write_b16 v200, v187 offset:0\n\t" \
  "Lbk14_%=:\n\t" \
  "s_mov_b64 exec, -1\n\t" \
  "v_mfma_f32_16x16x32_bf16 a[196:199], v[68:71], v[32:35], a[196:199]\n\t" \
  "v_mfma_f32_16x16x32_bf16 a[200:203], v[72:75], v[32:35], a[200:203]\n\t" \
  "v_mfma_f32_16x16x32_bf16 a[204:207], v[76:79], v[32:35], a[204:207]\n\t" \
  "s_waitcnt lgkmcnt(0)\n\t" \
  "s_barrier\n\t" \
  "v_mfma_f32_16x16x32_bf16 a[208:211], v[64:67], v[36:39], a[208:211]\n\t" \
  "s_cmp_lt_u32 s52, %[cmp]\n\t" \
  "s_cbranch_scc1 Lst15_%=\n\t" \
  "Lbk15_%=:\n\t" \
  "s_add_u32 %[sM], %[sM], 512\n\t" \
  "s_load_dwordx2 s[54:55], %[ws], %[sM] glc\n\t" \
  "ds_read_b128 v[16:19], v12 offset:0\n\t" \
  "v_mfma_f32_16x16x32_bf16 a[212:215], v[68:71], v[36:39], a[212:215]\n\t" \
  "ds_read_b128 v[20:23], v12 offset:1024\n\t" \
  "buffer_load_dwordx4 v[176:179], v12, %[srdX], %[sX] offen sc0 sc1\n\t" \
  "v_mfma_f32_16x16x32_bf16 a[216:219], v[72:75], v[36:39], a[216:219]\n\t" \
  "ds_read_b128 v[24:27], v12 offset:2048\n\t" \
  "v_mfma_f32_16x16x32_bf16 a[220:223], v[76:79], v[36:39], a[220:223]\n\t" \
  "ds_read_b128 v[28:31], v12 offset:3072\n\t" \
  "buffer_load_dwordx4 v[180:183], v12, %[srdX], %[sX] offen offset:1024 sc0 sc1\n\t" \
  "v_mfma_f32_16x16x32_bf16 a[224:227], v[64:67], v[40:43], a[224:227]\n\t" \
  "ds_read_b128 v[32:35], v12 offset:4096\n\t" \
  "v_mfma_f32_16x16x32_bf16 a[228:231], v[68:71], v[40:43], a[228:231]\n\t" \
  "ds_read_b128 v[36:39], v12 offset:5120\n\t" \
  "buffer_load_dwordx3 v[184:186], %[vL12], %[srdX], %[sX] offen offset:2048 sc0 sc1\n\t" \
  "v_mfma_f32_16x16x32_bf16 a[232:235], v[72:75], v[40:43], a[232:235]\n\t" \
  "v_mfma_f32_16x16x32_bf16 a[236:239], v[76:79], v[40:43], a[236:239]\n\t" \
  "buffer_load_dword v187, %[vL4], %[srdX], %[sX] offen offset:2816 sc0 sc1\n\t" \
  "v_mfma_f32_16x16x32_bf16 a[240:243], v[64:67], v[44:47], a[240:243]\n\t" \
  "s_add_u32 %[sX], %[sX], 12288\n\t" \
  "s_sub_u32 s81, s72, s76\n\t" \
  "s_cmp_ge_i32 s81, 0\n\t" \
  "s_cbranch_scc1 Lst16_%=\n\t" \
  "Lbk16_%=:\n\t" \
  "ds_read_b128 v[40:43], v12 offset:6144\n\t" \
  "v_mfma_f32_16x16x32_bf16 a[244:247], v[68:71], v[44:47], a[244:247]\n\t" \
  "v_mfma_f32_16x16x32_bf16 a[248:251], v[72:75], v[44:47], a[248:251]\n\t" \
  "v_mfma_f32_16x16x32_bf16 a[252:255], v[76:79], v[44:47], a[252:255]\n\t" \
  "s_waitcnt lgkmcnt(5)\n\t" \
  "v_mfma_f32_16x16x32_bf16 a[0:3], v[48:51], v[16:19], a[0:3]\n\t" \
  "ds_read_b128 v[44:47], v12 offset:7168\n\t" \
  "v_cvt_pk_bf16_f32 v64, v144, v148\n\t" \
  "v_mfma_f32_16x16x32_bf16 a[4:7], v[52:55], v[16:19], a[4:7]\n\t" \
  "v_cvt_pk_bf16_f32 v68, v145, v149\n\t" \
  "v_cvt_pk_bf16_f32 v72, v146, v150\n\t" \
  "v_mfma_f32_16x16x32_bf16 a[8:11], v[56:59], v[16:19], a[8:11]\n\t" \
  "v_cvt_pk_bf16_f32 v76, v147, v151\n\t" \
  "v_cvt_pk_bf16_f32 v65, v152, v156\n\t" \
  "v_mfma_f32_16x16x32_bf16 a[12:15], v[60:63], v[16:19], a[12:15]\n\t" \
  "v_cvt_pk_bf16_f32 v69, v153, v157\n\t" \
  "v_cvt_pk_bf16_f32 v73, v154, v158\n\t" \
  "v_mfma_f32_16x16x32_bf16 a[16:19], v[48:51], v[20:23], a[16:19]\n\t" \
  "ds_read_b128 v[16:19], v12 offset:8192\n\t" \
  "v_cvt_pk_bf16_f32 v77, v155, v159\n\t" \
  "v_mfma_f32_16x16x32_bf16 a[20:23], v[52:55], v[20:23], a[20:23]\n\t" \
  "v_cvt_pk_bf16_f32 v66, v160, v164\n\t" \
  "v_cvt_pk_bf16_f32 v70, v161, v165\n\t" \
  "v_mfma_f32_16x16x32_bf16 a[24:27], v[56:59], v[20:23], a[24:27]\n\t" \
  "v_cvt_pk_bf16_f32 v74, v162, v166\n\t" \
  "v_cvt_pk_bf16_f32 v78, v163, v167\n\t" \
  "v_mfma_f32_16x16x32_bf16 a[28:31], v[60:63], v[20:23], a[28:31]\n\t" \
  "s_waitcnt lgkmcnt(5)\n\t" \
  "v_cvt_pk_bf16_f32 v67, v168, v172\n\t" \
  "v_cvt_pk_bf16_f32 v71, v169, v173\n\t" \
  "v_mfma_f32_16x16x32_bf16 a[32:35], v[48:51], v[24:27], a[32:35]\n\t" \
  "ds_read_b128 v[20:23], v12 offset:9216\n\t" \
  "v_cvt_pk_bf16_f32 v75, v170, v174\n\t" \
  "v_mfma_f32_16x16x32_bf16 a[36:39], v[52:55], v[24:27], a[36:39]\n\t" \
  "v_cvt_pk_bf16_f32 v79, v171, v175\n\t" \
  "v_mfma_f32_16x16x32_bf16 a[40:43], v[56:59], v[24:27], a[40:43]\n\t" \
  "s_cbranch_vccnz Lst17_%=\n\t" \
  "s_waitcnt vmcnt(12)\n\t" \
  "Lbk17_%=:\n\t" \
  "buffer_load_dwordx4 v[144:147], v4, %[srdW], %[sW] offen\n\t" \
  "v_mfma_f32_16x16x32_bf16 a[44:47], v[60:63], v[24:27], a[44:47]\n\t" \
  "s_and_b32 s62, s57, 0x7f807f80\n\t" \
  "v_and_b32 v200, s63, v196\n\t" \
  "v_alignbit_b32 v201, v196, v196, 3\n\t" \
  "v_mfma_f32_16x16x32_bf16 a[48:51], v[48:51], v[28:31], a[48:51]\n\t" \
  "ds_read_b128 v[24:27], v12 offset:10240\n\t" \
  "v_perm_b32 v204, v200, v188, s84\n\t" \
  "v_mfma_f32_16x16x32_bf16 a[52:55], v[52:55], v[28:31], a[52:55]\n\t" \
  "v_and_b32 v201, s63, v201\n\t" \
  "v_perm_b32 v205, v200, v188, s85\n\t" \
  "v_mfma_f32_16x16x32_bf16 a[56:59], v[56:59], v[28:31], a[56:59]\n\t" \
  "v_add_u32 v204, s62, v204\n\t" \
  "v_perm_b32 v206, v201, v189, s84\n\t" \
  "v_mfma_f32_16x16x32_bf16 a[60:63], v[60:63], v[28:31], a[60:63]\n\t" \
  "s_waitcnt lgkmcnt(5)\n\t" \
  "buffer_load_dwordx4 v[148:151], v5, %[srdW], %[sW] offen\n\t" \
  "v_mfma_f32_16x16x32_bf16 a[64:67], v[48:51], v[32:35], a[64:67]\n\t" \
  "ds_read_b128 v[28:31], v12 offset:11264\n\t" \
  "v_add_u32 v205, s62, v205\n\t" \
  "v_mfma_f32_16x16x32_bf16 a[68:71], v[52:55], v[32:35], a[68:71]\n\t" \
  "v_perm_b32 v207, v201, v189, s85\n\t" \
  "v_add_u32 v206, s62, v206\n\t" \
  "v_mfma_f32_16x16x32_bf16 a[72:75], v[56:59], v[32:35], a[72:75]\n\t" \
  "v_add_u32 v207, s62, v207\n\t" \
  "ds_write_b128 v13, v[204:207] offset:16384\n\t" \
  "v_mfma_f32_16x16x32_bf16 a[76:79], v[60:63], v[32:35], a[76:79]\n\t" \
  "v_and_b32 v202, s63, v197\n\t" \
  "v_alignbit_b32 v203, v197, v197, 3\n\t" \
  "v_mfma_f32_16x16x32_bf16 a[80:83], v[48:51], v[36:39], a[80:83]\n\t" \
  "s_memrealtime s[72:73]\n\t" \
  "ds_read_b128 v[32:35], v12 offset:12288\n\t" \
  "v_perm_b32 v208, v202, v190, s84\n\t" \
  "v_mfma_f32_16x16x32_bf16 a[84:87], v[52:55], v[36:39], a[84:87]\n\t" \
  "buffer_load_dwordx4 v[152:155], v6, %[srdW], %[sW] offen\n\t" \
  "v_mfma_f32_16x16x32_bf16 a[88:91], v[56:59], v[36:39], a[88:91]\n\t" \
  "v_and_b32 v203, s63, v203\n\t" \
  "v_perm_b32 v209, v202, v190, s85\n\t" \
  "v_mfma_f32_16x16x32_bf16 a[92:95], v[60:63], v[36:39], a[92:95]\n\t" \
  "s_waitcnt lgkmcnt(6)\n\t" \
  "v_add_u32 v208, s62, v208\n\t" \
  "v_perm_b32 v210, v203, v191, s84\n\t" \
  "v_mfma_f32_16x16x32_bf16 a[96:99], v[48:51], v[40:43], a[96:99]\n\t" \
  "ds_read_b128 v[36:39], v12 offset:13312\n\t" \
  "v_add_u32 v209, s62, v209\n\t" \
  "v_mfma_f32_16x16x32_bf16 a[100:103], v[52:55], v[40:43], a[100:103]\n\t" \
  "v_perm_b32 v211, v203, v191, s85\n\t" \
  "v_add_u32 v210, s62, v210\n\t" \
  "v_mfma_f32_16x16x32_bf16 a[104:107], v[56:59], v[40:43], a[104:107]\n\t" \
  "buffer_load_dwordx4 v[156:159], v7, %[srdW], %[sW] offen\n\t" \
  "v_mfma_f32_16x16x32_bf16 a[108:111], v[60:63], v[40:43], a[108:111]\n\t" \
  "v_add_u32 v211, s62, v211\n\t" \
  "ds_write_b128 v13, v[208:211] offset:17408\n\t" \
  "v_mfma_f32_16x16x32_bf16 a[112:115], v[48:51], v[44:47], a[112:115]\n\t" \
  "ds_read_b128 v[40:43], v12 offset:14336\n\t" \
  "v_and_b32 v200, s63, v198\n\t" \
  "v_mfma_f32_16x16x32_bf16 a[116:119], v[52:55], v[44:47], a[116:119]\n\t" \
  "v_alignbit_b32 v201, v198, v198, 3\n\t" \
  "v_perm_b32 v212, v200, v192, s84\n\t" \
  "v_mfma_f32_16x16x32_bf16 a[120:123], v[56:59], v[44:47], a[120:123]\n\t" \
  "v_and_b32 v201, s63, v201\n\t" \
  "v_perm_b32 v213, v200, v192, s85\n\t" \
  "v_mfma_f32_16x16x32_bf16 a[124:127], v[60:63], v[44:47], a[124:127]\n\t" \
  "s_waitcnt lgkmcnt(7)\n\t" \
  "buffer_load_dwordx4 v[160:163], v8, %[srdW], %[sW] offen\n\t" \
  "v_mfma_f32_16x16x32_bf16 a[128:131], v[48:51], v[16:19], a[128:131]\n\t" \
  "ds_read_b128 v[44:47], v12 offset:15360\n\t" \
  "v_add_u32 v212, s62, v212\n\t" \
  "v_mfma_f32_16x16x32_bf16 a[132:135], v[52:55], v[16:19], a[132:135]\n\t" \
  "v_perm_b32 v214, v201, v193, s84\n\t" \
  "v_add_u32 v213, s62, v213\n\t" \
  "v_mfma_f32_16x16x32_bf16 a[136:139], v[56:59], v[16:19], a[136:139]\n\t" \
  "v_perm_b32 v215, v201, v193, s85\n\t" \
  "v_add_u32 v214, s62, v214\n\t" \
  "v_mfma_f32_16x16x32_bf16 a[140:143], v[60:63], v[16:19], a[140:143]\n\t" \
  "v_add_u32 v215, s62, v215\n\t" \
  "ds_write_b128 v13, v[212:215] offset:18432\n\t" \
  "v_mfma_f32_16x16x32_bf16 a[144:147], v[48:51], v[20:23], a[144:147]\n\t" \
  "buffer_load_dwordx4 v[164:167], v9, %[srdW], %[sW] offen\n\t" \
  "v_mfma_f32_16x16x32_bf16 a[148:151], v[52:55], v[20:23], a[148:151]\n\t" \
  "v_lshrrev_b32 v202, 5, v196\n\t" \
  "v_lshlrev_b32 v204, 2, v198\n\t" \
  "v_mfma_f32_16x16x32_bf16 a[152:155], v[56:59], v[20:23], a[152:155]\n\t" \
  "v_lshrrev_b32 v203, 5, v197\n\t" \
  "v_lshlrev_b32 v205, 1, v198\n\t" \
  "v_mfma_f32_16x16x32_bf16 a[156:159], v[60:63], v[20:23], a[156:159]\n\t" \
  "s_waitcnt lgkmcnt(7)\n\t" \
  "v_bfi_b32 v202, s64, v204, v202\n\t" \
  "v_bfi_b32 v203, s64, v205, v203\n\t" \
  "v_mfma_f32_16x16x32_bf16 a[160:163], v[48:51], v[24:27], a[160:163]\n\t" \
  "v_and_b32 v202, s63, v202\n\t" \
  "v_perm_b32 v204, v202, v194, s84\n\t" \
  "v_mfma_f32_16x16x32_bf16 a[164:167], v[52:55], v[24:27], a[164:167]\n\t" \
  "buffer_load_dwordx4 v[168:171], v10, %[srdW], %[sW] offen\n\t" \
  "v_mfma_f32_16x16x32_bf16 a[168:171], v[56:59], v[24:27], a[168:171]\n\t" \
  "v_and_b32 v203, s63, v203\n\t" \
  "v_perm_b32 v205, v202, v194, s85\n\t" \
  "v_mfma_f32_16x16x32_bf16 a[172:175], v[60:63], v[24:27], a[172:175]\n\t" \
  "v_add_u32 v204, s62, v204\n\t" \
  "v_perm_b32 v206, v203, v195, s84\n\t" \
  "v_mfma_f32_16x16x32_bf16 a[176:179], v[48:51], v[28:31], a[176:179]\n\t" \
  "v_add_u32 v205, s62, v205\n\t" \
  "v_perm_b32 v207, v203, v195, s85\n\t" \
  "v_mfma_f32_16x16x32_bf16 a[180:183], v[52:55], v[28:31], a[180:183]\n\t" \
  "v_add_u32 v206, s62, v206\n\t" \
  "v_add_u32 v207, s62, v207\n\t" \
  "v_mfma_f32_16x16x32_bf16 a[184:187], v[56:59], v[28:31], a[184:187]\n\t" \
  "buffer_load_dwordx4 v[172:175], v11, %[srdW], %[sW] offen\n\t" \
  "v_mfma_f32_16x16x32_bf16 a[188:191], v[60:63], v[28:31], a[188:191]\n\t" \
  "s_waitcnt lgkmcnt(4)\n\t" \
  "s_add_u32 %[sW], %[sW], 2097152\n\t" \
  "v_lshrrev_b32 v200, 16, v199\n\t" \
  "ds_write_b128 v13, v[204:207] offset:19456\n\t" \
  "v_mfma_f32_16x16x32_bf16 a[192:195], v[48:51], v[32:35], a[192:195]\n\t" \
  "s_bfm_b64 exec, s57, 0\n\t" \
  "s_cbranch_execz Lst18_%=\n\t" \
  "ds_write_b16 v200, v199 offset:16384\n\t" \
  "Lbk18_%=:\n\t" \
  "s_mov_b64 exec, -1\n\t" \
  "v_mfma_f32_16x16x32_bf16 a[196:199], v[52:55], v[32:35], a[196:199]\n\t" \
  "v_mfma_f32_16x16x32_bf16 a[200:203], v[56:59], v[32:35], a[200:203]\n\t" \
  "v_mfma_f32_16x16x32_bf16 a[204:207], v[60:63], v[32:35], a[204:207]\n\t" \
  "s_waitcnt lgkmcnt(0)\n\t" \
  "s_barrier\n\t" \
  "v_mfma_f32_16x16x32_bf16 a[208:211], v[48:51], v[36:39], a[208:211]\n\t" \
  "s_cmp_lt_u32 s54, %[cmp]\n\t" \
  "s_cbranch_scc1 Lst19_%=\n\t" \
  "Lbk19_%=:\n\t" \
  "s_add_u32 %[sM], %[sM], 512\n\t" \
  "s_load_dwordx2 s[56:57], %[ws], %[sM] glc\n\t" \
  "ds_read_b128 v[16:19], v12 offset:16384\n\t" \
  "v_mfma_f32_16x16x32_bf16 a[212:215], v[52:55], v[36:39], a[212:215]\n\t" \
  "ds_read_b128 v[20:23], v12 offset:17408\n\t" \
  "buffer_load_dwordx4 v[188:191], v12, %[srdX], %[sX] offen sc0 sc1\n\t" \
  "v_mfma_f32_16x16x32_bf16 a[216:219], v[56:59], v[36:39], a[216:219]\n\t" \
  "ds_read_b128 v[24:27], v12 offset:18432\n\t" \
  "v_mfma_f32_16x16x32_bf16 a[220:223], v[60:63], v[36:39], a[220:223]\n\t" \
  "ds_read_b128 v[28:31], v12 offset:19456\n\t" \
  "buffer_load_dwordx4 v[192:195], v12, %[srdX], %[sX] offen offset:1024 sc0 sc1\n\t" \
  "v_mfma_f32_16x16x32_bf16 a[224:227], v[48:51], v[40:43], a[224:227]\n\t" \
  "ds_read_b128 v[32:35], v12 offset:20480\n\t" \
  "v_mfma_f32_16x16x32_bf16 a[228:231], v[52:55], v[40:43], a[228:231]\n\t" \
  "ds_read_b128 v[36:39], v12 offset:21504\n\t" \
  "buffer_load_dwordx3 v[196:198], %[vL12], %[srdX], %[sX] offen offset:2048 sc0 sc1\n\t" \
  "v_mfma_f32_16x16x32_bf16 a[232:235], v[56:59], v[40:43], a[232:235]\n\t" \
  "v_mfma_f32_16x16x32_bf16 a[236:239], v[60:63], v[40:43], a[236:239]\n\t" \
  "buffer_load_dword v199, %[vL4], %[srdX], %[sX] offen offset:2816 sc0 sc1\n\t" \
  "v_mfma_f32_16x16x32_bf16 a[240:243], v[48:51], v[44:47], a[240:243]\n\t" \
  "s_add_u32 %[sX], %[sX], 12288\n\t" \
  "s_sub_u32 s81, s72, s76\n\t" \
  "s_cmp_ge_i32 s81, 0\n\t" \
  "s_cbranch_scc1 Lst20_%=\n\t" \
  "Lbk20_%=:\n\t" \
  "ds_read_b128 v[40:43], v12 offset:22528\n\t" \
  "v_mfma_f32_16x16x32_bf16 a[244:247], v[52:55], v[44:47], a[244:247]\n\t" \
  "v_mfma_f32_16x16x32_bf16 a[248:251], v[56:59], v[44:47], a[248:251]\n\t" \
  "v_mfma_f32_16x16x32_bf16 a[252:255], v[60:63], v[44:47], a[252:255]\n\t" \
  "s_waitcnt lgkmcnt(5)\n\t" \
  "v_mfma_f32_16x16x32_bf16 a[0:3], v[64:67], v[16:19], a[0:3]\n\t" \
  "ds_read_b128 v[44:47], v12 offset:23552\n\t" \
  "v_cvt_pk_bf16_f32 v48, v80, v84\n\t" \
  "v_mfma_f32_16x16x32_bf16 a[4:7], v[68:71], v[16:19], a[4:7]\n\t" \
  "v_cvt_pk_bf16_f32 v52, v81, v85\n\t" \
  "v_cvt_pk_bf16_f32 v56, v82, v86\n\t" \
  "v_mfma_f32_16x16x32_bf16 a[8:11], v[72:75], v[16:19], a[8:11]\n\t" \
  "v_cvt_pk_bf16_f32 v60, v83, v87\n\t" \
  "v_cvt_pk_bf16_f32 v49, v88, v92\n\t" \
  "v_mfma_f32_16x16x32_bf16 a[12:15], v[76:79], v[16:19], a[12:15]\n\t" \
  "v_cvt_pk_bf16_f32 v53, v89, v93\n\t" \
  "v_cvt_pk_bf16_f32 v57, v90, v94\n\t" \
  "v_mfma_f32_16x16x32_bf16 a[16:19], v[64:67], v[20:23], a[16:19]\n\t" \
  "ds_read_b128 v[16:19], v12 offset:24576\n\t" \
  "v_cvt_pk_bf16_f32 v61, v91, v95\n\t" \
  "v_mfma_f32_16x16x32_bf16 a[20:23], v[68:71], v[20:23], a[20:23]\n\t" \
  "v_cvt_pk_bf16_f32 v50, v96, v100\n\t" \
  "v_cvt_pk_bf16_f32 v54, v97, v101\n\t" \
  "v_mfma_f32_16x16x32_bf16 a[24:27], v[72:75], v[20:23], a[24:27]\n\t" \
  "v_cvt_pk_bf16_f32 v58, v98, v102\n\t" \
  "v_cvt_pk_bf16_f32 v62, v99, v103\n\t" \
  "v_mfma_f32_16x16x32_bf16 a[28:31], v[76:79], v[20:23], a[28:31]\n\t" \
  "s_waitcnt lgkmcnt(5)\n\t" \
  "v_cvt_pk_bf16_f32 v51, v104, v108\n\t" \
  "v_cvt_pk_bf16_f32 v55, v105, v109\n\t" \
  "v_mfma_f32_16x16x32_bf16 a[32:35], v[64:67], v[24:27], a[32:35]\n\t" \
  "ds_read_b128 v[20:23], v12 offset:25600\n\t" \
  "v_cvt_pk_bf16_f32 v59, v106, v110\n\t" \
  "v_mfma_f32_16x16x32_bf16 a[36:39], v[68:71], v[24:27], a[36:39]\n\t" \
  "v_cvt_pk_bf16_f32 v63, v107, v111\n\t" \
  "v_mfma_f32_16x16x32_bf16 a[40:43], v[72:75], v[24:27], a[40:43]\n\t" \
  "s_cbranch_vccnz Lst21_%=\n\t" \
  "s_waitcnt vmcnt(12)\n\t" \
  "Lbk21_%=:\n\t" \
  "buffer_load_dwordx4 v[80:83], v4, %[srdW], %[sW] offen\n\t" \
  "v_mfma_f32_16x16x32_bf16 a[44:47], v[76:79], v[24:27], a[44:47]\n\t" \
  "s_and_b32 s62, s53, 0x7f807f80\n\t" \
  "v_and_b32 v200, s63, v184\n\t" \
  "v_alignbit_b32 v201, v184, v184, 3\n\t" \
  "v_mfma_f32_16x16x32_bf16 a[48:51], v[64:67], v[28:31], a[48:51]\n\t" \
  "ds_read_b128 v[24:27], v12 offset:26624\n\t" \
  "v_perm_b32 v204, v200, v176, s84\n\t" \
  "v_mfma_f32_16x16x32_bf16 a[52:55], v[68:71], v[28:31], a[52:55]\n\t" \
  "v_and_b32 v201, s63, v201\n\t" \
  "v_perm_b32 v205, v200, v176, s85\n\t" \
  "v_mfma_f32_16x16x32_bf16 a[56:59], v[72:75], v[28:31], a[56:59]\n\t" \
  "v_add_u32 v204, s62, v204\n\t" \
  "v_perm_b32 v206, v201, v177, s84\n\t" \
  "v_mfma_f32_16x16x32_bf16 a[60:63], v[76:79], v[28:31], a[60:63]\n\t" \
  "s_waitcnt lgkmcnt(5)\n\t" \
  "buffer_load_dwordx4 v[84:87], v5, %[srdW], %[sW] offen\n\t" \
  "v_mfma_f32_16x16x32_bf16 a[64:67], v[64:67], v[32:35], a[64:67]\n\t" \
  "ds_read_b128 v[28:31], v12 offset:27648\n\t" \
  "v_add_u32 v205, s62, v205\n\t" \
  "v_mfma_f32_16x16x32_bf16 a[68:71], v[68:71], v[32:35], a[68:71]\n\t" \
  "v_perm_b32 v207, v201, v177, s85\n\t" \
  "v_add_u32 v206, s62, v206\n\t" \
  "v_mfma_f32_16x16x32_bf16 a[72:75], v[72:75], v[32:35], a[72:75]\n\t" \
  "v_add_u32 v207, s62, v207\n\t" \
  "ds_write_b128 v13, v[204:207] offset:0\n\t" \
  "v_mfma_f32_16x16x32_bf16 a[76:79], v[76:79], v[32:35], a[76:79]\n\t" \
  "v_and_b32 v202, s63, v185\n\t" \
  "v_alignbit_b32 v203, v185, v185, 3\n\t" \
  "v_mfma_f32_16x16x32_bf16 a[80:83], v[64:67], v[36:39], a[80:83]\n\t" \
  "s_memrealtime s[72:73]\n\t" \
  "ds_read_b128 v[32:35], v12 offset:28672\n\t" \
  "v_perm_b32 v208, v202, v178, s84\n\t" \
  "v_mfma_f32_16x16x32_bf16 a[84:87], v[68:71], v[36:39], a[84:87]\n\t" \
  "buffer_load_dwordx4 v[88:91], v6, %[srdW], %[sW] offen\n\t" \
  "v_mfma_f32_16x16x32_bf16 a[88:91], v[72:75], v[36:39], a[88:91]\n\t" \
  "v_and_b32 v203, s63, v203\n\t" \
  "v_perm_b32 v209, v202, v178, s85\n\t" \
  "v_mfma_f32_16x16x32_bf16 a[92:95], v[76:79], v[36:39], a[92:95]\n\t" \
  "s_waitcnt lgkmcnt(6)\n\t" \
  "v_add_u32 v208, s62, v208\n\t" \
  "v_perm_b32 v210, v203, v179, s84\n\t" \
  "v_mfma_f32_16x16x32_bf16 a[96:99], v[64:67], v[40:43], a[96:99]\n\t" \
  "ds_read_b128 v[36:39], v12 offset:29696\n\t" \
  "v_add_u32 v209, s62, v209\n\t" \
  "v_mfma_f32_16x16x32_bf16 a[100:103], v[68:71], v[40:43], a[100:103]\n\t" \
  "v_perm_b32 v211, v203, v179, s85\n\t" \
  "v_add_u32 v210, s62, v210\n\t" \
  "v_mfma_f32_16x16x32_bf16 a[104:107], v[72:75], v[40:43], a[104:107]\n\t" \
  "buffer_load_dwordx4 v[92:95], v7, %[srdW], %[sW] offen\n\t" \
  "v_mfma_f32_16x16x32_bf16 a[108:111], v[76:79], v[40:43], a[108:111]\n\t" \
  "v_add_u32 v211, s62, v211\n\t" \
  "ds_write_b128 v13, v[208:211] offset:1024\n\t" \
  "v_mfma_f32_16x16x32_bf16 a[112:115], v[64:67], v[44:47], a[112:115]\n\t" \
  "ds_read_b128 v[40:43], v12 offset:30720\n\t" \
  "v_and_b32 v200, s63, v186\n\t" \
  "v_mfma_f32_16x16x32_bf16 a[116:119], v[68:71], v[44:47], a[116:119]\n\t" \
  "v_alignbit_b32 v201, v186, v186, 3\n\t" \
  "v_perm_b32 v212, v200, v180, s84\n\t" \
  "v_mfma_f32_16x16x32_bf16 a[120:123], v[72:75], v[44:47], a[120:123]\n\t" \
  "v_and_b32 v201, s63, v201\n\t" \
  "v_perm_b32 v213, v200, v180, s85\n\t" \
  "v_mfma_f32_16x16x32_bf16 a[124:127], v[76:79], v[44:47], a[124:127]\n\t" \
  "s_waitcnt lgkmcnt(7)\n\t" \
  "buffer_load_dwordx4 v[96:99], v8, %[srdW], %[sW] offen\n\t" \
  "v_mfma_f32_16x16x32_bf16 a[128:131], v[64:67], v[16:19], a[128:131]\n\t" \
  "ds_read_b128 v[44:47], v12 offset:31744\n\t" \
  "v_add_u32 v212, s62, v212\n\t" \
  "v_mfma_f32_16x16x32_bf16 a[132:135], v[68:71], v[16:19], a[132:135]\n\t" \
  "v_perm_b32 v214, v201, v181, s84\n\t" \
  "v_add_u32 v213, s62, v213\n\t" \
  "v_mfma_f32_16x16x32_bf16 a[136:139], v[72:75], v[16:19], a[136:139]\n\t" \
  "v_perm_b32 v215, v201, v181, s85\n\t" \
  "v_add_u32 v214, s62, v214\n\t" \
  "v_mfma_f32_16x16x32_bf16 a[140:143], v[76:79], v[16:19], a[140:143]\n\t" \
  "v_add_u32 v215, s62, v215\n\t" \
  "ds_write_b128 v13, v[212:215] offset:2048\n\t" \
  "v_mfma_f32_16x16x32_bf16 a[144:147], v[64:67], v[20:23], a[144:147]\n\t" \
  "buffer_load_dwordx4 v[100:103], v9, %[srdW], %[sW] offen\n\t" \
  "v_mfma_f32_16x16x32_bf16 a[148:151], v[68:71], v[20:23], a[148:151]\n\t" \
  "v_lshrrev_b32 v202, 5, v184\n\t" \
  "v_lshlrev_b32 v204, 2, v186\n\t" \
  "v_mfma_f32_16x16x32_bf16 a[152:155], v[72:75], v[20:23], a[152:155]\n\t" \
  "v_lshrrev_b32 v203, 5, v185\n\t" \
  "v_lshlrev_b32 v205, 1, v186\n\t" \
  "v_mfma_f32_16x16x32_bf16 a[156:159], v[76:79], v[20:23], a[156:159]\n\t" \
  "s_waitcnt lgkmcnt(7)\n\t" \
  "v_bfi_b32 v202, s64, v204, v202\n\t" \
  "v_bfi_b32 v203, s64, v205, v203\n\t" \
  "v_mfma_f32_16x16x32_bf16 a[160:163], v[64:67], v[24:27], a[160:163]\n\t" \
  "v_and_b32 v202, s63, v202\n\t" \
  "v_perm_b32 v204, v202, v182, s84\n\t" \
  "v_mfma_f32_16x16x32_bf16 a[164:167], v[68:71], v[24:27], a[164:167]\n\t" \
  "buffer_load_dwordx4 v[104:107], v10, %[srdW], %[sW] offen\n\t" \
  "v_mfma_f32_16x16x32_bf16 a[168:171], v[72:75], v[24:27], a[168:171]\n\t" \
  "v_and_b32 v203, s63, v203\n\t" \
  "v_perm_b32 v205, v202, v182, s85\n\t" \
  "v_mfma_f32_16x16x32_bf16 a[172:175], v[76:79], v[24:27], a[172:175]\n\t" \
  "v_add_u32 v204, s62, v204\n\t" \
  "v_perm_b32 v206, v203, v183, s84\n\t" \
  "v_mfma_f32_16x16x32_bf16 a[176:179], v[64:67], v[28:31], a[176:179]\n\t" \
  "v_add_u32 v205, s62, v205\n\t" \
  "v_perm_b32 v207, v203, v183, s85\n\t" \
  "v_mfma_f32_16x16x32_bf16 a[180:183], v[68:71], v[28:31], a[180:183]\n\t" \
  "v_add_u32 v206, s62, v206\n\t" \
  "v_add_u32 v207, s62, v207\n\t" \
  "v_mfma_f32_16x16x32_bf16 a[184:187], v[72:75], v[28:31], a[184:187]\n\t" \
  "buffer_load_dwordx4 v[108:111], v11, %[srdW], %[sW] offen\n\t" \
  "v_mfma_f32_16x16x32_bf16 a[188:191], v[76:79], v[28:31], a[188:191]\n\t" \
  "s_waitcnt lgkmcnt(4)\n\t" \
  "s_add_u32 %[sW], %[sW], 2097152\n\t" \
  "v_lshrrev_b32 v200, 16, v187\n\t" \
  "ds_write_b128 v13, v[204:207] offset:3072\n\t" \
  "v_mfma_f32_16x16x32_bf16 a[192:195], v[64:67], v[32:35], a[192:195]\n\t" \
  "s_bfm_b64 exec, s53, 0\n\t" \
  "s_cbranch_execz Lst22_%=\n\t" \
  "ds_write_b16 v200, v187 offset:0\n\t" \
  "Lbk22_%=:\n\t" \
  "s_mov_b64 exec, -1\n\t" \
  "v_mfma_f32_16x16x32_bf16 a[196:199], v[68:71], v[32:35], a[196:199]\n\t" \
  "v_mfma_f32_16x16x32_bf16 a[200:203], v[72:75], v[32:35], a[200:203]\n\t" \
  "v_mfma_f32_16x16x32_bf16 a[204:207], v[76:79], v[32:35], a[204:207]\n\t" \
  "s_waitcnt lgkmcnt(0)\n\t" \
  "s_barrier\n\t" \
  "v_mfma_f32_16x16x32_bf16 a[208:211], v[64:67], v[36:39], a[208:211]\n\t" \
  "s_cmp_lt_u32 s56, %[cmp]\n\t" \
  "s_cbranch_scc1 Lst23_%=\n\t" \
  "Lbk23_%=:\n\t" \
  "s_add_u32 %[sM], %[sM], 512\n\t" \
  "s_load_dwordx2 s[52:53], %[ws], %[sM] glc\n\t" \
  "ds_read_b128 v[16:19], v12 offset:0\n\t" \
  "v_mfma_f32_16x16x32_bf16 a[212:215], v[68:71], v[36:39], a[212:215]\n\t" \
  "ds_read_b128 v[20:23], v12 offset:1024\n\t" \
  "buffer_load_dwordx4 v[176:179], v12, %[srdX], %[sX] offen sc0 sc1\n\t" \
  "v_mfma_f32_16x16x32_bf16 a[216:219], v[72:75], v[36:39], a[216:219]\n\t" \
  "ds_read_b128 v[24:27], v12 offset:2048\n\t" \
  "v_mfma_f32_16x16x32_bf16 a[220:223], v[76:79], v[36:39], a[220:223]\n\t" \
  "ds_read_b128 v[28:31], v12 offset:3072\n\t" \
  "buffer_load_dwordx4 v[180:183], v12, %[srdX], %[sX] offen offset:1024 sc0 sc1\n\t" \
  "v_mfma_f32_16x16x32_bf16 a[224:227], v[64:67], v[40:43], a[224:227]\n\t" \
  "ds_read_b128 v[32:35], v12 offset:4096\n\t" \
  "v_mfma_f32_16x16x32_bf16 a[228:231], v[68:71], v[40:43], a[228:231]\n\t" \
  "ds_read_b128 v[36:39], v12 offset:5120\n\t" \
  "buffer_load_dwordx3 v[184:186], %[vL12], %[srdX], %[sX] offen offset:2048 sc0 sc1\n\t" \
  "v_mfma_f32_16x16x32_bf16 a[232:235], v[72:75], v[40:43], a[232:235]\n\t" \
  "v_mfma_f32_16x16x32_bf16 a[236:239], v[76:79], v[40:43], a[236:239]\n\t" \
  "buffer_load_dword v187, %[vL4], %[srdX], %[sX] offen offset:2816 sc0 sc1\n\t" \
  "v_mfma_f32_16x16x32_bf16 a[240:243], v[64:67], v[44:47], a[240:243]\n\t" \
  "s_add_u32 %[sX], %[sX], 12288\n\t" \
  "s_sub_u32 s81, s72, s76\n\t" \
  "s_cmp_ge_i32 s81, 0\n\t" \
  "s_cbranch_scc1 Lst24_%=\n\t" \
  "Lbk24_%=:\n\t" \
  "ds_read_b128 v[40:43], v12 offset:6144\n\t" \
  "v_mfma_f32_16x16x32_bf16 a[244:247], v[68:71], v[44:47], a[244:247]\n\t" \
  "v_mfma_f32_16x16x32_bf16 a[248:251], v[72:75], v[44:47], a[248:251]\n\t" \
  "v_mfma_f32_16x16x32_bf16 a[252:255], v[76:79], v[44:47], a[252:255]\n\t" \
  "s_waitcnt lgkmcnt(5)\n\t" \
  "s_sub_u32 %[cnt], %[cnt], 1\n\t" \
  "s_cmp_lg_u32 %[cnt], 0\n\t" \
  "s_cbranch_scc1 Lloop_%=\n\t" \
  "v_mfma_f32_16x16x32_bf16 a[0:3], v[48:51], v[16:19], a[0:3]\n\t" \
  "ds_read_b128 v[44:47], v12 offset:7168\n\t" \
  "v_cvt_pk_bf16_f32 v64, v112, v116\n\t" \
  "v_mfma_f32_16x16x32_bf16 a[4:7], v[52:55], v[16:19], a[4:7]\n\t" \
  "v_cvt_pk_bf16_f32 v68, v113, v117\n\t" \
  "v_cvt_pk_bf16_f32 v72, v114, v118\n\t" \
  "buffer_load_dword v226, v225, %[srdW], 0 offen sc1\n\t" \
  "v_mfma_f32_16x16x32_bf16 a[8:11], v[56:59], v[16:19], a[8:11]\n\t" \
  "v_cvt_pk_bf16_f32 v76, v115, v119\n\t" \
  "v_cvt_pk_bf16_f32 v65, v120, v124\n\t" \
  "v_add_u32 v225, 12582912, v225\n\t" \
  "v_mfma_f32_16x16x32_bf16 a[12:15], v[60:63], v[16:19], a[12:15]\n\t" \
  "v_cvt_pk_bf16_f32 v69, v121, v125\n\t" \
  "v_cvt_pk_bf16_f32 v73, v122, v126\n\t" \
  "v_mfma_f32_16x16x32_bf16 a[16:19], v[48:51], v[20:23], a[16:19]\n\t" \
  "ds_read_b128 v[16:19], v12 offset:8192\n\t" \
  "v_cvt_pk_bf16_f32 v77, v123, v127\n\t" \
  "v_mfma_f32_16x16x32_bf16 a[20:23], v[52:55], v[20:23], a[20:23]\n\t" \
  "v_cvt_pk_bf16_f32 v66, v128, v132\n\t" \
  "v_cvt_pk_bf16_f32 v70, v129, v133\n\t" \
  "v_mfma_f32_16x16x32_bf16 a[24:27], v[56:59], v[20:23], a[24:27]\n\t" \
  "v_cvt_pk_bf16_f32 v74, v130, v134\n\t" \
  "v_cvt_pk_bf16_f32 v78, v131, v135\n\t" \
  "v_mfma_f32_16x16x32_bf16 a[28:31], v[60:63], v[20:23], a[28:31]\n\t" \
  "s_waitcnt lgkmcnt(5)\n\t" \
  "v_cvt_pk_bf16_f32 v67, v136, v140\n\t" \
  "v_cvt_pk_bf16_f32 v71, v137, v141\n\t" \
  "v_mfma_f32_16x16x32_bf16 a[32:35], v[48:51], v[24:27], a[32:35]\n\t" \
  "ds_read_b128 v[20:23], v12 offset:9216\n\t" \
  "v_cvt_pk_bf16_f32 v75, v138, v142\n\t" \
  "v_mfma_f32_16x16x32_bf16 a[36:39], v[52:55], v[24:27], a[36:39]\n\t" \
  "v_cvt_pk_bf16_f32 v79, v139, v143\n\t" \
  "v_mfma_f32_16x16x32_bf16 a[40:43], v[56:59], v[24:27], a[40:43]\n\t" \
  "s_cbranch_vccnz Lst25_%=\n\t" \
  "s_waitcnt vmcnt(13)\n\t" \
  "Lbk25_%=:\n\t" \
  "buffer_load_dwordx4 v[112:115], v4, %[srdW], %[sW] offen\n\t" \
  "v_mfma_f32_16x16x32_bf16 a[44:47], v[60:63], v[24:27], a[44:47]\n\t" \
  "s_and_b32 s62, s55, 0x7f807f80\n\t" \
  "v_and_b32 v200, s63, v196\n\t" \
  "v_alignbit_b32 v201, v196, v196, 3\n\t" \
  "v_mfma_f32_16x16x32_bf16 a[48:51], v[48:51], v[28:31], a[48:51]\n\t" \
  "ds_read_b128 v[24:27], v12 offset:10240\n\t" \
  "v_perm_b32 v204, v200, v188, s84\n\t" \
  "v_mfma_f32_16x16x32_bf16 a[52:55], v[52:55], v[28:31], a[52:55]\n\t" \
  "v_and_b32 v201, s63, v201\n\t" \
  "v_perm_b32 v205, v200, v188, s85\n\t" \
  "v_mfma_f32_16x16x32_bf16 a[56:59], v[56:59], v[28:31], a[56:59]\n\t" \
  "v_add_u32 v204, s62, v204\n\t" \
  "v_perm_b32 v206, v201, v189, s84\n\t" \
  "v_mfma_f32_16x16x32_bf16 a[60:63], v[60:63], v[28:31], a[60:63]\n\t" \
  "s_waitcnt lgkmcnt(5)\n\t" \
  "buffer_load_dwordx4 v[116:119], v5, %[srdW], %[sW] offen\n\t" \
  "v_mfma_f32_16x16x32_bf16 a[64:67], v[48:51], v[32:35], a[64:67]\n\t" \
  "ds_read_b128 v[28:31], v12 offset:11264\n\t" \
  "v_add_u32 v205, s62, v205\n\t" \
  "v_mfma_f32_16x16x32_bf16 a[68:71], v[52:55], v[32:35], a[68:71]\n\t" \
  "v_perm_b32 v207, v201, v189, s85\n\t" \
  "v_add_u32 v206, s62, v206\n\t" \
  "v_mfma_f32_16x16x32_bf16 a[72:75], v[56:59], v[32:35], a[72:75]\n\t" \
  "v_add_u32 v207, s62, v207\n\t" \
  "ds_write_b128 v13, v[204:207] offset:16384\n\t" \
  "v_mfma_f32_16x16x32_bf16 a[76:79], v[60:63], v[32:35], a[76:79]\n\t" \
  "v_and_b32 v202, s63, v197\n\t" \
  "v_alignbit_b32 v203, v197, v197, 3\n\t" \
  "v_mfma_f32_16x16x32_bf16 a[80:83], v[48:51], v[36:39], a[80:83]\n\t" \
  "s_memrealtime s[72:73]\n\t" \
  "ds_read_b128 v[32:35], v12 offset:12288\n\t" \
  "v_perm_b32 v208, v202, v190, s84\n\t" \
  "v_mfma_f32_16x16x32_bf16 a[84:87], v[52:55], v[36:39], a[84:87]\n\t" \
  "buffer_load_dwordx4 v[120:123], v6, %[srdW], %[sW] offen\n\t" \
  "v_mfma_f32_16x16x32_bf16 a[88:91], v[56:59], v[36:39], a[88:91]\n\t" \
  "v_and_b32 v203, s63, v203\n\t" \
  "v_perm_b32 v209, v202, v190, s85\n\t" \
  "v_mfma_f32_16x16x32_bf16 a[92:95], v[60:63], v[36:39], a[92:95]\n\t" \
  "s_waitcnt lgkmcnt(6)\n\t" \
  "v_add_u32 v208, s62, v208\n\t" \
  "v_perm_b32 v210, v203, v191, s84\n\t" \
  "v_mfma_f32_16x16x32_bf16 a[96:99], v[48:51], v[40:43], a[96:99]\n\t" \
  "ds_read_b128 v[36:39], v12 offset:13312\n\t" \
  "v_add_u32 v209, s62, v209\n\t" \
  "v_mfma_f32_16x16x32_bf16 a[100:103], v[52:55], v[40:43], a[100:103]\n\t" \
  "v_perm_b32 v211, v203, v191, s85\n\t" \
  "v_add_u32 v210, s62, v210\n\t" \
  "v_mfma_f32_16x16x32_bf16 a[104:107], v[56:59], v[40:43], a[104:107]\n\t" \
  "buffer_load_dwordx4 v[124:127], v7, %[srdW], %[sW] offen\n\t" \
  "v_mfma_f32_16x16x32_bf16 a[108:111], v[60:63], v[40:43], a[108:111]\n\t" \
  "v_add_u32 v211, s62, v211\n\t" \
  "ds_write_b128 v13, v[208:211] offset:17408\n\t" \
  "v_mfma_f32_16x16x32_bf16 a[112:115], v[48:51], v[44:47], a[112:115]\n\t" \
  "ds_read_b128 v[40:43], v12 offset:14336\n\t" \
  "v_and_b32 v200, s63, v198\n\t" \
  "v_mfma_f32_16x16x32_bf16 a[116:119], v[52:55], v[44:47], a[116:119]\n\t" \
  "v_alignbit_b32 v201, v198, v198, 3\n\t" \
  "v_perm_b32 v212, v200, v192, s84\n\t" \
  "v_mfma_f32_16x16x32_bf16 a[120:123], v[56:59], v[44:47], a[120:123]\n\t" \
  "v_and_b32 v201, s63, v201\n\t" \
  "v_perm_b32 v213, v200, v192, s85\n\t" \
  "v_mfma_f32_16x16x32_bf16 a[124:127], v[60:63], v[44:47], a[124:127]\n\t" \
  "s_waitcnt lgkmcnt(7)\n\t" \
  "buffer_load_dwordx4 v[128:131], v8, %[srdW], %[sW] offen\n\t" \
  "v_mfma_f32_16x16x32_bf16 a[128:131], v[48:51], v[16:19], a[128:131]\n\t" \
  "ds_read_b128 v[44:47], v12 offset:15360\n\t" \
  "v_add_u32 v212, s62, v212\n\t" \
  "v_mfma_f32_16x16x32_bf16 a[132:135], v[52:55], v[16:19], a[132:135]\n\t" \
  "v_perm_b32 v214, v201, v193, s84\n\t" \
  "v_add_u32 v213, s62, v213\n\t" \
  "v_mfma_f32_16x16x32_bf16 a[136:139], v[56:59], v[16:19], a[136:139]\n\t" \
  "v_perm_b32 v215, v201, v193, s85\n\t" \
  "v_add_u32 v214, s62, v214\n\t" \
  "v_mfma_f32_16x16x32_bf16 a[140:143], v[60:63], v[16:19], a[140:143]\n\t" \
  "v_add_u32 v215, s62, v215\n\t" \
  "ds_write_b128 v13, v[212:215] offset:18432\n\t" \
  "v_mfma_f32_16x16x32_bf16 a[144:147], v[48:51], v[20:23], a[144:147]\n\t" \
  "buffer_load_dwordx4 v[132:135], v9, %[srdW], %[sW] offen\n\t" \
  "v_mfma_f32_16x16x32_bf16 a[148:151], v[52:55], v[20:23], a[148:151]\n\t" \
  "v_lshrrev_b32 v202, 5, v196\n\t" \
  "v_lshlrev_b32 v204, 2, v198\n\t" \
  "v_mfma_f32_16x16x32_bf16 a[152:155], v[56:59], v[20:23], a[152:155]\n\t" \
  "v_lshrrev_b32 v203, 5, v197\n\t" \
  "v_lshlrev_b32 v205, 1, v198\n\t" \
  "v_mfma_f32_16x16x32_bf16 a[156:159], v[60:63], v[20:23], a[156:159]\n\t" \
  "s_waitcnt lgkmcnt(7)\n\t" \
  "v_bfi_b32 v202, s64, v204, v202\n\t" \
  "v_bfi_b32 v203, s64, v205, v203\n\t" \
  "v_mfma_f32_16x16x32_bf16 a[160:163], v[48:51], v[24:27], a[160:163]\n\t" \
  "v_and_b32 v202, s63, v202\n\t" \
  "v_perm_b32 v204, v202, v194, s84\n\t" \
  "v_mfma_f32_16x16x32_bf16 a[164:167], v[52:55], v[24:27], a[164:167]\n\t" \
  "buffer_load_dwordx4 v[136:139], v10, %[srdW], %[sW] offen\n\t" \
  "v_mfma_f32_16x16x32_bf16 a[168:171], v[56:59], v[24:27], a[168:171]\n\t" \
  "v_and_b32 v203, s63, v203\n\t" \
  "v_perm_b32 v205, v202, v194, s85\n\t" \
  "v_mfma_f32_16x16x32_bf16 a[172:175], v[60:63], v[24:27], a[172:175]\n\t" \
  "v_add_u32 v204, s62, v204\n\t" \
  "v_perm_b32 v206, v203, v195, s84\n\t" \
  "v_mfma_f32_16x16x32_bf16 a[176:179], v[48:51], v[28:31], a[176:179]\n\t" \
  "v_add_u32 v205, s62, v205\n\t" \
  "v_perm_b32 v207, v203, v195, s85\n\t" \
  "v_mfma_f32_16x16x32_bf16 a[180:183], v[52:55], v[28:31], a[180:183]\n\t" \
  "v_add_u32 v206, s62, v206\n\t" \
  "v_add_u32 v207, s62, v207\n\t" \
  "v_mfma_f32_16x16x32_bf16 a[184:187], v[56:59], v[28:31], a[184:187]\n\t" \
  "buffer_load_dwordx4 v[140:143], v11, %[srdW], %[sW] offen\n\t" \
  "v_mfma_f32_16x16x32_bf16 a[188:191], v[60:63], v[28:31], a[188:191]\n\t" \
  "s_waitcnt lgkmcnt(4)\n\t" \
  "s_add_u32 %[sW], %[sW], 2097152\n\t" \
  "v_lshrrev_b32 v200, 16, v199\n\t" \
  "ds_write_b128 v13, v[204:207] offset:19456\n\t" \
  "v_mfma_f32_16x16x32_bf16 a[192:195], v[48:51], v[32:35], a[192:195]\n\t" \
  "s_bfm_b64 exec, s55, 0\n\t" \
  "s_cbranch_execz Lst26_%=\n\t" \
  "ds_write_b16 v200, v199 offset:16384\n\t" \
  "Lbk26_%=:\n\t" \
  "s_mov_b64 exec, -1\n\t" \
  "v_mfma_f32_16x16x32_bf16 a[196:199], v[52:55], v[32:35], a[196:199]\n\t" \
  "v_mfma_f32_16x16x32_bf16 a[200:203], v[56:59], v[32:35], a[200:203]\n\t" \
  "v_mfma_f32_16x16x32_bf16 a[204:207], v[60:63], v[32:35], a[204:207]\n\t" \
  "s_waitcnt lgkmcnt(0)\n\t" \
  "s_barrier\n\t" \
  "v_mfma_f32_16x16x32_bf16 a[208:211], v[48:51], v[36:39], a[208:211]\n\t" \
  "s_cmp_lt_u32 s52, %[cmp]\n\t" \
  "s_cbranch_scc1 Lst27_%=\n\t" \
  "Lbk27_%=:\n\t" \
  "s_add_u32 %[sM], %[sM], 512\n\t" \
  "s_load_dwordx2 s[54:55], %[ws], %[sM] glc\n\t" \
  "ds_read_b128 v[16:19], v12 offset:16384\n\t" \
  "v_mfma_f32_16x16x32_bf16 a[212:215], v[52:55], v[36:39], a[212:215]\n\t" \
  "ds_read_b128 v[20:23], v12 offset:17408\n\t" \
  "buffer_load_dwordx4 v[188:191], v12, %[srdX], %[sX] offen sc0 sc1\n\t" \
  "v_mfma_f32_16x16x32_bf16 a[216:219], v[56:59], v[36:39], a[216:219]\n\t" \
  "ds_read_b128 v[24:27], v12 offset:18432\n\t" \
  "v_mfma_f32_16x16x32_bf16 a[220:223], v[60:63], v[36:39], a[220:223]\n\t" \
  "ds_read_b128 v[28:31], v12 offset:19456\n\t" \
  "buffer_load_dwordx4 v[192:195], v12, %[srdX], %[sX] offen offset:1024 sc0 sc1\n\t" \
  "v_mfma_f32_16x16x32_bf16 a[224:227], v[48:51], v[40:43], a[224:227]\n\t" \
  "ds_read_b128 v[32:35], v12 offset:20480\n\t" \
  "v_mfma_f32_16x16x32_bf16 a[228:231], v[52:55], v[40:43], a[228:231]\n\t" \
  "ds_read_b128 v[36:39], v12 offset:21504\n\t" \
  "buffer_load_dwordx3 v[196:198], %[vL12], %[srdX], %[sX] offen offset:2048 sc0 sc1\n\t" \
  "v_mfma_f32_16x16x32_bf16 a[232:235], v[56:59], v[40:43], a[232:235]\n\t" \
  "v_mfma_f32_16x16x32_bf16 a[236:239], v[60:63], v[40:43], a[236:239]\n\t" \
  "buffer_load_dword v199, %[vL4], %[srdX], %[sX] offen offset:2816 sc0 sc1\n\t" \
  "v_mfma_f32_16x16x32_bf16 a[240:243], v[48:51], v[44:47], a[240:243]\n\t" \
  "s_add_u32 %[sX], %[sX], 12288\n\t" \
  "s_sub_u32 s81, s72, s76\n\t" \
  "s_cmp_ge_i32 s81, 0\n\t" \
  "s_cbranch_scc1 Lst28_%=\n\t" \
  "Lbk28_%=:\n\t" \
  "ds_read_b128 v[40:43], v12 offset:22528\n\t" \
  "v_mfma_f32_16x16x32_bf16 a[244:247], v[52:55], v[44:47], a[244:247]\n\t" \
  "v_mfma_f32_16x16x32_bf16 a[248:251], v[56:59], v[44:47], a[248:251]\n\t" \
  "v_mfma_f32_16x16x32_bf16 a[252:255], v[60:63], v[44:47], a[252:255]\n\t" \
  "s_waitcnt lgkmcnt(5)\n\t" \
  "v_mfma_f32_16x16x32_bf16 a[0:3], v[64:67], v[16:19], a[0:3]\n\t" \
  "ds_read_b128 v[44:47], v12 offset:23552\n\t" \
  "v_cvt_pk_bf16_f32 v48, v144, v148\n\t" \
  "v_mfma_f32_16x16x32_bf16 a[4:7], v[68:71], v[16:19], a[4:7]\n\t" \
  "v_cvt_pk_bf16_f32 v52, v145, v149\n\t" \
  "v_cvt_pk_bf16_f32 v56, v146, v150\n\t" \
  "v_mfma_f32_16x16x32_bf16 a[8:11], v[72:75], v[16:19], a[8:11]\n\t" \
  "v_cvt_pk_bf16_f32 v60, v147, v151\n\t" \
  "v_cvt_pk_bf16_f32 v49, v152, v156\n\t" \
  "v_mfma_f32_16x16x32_bf16 a[12:15], v[76:79], v[16:19], a[12:15]\n\t" \
  "v_cvt_pk_bf16_f32 v53, v153, v157\n\t" \
  "v_cvt_pk_bf16_f32 v57, v154, v158\n\t" \
  "v_mfma_f32_16x16x32_bf16 a[16:19], v[64:67], v[20:23], a[16:19]\n\t" \
  "ds_read_b128 v[16:19], v12 offset:24576\n\t" \
  "v_cvt_pk_bf16_f32 v61, v155, v159\n\t" \
  "v_mfma_f32_16x16x32_bf16 a[20:23], v[68:71], v[20:23], a[20:23]\n\t" \
  "v_cvt_pk_bf16_f32 v50, v160, v164\n\t" \
  "v_cvt_pk_bf16_f32 v54, v161, v165\n\t" \
  "v_mfma_f32_16x16x32_bf16 a[24:27], v[72:75], v[20:23], a[24:27]\n\t" \
  "v_cvt_pk_bf16_f32 v58, v162, v166\n\t" \
  "v_cvt_pk_bf16_f32 v62, v163, v167\n\t" \
  "v_mfma_f32_16x16x32_bf16 a[28:31], v[76:79], v[20:23], a[28:31]\n\t" \
  "s_waitcnt lgkmcnt(5)\n\t" \
  "v_cvt_pk_bf16_f32 v51, v168, v172\n\t" \
  "v_cvt_pk_bf16_f32 v55, v169, v173\n\t" \
  "v_mfma_f32_16x16x32_bf16 a[32:35], v[64:67], v[24:27], a[32:35]\n\t" \
  "ds_read_b128 v[20:23], v12 offset:25600\n\t" \
  "v_cvt_pk_bf16_f32 v59, v170, v174\n\t" \
  "v_mfma_f32_16x16x32_bf16 a[36:39], v[68:71], v[24:27], a[36:39]\n\t" \
  "v_cvt_pk_bf16_f32 v63, v171, v175\n\t" \
  "v_mfma_f32_16x16x32_bf16 a[40:43], v[72:75], v[24:27], a[40:43]\n\t" \
  "s_cbranch_vccnz Lst29_%=\n\t" \
  "s_waitcnt vmcnt(13)\n\t" \
  "Lbk29_%=:\n\t" \
  "buffer_load_dwordx4 v[144:147], v4, %[srdW], %[sW] offen\n\t" \
  "v_mfma_f32_16x16x32_bf16 a[44:47], v[76:79], v[24:27], a[44:47]\n\t" \
  "s_and_b32 s62, s57, 0x7f807f80\n\t" \
  "v_and_b32 v200, s63, v184\n\t" \
  "v_alignbit_b32 v201, v184, v184, 3\n\t" \
  "v_mfma_f32_16x16x32_bf16 a[48:51], v[64:67], v[28:31], a[48:51]\n\t" \
  "ds_read_b128 v[24:27], v12 offset:26624\n\t" \
  "v_perm_b32 v204, v200, v176, s84\n\t" \
  "v_mfma_f32_16x16x32_bf16 a[52:55], v[68:71], v[28:31], a[52:55]\n\t" \
  "v_and_b32 v201, s63, v201\n\t" \
  "v_perm_b32 v205, v200, v176, s85\n\t" \
  "v_mfma_f32_16x16x32_bf16 a[56:59], v[72:75], v[28:31], a[56:59]\n\t" \
  "v_add_u32 v204, s62, v204\n\t" \
  "v_perm_b32 v206, v201, v177, s84\n\t" \
  "v_mfma_f32_16x16x32_bf16 a[60:63], v[76:79], v[28:31], a[60:63]\n\t" \
  "s_waitcnt lgkmcnt(5)\n\t" \
  "buffer_load_dwordx4 v[148:151], v5, %[srdW], %[sW] offen\n\t" \
  "v_mfma_f32_16x16x32_bf16 a[64:67], v[64:67], v[32:35], a[64:67]\n\t" \
  "ds_read_b128 v[28:31], v12 offset:27648\n\t" \
  "v_add_u32 v205, s62, v205\n\t" \
  "v_mfma_f32_16x16x32_bf16 a[68:71], v[68:71], v[32:35], a[68:71]\n\t" \
  "v_perm_b32 v207, v201, v177, s85\n\t" \
  "v_add_u32 v206, s62, v206\n\t" \
  "v_mfma_f32_16x16x32_bf16 a[72:75], v[72:75], v[32:35], a[72:75]\n\t" \
  "v_add_u32 v207, s62, v207\n\t" \
  "ds_write_b128 v13, v[204:207] offset:0\n\t" \
  "v_mfma_f32_16x16x32_bf16 a[76:79], v[76:79], v[32:35], a[76:79]\n\t" \
  "v_and_b32 v202, s63, v185\n\t" \
  "v_alignbit_b32 v203, v185, v185, 3\n\t" \
  "v_mfma_f32_16x16x32_bf16 a[80:83], v[64:67], v[36:39], a[80:83]\n\t" \
  "s_memrealtime s[72:73]\n\t" \
  "ds_read_b128 v[32:35], v12 offset:28672\n\t" \
  "v_perm_b32 v208, v202, v178, s84\n\t" \
  "v_mfma_f32_16x16x32_bf16 a[84:87], v[68:71], v[36:39], a[84:87]\n\t" \
  "buffer_load_dwordx4 v[152:155], v6, %[srdW], %[sW] offen\n\t" \
  "v_mfma_f32_16x16x32_bf16 a[88:91], v[72:75], v[36:39], a[88:91]\n\t" \
  "v_and_b32 v203, s63, v203\n\t" \
  "v_perm_b32 v209, v202, v178, s85\n\t" \
  "v_mfma_f32_16x16x32_bf16 a[92:95], v[76:79], v[36:39], a[92:95]\n\t" \
  "s_waitcnt lgkmcnt(6)\n\t" \
  "v_add_u32 v208, s62, v208\n\t" \
  "v_perm_b32 v210, v203, v179, s84\n\t" \
  "v_mfma_f32_16x16x32_bf16 a[96:99], v[64:67], v[40:43], a[96:99]\n\t" \
  "ds_read_b128 v[36:39], v12 offset:29696\n\t" \
  "v_add_u32 v209, s62, v209\n\t" \
  "v_mfma_f32_16x16x32_bf16 a[100:103], v[68:71], v[40:43], a[100:103]\n\t" \
  "v_perm_b32 v211, v203, v179, s85\n\t" \
  "v_add_u32 v210, s62, v210\n\t" \
  "v_mfma_f32_16x16x32_bf16 a[104:107], v[72:75], v[40:43], a[104:107]\n\t" \
  "buffer_load_dwordx4 v[156:159], v7, %[srdW], %[sW] offen\n\t" \
  "v_mfma_f32_16x16x32_bf16 a[108:111], v[76:79], v[40:43], a[108:111]\n\t" \
  "v_add_u32 v211, s62, v211\n\t" \
  "ds_write_b128 v13, v[208:211] offset:1024\n\t" \
  "v_mfma_f32_16x16x32_bf16 a[112:115], v[64:67], v[44:47], a[112:115]\n\t" \
  "ds_read_b128 v[40:43], v12 offset:30720\n\t" \
  "v_and_b32 v200, s63, v186\n\t" \
  "v_mfma_f32_16x16x32_bf16 a[116:119], v[68:71], v[44:47], a[116:119]\n\t" \
  "v_alignbit_b32 v201, v186, v186, 3\n\t" \
  "v_perm_b32 v212, v200, v180, s84\n\t" \
  "v_mfma_f32_16x16x32_bf16 a[120:123], v[72:75], v[44:47], a[120:123]\n\t" \
  "v_and_b32 v201, s63, v201\n\t" \
  "v_perm_b32 v213, v200, v180, s85\n\t" \
  "v_mfma_f32_16x16x32_bf16 a[124:127], v[76:79], v[44:47], a[124:127]\n\t" \
  "s_waitcnt lgkmcnt(7)\n\t" \
  "buffer_load_dwordx4 v[160:163], v8, %[srdW], %[sW] offen\n\t" \
  "v_mfma_f32_16x16x32_bf16 a[128:131], v[64:67], v[16:19], a[128:131]\n\t" \
  "ds_read_b128 v[44:47], v12 offset:31744\n\t" \
  "v_add_u32 v212, s62, v212\n\t" \
  "v_mfma_f32_16x16x32_bf16 a[132:135], v[68:71], v[16:19], a[132:135]\n\t" \
  "v_perm_b32 v214, v201, v181, s84\n\t" \
  "v_add_u32 v213, s62, v213\n\t" \
  "v_mfma_f32_16x16x32_bf16 a[136:139], v[72:75], v[16:19], a[136:139]\n\t" \
  "v_perm_b32 v215, v201, v181, s85\n\t" \
  "v_add_u32 v214, s62, v214\n\t" \
  "v_mfma_f32_16x16x32_bf16 a[140:143], v[76:79], v[16:19], a[140:143]\n\t" \
  "v_add_u32 v215, s62, v215\n\t" \
  "ds_write_b128 v13, v[212:215] offset:2048\n\t" \
  "v_mfma_f32_16x16x32_bf16 a[144:147], v[64:67], v[20:23], a[144:147]\n\t" \
  "buffer_load_dwordx4 v[164:167], v9, %[srdW], %[sW] offen\n\t" \
  "v_mfma_f32_16x16x32_bf16 a[148:151], v[68:71], v[20:23], a[148:151]\n\t" \
  "v_lshrrev_b32 v202, 5, v184\n\t" \
  "v_lshlrev_b32 v204, 2, v186\n\t" \
  "v_mfma_f32_16x16x32_bf16 a[152:155], v[72:75], v[20:23], a[152:155]\n\t" \
  "v_lshrrev_b32 v203, 5, v185\n\t" \
  "v_lshlrev_b32 v205, 1, v186\n\t" \
  "v_mfma_f32_16x16x32_bf16 a[156:159], v[76:79], v[20:23], a[156:159]\n\t" \
  "s_waitcnt lgkmcnt(7)\n\t" \
  "v_bfi_b32 v202, s64, v204, v202\n\t" \
  "v_bfi_b32 v203, s64, v205, v203\n\t" \
  "v_mfma_f32_16x16x32_bf16 a[160:163], v[64:67], v[24:27], a[160:163]\n\t" \
  "v_and_b32 v202, s63, v202\n\t" \
  "v_perm_b32 v204, v202, v182, s84\n\t" \
  "v_mfma_f32_16x16x32_bf16 a[164:167], v[68:71], v[24:27], a[164:167]\n\t" \
  "buffer_load_dwordx4 v[168:171], v10, %[srdW], %[sW] offen\n\t" \
  "v_mfma_f32_16x16x32_bf16 a[168:171], v[72:75], v[24:27], a[168:171]\n\t" \
  "v_and_b32 v203, s63, v203\n\t" \
  "v_perm_b32 v205, v202, v182, s85\n\t" \
  "v_mfma_f32_16x16x32_bf16 a[172:175], v[76:79], v[24:27], a[172:175]\n\t" \
  "v_add_u32 v204, s62, v204\n\t" \
  "v_perm_b32 v206, v203, v183, s84\n\t" \
  "v_mfma_f32_16x16x32_bf16 a[176:179], v[64:67], v[28:31], a[176:179]\n\t" \
  "v_add_u32 v205, s62, v205\n\t" \
  "v_perm_b32 v207, v203, v183, s85\n\t" \
  "v_mfma_f32_16x16x32_bf16 a[180:183], v[68:71], v[28:31], a[180:183]\n\t" \
  "v_add_u32 v206, s62, v206\n\t" \
  "v_add_u32 v207, s62, v207\n\t" \
  "v_mfma_f32_16x16x32_bf16 a[184:187], v[72:75], v[28:31], a[184:187]\n\t" \
  "buffer_load_dwordx4 v[172:175], v11, %[srdW], %[sW] offen\n\t" \
  "v_mfma_f32_16x16x32_bf16 a[188:191], v[76:79], v[28:31], a[188:191]\n\t" \
  "s_waitcnt lgkmcnt(4)\n\t" \
  "s_add_u32 %[sW], %[sW], 2097152\n\t" \
  "v_lshrrev_b32 v200, 16, v187\n\t" \
  "ds_write_b128 v13, v[204:207] offset:3072\n\t" \
  "v_mfma_f32_16x16x32_bf16 a[192:195], v[64:67], v[32:35], a[192:195]\n\t" \
  "s_bfm_b64 exec, s57, 0\n\t" \
  "s_cbranch_execz Lst30_%=\n\t" \
  "ds_write_b16 v200, v187 offset:0\n\t" \
  "Lbk30_%=:\n\t" \
  "s_mov_b64 exec, -1\n\t" \
  "v_mfma_f32_16x16x32_bf16 a[196:199], v[68:71], v[32:35], a[196:199]\n\t" \
  "v_mfma_f32_16x16x32_bf16 a[200:203], v[72:75], v[32:35], a[200:203]\n\t" \
  "v_mfma_f32_16x16x32_bf16 a[204:207], v[76:79], v[32:35], a[204:207]\n\t" \
  "s_waitcnt lgkmcnt(0)\n\t" \
  "s_barrier\n\t" \
  "v_mfma_f32_16x16x32_bf16 a[208:211], v[64:67], v[36:39], a[208:211]\n\t" \
  "s_cmp_lt_u32 s54, %[cmp]\n\t" \
  "s_cbranch_scc1 Lst31_%=\n\t" \
  "Lbk31_%=:\n\t" \
  "s_add_u32 %[sM], %[sM], 512\n\t" \
  "s_load_dwordx2 s[56:57], %[ws], %[sM] glc\n\t" \
  "ds_read_b128 v[16:19], v12 offset:0\n\t" \
  "v_mfma_f32_16x16x32_bf16 a[212:215], v[68:71], v[36:39], a[212:215]\n\t" \
  "ds_read_b128 v[20:23], v12 offset:1024\n\t" \
  "buffer_load_dwordx4 v[176:179], v12, %[srdX], %[sX] offen sc0 sc1\n\t" \
  "v_mfma_f32_16x16x32_bf16 a[216:219], v[72:75], v[36:39], a[216:219]\n\t" \
  "ds_read_b128 v[24:27], v12 offset:2048\n\t" \
  "v_mfma_f32_16x16x32_bf16 a[220:223], v[76:79], v[36:39], a[220:223]\n\t" \
  "ds_read_b128 v[28:31], v12 offset:3072\n\t" \
  "buffer_load_dwordx4 v[180:183], v12, %[srdX], %[sX] offen offset:1024 sc0 sc1\n\t" \
  "v_mfma_f32_16x16x32_bf16 a[224:227], v[64:67], v[40:43], a[224:227]\n\t" \
  "ds_read_b128 v[32:35], v12 offset:4096\n\t" \
  "v_mfma_f32_16x16x32_bf16 a[228:231], v[68:71], v[40:43], a[228:231]\n\t" \
  "ds_read_b128 v[36:39], v12 offset:5120\n\t" \
  "buffer_load_dwordx3 v[184:186], %[vL12], %[srdX], %[sX] offen offset:2048 sc0 sc1\n\t" \
  "v_mfma_f32_16x16x32_bf16 a[232:235], v[72:75], v[40:43], a[232:235]\n\t" \
  "v_mfma_f32_16x16x32_bf16 a[236:239], v[76:79], v[40:43], a[236:239]\n\t" \
  "buffer_load_dword v187, %[vL4], %[srdX], %[sX] offen offset:2816 sc0 sc1\n\t" \
  "v_mfma_f32_16x16x32_bf16 a[240:243], v[64:67], v[44:47], a[240:243]\n\t" \
  "s_add_u32 %[sX], %[sX], 12288\n\t" \
  "s_sub_u32 s81, s72, s76\n\t" \
  "s_cmp_ge_i32 s81, 0\n\t" \
  "s_cbranch_scc1 Lst32_%=\n\t" \
  "Lbk32_%=:\n\t" \
  "ds_read_b128 v[40:43], v12 offset:6144\n\t" \
  "v_mfma_f32_16x16x32_bf16 a[244:247], v[68:71], v[44:47], a[244:247]\n\t" \
  "v_mfma_f32_16x16x32_bf16 a[248:251], v[72:75], v[44:47], a[248:251]\n\t" \
  "v_mfma_f32_16x16x32_bf16 a[252:255], v[76:79], v[44:47], a[252:255]\n\t" \
  "s_waitcnt lgkmcnt(5)\n\t" \
  "v_mfma_f32_16x16x32_bf16 a[0:3], v[48:51], v[16:19], a[0:3]\n\t" \
  "ds_read_b128 v[44:47], v12 offset:7168\n\t" \
  "v_cvt_pk_bf16_f32 v64, v80, v84\n\t" \
  "v_mfma_f32_16x16x32_bf16 a[4:7], v[52:55], v[16:19], a[4:7]\n\t" \
  "v_cvt_pk_bf16_f32 v68, v81, v85\n\t" \
  "v_cvt_pk_bf16_f32 v72, v82, v86\n\t" \
  "v_mfma_f32_16x16x32_bf16 a[8:11], v[56:59], v[16:19], a[8:11]\n\t" \
  "v_cvt_pk_bf16_f32 v76, v83, v87\n\t" \
  "v_cvt_pk_bf16_f32 v65, v88, v92\n\t" \
  "v_mfma_f32_16x16x32_bf16 a[12:15], v[60:63], v[16:19], a[12:15]\n\t" \
  "v_cvt_pk_bf16_f32 v69, v89, v93\n\t" \
  "v_cvt_pk_bf16_f32 v73, v90, v94\n\t" \
  "v_mfma_f32_16x16x32_bf16 a[16:19], v[48:51], v[20:23], a[16:19]\n\t" \
  "ds_read_b128 v[16:19], v12 offset:8192\n\t" \
  "v_cvt_pk_bf16_f32 v77, v91, v95\n\t" \
  "v_mfma_f32_16x16x32_bf16 a[20:23], v[52:55], v[20:23], a[20:23]\n\t" \
  "v_cvt_pk_bf16_f32 v66, v96, v100\n\t" \
  "v_cvt_pk_bf16_f32 v70, v97, v101\n\t" \
  "v_mfma_f32_16x16x32_bf16 a[24:27], v[56:59], v[20:23], a[24:27]\n\t" \
  "v_cvt_pk_bf16_f32 v74, v98, v102\n\t" \
  "v_cvt_pk_bf16_f32 v78, v99, v103\n\t" \
  "v_mfma_f32_16x16x32_bf16 a[28:31], v[60:63], v[20:23], a[28:31]\n\t" \
  "s_waitcnt lgkmcnt(5)\n\t" \
  "v_cvt_pk_bf16_f32 v67, v104, v108\n\t" \
  "v_cvt_pk_bf16_f32 v71, v105, v109\n\t" \
  "v_mfma_f32_16x16x32_bf16 a[32:35], v[48:51], v[24:27], a[32:35]\n\t" \
  "ds_read_b128 v[20:23], v12 offset:9216\n\t" \
  "v_cvt_pk_bf16_f32 v75, v106, v110\n\t" \
  "v_mfma_f32_16x16x32_bf16 a[36:39], v[52:55], v[24:27], a[36:39]\n\t" \
  "v_cvt_pk_bf16_f32 v79, v107, v111\n\t" \
  "v_mfma_f32_16x16x32_bf16 a[40:43], v[56:59], v[24:27], a[40:43]\n\t" \
  "s_cbranch_vccnz Lst33_%=\n\t" \
  "s_waitcnt vmcnt(12)\n\t" \
  "Lbk33_%=:\n\t" \
  "buffer_load_dwordx4 v[80:83], v4, %[srdW], %[sW] offen\n\t" \
  "v_mfma_f32_16x16x32_bf16 a[44:47], v[60:63], v[24:27], a[44:47]\n\t" \
  "s_and_b32 s62, s53, 0x7f807f80\n\t" \
  "v_and_b32 v200, s63, v196\n\t" \
  "v_alignbit_b32 v201, v196, v196, 3\n\t" \
  "v_mfma_f32_16x16x32_bf16 a[48:51], v[48:51], v[28:31], a[48:51]\n\t" \
  "ds_read_b128 v[24:27], v12 offset:10240\n\t" \
  "v_perm_b32 v204, v200, v188, s84\n\t" \
  "v_mfma_f32_16x16x32_bf16 a[52:55], v[52:55], v[28:31], a[52:55]\n\t" \
  "v_and_b32 v201, s63, v201\n\t" \
  "v_perm_b32 v205, v200, v188, s85\n\t" \
  "v_mfma_f32_16x16x32_bf16 a[56:59], v[56:59], v[28:31], a[56:59]\n\t" \
  "v_add_u32 v204, s62, v204\n\t" \
  "v_perm_b32 v206, v201, v189, s84\n\t" \
  "v_mfma_f32_16x16x32_bf16 a[60:63], v[60:63], v[28:31], a[60:63]\n\t" \
  "s_waitcnt lgkmcnt(5)\n\t" \
  "buffer_load_dwordx4 v[84:87], v5, %[srdW], %[sW] offen\n\t" \
  "v_mfma_f32_16x16x32_bf16 a[64:67], v[48:51], v[32:35], a[64:67]\n\t" \
  "ds_read_b128 v[28:31], v12 offset:11264\n\t" \
  "v_add_u32 v205, s62, v205\n\t" \
  "v_mfma_f32_16x16x32_bf16 a[68:71], v[52:55], v[32:35], a[68:71]\n\t" \
  "v_perm_b32 v207, v201, v189, s85\n\t" \
  "v_add_u32 v206, s62, v206\n\t" \
  "v_mfma_f32_16x16x32_bf16 a[72:75], v[56:59], v[32:35], a[72:75]\n\t" \
  "v_add_u32 v207, s62, v207\n\t" \
  "ds_write_b128 v13, v[204:207] offset:16384\n\t" \
  "v_mfma_f32_16x16x32_bf16 a[76:79], v[60:63], v[32:35], a[76:79]\n\t" \
  "v_and_b32 v202, s63, v197\n\t" \
  "v_alignbit_b32 v203, v197, v197, 3\n\t" \
  "v_mfma_f32_16x16x32_bf16 a[80:83], v[48:51], v[36:39], a[80:83]\n\t" \
  "s_memrealtime s[72:73]\n\t" \
  "ds_read_b128 v[32:35], v12 offset:12288\n\t" \
  "v_perm_b32 v208, v202, v190, s84\n\t" \
  "v_mfma_f32_16x16x32_bf16 a[84:87], v[52:55], v[36:39], a[84:87]\n\t" \
  "buffer_load_dwordx4 v[88:91], v6, %[srdW], %[sW] offen\n\t" \
  "v_mfma_f32_16x16x32_bf16 a[88:91], v[56:59], v[36:39], a[88:91]\n\t" \
  "v_and_b32 v203, s63, v203\n\t" \
  "v_perm_b32 v209, v202, v190, s85\n\t" \
  "v_mfma_f32_16x16x32_bf16 a[92:95], v[60:63], v[36:39], a[92:95]\n\t" \
  "s_waitcnt lgkmcnt(6)\n\t" \
  "v_add_u32 v208, s62, v208\n\t" \
  "v_perm_b32 v210, v203, v191, s84\n\t" \
  "v_mfma_f32_16x16x32_bf16 a[96:99], v[48:51], v[40:43], a[96:99]\n\t" \
  "ds_read_b128 v[36:39], v12 offset:13312\n\t" \
  "v_add_u32 v209, s62, v209\n\t" \
  "v_mfma_f32_16x16x32_bf16 a[100:103], v[52:55], v[40:43], a[100:103]\n\t" \
  "v_perm_b32 v211, v203, v191, s85\n\t" \
  "v_add_u32 v210, s62, v210\n\t" \
  "v_mfma_f32_16x16x32_bf16 a[104:107], v[56:59], v[40:43], a[104:107]\n\t" \
  "buffer_load_dwordx4 v[92:95], v7, %[srdW], %[sW] offen\n\t" \
  "v_mfma_f32_16x16x32_bf16 a[108:111], v[60:63], v[40:43], a[108:111]\n\t" \
  "v_add_u32 v211, s62, v211\n\t" \
  "ds_write_b128 v13, v[208:211] offset:17408\n\t" \
  "v_mfma_f32_16x16x32_bf16 a[112:115], v[48:51], v[44:47], a[112:115]\n\t" \
  "ds_read_b128 v[40:43], v12 offset:14336\n\t" \
  "v_and_b32 v200, s63, v198\n\t" \
  "v_mfma_f32_16x16x32_bf16 a[116:119], v[52:55], v[44:47], a[116:119]\n\t" \
  "v_alignbit_b32 v201, v198, v198, 3\n\t" \
  "v_perm_b32 v212, v200, v192, s84\n\t" \
  "v_mfma_f32_16x16x32_bf16 a[120:123], v[56:59], v[44:47], a[120:123]\n\t" \
  "v_and_b32 v201, s63, v201\n\t" \
  "v_perm_b32 v213, v200, v192, s85\n\t" \
  "v_mfma_f32_16x16x32_bf16 a[124:127], v[60:63], v[44:47], a[124:127]\n\t" \
  "s_waitcnt lgkmcnt(7)\n\t" \
  "buffer_load_dwordx4 v[96:99], v8, %[srdW], %[sW] offen\n\t" \
  "v_mfma_f32_16x16x32_bf16 a[128:131], v[48:51], v[16:19], a[128:131]\n\t" \
  "ds_read_b128 v[44:47], v12 offset:15360\n\t" \
  "v_add_u32 v212, s62, v212\n\t" \
  "v_mfma_f32_16x16x32_bf16 a[132:135], v[52:55], v[16:19], a[132:135]\n\t" \
  "v_perm_b32 v214, v201, v193, s84\n\t" \
  "v_add_u32 v213, s62, v213\n\t" \
  "v_mfma_f32_16x16x32_bf16 a[136:139], v[56:59], v[16:19], a[136:139]\n\t" \
  "v_perm_b32 v215, v201, v193, s85\n\t" \
  "v_add_u32 v214, s62, v214\n\t" \
  "v_mfma_f32_16x16x32_bf16 a[140:143], v[60:63], v[16:19], a[140:143]\n\t" \
  "v_add_u32 v215, s62, v215\n\t" \
  "ds_write_b128 v13, v[212:215] offset:18432\n\t" \
  "v_mfma_f32_16x16x32_bf16 a[144:147], v[48:51], v[20:23], a[144:147]\n\t" \
  "buffer_load_dwordx4 v[100:103], v9, %[srdW], %[sW] offen\n\t" \
  "v_mfma_f32_16x16x32_bf16 a[148:151], v[52:55], v[20:23], a[148:151]\n\t" \
  "v_lshrrev_b32 v202, 5, v196\n\t" \
  "v_lshlrev_b32 v204, 2, v198\n\t" \
  "v_mfma_f32_16x16x32_bf16 a[152:155], v[56:59], v[20:23], a[152:155]\n\t" \
  "v_lshrrev_b32 v203, 5, v197\n\t" \
  "v_lshlrev_b32 v205, 1, v198\n\t" \
  "v_mfma_f32_16x16x32_bf16 a[156:159], v[60:63], v[20:23], a[156:159]\n\t" \
  "s_waitcnt lgkmcnt(7)\n\t" \
  "v_bfi_b32 v202, s64, v204, v202\n\t" \
  "v_bfi_b32 v203, s64, v205, v203\n\t" \
  "v_mfma_f32_16x16x32_bf16 a[160:163], v[48:51], v[24:27], a[160:163]\n\t" \
  "v_and_b32 v202, s63, v202\n\t" \
  "v_perm_b32 v204, v202, v194, s84\n\t" \
  "v_mfma_f32_16x16x32_bf16 a[164:167], v[52:55], v[24:27], a[164:167]\n\t" \
  "buffer_load_dwordx4 v[104:107], v10, %[srdW], %[sW] offen\n\t" \
  "v_mfma_f32_16x16x32_bf16 a[168:171], v[56:59], v[24:27], a[168:171]\n\t" \
  "v_and_b32 v203, s63, v203\n\t" \
  "v_perm_b32 v205, v202, v194, s85\n\t" \
  "v_mfma_f32_16x16x32_bf16 a[172:175], v[60:63], v[24:27], a[172:175]\n\t" \
  "v_add_u32 v204, s62, v204\n\t" \
  "v_perm_b32 v206, v203, v195, s84\n\t" \
  "v_mfma_f32_16x16x32_bf16 a[176:179], v[48:51], v[28:31], a[176:179]\n\t" \
  "v_add_u32 v205, s62, v205\n\t" \
  "v_perm_b32 v207, v203, v195, s85\n\t" \
  "v_mfma_f32_16x16x32_bf16 a[180:183], v[52:55], v[28:31], a[180:183]\n\t" \
  "v_add_u32 v206, s62, v206\n\t" \
  "v_add_u32 v207, s62, v207\n\t" \
  "v_mfma_f32_16x16x32_bf16 a[184:187], v[56:59], v[28:31], a[184:187]\n\t" \
  "buffer_load_dwordx4 v[108:111], v11, %[srdW], %[sW] offen\n\t" \
  "v_mfma_f32_16x16x32_bf16 a[188:191], v[60:63], v[28:31], a[188:191]\n\t" \
  "s_waitcnt lgkmcnt(4)\n\t" \
  "s_add_u32 %[sW], %[sW], 2097152\n\t" \
  "v_lshrrev_b32 v200, 16, v199\n\t" \
  "ds_write_b128 v13, v[204:207] offset:19456\n\t" \
  "v_mfma_f32_16x16x32_bf16 a[192:195], v[48:51], v[32:35], a[192:195]\n\t" \
  "s_bfm_b64 exec, s53, 0\n\t" \
  "s_cbranch_execz Lst34_%=\n\t" \
  "ds_write_b16 v200, v199 offset:16384\n\t" \
  "Lbk34_%=:\n\t" \
  "s_mov_b64 exec, -1\n\t" \
  "v_mfma_f32_16x16x32_bf16 a[196:199], v[52:55], v[32:35], a[196:199]\n\t" \
  "v_mfma_f32_16x16x32_bf16 a[200:203], v[56:59], v[32:35], a[200:203]\n\t" \
  "v_mfma_f32_16x16x32_bf16 a[204:207], v[60:63], v[32:35], a[204:207]\n\t" \
  "s_waitcnt lgkmcnt(0)\n\t" \
  "s_barrier\n\t" \
  "v_mfma_f32_16x16x32_bf16 a[208:211], v[48:51], v[36:39], a[208:211]\n\t" \
  "s_cmp_lt_u32 s56, %[cmp]\n\t" \
  "s_cbranch_scc1 Lst35_%=\n\t" \
  "Lbk35_%=:\n\t" \
  "s_add_u32 %[sM], %[sM], 512\n\t" \
  "s_load_dwordx2 s[52:53], %[ws], %[sM] glc\n\t" \
  "ds_read_b128 v[16:19], v12 offset:16384\n\t" \
  "v_mfma_f32_16x16x32_bf16 a[212:215], v[52:55], v[36:39], a[212:215]\n\t" \
  "ds_read_b128 v[20:23], v12 offset:17408\n\t" \
  "buffer_load_dwordx4 v[188:191], v12, %[srdX], %[sX] offen sc0 sc1\n\t" \
  "v_mfma_f32_16x16x32_bf16 a[216:219], v[56:59], v[36:39], a[216:219]\n\t" \
  "ds_read_b128 v[24:27], v12 offset:18432\n\t" \
  "v_mfma_f32_16x16x32_bf16 a[220:223], v[60:63], v[36:39], a[220:223]\n\t" \
  "ds_read_b128 v[28:31], v12 offset:19456\n\t" \
  "buffer_load_dwordx4 v[192:195], v12, %[srdX], %[sX] offen offset:1024 sc0 sc1\n\t" \
  "v_mfma_f32_16x16x32_bf16 a[224:227], v[48:51], v[40:43], a[224:227]\n\t" \
  "ds_read_b128 v[32:35], v12 offset:20480\n\t" \
  "v_mfma_f32_16x16x32_bf16 a[228:231], v[52:55], v[40:43], a[228:231]\n\t" \
  "ds_read_b128 v[36:39], v12 offset:21504\n\t" \
  "buffer_load_dwordx3 v[196:198], %[vL12], %[srdX], %[sX] offen offset:2048 sc0 sc1\n\t" \
  "v_mfma_f32_16x16x32_bf16 a[232:235], v[56:59], v[40:43], a[232:235]\n\t" \
  "v_mfma_f32_16x16x32_bf16 a[236:239], v[60:63], v[40:43], a[236:239]\n\t" \
  "buffer_load_dword v199, %[vL4], %[srdX], %[sX] offen offset:2816 sc0 sc1\n\t" \
  "v_mfma_f32_16x16x32_bf16 a[240:243], v[48:51], v[44:47], a[240:243]\n\t" \
  "s_add_u32 %[sX], %[sX], 12288\n\t" \
  "s_sub_u32 s81, s72, s76\n\t" \
  "s_cmp_ge_i32 s81, 0\n\t" \
  "s_cbranch_scc1 Lst36_%=\n\t" \
  "Lbk36_%=:\n\t" \
  "ds_read_b128 v[40:43], v12 offset:22528\n\t" \
  "v_mfma_f32_16x16x32_bf16 a[244:247], v[52:55], v[44:47], a[244:247]\n\t" \
  "v_mfma_f32_16x16x32_bf16 a[248:251], v[56:59], v[44:47], a[248:251]\n\t" \
  "v_mfma_f32_16x16x32_bf16 a[252:255], v[60:63], v[44:47], a[252:255]\n\t" \
  "s_waitcnt lgkmcnt(5)\n\t" \
  "v_mfma_f32_16x16x32_bf16 a[0:3], v[64:67], v[16:19], a[0:3]\n\t" \
  "ds_read_b128 v[44:47], v12 offset:23552\n\t" \
  "v_cvt_pk_bf16_f32 v48, v112, v116\n\t" \
  "v_mfma_f32_16x16x32_bf16 a[4:7], v[68:71], v[16:19], a[4:7]\n\t" \
  "v_cvt_pk_bf16_f32 v52, v113, v117\n\t" \
  "v_cvt_pk_bf16_f32 v56, v114, v118\n\t" \
  "v_mfma_f32_16x16x32_bf16 a[8:11], v[72:75], v[16:19], a[8:11]\n\t" \
  "v_cvt_pk_bf16_f32 v60, v115, v119\n\t" \
  "v_cvt_pk_bf16_f32 v49, v120, v124\n\t" \
  "v_mfma_f32_16x16x32_bf16 a[12:15], v[76:79], v[16:19], a[12:15]\n\t" \
  "v_cvt_pk_bf16_f32 v53, v121, v125\n\t" \
  "v_cvt_pk_bf16_f32 v57, v122, v126\n\t" \
  "v_mfma_f32_16x16x32_bf16 a[16:19], v[64:67], v[20:23], a[16:19]\n\t" \
  "ds_read_b128 v[16:19], v12 offset:24576\n\t" \
  "v_cvt_pk_bf16_f32 v61, v123, v127\n\t" \
  "v_mfma_f32_16x16x32_bf16 a[20:23], v[68:71], v[20:23], a[20:23]\n\t" \
  "v_cvt_pk_bf16_f32 v50, v128, v132\n\t" \
  "v_cvt_pk_bf16_f32 v54, v129, v133\n\t" \
  "v_mfma_f32_16x16x32_bf16 a[24:27], v[72:75], v[20:23], a[24:27]\n\t" \
  "v_cvt_pk_bf16_f32 v58, v130, v134\n\t" \
  "v_cvt_pk_bf16_f32 v62, v131, v135\n\t" \
  "v_mfma_f32_16x16x32_bf16 a[28:31], v[76:79], v[20:23], a[28:31]\n\t" \
  "s_waitcnt lgkmcnt(5)\n\t" \
  "v_cvt_pk_bf16_f32 v51, v136, v140\n\t" \
  "v_cvt_pk_bf16_f32 v55, v137, v141\n\t" \
  "v_mfma_f32_16x16x32_bf16 a[32:35], v[64:67], v[24:27], a[32:35]\n\t" \
  "ds_read_b128 v[20:23], v12 offset:25600\n\t" \
  "v_cvt_pk_bf16_f32 v59, v138, v142\n\t" \
  "v_mfma_f32_16x16x32_bf16 a[36:39], v[68:71], v[24:27], a[36:39]\n\t" \
  "v_cvt_pk_bf16_f32 v63, v139, v143\n\t" \
  "v_mfma_f32_16x16x32_bf16 a[40:43], v[72:75], v[24:27], a[40:43]\n\t" \
  "s_cbranch_vccnz Lst37_%=\n\t" \
  "s_waitcnt vmcnt(12)\n\t" \
  "Lbk37_%=:\n\t" \
  "buffer_load_dwordx4 v[112:115], v4, %[srdW], %[sW] offen\n\t" \
  "v_mfma_f32_16x16x32_bf16 a[44:47], v[76:79], v[24:27], a[44:47]\n\t" \
  "s_and_b32 s62, s55, 0x7f807f80\n\t" \
  "v_and_b32 v200, s63, v184\n\t" \
  "v_alignbit_b32 v201, v184, v184, 3\n\t" \
  "v_mfma_f32_16x16x32_bf16 a[48:51], v[64:67], v[28:31], a[48:51]\n\t" \
  "ds_read_b128 v[24:27], v12 offset:26624\n\t" \
  "v_perm_b32 v204, v200, v176, s84\n\t" \
  "v_mfma_f32_16x16x32_bf16 a[52:55], v[68:71], v[28:31], a[52:55]\n\t" \
  "v_and_b32 v201, s63, v201\n\t" \
  "v_perm_b32 v205, v200, v176, s85\n\t" \
  "v_mfma_f32_16x16x32_bf16 a[56:59], v[72:75], v[28:31], a[56:59]\n\t" \
  "v_add_u32 v204, s62, v204\n\t" \
  "v_perm_b32 v206, v201, v177, s84\n\t" \
  "v_mfma_f32_16x16x32_bf16 a[60:63], v[76:79], v[28:31], a[60:63]\n\t" \
  "s_waitcnt lgkmcnt(5)\n\t" \
  "buffer_load_dwordx4 v[116:119], v5, %[srdW], %[sW] offen\n\t" \
  "v_mfma_f32_16x16x32_bf16 a[64:67], v[64:67], v[32:35], a[64:67]\n\t" \
  "ds_read_b128 v[28:31], v12 offset:27648\n\t" \
  "v_add_u32 v205, s62, v205\n\t" \
  "v_mfma_f32_16x16x32_bf16 a[68:71], v[68:71], v[32:35], a[68:71]\n\t" \
  "v_perm_b32 v207, v201, v177, s85\n\t" \
  "v_add_u32 v206, s62, v206\n\t" \
  "v_mfma_f32_16x16x32_bf16 a[72:75], v[72:75], v[32:35], a[72:75]\n\t" \
  "v_add_u32 v207, s62, v207\n\t" \
  "ds_write_b128 v13, v[204:207] offset:0\n\t" \
  "v_mfma_f32_16x16x32_bf16 a[76:79], v[76:79], v[32:35], a[76:79]\n\t" \
  "v_and_b32 v202, s63, v185\n\t" \
  "v_alignbit_b32 v203, v185, v185, 3\n\t" \
  "v_mfma_f32_16x16x32_bf16 a[80:83], v[64:67], v[36:39], a[80:83]\n\t" \
  "s_memrealtime s[72:73]\n\t" \
  "ds_read_b128 v[32:35], v12 offset:28672\n\t" \
  "v_perm_b32 v208, v202, v178, s84\n\t" \
  "v_mfma_f32_16x16x32_bf16 a[84:87], v[68:71], v[36:39], a[84:87]\n\t" \
  "buffer_load_dwordx4 v[120:123], v6, %[srdW], %[sW] offen\n\t" \
  "v_mfma_f32_16x16x32_bf16 a[88:91], v[72:75], v[36:39], a[88:91]\n\t" \
  "v_and_b32 v203, s63, v203\n\t" \
  "v_perm_b32 v209, v202, v178, s85\n\t" \
  "v_mfma_f32_16x16x32_bf16 a[92:95], v[76:79], v[36:39], a[92:95]\n\t" \
  "s_waitcnt lgkmcnt(6)\n\t" \
  "v_add_u32 v208, s62, v208\n\t" \
  "v_perm_b32 v210, v203, v179, s84\n\t" \
  "v_mfma_f32_16x16x32_bf16 a[96:99], v[64:67], v[40:43], a[96:99]\n\t" \
  "ds_read_b128 v[36:39], v12 offset:29696\n\t" \
  "v_add_u32 v209, s62, v209\n\t" \
  "v_mfma_f32_16x16x32_bf16 a[100:103], v[68:71], v[40:43], a[100:103]\n\t" \
  "v_perm_b32 v211, v203, v179, s85\n\t" \
  "v_add_u32 v210, s62, v210\n\t" \
  "v_mfma_f32_16x16x32_bf16 a[104:107], v[72:75], v[40:43], a[104:107]\n\t" \
  "buffer_load_dwordx4 v[124:127], v7, %[srdW], %[sW] offen\n\t" \
  "v_mfma_f32_16x16x32_bf16 a[108:111], v[76:79], v[40:43], a[108:111]\n\t" \
  "v_add_u32 v211, s62, v211\n\t" \
  "ds_write_b128 v13, v[208:211] offset:1024\n\t" \
  "v_mfma_f32_16x16x32_bf16 a[112:115], v[64:67], v[44:47], a[112:115]\n\t" \
  "ds_read_b128 v[40:43], v12 offset:30720\n\t" \
  "v_and_b32 v200, s63, v186\n\t" \
  "v_mfma_f32_16x16x32_bf16 a[116:119], v[68:71], v[44:47], a[116:119]\n\t" \
  "v_alignbit_b32 v201, v186, v186, 3\n\t" \
  "v_perm_b32 v212, v200, v180, s84\n\t" \
  "v_mfma_f32_16x16x32_bf16 a[120:123], v[72:75], v[44:47], a[120:123]\n\t" \
  "v_and_b32 v201, s63, v201\n\t" \
  "v_perm_b32 v213, v200, v180, s85\n\t" \
  "v_mfma_f32_16x16x32_bf16 a[124:127], v[76:79], v[44:47], a[124:127]\n\t" \
  "s_waitcnt lgkmcnt(7)\n\t" \
  "buffer_load_dwordx4 v[128:131], v8, %[srdW], %[sW] offen\n\t" \
  "v_mfma_f32_16x16x32_bf16 a[128:131], v[64:67], v[16:19], a[128:131]\n\t" \
  "ds_read_b128 v[44:47], v12 offset:31744\n\t" \
  "v_add_u32 v212, s62, v212\n\t" \
  "v_mfma_f32_16x16x32_bf16 a[132:135], v[68:71], v[16:19], a[132:135]\n\t" \
  "v_perm_b32 v214, v201, v181, s84\n\t" \
  "v_add_u32 v213, s62, v213\n\t" \
  "v_mfma_f32_16x16x32_bf16 a[136:139], v[72:75], v[16:19], a[136:139]\n\t" \
  "v_perm_b32 v215, v201, v181, s85\n\t" \
  "v_add_u32 v214, s62, v214\n\t" \
  "v_mfma_f32_16x16x32_bf16 a[140:143], v[76:79], v[16:19], a[140:143]\n\t" \
  "v_add_u32 v215, s62, v215\n\t" \
  "ds_write_b128 v13, v[212:215] offset:2048\n\t" \
  "v_mfma_f32_16x16x32_bf16 a[144:147], v[64:67], v[20:23], a[144:147]\n\t" \
  "buffer_load_dwordx4 v[132:135], v9, %[srdW], %[sW] offen\n\t" \
  "v_mfma_f32_16x16x32_bf16 a[148:151], v[68:71], v[20:23], a[148:151]\n\t" \
  "v_lshrrev_b32 v202, 5, v184\n\t" \
  "v_lshlrev_b32 v204, 2, v186\n\t" \
  "v_mfma_f32_16x16x32_bf16 a[152:155], v[72:75], v[20:23], a[152:155]\n\t" \
  "v_lshrrev_b32 v203, 5, v185\n\t" \
  "v_lshlrev_b32 v205, 1, v186\n\t" \
  "v_mfma_f32_16x16x32_bf16 a[156:159], v[76:79], v[20:23], a[156:159]\n\t" \
  "s_waitcnt lgkmcnt(7)\n\t" \
  "v_bfi_b32 v202, s64, v204, v202\n\t" \
  "v_bfi_b32 v203, s64, v205, v203\n\t" \
  "v_mfma_f32_16x16x32_bf16 a[160:163], v[64:67], v[24:27], a[160:163]\n\t" \
  "v_and_b32 v202, s63, v202\n\t" \
  "v_perm_b32 v204, v202, v182, s84\n\t" \
  "v_mfma_f32_16x16x32_bf16 a[164:167], v[68:71], v[24:27], a[164:167]\n\t" \
  "buffer_load_dwordx4 v[136:139], v10, %[srdW], %[sW] offen\n\t" \
  "v_mfma_f32_16x16x32_bf16 a[168:171], v[72:75], v[24:27], a[168:171]\n\t" \
  "v_and_b32 v203, s63, v203\n\t" \
  "v_perm_b32 v205, v202, v182, s85\n\t" \
  "v_mfma_f32_16x16x32_bf16 a[172:175], v[76:79], v[24:27], a[172:175]\n\t" \
  "v_add_u32 v204, s62, v204\n\t" \
  "v_perm_b32 v206, v203, v183, s84\n\t" \
  "v_mfma_f32_16x16x32_bf16 a[176:179], v[64:67], v[28:31], a[176:179]\n\t" \
  "v_add_u32 v205, s62, v205\n\t" \
  "v_perm_b32 v207, v203, v183, s85\n\t" \
  "v_mfma_f32_16x16x32_bf16 a[180:183], v[68:71], v[28:31], a[180:183]\n\t" \
  "v_add_u32 v206, s62, v206\n\t" \
  "v_add_u32 v207, s62, v207\n\t" \
  "v_mfma_f32_16x16x32_bf16 a[184:187], v[72:75], v[28:31], a[184:187]\n\t" \
  "buffer_load_dwordx4 v[140:143], v11, %[srdW], %[sW] offen\n\t" \
  "v_mfma_f32_16x16x32_bf16 a[188:191], v[76:79], v[28:31], a[188:191]\n\t" \
  "s_waitcnt lgkmcnt(4)\n\t" \
  "s_add_u32 %[sW], %[sW], 2097152\n\t" \
  "v_lshrrev_b32 v200, 16, v187\n\t" \
  "ds_write_b128 v13, v[204:207] offset:3072\n\t" \
  "v_mfma_f32_16x16x32_bf16 a[192:195], v[64:67], v[32:35], a[192:195]\n\t" \
  "s_bfm_b64 exec, s55, 0\n\t" \
  "s_cbranch_execz Lst38_%=\n\t" \
  "ds_write_b16 v200, v187 offset:0\n\t" \
  "Lbk38_%=:\n\t" \
  "s_mov_b64 exec, -1\n\t" \
  "v_mfma_f32_16x16x32_bf16 a[196:199], v[68:71], v[32:35], a[196:199]\n\t" \
  "v_mfma_f32_16x16x32_bf16 a[200:203], v[72:75], v[32:35], a[200:203]\n\t" \
  "v_mfma_f32_16x16x32_bf16 a[204:207], v[76:79], v[32:35], a[204:207]\n\t" \
  "s_waitcnt lgkmcnt(0)\n\t" \
  "s_barrier\n\t" \
  "v_mfma_f32_16x16x32_bf16 a[208:211], v[64:67], v[36:39], a[208:211]\n\t" \
  "s_cmp_lt_u32 s52, %[cmp]\n\t" \
  "s_cbranch_scc1 Lst39_%=\n\t" \
  "Lbk39_%=:\n\t" \
  "s_add_u32 %[sM], %[sM], 512\n\t" \
  "s_load_dwordx2 s[54:55], %[ws], %[sM] glc\n\t" \
  "ds_read_b128 v[16:19], v12 offset:0\n\t" \
  "v_mfma_f32_16x16x32_bf16 a[212:215], v[68:71], v[36:39], a[212:215]\n\t" \
  "ds_read_b128 v[20:23], v12 offset:1024\n\t" \
  "buffer_load_dwordx4 v[176:179], v12, %[srdX], %[sX] offen sc0 sc1\n\t" \
  "v_mfma_f32_16x16x32_bf16 a[216:219], v[72:75], v[36:39], a[216:219]\n\t" \
  "ds_read_b128 v[24:27], v12 offset:2048\n\t" \
  "v_mfma_f32_16x16x32_bf16 a[220:223], v[76:79], v[36:39], a[220:223]\n\t" \
  "ds_read_b128 v[28:31], v12 offset:3072\n\t" \
  "buffer_load_dwordx4 v[180:183], v12, %[srdX], %[sX] offen offset:1024 sc0 sc1\n\t" \
  "v_mfma_f32_16x16x32_bf16 a[224:227], v[64:67], v[40:43], a[224:227]\n\t" \
  "ds_read_b128 v[32:35], v12 offset:4096\n\t" \
  "v_mfma_f32_16x16x32_bf16 a[228:231], v[68:71], v[40:43], a[228:231]\n\t" \
  "ds_read_b128 v[36:39], v12 offset:5120\n\t" \
  "buffer_load_dwordx3 v[184:186], %[vL12], %[srdX], %[sX] offen offset:2048 sc0 sc1\n\t" \
  "v_mfma_f32_16x16x32_bf16 a[232:235], v[72:75], v[40:43], a[232:235]\n\t" \
  "v_mfma_f32_16x16x32_bf16 a[236:239], v[76:79], v[40:43], a[236:239]\n\t" \
  "buffer_load_dword v187, %[vL4], %[srdX], %[sX] offen offset:2816 sc0 sc1\n\t" \
  "v_mfma_f32_16x16x32_bf16 a[240:243], v[64:67], v[44:47], a[240:243]\n\t" \
  "s_add_u32 %[sX], %[sX], 12288\n\t" \
  "s_sub_u32 s81, s72, s76\n\t" \
  "s_cmp_ge_i32 s81, 0\n\t" \
  "s_cbranch_scc1 Lst40_%=\n\t" \
  "Lbk40_%=:\n\t" \
  "ds_read_b128 v[40:43], v12 offset:6144\n\t" \
  "v_mfma_f32_16x16x32_bf16 a[244:247], v[68:71], v[44:47], a[244:247]\n\t" \
  "v_mfma_f32_16x16x32_bf16 a[248:251], v[72:75], v[44:47], a[248:251]\n\t" \
  "v_mfma_f32_16x16x32_bf16 a[252:255], v[76:79], v[44:47], a[252:255]\n\t" \
  "s_waitcnt lgkmcnt(5)\n\t" \
  "v_mfma_f32_16x16x32_bf16 a[0:3], v[48:51], v[16:19], a[0:3]\n\t" \
  "ds_read_b128 v[44:47], v12 offset:7168\n\t" \
  "v_cvt_pk_bf16_f32 v64, v144, v148\n\t" \
  "v_mfma_f32_16x16x32_bf16 a[4:7], v[52:55], v[16:19], a[4:7]\n\t" \
  "v_cvt_pk_bf16_f32 v68, v145, v149\n\t" \
  "v_cvt_pk_bf16_f32 v72, v146, v150\n\t" \
  "v_mfma_f32_16x16x32_bf16 a[8:11], v[56:59], v[16:19], a[8:11]\n\t" \
  "v_cvt_pk_bf16_f32 v76, v147, v151\n\t" \
  "v_cvt_pk_bf16_f32 v65, v152, v156\n\t" \
  "v_mfma_f32_16x16x32_bf16 a[12:15], v[60:63], v[16:19], a[12:15]\n\t" \
  "v_cvt_pk_bf16_f32 v69, v153, v157\n\t" \
  "v_cvt_pk_bf16_f32 v73, v154, v158\n\t" \
  "v_mfma_f32_16x16x32_bf16 a[16:19], v[48:51], v[20:23], a[16:19]\n\t" \
  "ds_read_b128 v[16:19], v12 offset:8192\n\t" \
  "v_cvt_pk_bf16_f32 v77, v155, v159\n\t" \
  "v_mfma_f32_16x16x32_bf16 a[20:23], v[52:55], v[20:23], a[20:23]\n\t" \
  "v_cvt_pk_bf16_f32 v66, v160, v164\n\t" \
  "v_cvt_pk_bf16_f32 v70, v161, v165\n\t" \
  "v_mfma_f32_16x16x32_bf16 a[24:27], v[56:59], v[20:23], a[24:27]\n\t" \
  "v_cvt_pk_bf16_f32 v74, v162, v166\n\t" \
  "v_cvt_pk_bf16_f32 v78, v163, v167\n\t" \
  "v_mfma_f32_16x16x32_bf16 a[28:31], v[60:63], v[20:23], a[28:31]\n\t" \
  "s_waitcnt lgkmcnt(5)\n\t" \
  "v_cvt_pk_bf16_f32 v67, v168, v172\n\t" \
  "v_cvt_pk_bf16_f32 v71, v169, v173\n\t" \
  "v_mfma_f32_16x16x32_bf16 a[32:35], v[48:51], v[24:27], a[32:35]\n\t" \
  "ds_read_b128 v[20:23], v12 offset:9216\n\t" \
  "v_cvt_pk_bf16_f32 v75, v170, v174\n\t" \
  "v_mfma_f32_16x16x32_bf16 a[36:39], v[52:55], v[24:27], a[36:39]\n\t" \
  "v_cvt_pk_bf16_f32 v79, v171, v175\n\t" \
  "v_mfma_f32_16x16x32_bf16 a[40:43], v[56:59], v[24:27], a[40:43]\n\t" \
  "s_cbranch_vccnz Lst41_%=\n\t" \
  "s_waitcnt vmcnt(12)\n\t" \
  "Lbk41_%=:\n\t" \
  "buffer_load_dwordx4 v[144:147], v4, %[srdW], %[sW] offen\n\t" \
  "v_mfma_f32_16x16x32_bf16 a[44:47], v[60:63], v[24:27], a[44:47]\n\t" \
  "s_and_b32 s62, s57, 0x7f807f80\n\t" \
  "v_and_b32 v200, s63, v196\n\t" \
  "v_alignbit_b32 v201, v196, v196, 3\n\t" \
  "v_mfma_f32_16x16x32_bf16 a[48:51], v[48:51], v[28:31], a[48:51]\n\t" \
  "ds_read_b128 v[24:27], v12 offset:10240\n\t" \
  "v_perm_b32 v204, v200, v188, s84\n\t" \
  "v_mfma_f32_16x16x32_bf16 a[52:55], v[52:55], v[28:31], a[52:55]\n\t" \
  "v_and_b32 v201, s63, v201\n\t" \
  "v_perm_b32 v205, v200, v188, s85\n\t" \
  "v_mfma_f32_16x16x32_bf16 a[56:59], v[56:59], v[28:31], a[56:59]\n\t" \
  "v_add_u32 v204, s62, v204\n\t" \
  "v_perm_b32 v206, v201, v189, s84\n\t" \
  "v_mfma_f32_16x16x32_bf16 a[60:63], v[60:63], v[28:31], a[60:63]\n\t" \
  "s_waitcnt lgkmcnt(5)\n\t" \
  "buffer_load_dwordx4 v[148:151], v5, %[srdW], %[sW] offen\n\t" \
  "v_mfma_f32_16x16x32_bf16 a[64:67], v[48:51], v[32:35], a[64:67]\n\t" \
  "ds_read_b128 v[28:31], v12 offset:11264\n\t" \
  "v_add_u32 v205, s62, v205\n\t" \
  "v_mfma_f32_16x16x32_bf16 a[68:71], v[52:55], v[32:35], a[68:71]\n\t" \
  "v_perm_b32 v207, v201, v189, s85\n\t" \
  "v_add_u32 v206, s62, v206\n\t" \
  "v_mfma_f32_16x16x32_bf16 a[72:75], v[56:59], v[32:35], a[72:75]\n\t" \
  "v_add_u32 v207, s62, v207\n\t" \
  "ds_write_b128 v13, v[204:207] offset:16384\n\t" \
  "v_mfma_f32_16x16x32_bf16 a[76:79], v[60:63], v[32:35], a[76:79]\n\t" \
  "v_and_b32 v202, s63, v197\n\t" \
  "v_alignbit_b32 v203, v197, v197, 3\n\t" \
  "v_mfma_f32_16x16x32_bf16 a[80:83], v[48:51], v[36:39], a[80:83]\n\t" \
  "s_memrealtime s[72:73]\n\t" \
  "ds_read_b128 v[32:35], v12 offset:12288\n\t" \
  "v_perm_b32 v208, v202, v190, s84\n\t" \
  "v_mfma_f32_16x16x32_bf16 a[84:87], v[52:55], v[36:39], a[84:87]\n\t" \
  "buffer_load_dwordx4 v[152:155], v6, %[srdW], %[sW] offen\n\t" \
  "v_mfma_f32_16x16x32_bf16 a[88:91], v[56:59], v[36:39], a[88:91]\n\t" \
  "v_and_b32 v203, s63, v203\n\t" \
  "v_perm_b32 v209, v202, v190, s85\n\t" \
  "v_mfma_f32_16x16x32_bf16 a[92:95], v[60:63], v[36:39], a[92:95]\n\t" \
  "s_waitcnt lgkmcnt(6)\n\t" \
  "v_add_u32 v208, s62, v208\n\t" \
  "v_perm_b32 v210, v203, v191, s84\n\t" \
  "v_mfma_f32_16x16x32_bf16 a[96:99], v[48:51], v[40:43], a[96:99]\n\t" \
  "ds_read_b128 v[36:39], v12 offset:13312\n\t" \
  "v_add_u32 v209, s62, v209\n\t" \
  "v_mfma_f32_16x16x32_bf16 a[100:103], v[52:55], v[40:43], a[100:103]\n\t" \
  "v_perm_b32 v211, v203, v191, s85\n\t" \
  "v_add_u32 v210, s62, v210\n\t" \
  "v_mfma_f32_16x16x32_bf16 a[104:107], v[56:59], v[40:43], a[104:107]\n\t" \
  "buffer_load_dwordx4 v[156:159], v7, %[srdW], %[sW] offen\n\t" \
  "v_mfma_f32_16x16x32_bf16 a[108:111], v[60:63], v[40:43], a[108:111]\n\t" \
  "v_add_u32 v211, s62, v211\n\t" \
  "ds_write_b128 v13, v[208:211] offset:17408\n\t" \
  "v_mfma_f32_16x16x32_bf16 a[112:115], v[48:51], v[44:47], a[112:115]\n\t" \
  "ds_read_b128 v[40:43], v12 offset:14336\n\t" \
  "v_and_b32 v200, s63, v198\n\t" \
  "v_mfma_f32_16x16x32_bf16 a[116:119], v[52:55], v[44:47], a[116:119]\n\t" \
  "v_alignbit_b32 v201, v198, v198, 3\n\t" \
  "v_perm_b32 v212, v200, v192, s84\n\t" \
  "v_mfma_f32_16x16x32_bf16 a[120:123], v[56:59], v[44:47], a[120:123]\n\t" \
  "v_and_b32 v201, s63, v201\n\t" \
  "v_perm_b32 v213, v200, v192, s85\n\t" \
  "v_mfma_f32_16x16x32_bf16 a[124:127], v[60:63], v[44:47], a[124:127]\n\t" \
  "s_waitcnt lgkmcnt(7)\n\t" \
  "buffer_load_dwordx4 v[160:163], v8, %[srdW], %[sW] offen\n\t" \
  "v_mfma_f32_16x16x32_bf16 a[128:131], v[48:51], v[16:19], a[128:131]\n\t" \
  "ds_read_b128 v[44:47], v12 offset:15360\n\t" \
  "v_add_u32 v212, s62, v212\n\t" \
  "v_mfma_f32_16x16x32_bf16 a[132:135], v[52:55], v[16:19], a[132:135]\n\t" \
  "v_perm_b32 v214, v201, v193, s84\n\t" \
  "v_add_u32 v213, s62, v213\n\t" \
  "v_mfma_f32_16x16x32_bf16 a[136:139], v[56:59], v[16:19], a[136:139]\n\t" \
  "v_perm_b32 v215, v201, v193, s85\n\t" \
  "v_add_u32 v214, s62, v214\n\t" \
  "v_mfma_f32_16x16x32_bf16 a[140:143], v[60:63], v[16:19], a[140:143]\n\t" \
  "v_add_u32 v215, s62, v215\n\t" \
  "ds_write_b128 v13, v[212:215] offset:18432\n\t" \
  "v_mfma_f32_16x16x32_bf16 a[144:147], v[48:51], v[20:23], a[144:147]\n\t" \
  "buffer_load_dwordx4 v[164:167], v9, %[srdW], %[sW] offen\n\t" \
  "v_mfma_f32_16x16x32_bf16 a[148:151], v[52:55], v[20:23], a[148:151]\n\t" \
  "v_lshrrev_b32 v202, 5, v196\n\t" \
  "v_lshlrev_b32 v204, 2, v198\n\t" \
  "v_mfma_f32_16x16x32_bf16 a[152:155], v[56:59], v[20:23], a[152:155]\n\t" \
  "v_lshrrev_b32 v203, 5, v197\n\t" \
  "v_lshlrev_b32 v205, 1, v198\n\t" \
  "v_mfma_f32_16x16x32_bf16 a[156:159], v[60:63], v[20:23], a[156:159]\n\t" \
  "s_waitcnt lgkmcnt(7)\n\t" \
  "v_bfi_b32 v202, s64, v204, v202\n\t" \
  "v_bfi_b32 v203, s64, v205, v203\n\t" \
  "v_mfma_f32_16x16x32_bf16 a[160:163], v[48:51], v[24:27], a[160:163]\n\t" \
  "v_and_b32 v202, s63, v202\n\t" \
  "v_perm_b32 v204, v202, v194, s84\n\t" \
  "v_mfma_f32_16x16x32_bf16 a[164:167], v[52:55], v[24:27], a[164:167]\n\t" \
  "buffer_load_dwordx4 v[168:171], v10, %[srdW], %[sW] offen\n\t" \
  "v_mfma_f32_16x16x32_bf16 a[168:171], v[56:59], v[24:27], a[168:171]\n\t" \
  "v_and_b32 v203, s63, v203\n\t" \
  "v_perm_b32 v205, v202, v194, s85\n\t" \
  "v_mfma_f32_16x16x32_bf16 a[172:175], v[60:63], v[24:27], a[172:175]\n\t" \
  "v_add_u32 v204, s62, v204\n\t" \
  "v_perm_b32 v206, v203, v195, s84\n\t" \
  "v_mfma_f32_16x16x32_bf16 a[176:179], v[48:51], v[28:31], a[176:179]\n\t" \
  "v_add_u32 v205, s62, v205\n\t" \
  "v_perm_b32 v207, v203, v195, s85\n\t" \
  "v_mfma_f32_16x16x32_bf16 a[180:183], v[52:55], v[28:31], a[180:183]\n\t" \
  "v_add_u32 v206, s62, v206\n\t" \
  "v_add_u32 v207, s62, v207\n\t" \
  "v_mfma_f32_16x16x32_bf16 a[184:187], v[56:59], v[28:31], a[184:187]\n\t" \
  "buffer_load_dwordx4 v[172:175], v11, %[srdW], %[sW] offen\n\t" \
  "v_mfma_f32_16x16x32_bf16 a[188:191], v[60:63], v[28:31], a[188:191]\n\t" \
  "s_waitcnt lgkmcnt(4)\n\t" \
  "s_add_u32 %[sW], %[sW], 2097152\n\t" \
  "v_lshrrev_b32 v200, 16, v199\n\t" \
  "ds_write_b128 v13, v[204:207] offset:19456\n\t" \
  "v_mfma_f32_16x16x32_bf16 a[192:195], v[48:51], v[32:35], a[192:195]\n\t" \
  "s_bfm_b64 exec, s57, 0\n\t" \
  "s_cbranch_execz Lst42_%=\n\t" \
  "ds_write_b16 v200, v199 offset:16384\n\t" \
  "Lbk42_%=:\n\t" \
  "s_mov_b64 exec, -1\n\t" \
  "v_mfma_f32_16x16x32_bf16 a[196:199], v[52:55], v[32:35], a[196:199]\n\t" \
  "v_mfma_f32_16x16x32_bf16 a[200:203], v[56:59], v[32:35], a[200:203]\n\t" \
  "v_mfma_f32_16x16x32_bf16 a[204:207], v[60:63], v[32:35], a[204:207]\n\t" \
  "s_waitcnt lgkmcnt(0)\n\t" \
  "s_barrier\n\t" \
  "v_mfma_f32_16x16x32_bf16 a[208:211], v[48:51], v[36:39], a[208:211]\n\t" \
  "s_cmp_lt_u32 s54, %[cmp]\n\t" \
  "s_cbranch_scc1 Lst43_%=\n\t" \
  "Lbk43_%=:\n\t" \
  "s_add_u32 %[sM], %[sM], 512\n\t" \
  "s_load_dwordx2 s[56:57], %[ws], %[sM] glc\n\t" \
  "ds_read_b128 v[16:19], v12 offset:16384\n\t" \
  "v_mfma_f32_16x16x32_bf16 a[212:215], v[52:55], v[36:39], a[212:215]\n\t" \
  "ds_read_b128 v[20:23], v12 offset:17408\n\t" \
  "buffer_load_dwordx4 v[188:191], v12, %[srdX], %[sX] offen sc0 sc1\n\t" \
  "v_mfma_f32_16x16x32_bf16 a[216:219], v[56:59], v[36:39], a[216:219]\n\t" \
  "ds_read_b128 v[24:27], v12 offset:18432\n\t" \
  "v_mfma_f32_16x16x32_bf16 a[220:223], v[60:63], v[36:39], a[220:223]\n\t" \
  "ds_read_b128 v[28:31], v12 offset:19456\n\t" \
  "buffer_load_dwordx4 v[192:195], v12, %[srdX], %[sX] offen offset:1024 sc0 sc1\n\t" \
  "v_mfma_f32_16x16x32_bf16 a[224:227], v[48:51], v[40:43], a[224:227]\n\t" \
  "ds_read_b128 v[32:35], v12 offset:20480\n\t" \
  "v_mfma_f32_16x16x32_bf16 a[228:231], v[52:55], v[40:43], a[228:231]\n\t" \
  "ds_read_b128 v[36:39], v12 offset:21504\n\t" \
  "buffer_load_dwordx3 v[196:198], %[vL12], %[srdX], %[sX] offen offset:2048 sc0 sc1\n\t" \
  "v_mfma_f32_16x16x32_bf16 a[232:235], v[56:59], v[40:43], a[232:235]\n\t" \
  "v_mfma_f32_16x16x32_bf16 a[236:239], v[60:63], v[40:43], a[236:239]\n\t" \
  "buffer_load_dword v199, %[vL4], %[srdX], %[sX] offen offset:2816 sc0 sc1\n\t" \
  "v_mfma_f32_16x16x32_bf16 a[240:243], v[48:51], v[44:47], a[240:243]\n\t" \
  "s_add_u32 %[sX], %[sX], 12288\n\t" \
  "s_sub_u32 s81, s72, s76\n\t" \
  "s_cmp_ge_i32 s81, 0\n\t" \
  "s_cbranch_scc1 Lst44_%=\n\t" \
  "Lbk44_%=:\n\t" \
  "ds_read_b128 v[40:43], v12 offset:22528\n\t" \
  "v_mfma_f32_16x16x32_bf16 a[244:247], v[52:55], v[44:47], a[244:247]\n\t" \
  "v_mfma_f32_16x16x32_bf16 a[248:251], v[56:59], v[44:47], a[248:251]\n\t" \
  "v_mfma_f32_16x16x32_bf16 a[252:255], v[60:63], v[44:47], a[252:255]\n\t" \
  "s_waitcnt lgkmcnt(5)\n\t" \
  "v_mfma_f32_16x16x32_bf16 a[0:3], v[64:67], v[16:19], a[0:3]\n\t" \
  "ds_read_b128 v[44:47], v12 offset:23552\n\t" \
  "v_cvt_pk_bf16_f32 v48, v80, v84\n\t" \
  "v_mfma_f32_16x16x32_bf16 a[4:7], v[68:71], v[16:19], a[4:7]\n\t" \
  "v_cvt_pk_bf16_f32 v52, v81, v85\n\t" \
  "v_cvt_pk_bf16_f32 v56, v82, v86\n\t" \
  "v_mfma_f32_16x16x32_bf16 a[8:11], v[72:75], v[16:19], a[8:11]\n\t" \
  "v_cvt_pk_bf16_f32 v60, v83, v87\n\t" \
  "v_cvt_pk_bf16_f32 v49, v88, v92\n\t" \
  "v_mfma_f32_16x16x32_bf16 a[12:15], v[76:79], v[16:19], a[12:15]\n\t" \
  "v_cvt_pk_bf16_f32 v53, v89, v93\n\t" \
  "v_cvt_pk_bf16_f32 v57, v90, v94\n\t" \
  "v_mfma_f32_16x16x32_bf16 a[16:19], v[64:67], v[20:23], a[16:19]\n\t" \
  "ds_read_b128 v[16:19], v12 offset:24576\n\t" \
  "v_cvt_pk_bf16_f32 v61, v91, v95\n\t" \
  "v_mfma_f32_16x16x32_bf16 a[20:23], v[68:71], v[20:23], a[20:23]\n\t" \
  "v_cvt_pk_bf16_f32 v50, v96, v100\n\t" \
  "v_cvt_pk_bf16_f32 v54, v97, v101\n\t" \
  "v_mfma_f32_16x16x32_bf16 a[24:27], v[72:75], v[20:23], a[24:27]\n\t" \
  "v_cvt_pk_bf16_f32 v58, v98, v102\n\t" \
  "v_cvt_pk_bf16_f32 v62, v99, v103\n\t" \
  "v_mfma_f32_16x16x32_bf16 a[28:31], v[76:79], v[20:23], a[28:31]\n\t" \
  "s_waitcnt lgkmcnt(5)\n\t" \
  "v_cvt_pk_bf16_f32 v51, v104, v108\n\t" \
  "v_cvt_pk_bf16_f32 v55, v105, v109\n\t" \
  "v_mfma_f32_16x16x32_bf16 a[32:35], v[64:67], v[24:27], a[32:35]\n\t" \
  "ds_read_b128 v[20:23], v12 offset:25600\n\t" \
  "v_cvt_pk_bf16_f32 v59, v106, v110\n\t" \
  "v_mfma_f32_16x16x32_bf16 a[36:39], v[68:71], v[24:27], a[36:39]\n\t" \
  "v_cvt_pk_bf16_f32 v63, v107, v111\n\t" \
  "v_mfma_f32_16x16x32_bf16 a[40:43], v[72:75], v[24:27], a[40:43]\n\t" \
  "s_cbranch_vccnz Lst45_%=\n\t" \
  "s_waitcnt vmcnt(12)\n\t" \
  "Lbk45_%=:\n\t" \
  "buffer_load_dwordx4 v[80:83], v4, %[srdW], %[sW] offen\n\t" \
  "v_mfma_f32_16x16x32_bf16 a[44:47], v[76:79], v[24:27], a[44:47]\n\t" \
  "s_and_b32 s62, s53, 0x7f807f80\n\t" \
  "v_and_b32 v200, s63, v184\n\t" \
  "v_alignbit_b32 v201, v184, v184, 3\n\t" \
  "v_mfma_f32_16x16x32_bf16 a[48:51], v[64:67], v[28:31], a[48:51]\n\t" \
  "ds_read_b128 v[24:27], v12 offset:26624\n\t" \
  "v_perm_b32 v204, v200, v176, s84\n\t" \
  "v_mfma_f32_16x16x32_bf16 a[52:55], v[68:71], v[28:31], a[52:55]\n\t" \
  "v_and_b32 v201, s63, v201\n\t" \
  "v_perm_b32 v205, v200, v176, s85\n\t" \
  "v_mfma_f32_16x16x32_bf16 a[56:59], v[72:75], v[28:31], a[56:59]\n\t" \
  "v_add_u32 v204, s62, v204\n\t" \
  "v_perm_b32 v206, v201, v177, s84\n\t" \
  "v_mfma_f32_16x16x32_bf16 a[60:63], v[76:79], v[28:31], a[60:63]\n\t" \
  "s_waitcnt lgkmcnt(5)\n\t" \
  "buffer_load_dwordx4 v[84:87], v5, %[srdW], %[sW] offen\n\t" \
  "v_mfma_f32_16x16x32_bf16 a[64:67], v[64:67], v[32:35], a[64:67]\n\t" \
  "ds_read_b128 v[28:31], v12 offset:27648\n\t" \
  "v_add_u32 v205, s62, v205\n\t" \
  "v_mfma_f32_16x16x32_bf16 a[68:71], v[68:71], v[32:35], a[68:71]\n\t" \
  "v_perm_b32 v207, v201, v177, s85\n\t" \
  "v_add_u32 v206, s62, v206\n\t" \
  "v_mfma_f32_16x16x32_bf16 a[72:75], v[72:75], v[32:35], a[72:75]\n\t" \
  "v_add_u32 v207, s62, v207\n\t" \
  "ds_write_b128 v13, v[204:207] offset:0\n\t" \
  "v_mfma_f32_16x16x32_bf16 a[76:79], v[76:79], v[32:35], a[76:79]\n\t" \
  "v_and_b32 v202, s63, v185\n\t" \
  "v_alignbit_b32 v203, v185, v185, 3\n\t" \
  "v_mfma_f32_16x16x32_bf16 a[80:83], v[64:67], v[36:39], a[80:83]\n\t" \
  "s_memrealtime s[72:73]\n\t" \
  "ds_read_b128 v[32:35], v12 offset:28672\n\t" \
  "v_perm_b32 v208, v202, v178, s84\n\t" \
  "v_mfma_f32_16x16x32_bf16 a[84:87], v[68:71], v[36:39], a[84:87]\n\t" \
  "buffer_load_dwordx4 v[88:91], v6, %[srdW], %[sW] offen\n\t" \
  "v_mfma_f32_16x16x32_bf16 a[88:91], v[72:75], v[36:39], a[88:91]\n\t" \
  "v_and_b32 v203, s63, v203\n\t" \
  "v_perm_b32 v209, v202, v178, s85\n\t" \
  "v_mfma_f32_16x16x32_bf16 a[92:95], v[76:79], v[36:39], a[92:95]\n\t" \
  "s_waitcnt lgkmcnt(6)\n\t" \
  "v_add_u32 v208, s62, v208\n\t" \
  "v_perm_b32 v210, v203, v179, s84\n\t" \
  "v_mfma_f32_16x16x32_bf16 a[96:99], v[64:67], v[40:43], a[96:99]\n\t" \
  "ds_read_b128 v[36:39], v12 offset:29696\n\t" \
  "v_add_u32 v209, s62, v209\n\t" \
  "v_mfma_f32_16x16x32_bf16 a[100:103], v[68:71], v[40:43], a[100:103]\n\t" \
  "v_perm_b32 v211, v203, v179, s85\n\t" \
  "v_add_u32 v210, s62, v210\n\t" \
  "v_mfma_f32_16x16x32_bf16 a[104:107], v[72:75], v[40:43], a[104:107]\n\t" \
  "buffer_load_dwordx4 v[92:95], v7, %[srdW], %[sW] offen\n\t" \
  "v_mfma_f32_16x16x32_bf16 a[108:111], v[76:79], v[40:43], a[108:111]\n\t" \
  "v_add_u32 v211, s62, v211\n\t" \
  "ds_write_b128 v13, v[208:211] offset:1024\n\t" \
  "v_mfma_f32_16x16x32_bf16 a[112:115], v[64:67], v[44:47], a[112:115]\n\t" \
  "ds_read_b128 v[40:43], v12 offset:30720\n\t" \
  "v_and_b32 v200, s63, v186\n\t" \
  "v_mfma_f32_16x16x32_bf16 a[116:119], v[68:71], v[44:47], a[116:119]\n\t" \
  "v_alignbit_b32 v201, v186, v186, 3\n\t" \
  "v_perm_b32 v212, v200, v180, s84\n\t" \
  "v_mfma_f32_16x16x32_bf16 a[120:123], v[72:75], v[44:47], a[120:123]\n\t" \
  "v_and_b32 v201, s63, v201\n\t" \
  "v_perm_b32 v213, v200, v180, s85\n\t" \
  "v_mfma_f32_16x16x32_bf16 a[124:127], v[76:79], v[44:47], a[124:127]\n\t" \
  "s_waitcnt lgkmcnt(7)\n\t" \
  "buffer_load_dwordx4 v[96:99], v8, %[srdW], %[sW] offen\n\t" \
  "v_mfma_f32_16x16x32_bf16 a[128:131], v[64:67], v[16:19], a[128:131]\n\t" \
  "ds_read_b128 v[44:47], v12 offset:31744\n\t" \
  "v_add_u32 v212, s62, v212\n\t" \
  "v_mfma_f32_16x16x32_bf16 a[132:135], v[68:71], v[16:19], a[132:135]\n\t" \
  "v_perm_b32 v214, v201, v181, s84\n\t" \
  "v_add_u32 v213, s62, v213\n\t" \
  "v_mfma_f32_16x16x32_bf16 a[136:139], v[72:75], v[16:19], a[136:139]\n\t" \
  "v_perm_b32 v215, v201, v181, s85\n\t" \
  "v_add_u32 v214, s62, v214\n\t" \
  "v_mfma_f32_16x16x32_bf16 a[140:143], v[76:79], v[16:19], a[140:143]\n\t" \
  "v_add_u32 v215, s62, v215\n\t" \
  "ds_write_b128 v13, v[212:215] offset:2048\n\t" \
  "v_mfma_f32_16x16x32_bf16 a[144:147], v[64:67], v[20:23], a[144:147]\n\t" \
  "buffer_load_dwordx4 v[100:103], v9, %[srdW], %[sW] offen\n\t" \
  "v_mfma_f32_16x16x32_bf16 a[148:151], v[68:71], v[20:23], a[148:151]\n\t" \
  "v_lshrrev_b32 v202, 5, v184\n\t" \
  "v_lshlrev_b32 v204, 2, v186\n\t" \
  "v_mfma_f32_16x16x32_bf16 a[152:155], v[72:75], v[20:23], a[152:155]\n\t" \
  "v_lshrrev_b32 v203, 5, v185\n\t" \
  "v_lshlrev_b32 v205, 1, v186\n\t" \
  "v_mfma_f32_16x16x32_bf16 a[156:159], v[76:79], v[20:23], a[156:159]\n\t" \
  "s_waitcnt lgkmcnt(7)\n\t" \
  "v_bfi_b32 v202, s64, v204, v202\n\t" \
  "v_bfi_b32 v203, s64, v205, v203\n\t" \
  "v_mfma_f32_16x16x32_bf16 a[160:163], v[64:67], v[24:27], a[160:163]\n\t" \
  "v_and_b32 v202, s63, v202\n\t" \
  "v_perm_b32 v204, v202, v182, s84\n\t" \
  "v_mfma_f32_16x16x32_bf16 a[164:167], v[68:71], v[24:27], a[164:167]\n\t" \
  "buffer_load_dwordx4 v[104:107], v10, %[srdW], %[sW] offen\n\t" \
  "v_mfma_f32_16x16x32_bf16 a[168:171], v[72:75], v[24:27], a[168:171]\n\t" \
  "v_and_b32 v203, s63, v203\n\t" \
  "v_perm_b32 v205, v202, v182, s85\n\t" \
  "v_mfma_f32_16x16x32_bf16 a[172:175], v[76:79], v[24:27], a[172:175]\n\t" \
  "v_add_u32 v204, s62, v204\n\t" \
  "v_perm_b32 v206, v203, v183, s84\n\t" \
  "v_mfma_f32_16x16x32_bf16 a[176:179], v[64:67], v[28:31], a[176:179]\n\t" \
  "v_add_u32 v205, s62, v205\n\t" \
  "v_perm_b32 v207, v203, v183, s85\n\t" \
  "v_mfma_f32_16x16x32_bf16 a[180:183], v[68:71], v[28:31], a[180:183]\n\t" \
  "v_add_u32 v206, s62, v206\n\t" \
  "v_add_u32 v207, s62, v207\n\t" \
  "v_mfma_f32_16x16x32_bf16 a[184:187], v[72:75], v[28:31], a[184:187]\n\t" \
  "buffer_load_dwordx4 v[108:111], v11, %[srdW], %[sW] offen\n\t" \
  "v_mfma_f32_16x16x32_bf16 a[188:191], v[76:79], v[28:31], a[188:191]\n\t" \
  "s_waitcnt lgkmcnt(4)\n\t" \
  "s_add_u32 %[sW], %[sW], 2097152\n\t" \
  "v_lshrrev_b32 v200, 16, v187\n\t" \
  "ds_write_b128 v13, v[204:207] offset:3072\n\t" \
  "v_mfma_f32_16x16x32_bf16 a[192:195], v[64:67], v[32:35], a[192:195]\n\t" \
  "s_bfm_b64 exec, s53, 0\n\t" \
  "s_cbranch_execz Lst46_%=\n\t" \
  "ds_write_b16 v200, v187 offset:0\n\t" \
  "Lbk46_%=:\n\t" \
  "s_mov_b64 exec, -1\n\t" \
  "v_mfma_f32_16x16x32_bf16 a[196:199], v[68:71], v[32:35], a[196:199]\n\t" \
  "v_mfma_f32_16x16x32_bf16 a[200:203], v[72:75], v[32:35], a[200:203]\n\t" \
  "v_mfma_f32_16x16x32_bf16 a[204:207], v[76:79], v[32:35], a[204:207]\n\t" \
  "s_waitcnt lgkmcnt(0)\n\t" \
  "s_barrier\n\t" \
  "v_mfma_f32_16x16x32_bf16 a[208:211], v[64:67], v[36:39], a[208:211]\n\t" \
  "s_cmp_lt_u32 s56, %[cmp]\n\t" \
  "s_cbranch_scc1 Lst47_%=\n\t" \
  "Lbk47_%=:\n\t" \
  "s_add_u32 %[sM], %[sM], 512\n\t" \
  "s_load_dwordx2 s[52:53], %[ws], %[sM] glc\n\t" \
  "ds_read_b128 v[16:19], v12 offset:0\n\t" \
  "v_mfma_f32_16x16x32_bf16 a[212:215], v[68:71], v[36:39], a[212:215]\n\t" \
  "ds_read_b128 v[20:23], v12 offset:1024\n\t" \
  "buffer_load_dwordx4 v[176:179], v12, %[srdX], %[sX] offen sc0 sc1\n\t" \
  "v_mfma_f32_16x16x32_bf16 a[216:219], v[72:75], v[36:39], a[216:219]\n\t" \
  "s_load_dword %[same], %[ws], %[sFP] glc\n\t" \
  "ds_read_b128 v[24:27], v12 offset:2048\n\t" \
  "v_mfma_f32_16x16x32_bf16 a[220:223], v[76:79], v[36:39], a[220:223]\n\t" \
  "ds_read_b128 v[28:31], v12 offset:3072\n\t" \
  "buffer_load_dwordx4 v[180:183], v12, %[srdX], %[sX] offen offset:1024 sc0 sc1\n\t" \
  "v_mfma_f32_16x16x32_bf16 a[224:227], v[64:67], v[40:43], a[224:227]\n\t" \
  "ds_read_b128 v[32:35], v12 offset:4096\n\t" \
  "v_mfma_f32_16x16x32_bf16 a[228:231], v[68:71], v[40:43], a[228:231]\n\t" \
  "ds_read_b128 v[36:39], v12 offset:5120\n\t" \
  "buffer_load_dwordx3 v[184:186], %[vL12], %[srdX], %[sX] offen offset:2048 sc0 sc1\n\t" \
  "v_mfma_f32_16x16x32_bf16 a[232:235], v[72:75], v[40:43], a[232:235]\n\t" \
  "v_mfma_f32_16x16x32_bf16 a[236:239], v[76:79], v[40:43], a[236:239]\n\t" \
  "buffer_load_dword v187, %[vL4], %[srdX], %[sX] offen offset:2816 sc0 sc1\n\t" \
  "v_mfma_f32_16x16x32_bf16 a[240:243], v[64:67], v[44:47], a[240:243]\n\t" \
  "s_add_u32 %[sX], %[sX], 12288\n\t" \
  "s_sub_u32 s81, s72, s76\n\t" \
  "s_cmp_ge_i32 s81, 0\n\t" \
  "s_cbranch_scc1 Lst48_%=\n\t" \
  "Lbk48_%=:\n\t" \
  "ds_read_b128 v[40:43], v12 offset:6144\n\t" \
  "v_mfma_f32_16x16x32_bf16 a[244:247], v[68:71], v[44:47], a[244:247]\n\t" \
  "v_mfma_f32_16x16x32_bf16 a[248:251], v[72:75], v[44:47], a[248:251]\n\t" \
  "v_mfma_f32_16x16x32_bf16 a[252:255], v[76:79], v[44:47], a[252:255]\n\t" \
  "s_waitcnt lgkmcnt(5)\n\t" \
  "v_mfma_f32_16x16x32_bf16 a[128:131], v[48:51], v[16:19], a[128:131]\n\t" \
  "ds_read_b128 v[44:47], v12 offset:7168\n\t" \
  "v_cvt_pk_bf16_f32 v64, v112, v116\n\t" \
  "v_mfma_f32_16x16x32_bf16 a[132:135], v[52:55], v[16:19], a[132:135]\n\t" \
  "v_cvt_pk_bf16_f32 v68, v113, v117\n\t" \
  "v_cvt_pk_bf16_f32 v72, v114, v118\n\t" \
  "v_mfma_f32_16x16x32_bf16 a[136:139], v[56:59], v[16:19], a[136:139]\n\t" \
  "v_cvt_pk_bf16_f32 v76, v115, v119\n\t" \
  "v_cvt_pk_bf16_f32 v65, v120, v124\n\t" \
  "v_mfma_f32_16x16x32_bf16 a[140:143], v[60:63], v[16:19], a[140:143]\n\t" \
  "v_cvt_pk_bf16_f32 v69, v121, v125\n\t" \
  "v_cvt_pk_bf16_f32 v73, v122, v126\n\t" \
  "v_mfma_f32_16x16x32_bf16 a[144:147], v[48:51], v[20:23], a[144:147]\n\t" \
  "ds_read_b128 v[16:19], v12 offset:8192\n\t" \
  "v_cvt_pk_bf16_f32 v77, v123, v127\n\t" \
  "v_mfma_f32_16x16x32_bf16 a[148:151], v[52:55], v[20:23], a[148:151]\n\t" \
  "v_cvt_pk_bf16_f32 v66, v128, v132\n\t" \
  "v_cvt_pk_bf16_f32 v70, v129, v133\n\t" \
  "v_mfma_f32_16x16x32_bf16 a[152:155], v[56:59], v[20:23], a[152:155]\n\t" \
  "v_cvt_pk_bf16_f32 v74, v130, v134\n\t" \
  "v_cvt_pk_bf16_f32 v78, v131, v135\n\t" \
  "v_mfma_f32_16x16x32_bf16 a[156:159], v[60:63], v[20:23], a[156:159]\n\t" \
  "s_waitcnt lgkmcnt(5)\n\t" \
  "v_cvt_pk_bf16_f32 v67, v136, v140\n\t" \
  "v_cvt_pk_bf16_f32 v71, v137, v141\n\t" \
  "v_mfma_f32_16x16x32_bf16 a[160:163], v[48:51], v[24:27], a[160:163]\n\t" \
  "ds_read_b128 v[20:23], v12 offset:9216\n\t" \
  "v_cvt_pk_bf16_f32 v75, v138, v142\n\t" \
  "v_mfma_f32_16x16x32_bf16 a[164:167], v[52:55], v[24:27], a[164:167]\n\t" \
  "v_cvt_pk_bf16_f32 v79, v139, v143\n\t" \
  "v_mfma_f32_16x16x32_bf16 a[168:171], v[56:59], v[24:27], a[168:171]\n\t" \
  "s_cbranch_vccnz Lst49_%=\n\t" \
  "s_waitcnt vmcnt(12)\n\t" \
  "Lbk49_%=:\n\t" \
  "s_and_b32 s62, s55, 0x7f807f80\n\t" \
  "v_and_b32 v200, s63, v196\n\t" \
  "v_mfma_f32_16x16x32_bf16 a[172:175], v[60:63], v[24:27], a[172:175]\n\t" \
  "v_alignbit_b32 v201, v196, v196, 3\n\t" \
  "v_perm_b32 v204, v200, v188, s84\n\t" \
  "v_mfma_f32_16x16x32_bf16 a[176:179], v[48:51], v[28:31], a[176:179]\n\t" \
  "ds_read_b128 v[24:27], v12 offset:10240\n\t" \
  "v_and_b32 v201, s63, v201\n\t" \
  "v_mfma_f32_16x16x32_bf16 a[180:183], v[52:55], v[28:31], a[180:183]\n\t" \
  "v_perm_b32 v205, v200, v188, s85\n\t" \
  "v_add_u32 v204, s62, v204\n\t" \
  "v_mfma_f32_16x16x32_bf16 a[184:187], v[56:59], v[28:31], a[184:187]\n\t" \
  "v_perm_b32 v206, v201, v189, s84\n\t" \
  "v_add_u32 v205, s62, v205\n\t" \
  "v_mfma_f32_16x16x32_bf16 a[188:191], v[60:63], v[28:31], a[188:191]\n\t" \
  "s_waitcnt lgkmcnt(5)\n\t" \
  "v_perm_b32 v207, v201, v189, s85\n\t" \
  "v_add_u32 v206, s62, v206\n\t" \
  "v_mfma_f32_16x16x32_bf16 a[192:195], v[48:51], v[32:35], a[192:195]\n\t" \
  "ds_read_b128 v[28:31], v12 offset:11264\n\t" \
  "v_add_u32 v207, s62, v207\n\t" \
  "v_mfma_f32_16x16x32_bf16 a[196:199], v[52:55], v[32:35], a[196:199]\n\t" \
  "ds_write_b128 v13, v[204:207] offset:16384\n\t" \
  "v_and_b32 v202, s63, v197\n\t" \
  "v_mfma_f32_16x16x32_bf16 a[200:203], v[56:59], v[32:35], a[200:203]\n\t" \
  "v_alignbit_b32 v203, v197, v197, 3\n\t" \
  "v_perm_b32 v208, v202, v190, s84\n\t" \
  "v_mfma_f32_16x16x32_bf16 a[204:207], v[60:63], v[32:35], a[204:207]\n\t" \
  "v_and_b32 v203, s63, v203\n\t" \
  "v_perm_b32 v209, v202, v190, s85\n\t" \
  "v_mfma_f32_16x16x32_bf16 a[208:211], v[48:51], v[36:39], a[208:211]\n\t" \
  "s_memrealtime s[72:73]\n\t" \
  "ds_read_b128 v[32:35], v12 offset:12288\n\t" \
  "v_add_u32 v208, s62, v208\n\t" \
  "v_mfma_f32_16x16x32_bf16 a[212:215], v[52:55], v[36:39], a[212:215]\n\t" \
  "v_perm_b32 v210, v203, v191, s84\n\t" \
  "v_add_u32 v209, s62, v209\n\t" \
  "v_mfma_f32_16x16x32_bf16 a[216:219], v[56:59], v[36:39], a[216:219]\n\t" \
  "v_perm_b32 v211, v203, v191, s85\n\t" \
  "v_add_u32 v210, s62, v210\n\t" \
  "v_mfma_f32_16x16x32_bf16 a[220:223], v[60:63], v[36:39], a[220:223]\n\t" \
  "s_waitcnt lgkmcnt(6)\n\t" \
  "v_add_u32 v211, s62, v211\n\t" \
  "ds_write_b128 v13, v[208:211] offset:17408\n\t" \
  "v_mfma_f32_16x16x32_bf16 a[224:227], v[48:51], v[40:43], a[224:227]\n\t" \
  "ds_read_b128 v[36:39], v12 offset:13312\n\t" \
  "v_and_b32 v200, s63, v198\n\t" \
  "v_mfma_f32_16x16x32_bf16 a[228:231], v[52:55], v[40:43], a[228:231]\n\t" \
  "v_alignbit_b32 v201, v198, v198, 3\n\t" \
  "v_perm_b32 v212, v200, v192, s84\n\t" \
  "v_mfma_f32_16x16x32_bf16 a[232:235], v[56:59], v[40:43], a[232:235]\n\t" \
  "v_and_b32 v201, s63, v201\n\t" \
  "v_perm_b32 v213, v200, v192, s85\n\t" \
  "v_mfma_f32_16x16x32_bf16 a[236:239], v[60:63], v[40:43], a[236:239]\n\t" \
  "v_add_u32 v212, s62, v212\n\t" \
  "v_perm_b32 v214, v201, v193, s84\n\t" \
  "v_mfma_f32_16x16x32_bf16 a[240:243], v[48:51], v[44:47], a[240:243]\n\t" \
  "ds_read_b128 v[40:43], v12 offset:14336\n\t" \
  "v_add_u32 v213, s62, v213\n\t" \
  "v_mfma_f32_16x16x32_bf16 a[244:247], v[52:55], v[44:47], a[244:247]\n\t" \
  "v_perm_b32 v215, v201, v193, s85\n\t" \
  "v_add_u32 v214, s62, v214\n\t" \
  "v_mfma_f32_16x16x32_bf16 a[248:251], v[56:59], v[44:47], a[248:251]\n\t" \
  "v_add_u32 v215, s62, v215\n\t" \
  "ds_write_b128 v13, v[212:215] offset:18432\n\t" \
  "v_mfma_f32_16x16x32_bf16 a[252:255], v[60:63], v[44:47], a[252:255]\n\t" \
  "s_waitcnt lgkmcnt(8)\n\t" \
  "v_lshrrev_b32 v202, 5, v196\n\t" \
  "v_lshlrev_b32 v204, 2, v198\n\t" \
  "v_mfma_f32_16x16x32_bf16 a[128:131], v[64:67], v[16:19], a[128:131]\n\t" \
  "ds_read_b128 v[44:47], v12 offset:15360\n\t" \
  "v_lshrrev_b32 v203, 5, v197\n\t" \
  "v_mfma_f32_16x16x32_bf16 a[132:135], v[68:71], v[16:19], a[132:135]\n\t" \
  "v_lshlrev_b32 v205, 1, v198\n\t" \
  "v_bfi_b32 v202, s64, v204, v202\n\t" \
  "v_mfma_f32_16x16x32_bf16 a[136:139], v[72:75], v[16:19], a[136:139]\n\t" \
  "v_bfi_b32 v203, s64, v205, v203\n\t" \
  "v_and_b32 v202, s63, v202\n\t" \
  "v_mfma_f32_16x16x32_bf16 a[140:143], v[76:79], v[16:19], a[140:143]\n\t" \
  "v_perm_b32 v204, v202, v194, s84\n\t" \
  "v_and_b32 v203, s63, v203\n\t" \
  "v_mfma_f32_16x16x32_bf16 a[144:147], v[64:67], v[20:23], a[144:147]\n\t" \
  "v_perm_b32 v205, v202, v194, s85\n\t" \
  "v_add_u32 v204, s62, v204\n\t" \
  "v_mfma_f32_16x16x32_bf16 a[148:151], v[68:71], v[20:23], a[148:151]\n\t" \
  "v_perm_b32 v206, v203, v195, s84\n\t" \
  "v_add_u32 v205, s62, v205\n\t" \
  "v_mfma_f32_16x16x32_bf16 a[152:155], v[72:75], v[20:23], a[152:155]\n\t" \
  "v_perm_b32 v207, v203, v195, s85\n\t" \
  "v_add_u32 v206, s62, v206\n\t" \
  "v_mfma_f32_16x16x32_bf16 a[156:159], v[76:79], v[20:23], a[156:159]\n\t" \
  "s_waitcnt lgkmcnt(7)\n\t" \
  "v_add_u32 v207, s62, v207\n\t" \
  "v_lshrrev_b32 v200, 16, v199\n\t" \
  "v_mfma_f32_16x16x32_bf16 a[160:163], v[64:67], v[24:27], a[160:163]\n\t" \
  "ds_write_b128 v13, v[204:207] offset:19456\n\t" \
  "v_cvt_pk_bf16_f32 v112, v144, v148\n\t" \
  "v_mfma_f32_16x16x32_bf16 a[164:167], v[68:71], v[24:27], a[164:167]\n\t" \
  "s_bfm_b64 exec, s55, 0\n\t" \
  "s_cbranch_execz Lst50_%=\n\t" \
  "ds_write_b16 v200, v199 offset:16384\n\t" \
  "Lbk50_%=:\n\t" \
  "s_mov_b64 exec, -1\n\t" \
  "v_mfma_f32_16x16x32_bf16 a[168:171], v[72:75], v[24:27], a[168:171]\n\t" \
  "v_cvt_pk_bf16_f32 v116, v145, v149\n\t" \
  "v_cvt_pk_bf16_f32 v120, v146, v150\n\t" \
  "v_mfma_f32_16x16x32_bf16 a[172:175], v[76:79], v[24:27], a[172:175]\n\t" \
  "v_cvt_pk_bf16_f32 v124, v147, v151\n\t" \
  "v_cvt_pk_bf16_f32 v113, v152, v156\n\t" \
  "v_mfma_f32_16x16x32_bf16 a[176:179], v[64:67], v[28:31], a[176:179]\n\t" \
  "v_cvt_pk_bf16_f32 v117, v153, v157\n\t" \
  "v_cvt_pk_bf16_f32 v121, v154, v158\n\t" \
  "v_mfma_f32_16x16x32_bf16 a[180:183], v[68:71], v[28:31], a[180:183]\n\t" \
  "v_cvt_pk_bf16_f32 v125, v155, v159\n\t" \
  "v_cvt_pk_bf16_f32 v114, v160, v164\n\t" \
  "v_mfma_f32_16x16x32_bf16 a[184:187], v[72:75], v[28:31], a[184:187]\n\t" \
  "v_cvt_pk_bf16_f32 v118, v161, v165\n\t" \
  "v_cvt_pk_bf16_f32 v122, v162, v166\n\t" \
  "v_mfma_f32_16x16x32_bf16 a[188:191], v[76:79], v[28:31], a[188:191]\n\t" \
  "s_waitcnt lgkmcnt(5)\n\t" \
  "v_cvt_pk_bf16_f32 v126, v163, v167\n\t" \
  "v_cvt_pk_bf16_f32 v115, v168, v172\n\t" \
  "v_mfma_f32_16x16x32_bf16 a[192:195], v[64:67], v[32:35], a[192:195]\n\t" \
  "v_cvt_pk_bf16_f32 v119, v169, v173\n\t" \
  "v_cvt_pk_bf16_f32 v123, v170, v174\n\t" \
  "v_mfma_f32_16x16x32_bf16 a[196:199], v[68:71], v[32:35], a[196:199]\n\t" \
  "v_cvt_pk_bf16_f32 v127, v171, v175\n\t" \
  "v_mfma_f32_16x16x32_bf16 a[200:203], v[72:75], v[32:35], a[200:203]\n\t" \
  "v_mfma_f32_16x16x32_bf16 a[204:207], v[76:79], v[32:35], a[204:207]\n\t" \
  "s_waitcnt lgkmcnt(0)\n\t" \
  "s_barrier\n\t" \
  "v_mfma_f32_16x16x32_bf16 a[208:211], v[64:67], v[36:39], a[208:211]\n\t" \
  "s_cmp_lt_u32 s52, %[cmp]\n\t" \
  "s_cbranch_scc1 Lst51_%=\n\t" \
  "Lbk51_%=:\n\t" \
  "s_add_u32 %[sM], %[sM], 512\n\t" \
  "ds_read_b128 v[16:19], v12 offset:16384\n\t" \
  "v_mfma_f32_16x16x32_bf16 a[212:215], v[68:71], v[36:39], a[212:215]\n\t" \
  "ds_read_b128 v[20:23], v12 offset:17408\n\t" \
  "buffer_load_dwordx4 v[188:191], v12, %[srdX], %[sX] offen sc0 sc1\n\t" \
  "v_mfma_f32_16x16x32_bf16 a[216:219], v[72:75], v[36:39], a[216:219]\n\t" \
  "ds_read_b128 v[24:27], v12 offset:18432\n\t" \
  "v_mfma_f32_16x16x32_bf16 a[220:223], v[76:79], v[36:39], a[220:223]\n\t" \
  "ds_read_b128 v[28:31], v12 offset:19456\n\t" \
  "buffer_load_dwordx4 v[192:195], v12, %[srdX], %[sX] offen offset:1024 sc0 sc1\n\t" \
  "v_mfma_f32_16x16x32_bf16 a[224:227], v[64:67], v[40:43], a[224:227]\n\t" \
  "ds_read_b128 v[32:35], v12 offset:20480\n\t" \
  "v_mfma_f32_16x16x32_bf16 a[228:231], v[68:71], v[40:43], a[228:231]\n\t" \
  "ds_read_b128 v[36:39], v12 offset:21504\n\t" \
  "buffer_load_dwordx3 v[196:198], %[vL12], %[srdX], %[sX] offen offset:2048 sc0 sc1\n\t" \
  "v_mfma_f32_16x16x32_bf16 a[232:235], v[72:75], v[40:43], a[232:235]\n\t" \
  "v_mfma_f32_16x16x32_bf16 a[236:239], v[76:79], v[40:43], a[236:239]\n\t" \
  "buffer_load_dword v199, %[vL4], %[srdX], %[sX] offen offset:2816 sc0 sc1\n\t" \
  "v_mfma_f32_16x16x32_bf16 a[240:243], v[64:67], v[44:47], a[240:243]\n\t" \
  "s_add_u32 %[sX], %[sX], 12288\n\t" \
  "s_sub_u32 s81, s72, s76\n\t" \
  "s_cmp_ge_i32 s81, 0\n\t" \
  "s_cbranch_scc1 Lst52_%=\n\t" \
  "Lbk52_%=:\n\t" \
  "ds_read_b128 v[40:43], v12 offset:22528\n\t" \
  "v_mfma_f32_16x16x32_bf16 a[244:247], v[68:71], v[44:47], a[244:247]\n\t" \
  "v_mfma_f32_16x16x32_bf16 a[248:251], v[72:75], v[44:47], a[248:251]\n\t" \
  "v_mfma_f32_16x16x32_bf16 a[252:255], v[76:79], v[44:47], a[252:255]\n\t" \
  "s_waitcnt lgkmcnt(5)\n\t" \
  "v_mfma_f32_16x16x32_bf16 a[128:131], v[112:115], v[16:19], a[128:131]\n\t" \
  "s_waitcnt vmcnt(8)\n\t" \
  "ds_read_b128 v[44:47], v12 offset:23552\n\t" \
  "v_cvt_pk_bf16_f32 v128, v80, v84\n\t" \
  "v_mfma_f32_16x16x32_bf16 a[132:135], v[116:119], v[16:19], a[132:135]\n\t" \
  "v_cvt_pk_bf16_f32 v132, v81, v85\n\t" \
  "v_cvt_pk_bf16_f32 v136, v82, v86\n\t" \
  "v_mfma_f32_16x16x32_bf16 a[136:139], v[120:123], v[16:19], a[136:139]\n\t" \
  "v_cvt_pk_bf16_f32 v140, v83, v87\n\t" \
  "v_cvt_pk_bf16_f32 v129, v88, v92\n\t" \
  "v_mfma_f32_16x16x32_bf16 a[140:143], v[124:127], v[16:19], a[140:143]\n\t" \
  "v_cvt_pk_bf16_f32 v133, v89, v93\n\t" \
  "v_cvt_pk_bf16_f32 v137, v90, v94\n\t" \
  "v_mfma_f32_16x16x32_bf16 a[144:147], v[112:115], v[20:23], a[144:147]\n\t" \
  "ds_read_b128 v[16:19], v12 offset:24576\n\t" \
  "v_cvt_pk_bf16_f32 v141, v91, v95\n\t" \
  "v_mfma_f32_16x16x32_bf16 a[148:151], v[116:119], v[20:23], a[148:151]\n\t" \
  "v_cvt_pk_bf16_f32 v130, v96, v100\n\t" \
  "v_cvt_pk_bf16_f32 v134, v97, v101\n\t" \
  "v_mfma_f32_16x16x32_bf16 a[152:155], v[120:123], v[20:23], a[152:155]\n\t" \
  "v_cvt_pk_bf16_f32 v138, v98, v102\n\t" \
  "v_cvt_pk_bf16_f32 v142, v99, v103\n\t" \
  "v_mfma_f32_16x16x32_bf16 a[156:159], v[124:127], v[20:23], a[156:159]\n\t" \
  "s_waitcnt lgkmcnt(5)\n\t" \
  "v_cvt_pk_bf16_f32 v131, v104, v108\n\t" \
  "v_cvt_pk_bf16_f32 v135, v105, v109\n\t" \
  "v_mfma_f32_16x16x32_bf16 a[160:163], v[112:115], v[24:27], a[160:163]\n\t" \
  "ds_read_b128 v[20:23], v12 offset:25600\n\t" \
  "v_cvt_pk_bf16_f32 v139, v106, v110\n\t" \
  "v_mfma_f32_16x16x32_bf16 a[164:167], v[116:119], v[24:27], a[164:167]\n\t" \
  "v_cvt_pk_bf16_f32 v143, v107, v111\n\t" \
  "v_mfma_f32_16x16x32_bf16 a[168:171], v[120:123], v[24:27], a[168:171]\n\t" \
  "s_cbranch_vccnz Lst53_%=\n\t" \
  "s_waitcnt vmcnt(4)\n\t" \
  "Lbk53_%=:\n\t" \
  "s_and_b32 s62, s57, 0x7f807f80\n\t" \
  "v_and_b32 v200, s63, v184\n\t" \
  "v_mfma_f32_16x16x32_bf16 a[172:175], v[124:127], v[24:27], a[172:175]\n\t" \
  "v_alignbit_b32 v201, v184, v184, 3\n\t" \
  "v_perm_b32 v204, v200, v176, s84\n\t" \
  "v_mfma_f32_16x16x32_bf16 a[176:179], v[112:115], v[28:31], a[176:179]\n\t" \
  "ds_read_b128 v[24:27], v12 offset:26624\n\t" \
  "v_and_b32 v201, s63, v201\n\t" \
  "v_mfma_f32_16x16x32_bf16 a[180:183], v[116:119], v[28:31], a[180:183]\n\t" \
  "v_perm_b32 v205, v200, v176, s85\n\t" \
  "v_add_u32 v204, s62, v204\n\t" \
  "v_mfma_f32_16x16x32_bf16 a[184:187], v[120:123], v[28:31], a[184:187]\n\t" \
  "v_perm_b32 v206, v201, v177, s84\n\t" \
  "v_add_u32 v205, s62, v205\n\t" \
  "v_mfma_f32_16x16x32_bf16 a[188:191], v[124:127], v[28:31], a[188:191]\n\t" \
  "s_waitcnt lgkmcnt(5)\n\t" \
  "v_perm_b32 v207, v201, v177, s85\n\t" \
  "v_add_u32 v206, s62, v206\n\t" \
  "v_mfma_f32_16x16x32_bf16 a[192:195], v[112:115], v[32:35], a[192:195]\n\t" \
  "ds_read_b128 v[28:31], v12 offset:27648\n\t" \
  "v_add_u32 v207, s62, v207\n\t" \
  "v_mfma_f32_16x16x32_bf16 a[196:199], v[116:119], v[32:35], a[196:199]\n\t" \
  "ds_write_b128 v13, v[204:207] offset:0\n\t" \
  "v_and_b32 v202, s63, v185\n\t" \
  "v_mfma_f32_16x16x32_bf16 a[200:203], v[120:123], v[32:35], a[200:203]\n\t" \
  "v_alignbit_b32 v203, v185, v185, 3\n\t" \
  "v_perm_b32 v208, v202, v178, s84\n\t" \
  "v_mfma_f32_16x16x32_bf16 a[204:207], v[124:127], v[32:35], a[204:207]\n\t" \
  "v_and_b32 v203, s63, v203\n\t" \
  "v_perm_b32 v209, v202, v178, s85\n\t" \
  "v_mfma_f32_16x16x32_bf16 a[208:211], v[112:115], v[36:39], a[208:211]\n\t" \
  "s_memrealtime s[72:73]\n\t" \
  "ds_read_b128 v[32:35], v12 offset:28672\n\t" \
  "v_add_u32 v208, s62, v208\n\t" \
  "v_mfma_f32_16x16x32_bf16 a[212:215], v[116:119], v[36:39], a[212:215]\n\t" \
  "v_perm_b32 v210, v203, v179, s84\n\t" \
  "v_add_u32 v209, s62, v209\n\t" \
  "v_mfma_f32_16x16x32_bf16 a[216:219], v[120:123], v[36:39], a[216:219]\n\t" \
  "v_perm_b32 v211, v203, v179, s85\n\t" \
  "v_add_u32 v210, s62, v210\n\t" \
  "v_mfma_f32_16x16x32_bf16 a[220:223], v[124:127], v[36:39], a[220:223]\n\t" \
  "s_waitcnt lgkmcnt(6)\n\t" \
  "v_add_u32 v211, s62, v211\n\t" \
  "ds_write_b128 v13, v[208:211] offset:1024\n\t" \
  "v_mfma_f32_16x16x32_bf16 a[224:227], v[112:115], v[40:43], a[224:227]\n\t" \
  "ds_read_b128 v[36:39], v12 offset:29696\n\t" \
  "v_and_b32 v200, s63, v186\n\t" \
  "v_mfma_f32_16x16x32_bf16 a[228:231], v[116:119], v[40:43], a[228:231]\n\t" \
  "v_alignbit_b32 v201, v186, v186, 3\n\t" \
  "v_perm_b32 v212, v200, v180, s84\n\t" \
  "v_mfma_f32_16x16x32_bf16 a[232:235], v[120:123], v[40:43], a[232:235]\n\t" \
  "v_and_b32 v201, s63, v201\n\t" \
  "v_perm_b32 v213, v200, v180, s85\n\t" \
  "v_mfma_f32_16x16x32_bf16 a[236:239], v[124:127], v[40:43], a[236:239]\n\t" \
  "v_add_u32 v212, s62, v212\n\t" \
  "v_perm_b32 v214, v201, v181, s84\n\t" \
  "v_mfma_f32_16x16x32_bf16 a[240:243], v[112:115], v[44:47], a[240:243]\n\t" \
  "ds_read_b128 v[40:43], v12 offset:30720\n\t" \
  "v_add_u32 v213, s62, v213\n\t" \
  "v_mfma_f32_16x16x32_bf16 a[244:247], v[116:119], v[44:47], a[244:247]\n\t" \
  "v_perm_b32 v215, v201, v181, s85\n\t" \
  "v_add_u32 v214, s62, v214\n\t" \
  "v_mfma_f32_16x16x32_bf16 a[248:251], v[120:123], v[44:47], a[248:251]\n\t" \
  "v_add_u32 v215, s62, v215\n\t" \
  "ds_write_b128 v13, v[212:215] offset:2048\n\t" \
  "v_mfma_f32_16x16x32_bf16 a[252:255], v[124:127], v[44:47], a[252:255]\n\t" \
  "s_waitcnt lgkmcnt(8)\n\t" \
  "v_lshrrev_b32 v202, 5, v184\n\t" \
  "v_lshlrev_b32 v204, 2, v186\n\t" \
  "v_mfma_f32_16x16x32_bf16 a[128:131], v[128:131], v[16:19], a[128:131]\n\t" \
  "ds_read_b128 v[44:47], v12 offset:31744\n\t" \
  "v_lshrrev_b32 v203, 5, v185\n\t" \
  "v_mfma_f32_16x16x32_bf16 a[132:135], v[132:135], v[16:19], a[132:135]\n\t" \
  "v_lshlrev_b32 v205, 1, v186\n\t" \
  "v_bfi_b32 v202, s64, v204, v202\n\t" \
  "v_mfma_f32_16x16x32_bf16 a[136:139], v[136:139], v[16:19], a[136:139]\n\t" \
  "v_bfi_b32 v203, s64, v205, v203\n\t" \
  "v_and_b32 v202, s63, v202\n\t" \
  "v_mfma_f32_16x16x32_bf16 a[140:143], v[140:143], v[16:19], a[140:143]\n\t" \
  "v_perm_b32 v204, v202, v182, s84\n\t" \
  "v_and_b32 v203, s63, v203\n\t" \
  "v_mfma_f32_16x16x32_bf16 a[144:147], v[128:131], v[20:23], a[144:147]\n\t" \
  "v_perm_b32 v205, v202, v182, s85\n\t" \
  "v_add_u32 v204, s62, v204\n\t" \
  "v_mfma_f32_16x16x32_bf16 a[148:151], v[132:135], v[20:23], a[148:151]\n\t" \
  "v_perm_b32 v206, v203, v183, s84\n\t" \
  "v_add_u32 v205, s62, v205\n\t" \
  "v_mfma_f32_16x16x32_bf16 a[152:155], v[136:139], v[20:23], a[152:155]\n\t" \
  "v_perm_b32 v207, v203, v183, s85\n\t" \
  "v_add_u32 v206, s62, v206\n\t" \
  "v_mfma_f32_16x16x32_bf16 a[156:159], v[140:143], v[20:23], a[156:159]\n\t" \
  "s_waitcnt lgkmcnt(7)\n\t" \
  "v_add_u32 v207, s62, v207\n\t" \
  "v_lshrrev_b32 v200, 16, v187\n\t" \
  "v_mfma_f32_16x16x32_bf16 a[160:163], v[128:131], v[24:27], a[160:163]\n\t" \
  "ds_write_b128 v13, v[204:207] offset:3072\n\t" \
  "v_mfma_f32_16x16x32_bf16 a[164:167], v[132:135], v[24:27], a[164:167]\n\t" \
  "s_bfm_b64 exec, s57, 0\n\t" \
  "s_cbranch_execz Lst54_%=\n\t" \
  "ds_write_b16 v200, v187 offset:0\n\t" \
  "Lbk54_%=:\n\t" \
  "s_mov_b64 exec, -1\n\t" \
  "v_mfma_f32_16x16x32_bf16 a[168:171], v[136:139], v[24:27], a[168:171]\n\t" \
  "v_mfma_f32_16x16x32_bf16 a[172:175], v[140:143], v[24:27], a[172:175]\n\t" \
  "v_mfma_f32_16x16x32_bf16 a[176:179], v[128:131], v[28:31], a[176:179]\n\t" \
  "v_mfma_f32_16x16x32_bf16 a[180:183], v[132:135], v[28:31], a[180:183]\n\t" \
  "v_mfma_f32_16x16x32_bf16 a[184:187], v[136:139], v[28:31], a[184:187]\n\t" \
  "v_mfma_f32_16x16x32_bf16 a[188:191], v[140:143], v[28:31], a[188:191]\n\t" \
  "s_waitcnt lgkmcnt(5)\n\t" \
  "v_mfma_f32_16x16x32_bf16 a[192:195], v[128:131], v[32:35], a[192:195]\n\t" \
  "v_mfma_f32_16x16x32_bf16 a[196:199], v[132:135], v[32:35], a[196:199]\n\t" \
  "v_mfma_f32_16x16x32_bf16 a[200:203], v[136:139], v[32:35], a[200:203]\n\t" \
  "v_mfma_f32_16x16x32_bf16 a[204:207], v[140:143], v[32:35], a[204:207]\n\t" \
  "s_waitcnt lgkmcnt(0)\n\t" \
  "s_barrier\n\t" \
  "v_mfma_f32_16x16x32_bf16 a[208:211], v[128:131], v[36:39], a[208:211]\n\t" \
  "s_add_u32 %[sM], %[sM], 512\n\t" \
  "s_sub_u32 s81, s72, s76\n\t" \
  "s_cmp_ge_i32 s81, 0\n\t" \
  "s_cbranch_scc1 Lst55_%=\n\t" \
  "Lbk55_%=:\n\t" \
  "ds_read_b128 v[16:19], v12 offset:0\n\t" \
  "v_mfma_f32_16x16x32_bf16 a[212:215], v[132:135], v[36:39], a[212:215]\n\t" \
  "ds_read_b128 v[20:23], v12 offset:1024\n\t" \
  "v_add_u32 v4, 0, v12\n\t" \
  "v_mfma_f32_16x16x32_bf16 a[216:219], v[136:139], v[36:39], a[216:219]\n\t" \
  "ds_read_b128 v[24:27], v12 offset:2048\n\t" \
  "v_add_u32 v5, 4096, v12\n\t" \
  "v_mfma_f32_16x16x32_bf16 a[220:223], v[140:143], v[36:39], a[220:223]\n\t" \
  "ds_read_b128 v[28:31], v12 offset:3072\n\t" \
  "v_add_u32 v6, 8192, v12\n\t" \
  "v_mfma_f32_16x16x32_bf16 a[224:227], v[128:131], v[40:43], a[224:227]\n\t" \
  "ds_read_b128 v[32:35], v12 offset:4096\n\t" \
  "v_add_u32 v7, 12288, v12\n\t" \
  "v_mfma_f32_16x16x32_bf16 a[228:231], v[132:135], v[40:43], a[228:231]\n\t" \
  "s_xor_b32 %[same], %[same], %[idv]\n\t" \
  "ds_read_b128 v[36:39], v12 offset:5120\n\t" \
  "v_mfma_f32_16x16x32_bf16 a[232:235], v[136:139], v[40:43], a[232:235]\n\t" \
  "v_mfma_f32_16x16x32_bf16 a[236:239], v[140:143], v[40:43], a[236:239]\n\t" \
  "v_mfma_f32_16x16x32_bf16 a[240:243], v[128:131], v[44:47], a[240:243]\n\t" \
  "ds_read_b128 v[40:43], v12 offset:6144\n\t" \
  "v_mfma_f32_16x16x32_bf16 a[244:247], v[132:135], v[44:47], a[244:247]\n\t" \
  "v_mfma_f32_16x16x32_bf16 a[248:251], v[136:139], v[44:47], a[248:251]\n\t" \
  "v_mfma_f32_16x16x32_bf16 a[252:255], v[140:143], v[44:47], a[252:255]\n\t" \
  "s_waitcnt lgkmcnt(5)\n\t" \
  "v_mfma_f32_16x16x32_bf16 a[0:3], v[48:51], v[16:19], a[0:3]\n\t" \
  "ds_read_b128 v[44:47], v12 offset:7168\n\t" \
  "v_mfma_f32_16x16x32_bf16 a[4:7], v[52:55], v[16:19], a[4:7]\n\t" \
  "v_accvgpr_read_b32 v216, a128\n\t" \
  "v_accvgpr_read_b32 v217, a129\n\t" \
  "v_accvgpr_read_b32 v218, a130\n\t" \
  "v_accvgpr_read_b32 v219, a131\n\t" \
  "v_cvt_pk_bf16_f32 v80, v216, v217\n\t" \
  "v_accvgpr_read_b32 v220, a132\n\t" \
  "v_mfma_f32_16x16x32_bf16 a[8:11], v[56:59], v[16:19], a[8:11]\n\t" \
  "v_accvgpr_read_b32 v221, a133\n\t" \
  "v_cvt_pk_bf16_f32 v81, v218, v219\n\t" \
  "v_accvgpr_read_b32 v222, a134\n\t" \
  "v_accvgpr_read_b32 v223, a135\n\t" \
  "v_cvt_pk_bf16_f32 v82, v220, v221\n\t" \
  "v_accvgpr_read_b32 v216, a136\n\t" \
  "v_mfma_f32_16x16x32_bf16 a[12:15], v[60:63], v[16:19], a[12:15]\n\t" \
  "v_accvgpr_read_b32 v217, a137\n\t" \
  "v_cvt_pk_bf16_f32 v83, v222, v223\n\t" \
  "v_accvgpr_read_b32 v218, a138\n\t" \
  "v_accvgpr_read_b32 v219, a139\n\t" \
  "v_cvt_pk_bf16_f32 v84, v216, v217\n\t" \
  "global_store_dwordx4 v4, v[80:83], %[exw] offset:0 sc1\n\t" \
  "v_mfma_f32_16x16x32_bf16 a[16:19], v[48:51], v[20:23], a[16:19]\n\t" \
  "ds_read_b128 v[16:19], v12 offset:8192\n\t" \
  "v_accvgpr_read_b32 v220, a140\n\t" \
  "v_accvgpr_read_b32 v221, a141\n\t" \
  "v_cvt_pk_bf16_f32 v85, v218, v219\n\t" \
  "v_accvgpr_read_b32 v222, a142\n\t" \
  "v_accvgpr_read_b32 v223, a143\n\t" \
  "v_mfma_f32_16x16x32_bf16 a[20:23], v[52:55], v[20:23], a[20:23]\n\t" \
  "v_cvt_pk_bf16_f32 v86, v220, v221\n\t" \
  "v_accvgpr_read_b32 v216, a144\n\t" \
  "v_accvgpr_read_b32 v217, a145\n\t" \
  "v_cvt_pk_bf16_f32 v87, v222, v223\n\t" \
  "v_accvgpr_read_b32 v218, a146\n\t" \
  "v_accvgpr_read_b32 v219, a147\n\t" \
  "v_mfma_f32_16x16x32_bf16 a[24:27], v[56:59], v[20:23], a[24:27]\n\t" \
  "v_cvt_pk_bf16_f32 v88, v216, v217\n\t" \
  "global_store_dwordx4 v4, v[84:87], %[exw] offset:1024 sc1\n\t" \
  "v_accvgpr_read_b32 v220, a148\n\t" \
  "v_accvgpr_read_b32 v221, a149\n\t" \
  "v_cvt_pk_bf16_f32 v89, v218, v219\n\t" \
  "v_mfma_f32_16x16x32_bf16 a[28:31], v[60:63], v[20:23], a[28:31]\n\t" \
  "s_waitcnt lgkmcnt(5)\n\t" \
  "v_accvgpr_read_b32 v222, a150\n\t" \
  "v_accvgpr_read_b32 v223, a151\n\t" \
  "v_cvt_pk_bf16_f32 v90, v220, v221\n\t" \
  "v_accvgpr_read_b32 v216, a152\n\t" \
  "v_accvgpr_read_b32 v217, a153\n\t" \
  "v_cvt_pk_bf16_f32 v91, v222, v223\n\t" \
  "v_mfma_f32_16x16x32_bf16 a[32:35], v[48:51], v[24:27], a[32:35]\n\t" \
  "ds_read_b128 v[20:23], v12 offset:9216\n\t" \
  "v_accvgpr_read_b32 v218, a154\n\t" \
  "v_accvgpr_read_b32 v219, a155\n\t" \
  "v_cvt_pk_bf16_f32 v92, v216, v217\n\t" \
  "global_store_dwordx4 v4, v[88:91], %[exw] offset:2048 sc1\n\t" \
  "v_mfma_f32_16x16x32_bf16 a[36:39], v[52:55], v[24:27], a[36:39]\n\t" \
  "v_accvgpr_read_b32 v220, a156\n\t" \
  "v_accvgpr_read_b32 v221, a157\n\t" \
  "v_cvt_pk_bf16_f32 v93, v218, v219\n\t" \
  "v_accvgpr_read_b32 v222, a158\n\t" \
  "v_accvgpr_read_b32 v223, a159\n\t" \
  "v_cvt_pk_bf16_f32 v94, v220, v221\n\t" \
  "v_mfma_f32_16x16x32_bf16 a[40:43], v[56:59], v[24:27], a[40:43]\n\t" \
  "v_accvgpr_read_b32 v216, a160\n\t" \
  "v_accvgpr_read_b32 v217, a161\n\t" \
  "v_cvt_pk_bf16_f32 v95, v222, v223\n\t" \
  "v_accvgpr_read_b32 v218, a162\n\t" \
  "v_accvgpr_read_b32 v219, a163\n\t" \
  "v_cvt_pk_bf16_f32 v96, v216, v217\n\t" \
  "v_mfma_f32_16x16x32_bf16 a[44:47], v[60:63], v[24:27], a[44:47]\n\t" \
  "global_store_dwordx4 v4, v[92:95], %[exw] offset:3072 sc1\n\t" \
  "v_accvgpr_read_b32 v220, a164\n\t" \
  "v_accvgpr_read_b32 v221, a165\n\t" \
  "v_cvt_pk_bf16_f32 v97, v218, v219\n\t" \
  "v_accvgpr_read_b32 v222, a166\n\t" \
  "v_mfma_f32_16x16x32_bf16 a[48:51], v[48:51], v[28:31], a[48:51]\n\t" \
  "ds_read_b128 v[24:27], v12 offset:10240\n\t" \
  "v_accvgpr_read_b32 v223, a167\n\t" \
  "v_cvt_pk_bf16_f32 v98, v220, v221\n\t" \
  "v_accvgpr_read_b32 v216, a168\n\t" \
  "v_accvgpr_read_b32 v217, a169\n\t" \
  "v_cvt_pk_bf16_f32 v99, v222, v223\n\t" \
  "v_mfma_f32_16x16x32_bf16 a[52:55], v[52:55], v[28:31], a[52:55]\n\t" \
  "v_accvgpr_read_b32 v218, a170\n\t" \
  "v_accvgpr_read_b32 v219, a171\n\t" \
  "v_cvt_pk_bf16_f32 v100, v216, v217\n\t" \
  "global_store_dwordx4 v5, v[96:99], %[exw] offset:0 sc1\n\t" \
  "v_accvgpr_read_b32 v220, a172\n\t" \
  "v_mfma_f32_16x16x32_bf16 a[56:59], v[56:59], v[28:31], a[56:59]\n\t" \
  "v_accvgpr_read_b32 v221, a173\n\t" \
  "v_cvt_pk_bf16_f32 v101, v218, v219\n\t" \
  "v_accvgpr_read_b32 v222, a174\n\t" \
  "v_accvgpr_read_b32 v223, a175\n\t" \
  "v_cvt_pk_bf16_f32 v102, v220, v221\n\t" \
  "v_accvgpr_read_b32 v216, a176\n\t" \
  "v_mfma_f32_16x16x32_bf16 a[60:63], v[60:63], v[28:31], a[60:63]\n\t" \
  "s_waitcnt lgkmcnt(5)\n\t" \
  "v_accvgpr_read_b32 v217, a177\n\t" \
  "v_cvt_pk_bf16_f32 v103, v222, v223\n\t" \
  "v_accvgpr_read_b32 v218, a178\n\t" \
  "v_accvgpr_read_b32 v219, a179\n\t" \
  "v_cvt_pk_bf16_f32 v104, v216, v217\n\t" \
  "global_store_dwordx4 v5, v[100:103], %[exw] offset:1024 sc1\n\t" \
  "v_mfma_f32_16x16x32_bf16 a[64:67], v[48:51], v[32:35], a[64:67]\n\t" \
  "ds_read_b128 v[28:31], v12 offset:11264\n\t" \
  "v_accvgpr_read_b32 v220, a180\n\t" \
  "v_accvgpr_read_b32 v221, a181\n\t" \
  "v_cvt_pk_bf16_f32 v105, v218, v219\n\t" \
  "v_accvgpr_read_b32 v222, a182\n\t" \
  "v_accvgpr_read_b32 v223, a183\n\t" \
  "v_mfma_f32_16x16x32_bf16 a[68:71], v[52:55], v[32:35], a[68:71]\n\t" \
  "v_cvt_pk_bf16_f32 v106, v220, v221\n\t" \
  "v_accvgpr_read_b32 v216, a184\n\t" \
  "v_accvgpr_read_b32 v217, a185\n\t" \
  "v_cvt_pk_bf16_f32 v107, v222, v223\n\t" \
  "v_accvgpr_read_b32 v218, a186\n\t" \
  "v_accvgpr_read_b32 v219, a187\n\t" \
  "v_mfma_f32_16x16x32_bf16 a[72:75], v[56:59], v[32:35], a[72:75]\n\t" \
  "v_cvt_pk_bf16_f32 v108, v216, v217\n\t" \
  "global_store_dwordx4 v5, v[104:107], %[exw] offset:2048 sc1\n\t" \
  "v_accvgpr_read_b32 v220, a188\n\t" \
  "v_accvgpr_read_b32 v221, a189\n\t" \
  "v_cvt_pk_bf16_f32 v109, v218, v219\n\t" \
  "v_mfma_f32_16x16x32_bf16 a[76:79], v[60:63], v[32:35], a[76:79]\n\t" \
  "v_accvgpr_read_b32 v222, a190\n\t" \
  "v_accvgpr_read_b32 v223, a191\n\t" \
  "v_cvt_pk_bf16_f32 v110, v220, v221\n\t" \
  "v_accvgpr_read_b32 v216, a192\n\t" \
  "v_accvgpr_read_b32 v217, a193\n\t" \
  "v_cvt_pk_bf16_f32 v111, v222, v223\n\t" \
  "v_mfma_f32_16x16x32_bf16 a[80:83], v[48:51], v[36:39], a[80:83]\n\t" \
  "ds_read_b128 v[32:35], v12 offset:12288\n\t" \
  "s_memrealtime s[72:73]\n\t" \
  "v_accvgpr_read_b32 v218, a194\n\t" \
  "v_accvgpr_read_b32 v219, a195\n\t" \
  "v_cvt_pk_bf16_f32 v144, v216, v217\n\t" \
  "global_store_dwordx4 v5, v[108:111], %[exw] offset:3072 sc1\n\t" \
  "v_mfma_f32_16x16x32_bf16 a[84:87], v[52:55], v[36:39], a[84:87]\n\t" \
  "v_accvgpr_read_b32 v220, a196\n\t" \
  "v_accvgpr_read_b32 v221, a197\n\t" \
  "v_cvt_pk_bf16_f32 v145, v218, v219\n\t" \
  "v_accvgpr_read_b32 v222, a198\n\t" \
  "v_accvgpr_read_b32 v223, a199\n\t" \
  "v_cvt_pk_bf16_f32 v146, v220, v221\n\t" \
  "v_mfma_f32_16x16x32_bf16 a[88:91], v[56:59], v[36:39], a[88:91]\n\t" \
  "v_accvgpr_read_b32 v216, a200\n\t" \
  "v_accvgpr_read_b32 v217, a201\n\t" \
  "v_cvt_pk_bf16_f32 v147, v222, v223\n\t" \
  "v_accvgpr_read_b32 v218, a202\n\t" \
  "v_accvgpr_read_b32 v219, a203\n\t" \
  "v_cvt_pk_bf16_f32 v148, v216, v217\n\t" \
  "v_mfma_f32_16x16x32_bf16 a[92:95], v[60:63], v[36:39], a[92:95]\n\t" \
  "s_waitcnt lgkmcnt(5)\n\t" \
  "global_store_dwordx4 v6, v[144:147], %[exw] offset:0 sc1\n\t" \
  "v_accvgpr_read_b32 v220, a204\n\t" \
  "v_accvgpr_read_b32 v221, a205\n\t" \
  "v_cvt_pk_bf16_f32 v149, v218, v219\n\t" \
  "v_accvgpr_read_b32 v222, a206\n\t" \
  "v_mfma_f32_16x16x32_bf16 a[96:99], v[48:51], v[40:43], a[96:99]\n\t" \
  "s_cbranch_vccnz Lst65_%=\n\t" \
  "s_waitcnt vmcnt(9)\n\t" \
  "Lbk65_%=:\n\t" \
  "s_and_b32 s62, s53, 0x7f807f80\n\t" \
  "ds_read_b128 v[36:39], v12 offset:13312\n\t" \
  "v_and_b32 v200, s63, v196\n\t" \
  "v_mfma_f32_16x16x32_bf16 a[100:103], v[52:55], v[40:43], a[100:103]\n\t" \
  "v_alignbit_b32 v201, v196, v196, 3\n\t" \
  "v_perm_b32 v204, v200, v188, s84\n\t" \
  "v_and_b32 v201, s63, v201\n\t" \
  "v_mfma_f32_16x16x32_bf16 a[104:107], v[56:59], v[40:43], a[104:107]\n\t" \
  "v_perm_b32 v205, v200, v188, s85\n\t" \
  "v_add_u32 v204, s62, v204\n\t" \
  "v_perm_b32 v206, v201, v189, s84\n\t" \
  "v_mfma_f32_16x16x32_bf16 a[108:111], v[60:63], v[40:43], a[108:111]\n\t" \
  "v_add_u32 v205, s62, v205\n\t" \
  "v_perm_b32 v207, v201, v189, s85\n\t" \
  "v_add_u32 v206, s62, v206\n\t" \
  "v_mfma_f32_16x16x32_bf16 a[112:115], v[48:51], v[44:47], a[112:115]\n\t" \
  "ds_read_b128 v[40:43], v12 offset:14336\n\t" \
  "v_add_u32 v207, s62, v207\n\t" \
  "ds_write_b128 v13, v[204:207] offset:16384\n\t" \
  "v_mfma_f32_16x16x32_bf16 a[116:119], v[52:55], v[44:47], a[116:119]\n\t" \
  "v_and_b32 v202, s63, v197\n\t" \
  "v_alignbit_b32 v203, v197, v197, 3\n\t" \
  "v_perm_b32 v208, v202, v190, s84\n\t" \
  "v_mfma_f32_16x16x32_bf16 a[120:123], v[56:59], v[44:47], a[120:123]\n\t" \
  "v_and_b32 v203, s63, v203\n\t" \
  "v_perm_b32 v209, v202, v190, s85\n\t" \
  "v_add_u32 v208, s62, v208\n\t" \
  "v_mfma_f32_16x16x32_bf16 a[124:127], v[60:63], v[44:47], a[124:127]\n\t" \
  "s_waitcnt lgkmcnt(6)\n\t" \
  "v_perm_b32 v210, v203, v191, s84\n\t" \
  "v_add_u32 v209, s62, v209\n\t" \
  "v_perm_b32 v211, v203, v191, s85\n\t" \
  "v_mfma_f32_16x16x32_bf16 a[0:3], v[64:67], v[16:19], a[0:3]\n\t" \
  "ds_read_b128 v[44:47], v12 offset:15360\n\t" \
  "v_add_u32 v210, s62, v210\n\t" \
  "v_add_u32 v211, s62, v211\n\t" \
  "v_mfma_f32_16x16x32_bf16 a[4:7], v[68:71], v[16:19], a[4:7]\n\t" \
  "ds_write_b128 v13, v[208:211] offset:17408\n\t" \
  "v_and_b32 v200, s63, v198\n\t" \
  "v_alignbit_b32 v201, v198, v198, 3\n\t" \
  "v_mfma_f32_16x16x32_bf16 a[8:11], v[72:75], v[16:19], a[8:11]\n\t" \
  "v_perm_b32 v212, v200, v192, s84\n\t" \
  "v_and_b32 v201, s63, v201\n\t" \
  "v_perm_b32 v213, v200, v192, s85\n\t" \
  "v_mfma_f32_16x16x32_bf16 a[12:15], v[76:79], v[16:19], a[12:15]\n\t" \
  "v_add_u32 v212, s62, v212\n\t" \
  "v_perm_b32 v214, v201, v193, s84\n\t" \
  "v_add_u32 v213, s62, v213\n\t" \
  "v_mfma_f32_16x16x32_bf16 a[16:19], v[64:67], v[20:23], a[16:19]\n\t" \
  "v_perm_b32 v215, v201, v193, s85\n\t" \
  "v_add_u32 v214, s62, v214\n\t" \
  "v_add_u32 v215, s62, v215\n\t" \
  "v_mfma_f32_16x16x32_bf16 a[20:23], v[68:71], v[20:23], a[20:23]\n\t" \
  "ds_write_b128 v13, v[212:215] offset:18432\n\t" \
  "v_lshrrev_b32 v202, 5, v196\n\t" \
  "v_lshlrev_b32 v204, 2, v198\n\t" \
  "v_mfma_f32_16x16x32_bf16 a[24:27], v[72:75], v[20:23], a[24:27]\n\t" \
  "v_lshrrev_b32 v203, 5, v197\n\t" \
  "v_lshlrev_b32 v205, 1, v198\n\t" \
  "v_bfi_b32 v202, s64, v204, v202\n\t" \
  "v_mfma_f32_16x16x32_bf16 a[28:31], v[76:79], v[20:23], a[28:31]\n\t" \
  "s_waitcnt lgkmcnt(7)\n\t" \
  "v_bfi_b32 v203, s64, v205, v203\n\t" \
  "v_and_b32 v202, s63, v202\n\t" \
  "v_perm_b32 v204, v202, v194, s84\n\t" \
  "v_mfma_f32_16x16x32_bf16 a[32:35], v[64:67], v[24:27], a[32:35]\n\t" \
  "v_and_b32 v203, s63, v203\n\t" \
  "v_perm_b32 v205, v202, v194, s85\n\t" \
  "v_add_u32 v204, s62, v204\n\t" \
  "v_mfma_f32_16x16x32_bf16 a[36:39], v[68:71], v[24:27], a[36:39]\n\t" \
  "v_perm_b32 v206, v203, v195, s84\n\t" \
  "v_add_u32 v205, s62, v205\n\t" \
  "v_perm_b32 v207, v203, v195, s85\n\t" \
  "v_mfma_f32_16x16x32_bf16 a[40:43], v[72:75], v[24:27], a[40:43]\n\t" \
  "v_add_u32 v206, s62, v206\n\t" \
  "v_add_u32 v207, s62, v207\n\t" \
  "v_lshrrev_b32 v200, 16, v199\n\t" \
  "v_mfma_f32_16x16x32_bf16 a[44:47], v[76:79], v[24:27], a[44:47]\n\t" \
  "ds_write_b128 v13, v[204:207] offset:19456\n\t" \
  "v_mfma_f32_16x16x32_bf16 a[48:51], v[64:67], v[28:31], a[48:51]\n\t" \
  "s_bfm_b64 exec, s53, 0\n\t" \
  "s_cbranch_execz Lst66_%=\n\t" \
  "ds_write_b16 v200, v199 offset:16384\n\t" \
  "Lbk66_%=:\n\t" \
  "s_mov_b64 exec, -1\n\t" \
  "v_mfma_f32_16x16x32_bf16 a[52:55], v[68:71], v[28:31], a[52:55]\n\t" \
  "v_mfma_f32_16x16x32_bf16 a[56:59], v[72:75], v[28:31], a[56:59]\n\t" \
  "v_mfma_f32_16x16x32_bf16 a[60:63], v[76:79], v[28:31], a[60:63]\n\t" \
  "s_waitcnt lgkmcnt(7)\n\t" \
  "v_mfma_f32_16x16x32_bf16 a[64:67], v[64:67], v[32:35], a[64:67]\n\t" \
  "v_mfma_f32_16x16x32_bf16 a[68:71], v[68:71], v[32:35], a[68:71]\n\t" \
  "v_mfma_f32_16x16x32_bf16 a[72:75], v[72:75], v[32:35], a[72:75]\n\t" \
  "v_mfma_f32_16x16x32_bf16 a[76:79], v[76:79], v[32:35], a[76:79]\n\t" \
  "s_waitcnt lgkmcnt(0)\n\t" \
  "s_barrier\n\t" \
  "v_mfma_f32_16x16x32_bf16 a[80:83], v[64:67], v[36:39], a[80:83]\n\t" \
  "s_add_u32 %[sM], %[sM], 512\n\t" \
  "s_sub_u32 s81, s72, s76\n\t" \
  "s_cmp_ge_i32 s81, 0\n\t" \
  "s_cbranch_scc1 Lst67_%=\n\t" \
  "Lbk67_%=:\n\t" \
  "ds_read_b128 v[16:19], v12 offset:16384\n\t" \
  "v_mfma_f32_16x16x32_bf16 a[84:87], v[68:71], v[36:39], a[84:87]\n\t" \
  "ds_read_b128 v[20:23], v12 offset:17408\n\t" \
  "v_accvgpr_read_b32 v223, a207\n\t" \
  "v_cvt_pk_bf16_f32 v150, v220, v221\n\t" \
  "v_mfma_f32_16x16x32_bf16 a[88:91], v[72:75], v[36:39], a[88:91]\n\t" \
  "ds_read_b128 v[24:27], v12 offset:18432\n\t" \
  "v_accvgpr_read_b32 v216, a208\n\t" \
  "v_accvgpr_read_b32 v217, a209\n\t" \
  "v_mfma_f32_16x16x32_bf16 a[92:95], v[76:79], v[36:39], a[92:95]\n\t" \
  "ds_read_b128 v[28:31], v12 offset:19456\n\t" \
  "v_cvt_pk_bf16_f32 v151, v222, v223\n\t" \
  "v_accvgpr_read_b32 v218, a210\n\t" \
  "v_mfma_f32_16x16x32_bf16 a[96:99], v[64:67], v[40:43], a[96:99]\n\t" \
  "ds_read_b128 v[32:35], v12 offset:20480\n\t" \
  "v_accvgpr_read_b32 v219, a211\n\t" \
  "v_cvt_pk_bf16_f32 v152, v216, v217\n\t" \
  "v_mfma_f32_16x16x32_bf16 a[100:103], v[68:71], v[40:43], a[100:103]\n\t" \
  "ds_read_b128 v[36:39], v12 offset:21504\n\t" \
  "global_store_dwordx4 v6, v[148:151], %[exw] offset:1024 sc1\n\t" \
  "v_mfma_f32_16x16x32_bf16 a[104:107], v[72:75], v[40:43], a[104:107]\n\t" \
  "v_accvgpr_read_b32 v220, a212\n\t" \
  "v_accvgpr_read_b32 v221, a213\n\t" \
  "v_cvt_pk_bf16_f32 v153, v218, v219\n\t" \
  "v_mfma_f32_16x16x32_bf16 a[108:111], v[76:79], v[40:43], a[108:111]\n\t" \
  "v_accvgpr_read_b32 v222, a214\n\t" \
  "v_accvgpr_read_b32 v223, a215\n\t" \
  "v_cvt_pk_bf16_f32 v154, v220, v221\n\t" \
  "v_mfma_f32_16x16x32_bf16 a[112:115], v[64:67], v[44:47], a[112:115]\n\t" \
  "ds_read_b128 v[40:43], v12 offset:22528\n\t" \
  "v_accvgpr_read_b32 v216, a216\n\t" \
  "v_accvgpr_read_b32 v217, a217\n\t" \
  "v_mfma_f32_16x16x32_bf16 a[116:119], v[68:71], v[44:47], a[116:119]\n\t" \
  "v_cvt_pk_bf16_f32 v155, v222, v223\n\t" \
  "v_accvgpr_read_b32 v218, a218\n\t" \
  "v_accvgpr_read_b32 v219, a219\n\t" \
  "v_mfma_f32_16x16x32_bf16 a[120:123], v[72:75], v[44:47], a[120:123]\n\t" \
  "v_cvt_pk_bf16_f32 v156, v216, v217\n\t" \
  "global_store_dwordx4 v6, v[152:155], %[exw] offset:2048 sc1\n\t" \
  "v_mfma_f32_16x16x32_bf16 a[124:127], v[76:79], v[44:47], a[124:127]\n\t" \
  "s_waitcnt lgkmcnt(5)\n\t" \
  "v_accvgpr_read_b32 v220, a220\n\t" \
  "v_accvgpr_read_b32 v221, a221\n\t" \
  "v_cvt_pk_bf16_f32 v157, v218, v219\n\t" \
  "v_mfma_f32_16x16x32_bf16 a[0:3], v[112:115], v[16:19], a[0:3]\n\t" \
  "ds_read_b128 v[44:47], v12 offset:23552\n\t" \
  "v_accvgpr_read_b32 v222, a222\n\t" \
  "v_accvgpr_read_b32 v223, a223\n\t" \
  "s_xor_b32 s82, %[sEF], 2048\n\t" \
  "s_load_dword s83, %[ws], s82 glc\n\t" \
  "v_mfma_f32_16x16x32_bf16 a[4:7], v[116:119], v[16:19], a[4:7]\n\t" \
  "v_cvt_pk_bf16_f32 v158, v220, v221\n\t" \
  "v_accvgpr_read_b32 v216, a224\n\t" \
  "v_accvgpr_read_b32 v217, a225\n\t" \
  "v_mfma_f32_16x16x32_bf16 a[8:11], v[120:123], v[16:19], a[8:11]\n\t" \
  "v_cvt_pk_bf16_f32 v159, v222, v223\n\t" \
  "v_accvgpr_read_b32 v218, a226\n\t" \
  "v_accvgpr_read_b32 v219, a227\n\t" \
  "v_mfma_f32_16x16x32_bf16 a[12:15], v[124:127], v[16:19], a[12:15]\n\t" \
  "v_cvt_pk_bf16_f32 v160, v216, v217\n\t" \
  "global_store_dwordx4 v6, v[156:159], %[exw] offset:3072 sc1\n\t" \
  "v_mfma_f32_16x16x32_bf16 a[16:19], v[112:115], v[20:23], a[16:19]\n\t" \
  "ds_read_b128 v[16:19], v12 offset:24576\n\t" \
  "v_accvgpr_read_b32 v220, a228\n\t" \
  "v_accvgpr_read_b32 v221, a229\n\t" \
  "v_mfma_f32_16x16x32_bf16 a[20:23], v[116:119], v[20:23], a[20:23]\n\t" \
  "v_cvt_pk_bf16_f32 v161, v218, v219\n\t" \
  "v_accvgpr_read_b32 v222, a230\n\t" \
  "v_accvgpr_read_b32 v223, a231\n\t" \
  "v_mfma_f32_16x16x32_bf16 a[24:27], v[120:123], v[20:23], a[24:27]\n\t" \
  "v_cvt_pk_bf16_f32 v162, v220, v221\n\t" \
  "v_accvgpr_read_b32 v216, a232\n\t" \
  "v_accvgpr_read_b32 v217, a233\n\t" \
  "v_mfma_f32_16x16x32_bf16 a[28:31], v[124:127], v[20:23], a[28:31]\n\t" \
  "s_waitcnt lgkmcnt(5)\n\t" \
  "v_cvt_pk_bf16_f32 v163, v222, v223\n\t" \
  "v_accvgpr_read_b32 v218, a234\n\t" \
  "v_accvgpr_read_b32 v219, a235\n\t" \
  "v_mfma_f32_16x16x32_bf16 a[32:35], v[112:115], v[24:27], a[32:35]\n\t" \
  "ds_read_b128 v[20:23], v12 offset:25600\n\t" \
  "v_cvt_pk_bf16_f32 v164, v216, v217\n\t" \
  "global_store_dwordx4 v7, v[160:163], %[exw] offset:0 sc1\n\t" \
  "v_mfma_f32_16x16x32_bf16 a[36:39], v[116:119], v[24:27], a[36:39]\n\t" \
  "v_accvgpr_read_b32 v220, a236\n\t" \
  "v_accvgpr_read_b32 v221, a237\n\t" \
  "v_cvt_pk_bf16_f32 v165, v218, v219\n\t" \
  "v_mfma_f32_16x16x32_bf16 a[40:43], v[120:123], v[24:27], a[40:43]\n\t" \
  "v_accvgpr_read_b32 v222, a238\n\t" \
  "v_accvgpr_read_b32 v223, a239\n\t" \
  "v_cvt_pk_bf16_f32 v166, v220, v221\n\t" \
  "v_mfma_f32_16x16x32_bf16 a[44:47], v[124:127], v[24:27], a[44:47]\n\t" \
  "v_accvgpr_read_b32 v216, a240\n\t" \
  "v_accvgpr_read_b32 v217, a241\n\t" \
  "v_cvt_pk_bf16_f32 v167, v222, v223\n\t" \
  "v_mfma_f32_16x16x32_bf16 a[48:51], v[112:115], v[28:31], a[48:51]\n\t" \
  "ds_read_b128 v[24:27], v12 offset:26624\n\t" \
  "v_accvgpr_read_b32 v218, a242\n\t" \
  "v_accvgpr_read_b32 v219, a243\n\t" \
  "v_mfma_f32_16x16x32_bf16 a[52:55], v[116:119], v[28:31], a[52:55]\n\t" \
  "v_cvt_pk_bf16_f32 v168, v216, v217\n\t" \
  "global_store_dwordx4 v7, v[164:167], %[exw] offset:1024 sc1\n\t" \
  "v_mfma_f32_16x16x32_bf16 a[56:59], v[120:123], v[28:31], a[56:59]\n\t" \
  "v_accvgpr_read_b32 v220, a244\n\t" \
  "v_accvgpr_read_b32 v221, a245\n\t" \
  "v_cvt_pk_bf16_f32 v169, v218, v219\n\t" \
  "v_mfma_f32_16x16x32_bf16 a[60:63], v[124:127], v[28:31], a[60:63]\n\t" \
  "s_waitcnt lgkmcnt(5)\n\t" \
  "v_accvgpr_read_b32 v222, a246\n\t" \
  "v_accvgpr_read_b32 v223, a247\n\t" \
  "v_cvt_pk_bf16_f32 v170, v220, v221\n\t" \
  "v_mfma_f32_16x16x32_bf16 a[64:67], v[112:115], v[32:35], a[64:67]\n\t" \
  "ds_read_b128 v[28:31], v12 offset:27648\n\t" \
  "v_accvgpr_read_b32 v216, a248\n\t" \
  "v_accvgpr_read_b32 v217, a249\n\t" \
  "v_mfma_f32_16x16x32_bf16 a[68:71], v[116:119], v[32:35], a[68:71]\n\t" \
  "v_cvt_pk_bf16_f32 v171, v222, v223\n\t" \
  "v_accvgpr_read_b32 v218, a250\n\t" \
  "v_accvgpr_read_b32 v219, a251\n\t" \
  "v_mfma_f32_16x16x32_bf16 a[72:75], v[120:123], v[32:35], a[72:75]\n\t" \
  "v_cvt_pk_bf16_f32 v172, v216, v217\n\t" \
  "global_store_dwordx4 v7, v[168:171], %[exw] offset:2048 sc1\n\t" \
  "v_mfma_f32_16x16x32_bf16 a[76:79], v[124:127], v[32:35], a[76:79]\n\t" \
  "v_accvgpr_read_b32 v220, a252\n\t" \
  "v_accvgpr_read_b32 v221, a253\n\t" \
  "v_cvt_pk_bf16_f32 v173, v218, v219\n\t" \
  "v_mfma_f32_16x16x32_bf16 a[80:83], v[112:115], v[36:39], a[80:83]\n\t" \
  "ds_read_b128 v[32:35], v12 offset:28672\n\t" \
  "v_accvgpr_read_b32 v222, a254\n\t" \
  "v_accvgpr_read_b32 v223, a255\n\t" \
  "v_mfma_f32_16x16x32_bf16 a[84:87], v[116:119], v[36:39], a[84:87]\n\t" \
  "v_cvt_pk_bf16_f32 v174, v220, v221\n\t" \
  "v_cvt_pk_bf16_f32 v175, v222, v223\n\t" \
  "s_nop 0\n\t" \
  "global_store_dwordx4 v7, v[172:175], %[exw] offset:3072 sc1\n\t" \
  "v_mfma_f32_16x16x32_bf16 a[88:91], v[120:123], v[36:39], a[88:91]\n\t" \
  "v_mfma_f32_16x16x32_bf16 a[92:95], v[124:127], v[36:39], a[92:95]\n\t" \
  "s_waitcnt lgkmcnt(5)\n\t" \
  "v_mfma_f32_16x16x32_bf16 a[96:99], v[112:115], v[40:43], a[96:99]\n\t" \
  "ds_read_b128 v[36:39], v12 offset:29696\n\t" \
  "v_mfma_f32_16x16x32_bf16 a[100:103], v[116:119], v[40:43], a[100:103]\n\t" \
  "v_mfma_f32_16x16x32_bf16 a[104:107], v[120:123], v[40:43], a[104:107]\n\t" \
  "v_mfma_f32_16x16x32_bf16 a[108:111], v[124:127], v[40:43], a[108:111]\n\t" \
  "v_mfma_f32_16x16x32_bf16 a[112:115], v[112:115], v[44:47], a[112:115]\n\t" \
  "ds_read_b128 v[40:43], v12 offset:30720\n\t" \
  "v_mfma_f32_16x16x32_bf16 a[116:119], v[116:119], v[44:47], a[116:119]\n\t" \
  "v_mfma_f32_16x16x32_bf16 a[120:123], v[120:123], v[44:47], a[120:123]\n\t" \
  "v_mfma_f32_16x16x32_bf16 a[124:127], v[124:127], v[44:47], a[124:127]\n\t" \
  "s_waitcnt lgkmcnt(5)\n\t" \
  "v_mfma_f32_16x16x32_bf16 a[0:3], v[128:131], v[16:19], a[0:3]\n\t" \
  "ds_read_b128 v[44:47], v12 offset:31744\n\t" \
  "v_mfma_f32_16x16x32_bf16 a[4:7], v[132:135], v[16:19], a[4:7]\n\t" \
  "v_mfma_f32_16x16x32_bf16 a[8:11], v[136:139], v[16:19], a[8:11]\n\t" \
  "v_mfma_f32_16x16x32_bf16 a[12:15], v[140:143], v[16:19], a[12:15]\n\t" \
  "v_mfma_f32_16x16x32_bf16 a[16:19], v[128:131], v[20:23], a[16:19]\n\t" \
  "v_mfma_f32_16x16x32_bf16 a[20:23], v[132:135], v[20:23], a[20:23]\n\t" \
  "v_mfma_f32_16x16x32_bf16 a[24:27], v[136:139], v[20:23], a[24:27]\n\t" \
  "v_mfma_f32_16x16x32_bf16 a[28:31], v[140:143], v[20:23], a[28:31]\n\t" \
  "s_waitcnt lgkmcnt(4)\n\t" \
  "v_mfma_f32_16x16x32_bf16 a[32:35], v[128:131], v[24:27], a[32:35]\n\t" \
  "v_mfma_f32_16x16x32_bf16 a[36:39], v[132:135], v[24:27], a[36:39]\n\t" \
  "v_mfma_f32_16x16x32_bf16 a[40:43], v[136:139], v[24:27], a[40:43]\n\t" \
  "v_mfma_f32_16x16x32_bf16 a[44:47], v[140:143], v[24:27], a[44:47]\n\t" \
  "v_mfma_f32_16x16x32_bf16 a[48:51], v[128:131], v[28:31], a[48:51]\n\t" \
  "s_waitcnt vmcnt(0)\n\t" \
  "s_mov_b64 exec, 1\n\t" \
  "v_mov_b32 v234, %[sEF]\n\t" \
  "v_mov_b32 v235, %[call]\n\t" \
  "global_store_dword v234, v235, %[ws] sc0 sc1\n\t" \
  "s_mov_b64 exec, -1\n\t" \
  "v_mfma_f32_16x16x32_bf16 a[52:55], v[132:135], v[28:31], a[52:55]\n\t" \
  "s_waitcnt lgkmcnt(0)\n\t" \
  "s_cmp_ge_u32 s83, %[cmp]\n\t" \
  "s_cbranch_scc0 Ler0_%=\n\t" \
  "global_load_dwordx4 v[48:51], v4, %[exr] offset:0 sc1\n\t" \
  "global_load_dwordx4 v[52:55], v4, %[exr] offset:1024 sc1\n\t" \
  "Ler0_%=:\n\t" \
  "v_mfma_f32_16x16x32_bf16 a[56:59], v[136:139], v[28:31], a[56:59]\n\t" \
  "s_cbranch_scc0 Ler1_%=\n\t" \
  "global_load_dwordx4 v[56:59], v4, %[exr] offset:2048 sc1\n\t" \
  "global_load_dwordx4 v[60:63], v4, %[exr] offset:3072 sc1\n\t" \
  "Ler1_%=:\n\t" \
  "v_mfma_f32_16x16x32_bf16 a[60:63], v[140:143], v[28:31], a[60:63]\n\t" \
  "s_cbranch_scc0 Ler2_%=\n\t" \
  "global_load_dwordx4 v[64:67], v5, %[exr] offset:0 sc1\n\t" \
  "global_load_dwordx4 v[68:71], v5, %[exr] offset:1024 sc1\n\t" \
  "Ler2_%=:\n\t" \
  "v_mfma_f32_16x16x32_bf16 a[64:67], v[128:131], v[32:35], a[64:67]\n\t" \
  "s_cbranch_scc0 Ler3_%=\n\t" \
  "global_load_dwordx4 v[72:75], v5, %[exr] offset:2048 sc1\n\t" \
  "global_load_dwordx4 v[76:79], v5, %[exr] offset:3072 sc1\n\t" \
  "Ler3_%=:\n\t" \
  "v_mfma_f32_16x16x32_bf16 a[68:71], v[132:135], v[32:35], a[68:71]\n\t" \
  "s_cbranch_scc0 Ler4_%=\n\t" \
  "global_load_dwordx4 v[80:83], v6, %[exr] offset:0 sc1\n\t" \
  "global_load_dwordx4 v[84:87], v6, %[exr] offset:1024 sc1\n\t" \
  "Ler4_%=:\n\t" \
  "v_mfma_f32_16x16x32_bf16 a[72:75], v[136:139], v[32:35], a[72:75]\n\t" \
  "s_cbranch_scc0 Ler5_%=\n\t" \
  "global_load_dwordx4 v[88:91], v6, %[exr] offset:2048 sc1\n\t" \
  "global_load_dwordx4 v[92:95], v6, %[exr] offset:3072 sc1\n\t" \
  "Ler5_%=:\n\t" \
  "v_mfma_f32_16x16x32_bf16 a[76:79], v[140:143], v[32:35], a[76:79]\n\t" \
  "s_waitcnt lgkmcnt(0)\n\t" \
  "s_cbranch_scc0 Ler6_%=\n\t" \
  "global_load_dwordx4 v[96:99], v7, %[exr] offset:0 sc1\n\t" \
  "global_load_dwordx4 v[100:103], v7, %[exr] offset:1024 sc1\n\t" \
  "Ler6_%=:\n\t" \
  "v_mfma_f32_16x16x32_bf16 a[80:83], v[128:131], v[36:39], a[80:83]\n\t" \
  "s_cbranch_scc0 Ler7_%=\n\t" \
  "global_load_dwordx4 v[104:107], v7, %[exr] offset:2048 sc1\n\t" \
  "global_load_dwordx4 v[108:111], v7, %[exr] offset:3072 sc1\n\t" \
  "Ler7_%=:\n\t" \
  "v_mfma_f32_16x16x32_bf16 a[84:87], v[132:135], v[36:39], a[84:87]\n\t" \
  "v_mfma_f32_16x16x32_bf16 a[88:91], v[136:139], v[36:39], a[88:91]\n\t" \
  "v_mfma_f32_16x16x32_bf16 a[92:95], v[140:143], v[36:39], a[92:95]\n\t" \
  "v_mfma_f32_16x16x32_bf16 a[96:99], v[128:131], v[40:43], a[96:99]\n\t" \
  "v_mfma_f32_16x16x32_bf16 a[100:103], v[132:135], v[40:43], a[100:103]\n\t" \
  "v_mfma_f32_16x16x32_bf16 a[104:107], v[136:139], v[40:43], a[104:107]\n\t" \
  "v_mfma_f32_16x16x32_bf16 a[108:111], v[140:143], v[40:43], a[108:111]\n\t" \
  "v_mfma_f32_16x16x32_bf16 a[112:115], v[128:131], v[44:47], a[112:115]\n\t" \
  "v_mfma_f32_16x16x32_bf16 a[116:119], v[132:135], v[44:47], a[116:119]\n\t" \
  "v_mfma_f32_16x16x32_bf16 a[120:123], v[136:139], v[44:47], a[120:123]\n\t" \
  "v_mfma_f32_16x16x32_bf16 a[124:127], v[140:143], v[44:47], a[124:127]\n\t" \
  "Ldr_%=:\n\t" \
  "s_cmp_ge_u32 s77, 8\n\t" \
  "s_cbranch_scc1 Ldre_%=\n\t" \
  "s_memrealtime s[72:73]\n\t" \
  "s_waitcnt lgkmcnt(0)\n\t" \
  "s_sub_u32 s81, s72, s76\n\t" \
  "s_cmp_ge_i32 s81, 0\n\t" \
  "s_cbranch_scc0 Ldr_%=\n\t" \
  "s_getpc_b64 s[74:75]\n\t" \
  "s_branch Lsvcc_%=\n\t" \
  "s_branch Ldr_%=\n\t" \
  "Ldre_%=:\n\t" \
  "s_cmp_ge_u32 s83, %[cmp]\n\t" \
  "s_cbranch_scc1 Lew3_%=\n\t" \
  "s_xor_b32 s82, %[sEF], 2048\n\t" \
  "Lew_%=:\n\t" \
  "s_load_dword s81, %[ws], s82 glc\n\t" \
  "s_memrealtime s[72:73]\n\t" \
  "s_waitcnt lgkmcnt(0)\n\t" \
  "s_cmp_ge_u32 s81, %[cmp]\n\t" \
  "s_cbranch_scc1 Lew2_%=\n\t" \
  "s_sub_u32 s81, s72, %[t0]\n\t" \
  "s_cmp_gt_u32 s81, %[dl]\n\t" \
  "s_cbranch_scc0 Lew_%=\n\t" \
  "v_mov_b32 v234, 0\n\t" \
  "v_mov_b32 v235, %[call]\n\t" \
  "global_store_dword v234, v235, %[ws] offset:128 sc1\n\t" \
  "s_nop 1\n\t" \
  "Lew2_%=:\n\t" \
  "global_load_dwordx4 v[48:51], v4, %[exr] offset:0 sc1\n\t" \
  "global_load_dwordx4 v[52:55], v4, %[exr] offset:1024 sc1\n\t" \
  "global_load_dwordx4 v[56:59], v4, %[exr] offset:2048 sc1\n\t" \
  "global_load_dwordx4 v[60:63], v4, %[exr] offset:3072 sc1\n\t" \
  "global_load_dwordx4 v[64:67], v5, %[exr] offset:0 sc1\n\t" \
  "global_load_dwordx4 v[68:71], v5, %[exr] offset:1024 sc1\n\t" \
  "global_load_dwordx4 v[72:75], v5, %[exr] offset:2048 sc1\n\t" \
  "global_load_dwordx4 v[76:79], v5, %[exr] offset:3072 sc1\n\t" \
  "global_load_dwordx4 v[80:83], v6, %[exr] offset:0 sc1\n\t" \
  "global_load_dwordx4 v[84:87], v6, %[exr] offset:1024 sc1\n\t" \
  "global_load_dwordx4 v[88:91], v6, %[exr] offset:2048 sc1\n\t" \
  "global_load_dwordx4 v[92:95], v6, %[exr] offset:3072 sc1\n\t" \
  "global_load_dwordx4 v[96:99], v7, %[exr] offset:0 sc1\n\t" \
  "global_load_dwordx4 v[100:103], v7, %[exr] offset:1024 sc1\n\t" \
  "global_load_dwordx4 v[104:107], v7, %[exr] offset:2048 sc1\n\t" \
  "global_load_dwordx4 v[108:111], v7, %[exr] offset:3072 sc1\n\t" \
  "Lew3_%=:\n\t" \
  "v_accvgpr_read_b32 v112, a0\n\t" \
  "v_accvgpr_read_b32 v113, a1\n\t" \
  "v_accvgpr_read_b32 v114, a2\n\t" \
  "v_accvgpr_read_b32 v115, a3\n\t" \
  "v_accvgpr_read_b32 v116, a4\n\t" \
  "v_accvgpr_read_b32 v117, a5\n\t" \
  "v_accvgpr_read_b32 v118, a6\n\t" \
  "v_accvgpr_read_b32 v119, a7\n\t" \
  "v_accvgpr_read_b32 v120, a8\n\t" \
  "v_accvgpr_read_b32 v121, a9\n\t" \
  "v_accvgpr_read_b32 v122, a10\n\t" \
  "v_accvgpr_read_b32 v123, a11\n\t" \
  "v_accvgpr_read_b32 v124, a12\n\t" \
  "v_accvgpr_read_b32 v125, a13\n\t" \
  "v_accvgpr_read_b32 v126, a14\n\t" \
  "v_accvgpr_read_b32 v127, a15\n\t" \
  "v_accvgpr_read_b32 v128, a16\n\t" \
  "v_accvgpr_read_b32 v129, a17\n\t" \
  "v_accvgpr_read_b32 v130, a18\n\t" \
  "v_accvgpr_read_b32 v131, a19\n\t" \
  "v_accvgpr_read_b32 v132, a20\n\t" \
  "v_accvgpr_read_b32 v133, a21\n\t" \
  "v_accvgpr_read_b32 v134, a22\n\t" \
  "v_accvgpr_read_b32 v135, a23\n\t" \
  "v_accvgpr_read_b32 v136, a24\n\t" \
  "v_accvgpr_read_b32 v137, a25\n\t" \
  "v_accvgpr_read_b32 v138, a26\n\t" \
  "v_accvgpr_read_b32 v139, a27\n\t" \
  "v_accvgpr_read_b32 v140, a28\n\t" \
  "v_accvgpr_read_b32 v141, a29\n\t" \
  "v_accvgpr_read_b32 v142, a30\n\t" \
  "v_accvgpr_read_b32 v143, a31\n\t" \
  "v_accvgpr_read_b32 v144, a32\n\t" \
  "v_accvgpr_read_b32 v145, a33\n\t" \
  "v_accvgpr_read_b32 v146, a34\n\t" \
  "v_accvgpr_read_b32 v147, a35\n\t" \
  "v_accvgpr_read_b32 v148, a36\n\t" \
  "v_accvgpr_read_b32 v149, a37\n\t" \
  "v_accvgpr_read_b32 v150, a38\n\t" \
  "v_accvgpr_read_b32 v151, a39\n\t" \
  "v_accvgpr_read_b32 v152, a40\n\t" \
  "v_accvgpr_read_b32 v153, a41\n\t" \
  "v_accvgpr_read_b32 v154, a42\n\t" \
  "v_accvgpr_read_b32 v155, a43\n\t" \
  "v_accvgpr_read_b32 v156, a44\n\t" \
  "v_accvgpr_read_b32 v157, a45\n\t" \
  "v_accvgpr_read_b32 v158, a46\n\t" \
  "v_accvgpr_read_b32 v159, a47\n\t" \
  "v_accvgpr_read_b32 v160, a48\n\t" \
  "v_accvgpr_read_b32 v161, a49\n\t" \
  "v_accvgpr_read_b32 v162, a50\n\t" \
  "v_accvgpr_read_b32 v163, a51\n\t" \
  "v_accvgpr_read_b32 v164, a52\n\t" \
  "v_accvgpr_read_b32 v165, a53\n\t" \
  "v_accvgpr_read_b32 v166, a54\n\t" \
  "v_accvgpr_read_b32 v167, a55\n\t" \
  "v_accvgpr_read_b32 v168, a56\n\t" \
  "v_accvgpr_read_b32 v169, a57\n\t" \
  "v_accvgpr_read_b32 v170, a58\n\t" \
  "v_accvgpr_read_b32 v171, a59\n\t" \
  "v_accvgpr_read_b32 v172, a60\n\t" \
  "v_accvgpr_read_b32 v173, a61\n\t" \
  "v_accvgpr_read_b32 v174, a62\n\t" \
  "v_accvgpr_read_b32 v175, a63\n\t" \
  "v_accvgpr_read_b32 v176, a64\n\t" \
  "v_accvgpr_read_b32 v177, a65\n\t" \
  "v_accvgpr_read_b32 v178, a66\n\t" \
  "v_accvgpr_read_b32 v179, a67\n\t" \
  "v_accvgpr_read_b32 v180, a68\n\t" \
  "v_accvgpr_read_b32 v181, a69\n\t" \
  "v_accvgpr_read_b32 v182, a70\n\t" \
  "v_accvgpr_read_b32 v183, a71\n\t" \
  "v_accvgpr_read_b32 v184, a72\n\t" \
  "v_accvgpr_read_b32 v185, a73\n\t" \
  "v_accvgpr_read_b32 v186, a74\n\t" \
  "v_accvgpr_read_b32 v187, a75\n\t" \
  "v_accvgpr_read_b32 v188, a76\n\t" \
  "v_accvgpr_read_b32 v189, a77\n\t" \
  "v_accvgpr_read_b32 v190, a78\n\t" \
  "v_accvgpr_read_b32 v191, a79\n\t" \
  "v_accvgpr_read_b32 v192, a80\n\t" \
  "v_accvgpr_read_b32 v193, a81\n\t" \
  "v_accvgpr_read_b32 v194, a82\n\t" \
  "v_accvgpr_read_b32 v195, a83\n\t" \
  "v_accvgpr_read_b32 v196, a84\n\t" \
  "v_accvgpr_read_b32 v197, a85\n\t" \
  "v_accvgpr_read_b32 v198, a86\n\t" \
  "v_accvgpr_read_b32 v199, a87\n\t" \
  "v_accvgpr_read_b32 v200, a88\n\t" \
  "v_accvgpr_read_b32 v201, a89\n\t" \
  "v_accvgpr_read_b32 v202, a90\n\t" \
  "v_accvgpr_read_b32 v203, a91\n\t" \
  "v_accvgpr_read_b32 v204, a92\n\t" \
  "v_accvgpr_read_b32 v205, a93\n\t" \
  "v_accvgpr_read_b32 v206, a94\n\t" \
  "v_accvgpr_read_b32 v207, a95\n\t" \
  "v_accvgpr_read_b32 v208, a96\n\t" \
  "v_accvgpr_read_b32 v209, a97\n\t" \
  "v_accvgpr_read_b32 v210, a98\n\t" \
  "v_accvgpr_read_b32 v211, a99\n\t" \
  "v_accvgpr_read_b32 v212, a100\n\t" \
  "v_accvgpr_read_b32 v213, a101\n\t" \
  "v_accvgpr_read_b32 v214, a102\n\t" \
  "v_accvgpr_read_b32 v215, a103\n\t" \
  "v_accvgpr_read_b32 v216, a104\n\t" \
  "v_accvgpr_read_b32 v217, a105\n\t" \
  "v_accvgpr_read_b32 v218, a106\n\t" \
  "v_accvgpr_read_b32 v219, a107\n\t" \
  "v_accvgpr_read_b32 v220, a108\n\t" \
  "v_accvgpr_read_b32 v221, a109\n\t" \
  "v_accvgpr_read_b32 v222, a110\n\t" \
  "v_accvgpr_read_b32 v223, a111\n\t" \
  "v_accvgpr_read_b32 v224, a112\n\t" \
  "v_accvgpr_read_b32 v225, a113\n\t" \
  "v_accvgpr_read_b32 v226, a114\n\t" \
  "v_accvgpr_read_b32 v227, a115\n\t" \
  "v_accvgpr_read_b32 v228, a116\n\t" \
  "v_accvgpr_read_b32 v229, a117\n\t" \
  "v_accvgpr_read_b32 v230, a118\n\t" \
  "v_accvgpr_read_b32 v231, a119\n\t" \
  "v_accvgpr_read_b32 v232, a120\n\t" \
  "v_accvgpr_read_b32 v233, a121\n\t" \
  "v_accvgpr_read_b32 v234, a122\n\t" \
  "v_accvgpr_read_b32 v235, a123\n\t" \
  "v_accvgpr_read_b32 v236, a124\n\t" \
  "v_accvgpr_read_b32 v237, a125\n\t" \
  "v_accvgpr_read_b32 v238, a126\n\t" \
  "v_accvgpr_read_b32 v239, a127\n\t" \
  "v_mov_b32 %[m], 0\n\t" \
  "s_waitcnt vmcnt(15)\n\t" \
  "v_lshlrev_b32 v16, 16, v48\n\t" \
  "v_and_b32 v17, 0xffff0000, v48\n\t" \
  "v_lshlrev_b32 v18, 16, v49\n\t" \
  "v_and_b32 v19, 0xffff0000, v49\n\t" \
  "v_lshlrev_b32 v20, 16, v50\n\t" \
  "v_and_b32 v21, 0xffff0000, v50\n\t" \
  "v_lshlrev_b32 v22, 16, v51\n\t" \
  "v_and_b32 v23, 0xffff0000, v51\n\t" \
  "v_pk_add_f32 v[112:113], v[112:113], v[16:17]\n\t" \
  "v_pk_add_f32 v[114:115], v[114:115], v[18:19]\n\t" \
  "v_pk_add_f32 v[116:117], v[116:117], v[20:21]\n\t" \
  "v_pk_add_f32 v[118:119], v[118:119], v[22:23]\n\t" \
  "v_max3_f32 %[m], v112, v113, %[m]\n\t" \
  "v_max3_f32 %[m], v114, v115, %[m]\n\t" \
  "v_max3_f32 %[m], v116, v117, %[m]\n\t" \
  "v_max3_f32 %[m], v118, v119, %[m]\n\t" \
  "s_waitcnt vmcnt(14)\n\t" \
  "v_lshlrev_b32 v16, 16, v52\n\t" \
  "v_and_b32 v17, 0xffff0000, v52\n\t" \
  "v_lshlrev_b32 v18, 16, v53\n\t" \
  "v_and_b32 v19, 0xffff0000, v53\n\t" \
  "v_lshlrev_b32 v20, 16, v54\n\t" \
  "v_and_b32 v21, 0xffff0000, v54\n\t" \
  "v_lshlrev_b32 v22, 16, v55\n\t" \
  "v_and_b32 v23, 0xffff0000, v55\n\t" \
  "v_pk_add_f32 v[120:121], v[120:121], v[16:17]\n\t" \
  "v_pk_add_f32 v[122:123], v[122:123], v[18:19]\n\t" \
  "v_pk_add_f32 v[124:125], v[124:125], v[20:21]\n\t" \
  "v_pk_add_f32 v[126:127], v[126:127], v[22:23]\n\t" \
  "v_max3_f32 %[m], v120, v121, %[m]\n\t" \
  "v_max3_f32 %[m], v122, v123, %[m]\n\t" \
  "v_max3_f32 %[m], v124, v125, %[m]\n\t" \
  "v_max3_f32 %[m], v126, v127, %[m]\n\t" \
  "s_waitcnt vmcnt(13)\n\t" \
  "v_lshlrev_b32 v16, 16, v56\n\t" \
  "v_and_b32 v17, 0xffff0000, v56\n\t" \
  "v_lshlrev_b32 v18, 16, v57\n\t" \
  "v_and_b32 v19, 0xffff0000, v57\n\t" \
  "v_lshlrev_b32 v20, 16, v58\n\t" \
  "v_and_b32 v21, 0xffff0000, v58\n\t" \
  "v_lshlrev_b32 v22, 16, v59\n\t" \
  "v_and_b32 v23, 0xffff0000, v59\n\t" \
  "v_pk_add_f32 v[128:129], v[128:129], v[16:17]\n\t" \
  "v_pk_add_f32 v[130:131], v[130:131], v[18:19]\n\t" \
  "v_pk_add_f32 v[132:133], v[132:133], v[20:21]\n\t" \
  "v_pk_add_f32 v[134:135], v[134:135], v[22:23]\n\t" \
  "v_max3_f32 %[m], v128, v129, %[m]\n\t" \
  "v_max3_f32 %[m], v130, v131, %[m]\n\t" \
  "v_max3_f32 %[m], v132, v133, %[m]\n\t" \
  "v_max3_f32 %[m], v134, v135, %[m]\n\t" \
  "s_waitcnt vmcnt(12)\n\t" \
  "v_lshlrev_b32 v16, 16, v60\n\t" \
  "v_and_b32 v17, 0xffff0000, v60\n\t" \
  "v_lshlrev_b32 v18, 16, v61\n\t" \
  "v_and_b32 v19, 0xffff0000, v61\n\t" \
  "v_lshlrev_b32 v20, 16, v62\n\t" \
  "v_and_b32 v21, 0xffff0000, v62\n\t" \
  "v_lshlrev_b32 v22, 16, v63\n\t" \
  "v_and_b32 v23, 0xffff0000, v63\n\t" \
  "v_pk_add_f32 v[136:137], v[136:137], v[16:17]\n\t" \
  "v_pk_add_f32 v[138:139], v[138:139], v[18:19]\n\t" \
  "v_pk_add_f32 v[140:141], v[140:141], v[20:21]\n\t" \
  "v_pk_add_f32 v[142:143], v[142:143], v[22:23]\n\t" \
  "v_max3_f32 %[m], v136, v137, %[m]\n\t" \
  "v_max3_f32 %[m], v138, v139, %[m]\n\t" \
  "v_max3_f32 %[m], v140, v141, %[m]\n\t" \
  "v_max3_f32 %[m], v142, v143, %[m]\n\t" \
  "s_waitcnt vmcnt(11)\n\t" \
  "v_lshlrev_b32 v16, 16, v64\n\t" \
  "v_and_b32 v17, 0xffff0000, v64\n\t" \
  "v_lshlrev_b32 v18, 16, v65\n\t" \
  "v_and_b32 v19, 0xffff0000, v65\n\t" \
  "v_lshlrev_b32 v20, 16, v66\n\t" \
  "v_and_b32 v21, 0xffff0000, v66\n\t" \
  "v_lshlrev_b32 v22, 16, v67\n\t" \
  "v_and_b32 v23, 0xffff0000, v67\n\t" \
  "v_pk_add_f32 v[144:145], v[144:145], v[16:17]\n\t" \
  "v_pk_add_f32 v[146:147], v[146:147], v[18:19]\n\t" \
  "v_pk_add_f32 v[148:149], v[148:149], v[20:21]\n\t" \
  "v_pk_add_f32 v[150:151], v[150:151], v[22:23]\n\t" \
  "v_max3_f32 %[m], v144, v145, %[m]\n\t" \
  "v_max3_f32 %[m], v146, v147, %[m]\n\t" \
  "v_max3_f32 %[m], v148, v149, %[m]\n\t" \
  "v_max3_f32 %[m], v150, v151, %[m]\n\t" \
  "s_waitcnt vmcnt(10)\n\t" \
  "v_lshlrev_b32 v16, 16, v68\n\t" \
  "v_and_b32 v17, 0xffff0000, v68\n\t" \
  "v_lshlrev_b32 v18, 16, v69\n\t" \
  "v_and_b32 v19, 0xffff0000, v69\n\t" \
  "v_lshlrev_b32 v20, 16, v70\n\t" \
  "v_and_b32 v21, 0xffff0000, v70\n\t" \
  "v_lshlrev_b32 v22, 16, v71\n\t" \
  "v_and_b32 v23, 0xffff0000, v71\n\t" \
  "v_pk_add_f32 v[152:153], v[152:153], v[16:17]\n\t" \
  "v_pk_add_f32 v[154:155], v[154:155], v[18:19]\n\t" \
  "v_pk_add_f32 v[156:157], v[156:157], v[20:21]\n\t" \
  "v_pk_add_f32 v[158:159], v[158:159], v[22:23]\n\t" \
  "v_max3_f32 %[m], v152, v153, %[m]\n\t" \
  "v_max3_f32 %[m], v154, v155, %[m]\n\t" \
  "v_max3_f32 %[m], v156, v157, %[m]\n\t" \
  "v_max3_f32 %[m], v158, v159, %[m]\n\t" \
  "s_waitcnt vmcnt(9)\n\t" \
  "v_lshlrev_b32 v16, 16, v72\n\t" \
  "v_and_b32 v17, 0xffff0000, v72\n\t" \
  "v_lshlrev_b32 v18, 16, v73\n\t" \
  "v_and_b32 v19, 0xffff0000, v73\n\t" \
  "v_lshlrev_b32 v20, 16, v74\n\t" \
  "v_and_b32 v21, 0xffff0000, v74\n\t" \
  "v_lshlrev_b32 v22, 16, v75\n\t" \
  "v_and_b32 v23, 0xffff0000, v75\n\t" \
  "v_pk_add_f32 v[160:161], v[160:161], v[16:17]\n\t" \
  "v_pk_add_f32 v[162:163], v[162:163], v[18:19]\n\t" \
  "v_pk_add_f32 v[164:165], v[164:165], v[20:21]\n\t" \
  "v_pk_add_f32 v[166:167], v[166:167], v[22:23]\n\t" \
  "v_max3_f32 %[m], v160, v161, %[m]\n\t" \
  "v_max3_f32 %[m], v162, v163, %[m]\n\t" \
  "v_max3_f32 %[m], v164, v165, %[m]\n\t" \
  "v_max3_f32 %[m], v166, v167, %[m]\n\t" \
  "s_waitcnt vmcnt(8)\n\t" \
  "v_lshlrev_b32 v16, 16, v76\n\t" \
  "v_and_b32 v17, 0xffff0000, v76\n\t" \
  "v_lshlrev_b32 v18, 16, v77\n\t" \
  "v_and_b32 v19, 0xffff0000, v77\n\t" \
  "v_lshlrev_b32 v20, 16, v78\n\t" \
  "v_and_b32 v21, 0xffff0000, v78\n\t" \
  "v_lshlrev_b32 v22, 16, v79\n\t" \
  "v_and_b32 v23, 0xffff0000, v79\n\t" \
  "v_pk_add_f32 v[168:169], v[168:169], v[16:17]\n\t" \
  "v_pk_add_f32 v[170:171], v[170:171], v[18:19]\n\t" \
  "v_pk_add_f32 v[172:173], v[172:173], v[20:21]\n\t" \
  "v_pk_add_f32 v[174:175], v[174:175], v[22:23]\n\t" \
  "v_max3_f32 %[m], v168, v169, %[m]\n\t" \
  "v_max3_f32 %[m], v170, v171, %[m]\n\t" \
  "v_max3_f32 %[m], v172, v173, %[m]\n\t" \
  "v_max3_f32 %[m], v174, v175, %[m]\n\t" \
  "s_waitcnt vmcnt(7)\n\t" \
  "v_lshlrev_b32 v16, 16, v80\n\t" \
  "v_and_b32 v17, 0xffff0000, v80\n\t" \
  "v_lshlrev_b32 v18, 16, v81\n\t" \
  "v_and_b32 v19, 0xffff0000, v81\n\t" \
  "v_lshlrev_b32 v20, 16, v82\n\t" \
  "v_and_b32 v21, 0xffff0000, v82\n\t" \
  "v_lshlrev_b32 v22, 16, v83\n\t" \
  "v_and_b32 v23, 0xffff0000, v83\n\t" \
  "v_pk_add_f32 v[176:177], v[176:177], v[16:17]\n\t" \
  "v_pk_add_f32 v[178:179], v[178:179], v[18:19]\n\t" \
  "v_pk_add_f32 v[180:181], v[180:181], v[20:21]\n\t" \
  "v_pk_add_f32 v[182:183], v[182:183], v[22:23]\n\t" \
  "v_max3_f32 %[m], v176, v177, %[m]\n\t" \
  "v_max3_f32 %[m], v178, v179, %[m]\n\t" \
  "v_max3_f32 %[m], v180, v181, %[m]\n\t" \
  "v_max3_f32 %[m], v182, v183, %[m]\n\t" \
  "s_waitcnt vmcnt(6)\n\t" \
  "v_lshlrev_b32 v16, 16, v84\n\t" \
  "v_and_b32 v17, 0xffff0000, v84\n\t" \
  "v_lshlrev_b32 v18, 16, v85\n\t" \
  "v_and_b32 v19, 0xffff0000, v85\n\t" \
  "v_lshlrev_b32 v20, 16, v86\n\t" \
  "v_and_b32 v21, 0xffff0000, v86\n\t" \
  "v_lshlrev_b32 v22, 16, v87\n\t" \
  "v_and_b32 v23, 0xffff0000, v87\n\t" \
  "v_pk_add_f32 v[184:185], v[184:185], v[16:17]\n\t" \
  "v_pk_add_f32 v[186:187], v[186:187], v[18:19]\n\t" \
  "v_pk_add_f32 v[188:189], v[188:189], v[20:21]\n\t" \
  "v_pk_add_f32 v[190:191], v[190:191], v[22:23]\n\t" \
  "v_max3_f32 %[m], v184, v185, %[m]\n\t" \
  "v_max3_f32 %[m], v186, v187, %[m]\n\t" \
  "v_max3_f32 %[m], v188, v189, %[m]\n\t" \
  "v_max3_f32 %[m], v190, v191, %[m]\n\t" \
  "s_waitcnt vmcnt(5)\n\t" \
  "v_lshlrev_b32 v16, 16, v88\n\t" \
  "v_and_b32 v17, 0xffff0000, v88\n\t" \
  "v_lshlrev_b32 v18, 16, v89\n\t" \
  "v_and_b32 v19, 0xffff0000, v89\n\t" \
  "v_lshlrev_b32 v20, 16, v90\n\t" \
  "v_and_b32 v21, 0xffff0000, v90\n\t" \
  "v_lshlrev_b32 v22, 16, v91\n\t" \
  "v_and_b32 v23, 0xffff0000, v91\n\t" \
  "v_pk_add_f32 v[192:193], v[192:193], v[16:17]\n\t" \
  "v_pk_add_f32 v[194:195], v[194:195], v[18:19]\n\t" \
  "v_pk_add_f32 v[196:197], v[196:197], v[20:21]\n\t" \
  "v_pk_add_f32 v[198:199], v[198:199], v[22:23]\n\t" \
  "v_max3_f32 %[m], v192, v193, %[m]\n\t" \
  "v_max3_f32 %[m], v194, v195, %[m]\n\t" \
  "v_max3_f32 %[m], v196, v197, %[m]\n\t" \
  "v_max3_f32 %[m], v198, v199, %[m]\n\t" \
  "s_waitcnt vmcnt(4)\n\t" \
  "v_lshlrev_b32 v16, 16, v92\n\t" \
  "v_and_b32 v17, 0xffff0000, v92\n\t" \
  "v_lshlrev_b32 v18, 16, v93\n\t" \
  "v_and_b32 v19, 0xffff0000, v93\n\t" \
  "v_lshlrev_b32 v20, 16, v94\n\t" \
  "v_and_b32 v21, 0xffff0000, v94\n\t" \
  "v_lshlrev_b32 v22, 16, v95\n\t" \
  "v_and_b32 v23, 0xffff0000, v95\n\t" \
  "v_pk_add_f32 v[200:201], v[200:201], v[16:17]\n\t" \
  "v_pk_add_f32 v[202:203], v[202:203], v[18:19]\n\t" \
  "v_pk_add_f32 v[204:205], v[204:205], v[20:21]\n\t" \
  "v_pk_add_f32 v[206:207], v[206:207], v[22:23]\n\t" \
  "v_max3_f32 %[m], v200, v201, %[m]\n\t" \
  "v_max3_f32 %[m], v202, v203, %[m]\n\t" \
  "v_max3_f32 %[m], v204, v205, %[m]\n\t" \
  "v_max3_f32 %[m], v206, v207, %[m]\n\t" \
  "s_waitcnt vmcnt(3)\n\t" \
  "v_lshlrev_b32 v16, 16, v96\n\t" \
  "v_and_b32 v17, 0xffff0000, v96\n\t" \
  "v_lshlrev_b32 v18, 16, v97\n\t" \
  "v_and_b32 v19, 0xffff0000, v97\n\t" \
  "v_lshlrev_b32 v20, 16, v98\n\t" \
  "v_and_b32 v21, 0xffff0000, v98\n\t" \
  "v_lshlrev_b32 v22, 16, v99\n\t" \
  "v_and_b32 v23, 0xffff0000, v99\n\t" \
  "v_pk_add_f32 v[208:209], v[208:209], v[16:17]\n\t" \
  "v_pk_add_f32 v[210:211], v[210:211], v[18:19]\n\t" \
  "v_pk_add_f32 v[212:213], v[212:213], v[20:21]\n\t" \
  "v_pk_add_f32 v[214:215], v[214:215], v[22:23]\n\t" \
  "v_max3_f32 %[m], v208, v209, %[m]\n\t" \
  "v_max3_f32 %[m], v210, v211, %[m]\n\t" \
  "v_max3_f32 %[m], v212, v213, %[m]\n\t" \
  "v_max3_f32 %[m], v214, v215, %[m]\n\t" \
  "s_waitcnt vmcnt(2)\n\t" \
  "v_lshlrev_b32 v16, 16, v100\n\t" \
  "v_and_b32 v17, 0xffff0000, v100\n\t" \
  "v_lshlrev_b32 v18, 16, v101\n\t" \
  "v_and_b32 v19, 0xffff0000, v101\n\t" \
  "v_lshlrev_b32 v20, 16, v102\n\t" \
  "v_and_b32 v21, 0xffff0000, v102\n\t" \
  "v_lshlrev_b32 v22, 16, v103\n\t" \
  "v_and_b32 v23, 0xffff0000, v103\n\t" \
  "v_pk_add_f32 v[216:217], v[216:217], v[16:17]\n\t" \
  "v_pk_add_f32 v[218:219], v[218:219], v[18:19]\n\t" \
  "v_pk_add_f32 v[220:221], v[220:221], v[20:21]\n\t" \
  "v_pk_add_f32 v[222:223], v[222:223], v[22:23]\n\t" \
  "v_max3_f32 %[m], v216, v217, %[m]\n\t" \
  "v_max3_f32 %[m], v218, v219, %[m]\n\t" \
  "v_max3_f32 %[m], v220, v221, %[m]\n\t" \
  "v_max3_f32 %[m], v222, v223, %[m]\n\t" \
  "s_waitcnt vmcnt(1)\n\t" \
  "v_lshlrev_b32 v16, 16, v104\n\t" \
  "v_and_b32 v17, 0xffff0000, v104\n\t" \
  "v_lshlrev_b32 v18, 16, v105\n\t" \
  "v_and_b32 v19, 0xffff0000, v105\n\t" \
  "v_lshlrev_b32 v20, 16, v106\n\t" \
  "v_and_b32 v21, 0xffff0000, v106\n\t" \
  "v_lshlrev_b32 v22, 16, v107\n\t" \
  "v_and_b32 v23, 0xffff0000, v107\n\t" \
  "v_pk_add_f32 v[224:225], v[224:225], v[16:17]\n\t" \
  "v_pk_add_f32 v[226:227], v[226:227], v[18:19]\n\t" \
  "v_pk_add_f32 v[228:229], v[228:229], v[20:21]\n\t" \
  "v_pk_add_f32 v[230:231], v[230:231], v[22:23]\n\t" \
  "v_max3_f32 %[m], v224, v225, %[m]\n\t" \
  "v_max3_f32 %[m], v226, v227, %[m]\n\t" \
  "v_max3_f32 %[m], v228, v229, %[m]\n\t" \
  "v_max3_f32 %[m], v230, v231, %[m]\n\t" \
  "s_waitcnt vmcnt(0)\n\t" \
  "v_lshlrev_b32 v16, 16, v108\n\t" \
  "v_and_b32 v17, 0xffff0000, v108\n\t" \
  "v_lshlrev_b32 v18, 16, v109\n\t" \
  "v_and_b32 v19, 0xffff0000, v109\n\t" \
  "v_lshlrev_b32 v20, 16, v110\n\t" \
  "v_and_b32 v21, 0xffff0000, v110\n\t" \
  "v_lshlrev_b32 v22, 16, v111\n\t" \
  "v_and_b32 v23, 0xffff0000, v111\n\t" \
  "v_pk_add_f32 v[232:233], v[232:233], v[16:17]\n\t" \
  "v_pk_add_f32 v[234:235], v[234:235], v[18:19]\n\t" \
  "v_pk_add_f32 v[236:237], v[236:237], v[20:21]\n\t" \
  "v_pk_add_f32 v[238:239], v[238:239], v[22:23]\n\t" \
  "v_max3_f32 %[m], v232, v233, %[m]\n\t" \
  "v_max3_f32 %[m], v234, v235, %[m]\n\t" \
  "v_max3_f32 %[m], v236, v237, %[m]\n\t" \
  "v_max3_f32 %[m], v238, v239, %[m]\n\t" \
  "s_nop 1\n\t" \
  "s_branch Lend_%=\n\t" \
  "Lst0_%=:\n\t" \
  "s_waitcnt lgkmcnt(0)\n\t" \
  "s_bitcmp1_b32 s53, 31\n\t" \
  "s_cbranch_scc0 Lbk0_%=\n\t" \
  "s_mov_b64 exec, -1\n\t" \
  "s_add_u32 s83, %[sM0], 0\n\t" \
  "s_getpc_b64 s[74:75]\n\t" \
  "s_branch Lraw_%=\n\t" \
  "s_branch Lbk0_%=\n\t" \
  "Lst1_%=:\n\t" \
  "s_cmp_ge_u32 s68, 4\n\t" \
  "s_waitcnt vmcnt(21)\n\t" \
  "s_cbranch_scc1 Lsu1_%=\n\t" \
  "Lsu1_%=:\n\t" \
  "s_lshr_b32 s68, s68, 1\n\t" \
  "s_and_b32 s68, s68, 5\n\t" \
  "s_cselect_b64 vcc, -1, 0\n\t" \
  "s_branch Lbk1_%=\n\t" \
  "Lst2_%=:\n\t" \
  "s_waitcnt lgkmcnt(0)\n\t" \
  "s_bitcmp1_b32 s55, 31\n\t" \
  "s_cbranch_scc0 Lbk2_%=\n\t" \
  "s_mov_b64 exec, -1\n\t" \
  "s_add_u32 s83, %[sM0], 16384\n\t" \
  "s_getpc_b64 s[74:75]\n\t" \
  "s_branch Lraw_%=\n\t" \
  "s_branch Lbk2_%=\n\t" \
  "Lst3_%=:\n\t" \
  "s_getpc_b64 s[86:87]\n\t" \
  "s_branch Lspin_%=\n\t" \
  "s_mov_b32 s53, s59\n\t" \
  "s_branch Lbk3_%=\n\t" \
  "Lst4_%=:\n\t" \
  "s_getpc_b64 s[74:75]\n\t" \
  "s_branch Lsvca_%=\n\t" \
  "s_branch Lbk4_%=\n\t" \
  "Lst5_%=:\n\t" \
  "s_cmp_ge_u32 s68, 4\n\t" \
  "s_waitcnt vmcnt(21)\n\t" \
  "s_cbranch_scc1 Lsu5_%=\n\t" \
  "Lsu5_%=:\n\t" \
  "s_lshr_b32 s68, s68, 1\n\t" \
  "s_and_b32 s68, s68, 5\n\t" \
  "s_cselect_b64 vcc, -1, 0\n\t" \
  "s_branch Lbk5_%=\n\t" \
  "Lst6_%=:\n\t" \
  "s_waitcnt lgkmcnt(0)\n\t" \
  "s_bitcmp1_b32 s57, 31\n\t" \
  "s_cbranch_scc0 Lbk6_%=\n\t" \
  "s_mov_b64 exec, -1\n\t" \
  "s_add_u32 s83, %[sM0], 0\n\t" \
  "s_getpc_b64 s[74:75]\n\t" \
  "s_branch Lraw_%=\n\t" \
  "s_branch Lbk6_%=\n\t" \
  "Lst7_%=:\n\t" \
  "s_getpc_b64 s[86:87]\n\t" \
  "s_branch Lspin_%=\n\t" \
  "s_mov_b32 s55, s59\n\t" \
  "s_branch Lbk7_%=\n\t" \
  "Lst8_%=:\n\t" \
  "s_getpc_b64 s[74:75]\n\t" \
  "s_branch Lsvca_%=\n\t" \
  "s_branch Lbk8_%=\n\t" \
  "Lst9_%=:\n\t" \
  "s_cmp_ge_u32 s68, 4\n\t" \
  "s_waitcnt vmcnt(20)\n\t" \
  "s_cbranch_scc1 Lsu9_%=\n\t" \
  "Lsu9_%=:\n\t" \
  "s_lshr_b32 s68, s68, 1\n\t" \
  "s_and_b32 s68, s68, 5\n\t" \
  "s_cselect_b64 vcc, -1, 0\n\t" \
  "s_branch Lbk9_%=\n\t" \
  "Lst10_%=:\n\t" \
  "s_waitcnt lgkmcnt(0)\n\t" \
  "s_bitcmp1_b32 s53, 31\n\t" \
  "s_cbranch_scc0 Lbk10_%=\n\t" \
  "s_mov_b64 exec, -1\n\t" \
  "s_add_u32 s83, %[sM0], 16384\n\t" \
  "s_getpc_b64 s[74:75]\n\t" \
  "s_branch Lraw_%=\n\t" \
  "s_branch Lbk10_%=\n\t" \
  "Lst11_%=:\n\t" \
  "s_getpc_b64 s[86:87]\n\t" \
  "s_branch Lspin_%=\n\t" \
  "s_mov_b32 s57, s59\n\t" \
  "s_branch Lbk11_%=\n\t" \
  "Lst12_%=:\n\t" \
  "s_getpc_b64 s[74:75]\n\t" \
  "s_branch Lsvca_%=\n\t" \
  "s_branch Lbk12_%=\n\t" \
  "Lst13_%=:\n\t" \
  "s_cmp_ge_u32 s68, 4\n\t" \
  "s_waitcnt vmcnt(20)\n\t" \
  "s_cbranch_scc1 Lsu13_%=\n\t" \
  "Lsu13_%=:\n\t" \
  "s_lshr_b32 s68, s68, 1\n\t" \
  "s_and_b32 s68, s68, 5\n\t" \
  "s_cselect_b64 vcc, -1, 0\n\t" \
  "s_branch Lbk13_%=\n\t" \
  "Lst14_%=:\n\t" \
  "s_waitcnt lgkmcnt(0)\n\t" \
  "s_bitcmp1_b32 s55, 31\n\t" \
  "s_cbranch_scc0 Lbk14_%=\n\t" \
  "s_mov_b64 exec, -1\n\t" \
  "s_add_u32 s83, %[sM0], 0\n\t" \
  "s_getpc_b64 s[74:75]\n\t" \
  "s_branch Lraw_%=\n\t" \
  "s_branch Lbk14_%=\n\t" \
  "Lst15_%=:\n\t" \
  "s_getpc_b64 s[86:87]\n\t" \
  "s_branch Lspin_%=\n\t" \
  "s_mov_b32 s53, s59\n\t" \
  "s_branch Lbk15_%=\n\t" \
  "Lst16_%=:\n\t" \
  "s_getpc_b64 s[74:75]\n\t" \
  "s_branch Lsvca_%=\n\t" \
  "s_branch Lbk16_%=\n\t" \
  "Lst17_%=:\n\t" \
  "s_cmp_ge_u32 s68, 4\n\t" \
  "s_waitcnt vmcnt(20)\n\t" \
  "s_cbranch_scc1 Lsu17_%=\n\t" \
  "Lsu17_%=:\n\t" \
  "s_lshr_b32 s68, s68, 1\n\t" \
  "s_and_b32 s68, s68, 5\n\t" \
  "s_cselect_b64 vcc, -1, 0\n\t" \
  "s_branch Lbk17_%=\n\t" \
  "Lst18_%=:\n\t" \
  "s_waitcnt lgkmcnt(0)\n\t" \
  "s_bitcmp1_b32 s57, 31\n\t" \
  "s_cbranch_scc0 Lbk18_%=\n\t" \
  "s_mov_b64 exec, -1\n\t" \
  "s_add_u32 s83, %[sM0], 16384\n\t" \
  "s_getpc_b64 s[74:75]\n\t" \
  "s_branch Lraw_%=\n\t" \
  "s_branch Lbk18_%=\n\t" \
  "Lst19_%=:\n\t" \
  "s_getpc_b64 s[86:87]\n\t" \
  "s_branch Lspin_%=\n\t" \
  "s_mov_b32 s55, s59\n\t" \
  "s_branch Lbk19_%=\n\t" \
  "Lst20_%=:\n\t" \
  "s_getpc_b64 s[74:75]\n\t" \
  "s_branch Lsvca_%=\n\t" \
  "s_branch Lbk20_%=\n\t" \
  "Lst21_%=:\n\t" \
  "s_cmp_ge_u32 s68, 4\n\t" \
  "s_waitcnt vmcnt(20)\n\t" \
  "s_cbranch_scc1 Lsu21_%=\n\t" \
  "Lsu21_%=:\n\t" \
  "s_lshr_b32 s68, s68, 1\n\t" \
  "s_and_b32 s68, s68, 5\n\t" \
  "s_cselect_b64 vcc, -1, 0\n\t" \
  "s_branch Lbk21_%=\n\t" \
  "Lst22_%=:\n\t" \
  "s_waitcnt lgkmcnt(0)\n\t" \
  "s_bitcmp1_b32 s53, 31\n\t" \
  "s_cbranch_scc0 Lbk22_%=\n\t" \
  "s_mov_b64 exec, -1\n\t" \
  "s_add_u32 s83, %[sM0], 0\n\t" \
  "s_getpc_b64 s[74:75]\n\t" \
  "s_branch Lraw_%=\n\t" \
  "s_branch Lbk22_%=\n\t" \
  "Lst23_%=:\n\t" \
  "s_getpc_b64 s[86:87]\n\t" \
  "s_branch Lspin_%=\n\t" \
  "s_mov_b32 s57, s59\n\t" \
  "s_branch Lbk23_%=\n\t" \
  "Lst24_%=:\n\t" \
  "s_getpc_b64 s[74:75]\n\t" \
  "s_branch Lsvca_%=\n\t" \
  "s_branch Lbk24_%=\n\t" \
  "Lst25_%=:\n\t" \
  "s_cmp_ge_u32 s68, 4\n\t" \
  "s_waitcnt vmcnt(21)\n\t" \
  "s_cbranch_scc1 Lsu25_%=\n\t" \
  "Lsu25_%=:\n\t" \
  "s_lshr_b32 s68, s68, 1\n\t" \
  "s_and_b32 s68, s68, 5\n\t" \
  "s_cselect_b64 vcc, -1, 0\n\t" \
  "s_branch Lbk25_%=\n\t" \
  "Lst26_%=:\n\t" \
  "s_waitcnt lgkmcnt(0)\n\t" \
  "s_bitcmp1_b32 s55, 31\n\t" \
  "s_cbranch_scc0 Lbk26_%=\n\t" \
  "s_mov_b64 exec, -1\n\t" \
  "s_add_u32 s83, %[sM0], 16384\n\t" \
  "s_getpc_b64 s[74:75]\n\t" \
  "s_branch Lraw_%=\n\t" \
  "s_branch Lbk26_%=\n\t" \
  "Lst27_%=:\n\t" \
  "s_getpc_b64 s[86:87]\n\t" \
  "s_branch Lspin_%=\n\t" \
  "s_mov_b32 s53, s59\n\t" \
  "s_branch Lbk27_%=\n\t" \
  "Lst28_%=:\n\t" \
  "s_getpc_b64 s[74:75]\n\t" \
  "s_branch Lsvca_%=\n\t" \
  "s_branch Lbk28_%=\n\t" \
  "Lst29_%=:\n\t" \
  "s_cmp_ge_u32 s68, 4\n\t" \
  "s_waitcnt vmcnt(21)\n\t" \
  "s_cbranch_scc1 Lsu29_%=\n\t" \
  "Lsu29_%=:\n\t" \
  "s_lshr_b32 s68, s68, 1\n\t" \
  "s_and_b32 s68, s68, 5\n\t" \
  "s_cselect_b64 vcc, -1, 0\n\t" \
  "s_branch Lbk29_%=\n\t" \
  "Lst30_%=:\n\t" \
  "s_waitcnt lgkmcnt(0)\n\t" \
  "s_bitcmp1_b32 s57, 31\n\t" \
  "s_cbranch_scc0 Lbk30_%=\n\t" \
  "s_mov_b64 exec, -1\n\t" \
  "s_add_u32 s83, %[sM0], 0\n\t" \
  "s_getpc_b64 s[74:75]\n\t" \
  "s_branch Lraw_%=\n\t" \
  "s_branch Lbk30_%=\n\t" \
  "Lst31_%=:\n\t" \
  "s_getpc_b64 s[86:87]\n\t" \
  "s_branch Lspin_%=\n\t" \
  "s_mov_b32 s55, s59\n\t" \
  "s_branch Lbk31_%=\n\t" \
  "Lst32_%=:\n\t" \
  "s_getpc_b64 s[74:75]\n\t" \
  "s_branch Lsvca_%=\n\t" \
  "s_branch Lbk32_%=\n\t" \
  "Lst33_%=:\n\t" \
  "s_cmp_ge_u32 s68, 4\n\t" \
  "s_waitcnt vmcnt(20)\n\t" \
  "s_cbranch_scc1 Lsu33_%=\n\t" \
  "Lsu33_%=:\n\t" \
  "s_lshr_b32 s68, s68, 1\n\t" \
  "s_and_b32 s68, s68, 5\n\t" \
  "s_cselect_b64 vcc, -1, 0\n\t" \
  "s_branch Lbk33_%=\n\t" \
  "Lst34_%=:\n\t" \
  "s_waitcnt lgkmcnt(0)\n\t" \
  "s_bitcmp1_b32 s53, 31\n\t" \
  "s_cbranch_scc0 Lbk34_%=\n\t" \
  "s_mov_b64 exec, -1\n\t" \
  "s_add_u32 s83, %[sM0], 16384\n\t" \
  "s_getpc_b64 s[74:75]\n\t" \
  "s_branch Lraw_%=\n\t" \
  "s_branch Lbk34_%=\n\t" \
  "Lst35_%=:\n\t" \
  "s_getpc_b64 s[86:87]\n\t" \
  "s_branch Lspin_%=\n\t" \
  "s_mov_b32 s57, s59\n\t" \
  "s_branch Lbk35_%=\n\t" \
  "Lst36_%=:\n\t" \
  "s_getpc_b64 s[74:75]\n\t" \
  "s_branch Lsvca_%=\n\t" \
  "s_branch Lbk36_%=\n\t" \
  "Lst37_%=:\n\t" \
  "s_cmp_ge_u32 s68, 4\n\t" \
  "s_waitcnt vmcnt(20)\n\t" \
  "s_cbranch_scc1 Lsu37_%=\n\t" \
  "Lsu37_%=:\n\t" \
  "s_lshr_b32 s68, s68, 1\n\t" \
  "s_and_b32 s68, s68, 5\n\t" \
  "s_cselect_b64 vcc, -1, 0\n\t" \
  "s_branch Lbk37_%=\n\t" \
  "Lst38_%=:\n\t" \
  "s_waitcnt lgkmcnt(0)\n\t" \
  "s_bitcmp1_b32 s55, 31\n\t" \
  "s_cbranch_scc0 Lbk38_%=\n\t" \
  "s_mov_b64 exec, -1\n\t" \
  "s_add_u32 s83, %[sM0], 0\n\t" \
  "s_getpc_b64 s[74:75]\n\t" \
  "s_branch Lraw_%=\n\t" \
  "s_branch Lbk38_%=\n\t" \
  "Lst39_%=:\n\t" \
  "s_getpc_b64 s[86:87]\n\t" \
  "s_branch Lspin_%=\n\t" \
  "s_mov_b32 s53, s59\n\t" \
  "s_branch Lbk39_%=\n\t" \
  "Lst40_%=:\n\t" \
  "s_getpc_b64 s[74:75]\n\t" \
  "s_branch Lsvca_%=\n\t" \
  "s_branch Lbk40_%=\n\t" \
  "Lst41_%=:\n\t" \
  "s_cmp_ge_u32 s68, 4\n\t" \
  "s_waitcnt vmcnt(20)\n\t" \
  "s_cbranch_scc1 Lsu41_%=\n\t" \
  "Lsu41_%=:\n\t" \
  "s_lshr_b32 s68, s68, 1\n\t" \
  "s_and_b32 s68, s68, 5\n\t" \
  "s_cselect_b64 vcc, -1, 0\n\t" \
  "s_branch Lbk41_%=\n\t" \
  "Lst42_%=:\n\t" \
  "s_waitcnt lgkmcnt(0)\n\t" \
  "s_bitcmp1_b32 s57, 31\n\t" \
  "s_cbranch_scc0 Lbk42_%=\n\t" \
  "s_mov_b64 exec, -1\n\t" \
  "s_add_u32 s83, %[sM0], 16384\n\t" \
  "s_getpc_b64 s[74:75]\n\t" \
  "s_branch Lraw_%=\n\t" \
  "s_branch Lbk42_%=\n\t" \
  "Lst43_%=:\n\t" \
  "s_getpc_b64 s[86:87]\n\t" \
  "s_branch Lspin_%=\n\t" \
  "s_mov_b32 s55, s59\n\t" \
  "s_branch Lbk43_%=\n\t" \
  "Lst44_%=:\n\t" \
  "s_getpc_b64 s[74:75]\n\t" \
  "s_branch Lsvca_%=\n\t" \
  "s_branch Lbk44_%=\n\t" \
  "Lst45_%=:\n\t" \
  "s_cmp_ge_u32 s68, 4\n\t" \
  "s_waitcnt vmcnt(20)\n\t" \
  "s_cbranch_scc1 Lsu45_%=\n\t" \
  "Lsu45_%=:\n\t" \
  "s_lshr_b32 s68, s68, 1\n\t" \
  "s_and_b32 s68, s68, 5\n\t" \
  "s_cselect_b64 vcc, -1, 0\n\t" \
  "s_branch Lbk45_%=\n\t" \
  "Lst46_%=:\n\t" \
  "s_waitcnt lgkmcnt(0)\n\t" \
  "s_bitcmp1_b32 s53, 31\n\t" \
  "s_cbranch_scc0 Lbk46_%=\n\t" \
  "s_mov_b64 exec, -1\n\t" \
  "s_add_u32 s83, %[sM0], 0\n\t" \
  "s_getpc_b64 s[74:75]\n\t" \
  "s_branch Lraw_%=\n\t" \
  "s_branch Lbk46_%=\n\t" \
  "Lst47_%=:\n\t" \
  "s_getpc_b64 s[86:87]\n\t" \
  "s_branch Lspin_%=\n\t" \
  "s_mov_b32 s57, s59\n\t" \
  "s_branch Lbk47_%=\n\t" \
  "Lst48_%=:\n\t" \
  "s_getpc_b64 s[74:75]\n\t" \
  "s_branch Lsvca_%=\n\t" \
  "s_branch Lbk48_%=\n\t" \
  "Lst49_%=:\n\t" \
  "s_cmp_ge_u32 s68, 4\n\t" \
  "s_waitcnt vmcnt(20)\n\t" \
  "s_cbranch_scc1 Lsu49_%=\n\t" \
  "Lsu49_%=:\n\t" \
  "s_lshr_b32 s68, s68, 1\n\t" \
  "s_and_b32 s68, s68, 5\n\t" \
  "s_cselect_b64 vcc, -1, 0\n\t" \
  "s_branch Lbk49_%=\n\t" \
  "Lst50_%=:\n\t" \
  "s_waitcnt lgkmcnt(0)\n\t" \
  "s_bitcmp1_b32 s55, 31\n\t" \
  "s_cbranch_scc0 Lbk50_%=\n\t" \
  "s_mov_b64 exec, -1\n\t" \
  "s_add_u32 s83, %[sM0], 16384\n\t" \
  "s_getpc_b64 s[74:75]\n\t" \
  "s_branch Lraw_%=\n\t" \
  "s_branch Lbk50_%=\n\t" \
  "Lst51_%=:\n\t" \
  "s_getpc_b64 s[86:87]\n\t" \
  "s_branch Lspin_%=\n\t" \
  "s_mov_b32 s53, s59\n\t" \
  "s_branch Lbk51_%=\n\t" \
  "Lst52_%=:\n\t" \
  "s_getpc_b64 s[74:75]\n\t" \
  "s_branch Lsvcb_%=\n\t" \
  "s_branch Lbk52_%=\n\t" \
  "Lst53_%=:\n\t" \
  "s_cmp_ge_u32 s68, 4\n\t" \
  "s_waitcnt vmcnt(12)\n\t" \
  "s_cbranch_scc1 Lsu53_%=\n\t" \
  "Lsu53_%=:\n\t" \
  "s_lshr_b32 s68, s68, 1\n\t" \
  "s_and_b32 s68, s68, 5\n\t" \
  "s_cselect_b64 vcc, -1, 0\n\t" \
  "s_branch Lbk53_%=\n\t" \
  "Lst54_%=:\n\t" \
  "s_waitcnt lgkmcnt(0)\n\t" \
  "s_bitcmp1_b32 s57, 31\n\t" \
  "s_cbranch_scc0 Lbk54_%=\n\t" \
  "s_mov_b64 exec, -1\n\t" \
  "s_add_u32 s83, %[sM0], 0\n\t" \
  "s_getpc_b64 s[74:75]\n\t" \
  "s_branch Lraw_%=\n\t" \
  "s_branch Lbk54_%=\n\t" \
  "Lst55_%=:\n\t" \
  "s_getpc_b64 s[74:75]\n\t" \
  "s_branch Lsvcb_%=\n\t" \
  "s_branch Lbk55_%=\n\t" \
  "Lst65_%=:\n\t" \
  "s_cmp_ge_u32 s68, 4\n\t" \
  "s_waitcnt vmcnt(17)\n\t" \
  "s_cbranch_scc1 Lsu65_%=\n\t" \
  "Lsu65_%=:\n\t" \
  "s_lshr_b32 s68, s68, 1\n\t" \
  "s_and_b32 s68, s68, 5\n\t" \
  "s_cselect_b64 vcc, -1, 0\n\t" \
  "s_branch Lbk65_%=\n\t" \
  "Lst66_%=:\n\t" \
  "s_waitcnt lgkmcnt(0)\n\t" \
  "s_bitcmp1_b32 s53, 31\n\t" \
  "s_cbranch_scc0 Lbk66_%=\n\t" \
  "s_mov_b64 exec, -1\n\t" \
  "s_add_u32 s83, %[sM0], 16384\n\t" \
  "s_getpc_b64 s[74:75]\n\t" \
  "s_branch Lraw_%=\n\t" \
  "s_branch Lbk66_%=\n\t" \
  "Lst67_%=:\n\t" \
  "s_getpc_b64 s[74:75]\n\t" \
  "s_branch Lsvcb_%=\n\t" \
  "s_branch Lbk67_%=\n\t" \
  "Lsvcc_%=:\n\t" \
  "s_bitcmp1_b32 s77, 0\n\t" \
  "s_cbranch_scc1 Lsvc0_%=\n\t" \
  "s_mov_b32 s69, 1\n\t" \
  "s_branch Lpush_%=\n\t" \
  "Lsvcb_%=:\n\t" \
  "s_bitcmp1_b32 s77, 0\n\t" \
  "s_cbranch_scc0 Lpusha_%=\n\t" \
  "Lsvc0_%=:\n\t" \
  "s_waitcnt vmcnt(0)\n\t" \
  "s_branch Lflag_%=\n\t" \
  "Lsvca_%=:\n\t" \
  "s_bitcmp1_b32 s77, 0\n\t" \
  "s_cbranch_scc0 Lpusha_%=\n\t" \
  "s_sub_u32 s82, 2, s69\n\t" \
  "s_or_b32 s82, s82, s68\n\t" \
  "s_waitcnt vmcnt(24)\n\t" \
  "s_cbranch_scc0 Lsvc1_%=\n\t" \
  "s_waitcnt vmcnt(12)\n\t" \
  "Lsvc1_%=:\n\t" \
  "Lflag_%=:\n\t" \
  "s_lshl_b32 s81, s77, 1\n\t" \
  "s_add_u32 s81, s81, 49230\n\t" \
  "s_mov_b64 exec, 1\n\t" \
  "v_mov_b32 v234, s81\n\t" \
  "v_mov_b32 v235, 1\n\t" \
  "ds_add_rtn_u32 v234, v234, v235\n\t" \
  "ds_read_b32 v238, v224\n\t" \
  "ds_read_b32 v239, v224\n\t" \
  "s_waitcnt lgkmcnt(2)\n\t" \
  "v_readfirstlane_b32 s81, v234\n\t" \
  "s_mov_b64 exec, 0xff\n\t" \
  "v_mov_b32 v234, %[call]\n\t" \
  "v_mov_b32 v235, s65\n\t" \
  "s_cmp_eq_u32 s81, 3\n\t" \
  "s_cbranch_scc0 Lflag2_%=\n\t" \
  "global_store_dwordx2 v[232:233], v[234:235], off sc0 sc1\n\t" \
  "Lflag2_%=:\n\t" \
  "s_mov_b64 exec, -1\n\t" \
  "s_add_u32 s77, s77, 1\n\t" \
  "v_add_co_u32 v232, s[66:67], %[fi], v232\n\t" \
  "s_add_u32 s80, s80, 1360\n\t" \
  "s_add_u32 s76, s72, 0x7fffffff\n\t" \
  "v_addc_co_u32 v233, s[66:67], 0, v233, s[66:67]\n\t" \
  "s_bitcmp1_b32 s71, s77\n\t" \
  "s_cselect_b32 s81, 0, 0\n\t" \
  "s_add_u32 s81, s81, s80\n\t" \
  "s_cmp_lt_u32 s77, 8\n\t" \
  "s_cselect_b32 s76, s81, s76\n\t" \
  "s_add_u32 s74, s74, 4\n\t" \
  "s_addc_u32 s75, s75, 0\n\t" \
  "s_setpc_b64 s[74:75]\n\t" \
  "Lpusha_%=:\n\t" \
  "s_mov_b32 s69, 2\n\t" \
  "Lpush_%=:\n\t" \
  "ds_read_b96 v[234:236], v224\n\t" \
  "ds_read_b128 v[228:231], v14\n\t" \
  "ds_read_b32 v238, v224\n\t" \
  "ds_read_b32 v239, v224\n\t" \
  "s_waitcnt lgkmcnt(2)\n\t" \
  "v_readfirstlane_b32 s78, v234\n\t" \
  "v_readfirstlane_b32 s79, v235\n\t" \
  "v_readfirstlane_b32 s65, v236\n\t" \
  "v_add_u32 v14, 4096, v14\n\t" \
  "v_add_u32 v224, 64, v224\n\t" \
  "s_cmp_eq_u64 s[78:79], 0\n\t" \
  "s_cbranch_scc1 Lpush2_%=\n\t" \
  "Lpush3_%=:\n\t" \
  "s_mov_b64 exec, s[78:79]\n\t" \
  "global_store_dwordx4 v15, v[228:231], s[36:37] sc0 sc1\n\t" \
  "global_store_dwordx4 v15, v[228:231], s[38:39] sc0 sc1\n\t" \
  "global_store_dwordx4 v15, v[228:231], s[40:41] sc0 sc1\n\t" \
  "global_store_dwordx4 v15, v[228:231], s[42:43] sc0 sc1\n\t" \
  "global_store_dwordx4 v15, v[228:231], s[44:45] sc0 sc1\n\t" \
  "global_store_dwordx4 v15, v[228:231], s[46:47] sc0 sc1\n\t" \
  "global_store_dwordx4 v15, v[228:231], s[48:49] sc0 sc1\n\t" \
  "global_store_dwordx4 v15, v[228:231], s[50:51] sc0 sc1\n\t" \
  "s_lshl_b32 s68, s69, 2\n\t" \
  "s_mov_b64 vcc, -1\n\t" \
  "Lpush6_%=:\n\t" \
  "s_mov_b64 exec, -1\n\t" \
  "s_mov_b32 s70, 12\n\t" \
  "s_add_u32 s77, s77, 1\n\t" \
  "s_add_u32 s76, s72, 50\n\t" \
  "v_add_u32 v15, %[sInc], v15\n\t" \
  "s_add_u32 s74, s74, 4\n\t" \
  "s_addc_u32 s75, s75, 0\n\t" \
  "s_setpc_b64 s[74:75]\n\t" \
  "Lpush2_%=:\n\t" \
  "s_add_u32 s77, s77, 1\n\t" \
  "v_add_u32 v15, %[sInc], v15\n\t" \
  "s_branch Lflag_%=\n\t" \
  "Lspin_%=:\n\t" \
  "Lspin1_%=:\n\t" \
  "s_load_dwordx2 s[58:59], %[ws], %[sM] glc\n\t" \
  "s_waitcnt lgkmcnt(0)\n\t" \
  "s_memrealtime s[72:73]\n\t" \
  "s_waitcnt lgkmcnt(0)\n\t" \
  "s_sub_u32 s81, s72, s76\n\t" \
  "s_cmp_ge_i32 s81, 0\n\t" \
  "s_cbranch_scc0 Lspin2_%=\n\t" \
  "s_getpc_b64 s[74:75]\n\t" \
  "s_branch Lsvcc_%=\n\t" \
  "Lspin2_%=:\n\t" \
  "s_cmp_ge_u32 s58, %[cmp]\n\t" \
  "s_cbranch_scc1 Lspin3_%=\n\t" \
  "s_sub_u32 s81, s72, %[t0]\n\t" \
  "s_cmp_gt_u32 s81, %[dl]\n\t" \
  "s_cbranch_scc0 Lspin1_%=\n\t" \
  "s_mov_b32 %[cmp], 0\n\t" \
  "v_mov_b32 v234, 0\n\t" \
  "v_mov_b32 v235, %[call]\n\t" \
  "global_store_dword v234, v235, %[ws] offset:128 sc1\n\t" \
  "Lspin3_%=:\n\t" \
  "s_add_u32 s86, s86, 4\n\t" \
  "s_addc_u32 s87, s87, 0\n\t" \
  "s_setpc_b64 s[86:87]\n\t" \
  "Lraw_%=:\n\t" \
  "s_sub_u32 s81, %[sM], 1049600\n\t" \
  "s_lshr_b32 s81, s81, 7\n\t" \
  "s_add_u32 s81, s81, %[cq]\n\t" \
  "s_lshl_b32 s82, s81, 10\n\t" \
  "s_mul_i32 s81, s81, 3072\n\t" \
  "s_add_u32 s81, s81, 2097152\n\t" \
  "s_add_u32 s82, s82, 27262976\n\t" \
  "s_mov_b32 m0, s83\n\t" \
  "s_waitcnt lgkmcnt(0)\n\t" \
  "buffer_load_dwordx4 v12, %[srdX], s81 offen offset:0 sc0 sc1 lds\n\t" \
  "buffer_load_dwordx4 v12, %[srdX], s81 offen offset:1024 sc0 sc1 lds\n\t" \
  "buffer_load_dwordx4 v12, %[srdX], s81 offen offset:2048 sc0 sc1 lds\n\t" \
  "s_add_u32 m0, s83, 3072\n\t" \
  "s_nop 0\n\t" \
  "buffer_load_dwordx4 v12, %[srdX], s82 offen sc0 sc1 lds\n\t" \
  "s_waitcnt vmcnt(0)\n\t" \
  "s_add_u32 s74, s74, 4\n\t" \
  "s_addc_u32 s75, s75, 0\n\t" \
  "s_setpc_b64 s[74:75]\n\t" \
  "Lend_%=:\n\t"
#define RELU_ASM \
  "v_max_f32 v112, 0, v112\n\t" \
  "v_max_f32 v113, 0, v113\n\t" \
  "v_max_f32 v114, 0, v114\n\t" \
  "v_max_f32 v115, 0, v115\n\t" \
  "v_max_f32 v116, 0, v116\n\t" \
  "v_max_f32 v117, 0, v117\n\t" \
  "v_max_f32 v118, 0, v118\n\t" \
  "v_max_f32 v119, 0, v119\n\t" \
  "v_max_f32 v120, 0, v120\n\t" \
  "v_max_f32 v121, 0, v121\n\t" \
  "v_max_f32 v122, 0, v122\n\t" \
  "v_max_f32 v123, 0, v123\n\t" \
  "v_max_f32 v124, 0, v124\n\t" \
  "v_max_f32 v125, 0, v125\n\t" \
  "v_max_f32 v126, 0, v126\n\t" \
  "v_max_f32 v127, 0, v127\n\t" \
  "v_max_f32 v128, 0, v128\n\t" \
  "v_max_f32 v129, 0, v129\n\t" \
  "v_max_f32 v130, 0, v130\n\t" \
  "v_max_f32 v131, 0, v131\n\t" \
  "v_max_f32 v132, 0, v132\n\t" \
  "v_max_f32 v133, 0, v133\n\t" \
  "v_max_f32 v134, 0, v134\n\t" \
  "v_max_f32 v135, 0, v135\n\t" \
  "v_max_f32 v136, 0, v136\n\t" \
  "v_max_f32 v137, 0, v137\n\t" \
  "v_max_f32 v138, 0, v138\n\t" \
  "v_max_f32 v139, 0, v139\n\t" \
  "v_max_f32 v140, 0, v140\n\t" \
  "v_max_f32 v141, 0, v141\n\t" \
  "v_max_f32 v142, 0, v142\n\t" \
  "v_max_f32 v143, 0, v143\n\t" \
  "v_max_f32 v144, 0, v144\n\t" \
  "v_max_f32 v145, 0, v145\n\t" \
  "v_max_f32 v146, 0, v146\n\t" \
  "v_max_f32 v147, 0, v147\n\t" \
  "v_max_f32 v148, 0, v148\n\t" \
  "v_max_f32 v149, 0, v149\n\t" \
  "v_max_f32 v150, 0, v150\n\t" \
  "v_max_f32 v151, 0, v151\n\t" \
  "v_max_f32 v152, 0, v152\n\t" \
  "v_max_f32 v153, 0, v153\n\t" \
  "v_max_f32 v154, 0, v154\n\t" \
  "v_max_f32 v155, 0, v155\n\t" \
  "v_max_f32 v156, 0, v156\n\t" \
  "v_max_f32 v157, 0, v157\n\t" \
  "v_max_f32 v158, 0, v158\n\t" \
  "v_max_f32 v159, 0, v159\n\t" \
  "v_max_f32 v160, 0, v160\n\t" \
  "v_max_f32 v161, 0, v161\n\t" \
  "v_max_f32 v162, 0, v162\n\t" \
  "v_max_f32 v163, 0, v163\n\t" \
  "v_max_f32 v164, 0, v164\n\t" \
  "v_max_f32 v165, 0, v165\n\t" \
  "v_max_f32 v166, 0, v166\n\t" \
  "v_max_f32 v167, 0, v167\n\t" \
  "v_max_f32 v168, 0, v168\n\t" \
  "v_max_f32 v169, 0, v169\n\t" \
  "v_max_f32 v170, 0, v170\n\t" \
  "v_max_f32 v171, 0, v171\n\t" \
  "v_max_f32 v172, 0, v172\n\t" \
  "v_max_f32 v173, 0, v173\n\t" \
  "v_max_f32 v174, 0, v174\n\t" \
  "v_max_f32 v175, 0, v175\n\t" \
  "v_max_f32 v176, 0, v176\n\t" \
  "v_max_f32 v177, 0, v177\n\t" \
  "v_max_f32 v178, 0, v178\n\t" \
  "v_max_f32 v179, 0, v179\n\t" \
  "v_max_f32 v180, 0, v180\n\t" \
  "v_max_f32 v181, 0, v181\n\t" \
  "v_max_f32 v182, 0, v182\n\t" \
  "v_max_f32 v183, 0, v183\n\t" \
  "v_max_f32 v184, 0, v184\n\t" \
  "v_max_f32 v185, 0, v185\n\t" \
  "v_max_f32 v186, 0, v186\n\t" \
  "v_max_f32 v187, 0, v187\n\t" \
  "v_max_f32 v188, 0, v188\n\t" \
  "v_max_f32 v189, 0, v189\n\t" \
  "v_max_f32 v190, 0, v190\n\t" \
  "v_max_f32 v191, 0, v191\n\t" \
  "v_max_f32 v192, 0, v192\n\t" \
  "v_max_f32 v193, 0, v193\n\t" \
  "v_max_f32 v194, 0, v194\n\t" \
  "v_max_f32 v195, 0, v195\n\t" \
  "v_max_f32 v196, 0, v196\n\t" \
  "v_max_f32 v197, 0, v197\n\t" \
  "v_max_f32 v198, 0, v198\n\t" \
  "v_max_f32 v199, 0, v199\n\t" \
  "v_max_f32 v200, 0, v200\n\t" \
  "v_max_f32 v201, 0, v201\n\t" \
  "v_max_f32 v202, 0, v202\n\t" \
  "v_max_f32 v203, 0, v203\n\t" \
  "v_max_f32 v204, 0, v204\n\t" \
  "v_max_f32 v205, 0, v205\n\t" \
  "v_max_f32 v206, 0, v206\n\t" \
  "v_max_f32 v207, 0, v207\n\t" \
  "v_max_f32 v208, 0, v208\n\t" \
  "v_max_f32 v209, 0, v209\n\t" \
  "v_max_f32 v210, 0, v210\n\t" \
  "v_max_f32 v211, 0, v211\n\t" \
  "v_max_f32 v212, 0, v212\n\t" \
  "v_max_f32 v213, 0, v213\n\t" \
  "v_max_f32 v214, 0, v214\n\t" \
  "v_max_f32 v215, 0, v215\n\t" \
  "v_max_f32 v216, 0, v216\n\t" \
  "v_max_f32 v217, 0, v217\n\t" \
  "v_max_f32 v218, 0, v218\n\t" \
  "v_max_f32 v219, 0, v219\n\t" \
  "v_max_f32 v220, 0, v220\n\t" \
  "v_max_f32 v221, 0, v221\n\t" \
  "v_max_f32 v222, 0, v222\n\t" \
  "v_max_f32 v223, 0, v223\n\t" \
  "v_max_f32 v224, 0, v224\n\t" \
  "v_max_f32 v225, 0, v225\n\t" \
  "v_max_f32 v226, 0, v226\n\t" \
  "v_max_f32 v227, 0, v227\n\t" \
  "v_max_f32 v228, 0, v228\n\t" \
  "v_max_f32 v229, 0, v229\n\t" \
  "v_max_f32 v230, 0, v230\n\t" \
  "v_max_f32 v231, 0, v231\n\t" \
  "v_max_f32 v232, 0, v232\n\t" \
  "v_max_f32 v233, 0, v233\n\t" \
  "v_max_f32 v234, 0, v234\n\t" \
  "v_max_f32 v235, 0, v235\n\t" \
  "v_max_f32 v236, 0, v236\n\t" \
  "v_max_f32 v237, 0, v237\n\t" \
  "v_max_f32 v238, 0, v238\n\t" \
  "v_max_f32 v239, 0, v239\n\t"
#define QUANT_ASM \
  "v_mov_b32 v4, %[inv]\n\t" \
  "v_mov_b32 v5, %[inv]\n\t" \
  "v_mov_b32 v6, %[mg]\n\t" \
  "v_mov_b32 v7, %[mg]\n\t" \
  "v_mov_b32 v8, %[vo]\n\t" \
  "v_add_u32 v9, 32768, %[vo]\n\t" \
  "v_add_u32 v10, 65536, %[vo]\n\t" \
  "v_add_u32 v11, 98304, %[vo]\n\t" \
  "v_add_u32 v12, 131072, %[vo]\n\t" \
  "v_add_u32 v13, 163840, %[vo]\n\t" \
  "v_add_u32 v14, 196608, %[vo]\n\t" \
  "v_add_u32 v15, 229376, %[vo]\n\t" \
  "s_mov_b32 s36, 0x05040100\n\t" \
  "v_pk_fma_f32 v[112:113], v[112:113], v[4:5], v[6:7]\n\t" \
  "v_pk_fma_f32 v[114:115], v[114:115], v[4:5], v[6:7]\n\t" \
  "v_pk_fma_f32 v[116:117], v[116:117], v[4:5], v[6:7]\n\t" \
  "v_pk_fma_f32 v[118:119], v[118:119], v[4:5], v[6:7]\n\t" \
  "v_pk_fma_f32 v[120:121], v[120:121], v[4:5], v[6:7]\n\t" \
  "v_pk_fma_f32 v[122:123], v[122:123], v[4:5], v[6:7]\n\t" \
  "v_pk_fma_f32 v[124:125], v[124:125], v[4:5], v[6:7]\n\t" \
  "v_pk_fma_f32 v[126:127], v[126:127], v[4:5], v[6:7]\n\t" \
  "v_lshl_add_u32 v112, v116, 8, v112\n\t" \
  "v_lshl_add_u32 v120, v124, 8, v120\n\t" \
  "v_lshl_add_u32 v113, v117, 8, v113\n\t" \
  "v_lshl_add_u32 v121, v125, 8, v121\n\t" \
  "v_lshl_add_u32 v114, v118, 8, v114\n\t" \
  "v_lshl_add_u32 v122, v126, 8, v122\n\t" \
  "v_lshl_add_u32 v115, v119, 8, v115\n\t" \
  "v_lshl_add_u32 v123, v127, 8, v123\n\t" \
  "v_perm_b32 v16, v120, v112, s36\n\t" \
  "v_perm_b32 v17, v121, v113, s36\n\t" \
  "v_perm_b32 v18, v122, v114, s36\n\t" \
  "v_perm_b32 v19, v123, v115, s36\n\t" \
  "global_store_dwordx4 v8, v[16:19], %[outp] nt\n\t" \
  "v_pk_fma_f32 v[128:129], v[128:129], v[4:5], v[6:7]\n\t" \
  "v_pk_fma_f32 v[130:131], v[130:131], v[4:5], v[6:7]\n\t" \
  "v_pk_fma_f32 v[132:133], v[132:133], v[4:5], v[6:7]\n\t" \
  "v_pk_fma_f32 v[134:135], v[134:135], v[4:5], v[6:7]\n\t" \
  "v_pk_fma_f32 v[136:137], v[136:137], v[4:5], v[6:7]\n\t" \
  "v_pk_fma_f32 v[138:139], v[138:139], v[4:5], v[6:7]\n\t" \
  "v_pk_fma_f32 v[140:141], v[140:141], v[4:5], v[6:7]\n\t" \
  "v_pk_fma_f32 v[142:143], v[142:143], v[4:5], v[6:7]\n\t" \
  "v_lshl_add_u32 v128, v132, 8, v128\n\t" \
  "v_lshl_add_u32 v136, v140, 8, v136\n\t" \
  "v_lshl_add_u32 v129, v133, 8, v129\n\t" \
  "v_lshl_add_u32 v137, v141, 8, v137\n\t" \
  "v_lshl_add_u32 v130, v134, 8, v130\n\t" \
  "v_lshl_add_u32 v138, v142, 8, v138\n\t" \
  "v_lshl_add_u32 v131, v135, 8, v131\n\t" \
  "v_lshl_add_u32 v139, v143, 8, v139\n\t" \
  "v_perm_b32 v20, v136, v128, s36\n\t" \
  "v_perm_b32 v21, v137, v129, s36\n\t" \
  "v_perm_b32 v22, v138, v130, s36\n\t" \
  "v_perm_b32 v23, v139, v131, s36\n\t" \
  "global_store_dwordx4 v9, v[20:23], %[outp] nt\n\t" \
  "v_pk_fma_f32 v[144:145], v[144:145], v[4:5], v[6:7]\n\t" \
  "v_pk_fma_f32 v[146:147], v[146:147], v[4:5], v[6:7]\n\t" \
  "v_pk_fma_f32 v[148:149], v[148:149], v[4:5], v[6:7]\n\t" \
  "v_pk_fma_f32 v[150:151], v[150:151], v[4:5], v[6:7]\n\t" \
  "v_pk_fma_f32 v[152:153], v[152:153], v[4:5], v[6:7]\n\t" \
  "v_pk_fma_f32 v[154:155], v[154:155], v[4:5], v[6:7]\n\t" \
  "v_pk_fma_f32 v[156:157], v[156:157], v[4:5], v[6:7]\n\t" \
  "v_pk_fma_f32 v[158:159], v[158:159], v[4:5], v[6:7]\n\t" \
  "v_lshl_add_u32 v144, v148, 8, v144\n\t" \
  "v_lshl_add_u32 v152, v156, 8, v152\n\t" \
  "v_lshl_add_u32 v145, v149, 8, v145\n\t" \
  "v_lshl_add_u32 v153, v157, 8, v153\n\t" \
  "v_lshl_add_u32 v146, v150, 8, v146\n\t" \
  "v_lshl_add_u32 v154, v158, 8, v154\n\t" \
  "v_lshl_add_u32 v147, v151, 8, v147\n\t" \
  "v_lshl_add_u32 v155, v159, 8, v155\n\t" \
  "v_perm_b32 v24, v152, v144, s36\n\t" \
  "v_perm_b32 v25, v153, v145, s36\n\t" \
  "v_perm_b32 v26, v154, v146, s36\n\t" \
  "v_perm_b32 v27, v155, v147, s36\n\t" \
  "global_store_dwordx4 v10, v[24:27], %[outp] nt\n\t" \
  "v_pk_fma_f32 v[160:161], v[160:161], v[4:5], v[6:7]\n\t" \
  "v_pk_fma_f32 v[162:163], v[162:163], v[4:5], v[6:7]\n\t" \
  "v_pk_fma_f32 v[164:165], v[164:165], v[4:5], v[6:7]\n\t" \
  "v_pk_fma_f32 v[166:167], v[166:167], v[4:5], v[6:7]\n\t" \
  "v_pk_fma_f32 v[168:169], v[168:169], v[4:5], v[6:7]\n\t" \
  "v_pk_fma_f32 v[170:171], v[170:171], v[4:5], v[6:7]\n\t" \
  "v_pk_fma_f32 v[172:173], v[172:173], v[4:5], v[6:7]\n\t" \
  "v_pk_fma_f32 v[174:175], v[174:175], v[4:5], v[6:7]\n\t" \
  "v_lshl_add_u32 v160, v164, 8, v160\n\t" \
  "v_lshl_add_u32 v168, v172, 8, v168\n\t" \
  "v_lshl_add_u32 v161, v165, 8, v161\n\t" \
  "v_lshl_add_u32 v169, v173, 8, v169\n\t" \
  "v_lshl_add_u32 v162, v166, 8, v162\n\t" \
  "v_lshl_add_u32 v170, v174, 8, v170\n\t" \
  "v_lshl_add_u32 v163, v167, 8, v163\n\t" \
  "v_lshl_add_u32 v171, v175, 8, v171\n\t" \
  "v_perm_b32 v28, v168, v160, s36\n\t" \
  "v_perm_b32 v29, v169, v161, s36\n\t" \
  "v_perm_b32 v30, v170, v162, s36\n\t" \
  "v_perm_b32 v31, v171, v163, s36\n\t" \
  "global_store_dwordx4 v11, v[28:31], %[outp] nt\n\t" \
  "v_pk_fma_f32 v[176:177], v[176:177], v[4:5], v[6:7]\n\t" \
  "v_pk_fma_f32 v[178:179], v[178:179], v[4:5], v[6:7]\n\t" \
  "v_pk_fma_f32 v[180:181], v[180:181], v[4:5], v[6:7]\n\t" \
  "v_pk_fma_f32 v[182:183], v[182:183], v[4:5], v[6:7]\n\t" \
  "v_pk_fma_f32 v[184:185], v[184:185], v[4:5], v[6:7]\n\t" \
  "v_pk_fma_f32 v[186:187], v[186:187], v[4:5], v[6:7]\n\t" \
  "v_pk_fma_f32 v[188:189], v[188:189], v[4:5], v[6:7]\n\t" \
  "v_pk_fma_f32 v[190:191], v[190:191], v[4:5], v[6:7]\n\t" \
  "v_lshl_add_u32 v176, v180, 8, v176\n\t" \
  "v_lshl_add_u32 v184, v188, 8, v184\n\t" \
  "v_lshl_add_u32 v177, v181, 8, v177\n\t" \
  "v_lshl_add_u32 v185, v189, 8, v185\n\t" \
  "v_lshl_add_u32 v178, v182, 8, v178\n\t" \
  "v_lshl_add_u32 v186, v190, 8, v186\n\t" \
  "v_lshl_add_u32 v179, v183, 8, v179\n\t" \
  "v_lshl_add_u32 v187, v191, 8, v187\n\t" \
  "v_perm_b32 v32, v184, v176, s36\n\t" \
  "v_perm_b32 v33, v185, v177, s36\n\t" \
  "v_perm_b32 v34, v186, v178, s36\n\t" \
  "v_perm_b32 v35, v187, v179, s36\n\t" \
  "global_store_dwordx4 v12, v[32:35], %[outp] nt\n\t" \
  "v_pk_fma_f32 v[192:193], v[192:193], v[4:5], v[6:7]\n\t" \
  "v_pk_fma_f32 v[194:195], v[194:195], v[4:5], v[6:7]\n\t" \
  "v_pk_fma_f32 v[196:197], v[196:197], v[4:5], v[6:7]\n\t" \
  "v_pk_fma_f32 v[198:199], v[198:199], v[4:5], v[6:7]\n\t" \
  "v_pk_fma_f32 v[200:201], v[200:201], v[4:5], v[6:7]\n\t" \
  "v_pk_fma_f32 v[202:203], v[202:203], v[4:5], v[6:7]\n\t" \
  "v_pk_fma_f32 v[204:205], v[204:205], v[4:5], v[6:7]\n\t" \
  "v_pk_fma_f32 v[206:207], v[206:207], v[4:5], v[6:7]\n\t" \
  "v_lshl_add_u32 v192, v196, 8, v192\n\t" \
  "v_lshl_add_u32 v200, v204, 8, v200\n\t" \
  "v_lshl_add_u32 v193, v197, 8, v193\n\t" \
  "v_lshl_add_u32 v201, v205, 8, v201\n\t" \
  "v_lshl_add_u32 v194, v198, 8, v194\n\t" \
  "v_lshl_add_u32 v202, v206, 8, v202\n\t" \
  "v_lshl_add_u32 v195, v199, 8, v195\n\t" \
  "v_lshl_add_u32 v203, v207, 8, v203\n\t" \
  "v_perm_b32 v36, v200, v192, s36\n\t" \
  "v_perm_b32 v37, v201, v193, s36\n\t" \
  "v_perm_b32 v38, v202, v194, s36\n\t" \
  "v_perm_b32 v39, v203, v195, s36\n\t" \
  "global_store_dwordx4 v13, v[36:39], %[outp] nt\n\t" \
  "v_pk_fma_f32 v[208:209], v[208:209], v[4:5], v[6:7]\n\t" \
  "v_pk_fma_f32 v[210:211], v[210:211], v[4:5], v[6:7]\n\t" \
  "v_pk_fma_f32 v[212:213], v[212:213], v[4:5], v[6:7]\n\t" \
  "v_pk_fma_f32 v[214:215], v[214:215], v[4:5], v[6:7]\n\t" \
  "v_pk_fma_f32 v[216:217], v[216:217], v[4:5], v[6:7]\n\t" \
  "v_pk_fma_f32 v[218:219], v[218:219], v[4:5], v[6:7]\n\t" \
  "v_pk_fma_f32 v[220:221], v[220:221], v[4:5], v[6:7]\n\t" \
  "v_pk_fma_f32 v[222:223], v[222:223], v[4:5], v[6:7]\n\t" \
  "v_lshl_add_u32 v208, v212, 8, v208\n\t" \
  "v_lshl_add_u32 v216, v220, 8, v216\n\t" \
  "v_lshl_add_u32 v209, v213, 8, v209\n\t" \
  "v_lshl_add_u32 v217, v221, 8, v217\n\t" \
  "v_lshl_add_u32 v210, v214, 8, v210\n\t" \
  "v_lshl_add_u32 v218, v222, 8, v218\n\t" \
  "v_lshl_add_u32 v211, v215, 8, v211\n\t" \
  "v_lshl_add_u32 v219, v223, 8, v219\n\t" \
  "v_perm_b32 v40, v216, v208, s36\n\t" \
  "v_perm_b32 v41, v217, v209, s36\n\t" \
  "v_perm_b32 v42, v218, v210, s36\n\t" \
  "v_perm_b32 v43, v219, v211, s36\n\t" \
  "global_store_dwordx4 v14, v[40:43], %[outp] nt\n\t" \
  "v_pk_fma_f32 v[224:225], v[224:225], v[4:5], v[6:7]\n\t" \
  "v_pk_fma_f32 v[226:227], v[226:227], v[4:5], v[6:7]\n\t" \
  "v_pk_fma_f32 v[228:229], v[228:229], v[4:5], v[6:7]\n\t" \
  "v_pk_fma_f32 v[230:231], v[230:231], v[4:5], v[6:7]\n\t" \
  "v_pk_fma_f32 v[232:233], v[232:233], v[4:5], v[6:7]\n\t" \
  "v_pk_fma_f32 v[234:235], v[234:235], v[4:5], v[6:7]\n\t" \
  "v_pk_fma_f32 v[236:237], v[236:237], v[4:5], v[6:7]\n\t" \
  "v_pk_fma_f32 v[238:239], v[238:239], v[4:5], v[6:7]\n\t" \
  "v_lshl_add_u32 v224, v228, 8, v224\n\t" \
  "v_lshl_add_u32 v232, v236, 8, v232\n\t" \
  "v_lshl_add_u32 v225, v229, 8, v225\n\t" \
  "v_lshl_add_u32 v233, v237, 8, v233\n\t" \
  "v_lshl_add_u32 v226, v230, 8, v226\n\t" \
  "v_lshl_add_u32 v234, v238, 8, v234\n\t" \
  "v_lshl_add_u32 v227, v231, 8, v227\n\t" \
  "v_lshl_add_u32 v235, v239, 8, v235\n\t" \
  "v_perm_b32 v44, v232, v224, s36\n\t" \
  "v_perm_b32 v45, v233, v225, s36\n\t" \
  "v_perm_b32 v46, v234, v226, s36\n\t" \
  "v_perm_b32 v47, v235, v227, s36\n\t" \
  "global_store_dwordx4 v15, v[44:47], %[outp] nt\n\t"
#define ACLOB "a0","a1","a2","a3","a4","a5","a6","a7","a8","a9","a10","a11","a12","a13","a14","a15","a16","a17","a18","a19","a20","a21","a22","a23","a24","a25","a26","a27","a28","a29","a30","a31","a32","a33","a34","a35","a36","a37","a38","a39","a40","a41","a42","a43","a44","a45","a46","a47","a48","a49","a50","a51","a52","a53","a54","a55","a56","a57","a58","a59","a60","a61","a62","a63","a64","a65","a66","a67","a68","a69","a70","a71","a72","a73","a74","a75","a76","a77","a78","a79","a80","a81","a82","a83","a84","a85","a86","a87","a88","a89","a90","a91","a92","a93","a94","a95","a96","a97","a98","a99","a100","a101","a102","a103","a104","a105","a106","a107","a108","a109","a110","a111","a112","a113","a114","a115","a116","a117","a118","a119","a120","a121","a122","a123","a124","a125","a126","a127","a128","a129","a130","a131","a132","a133","a134","a135","a136","a137","a138","a139","a140","a141","a142","a143","a144","a145","a146","a147","a148","a149","a150","a151","a152","a153","a154","a155","a156","a157","a158","a159","a160","a161","a162","a163","a164","a165","a166","a167","a168","a169","a170","a171","a172","a173","a174","a175","a176","a177","a178","a179","a180","a181","a182","a183","a184","a185","a186","a187","a188","a189","a190","a191","a192","a193","a194","a195","a196","a197","a198","a199","a200","a201","a202","a203","a204","a205","a206","a207","a208","a209","a210","a211","a212","a213","a214","a215","a216","a217","a218","a219","a220","a221","a222","a223","a224","a225","a226","a227","a228","a229","a230","a231","a232","a233","a234","a235","a236","a237","a238","a239","a240","a241","a242","a243","a244","a245","a246","a247","a248","a249","a250","a251","a252","a253","a254","a255"
#define VCLOB_MAIN "v4","v5","v6","v7","v8","v9","v10","v11","v12","v13","v14","v15","v16","v17","v18","v19","v20","v21","v22","v23","v24","v25","v26","v27","v28","v29","v30","v31","v32","v33","v34","v35","v36","v37","v38","v39","v40","v41","v42","v43","v44","v45","v46","v47","v48","v49","v50","v51","v52","v53","v54","v55","v56","v57","v58","v59","v60","v61","v62","v63","v64","v65","v66","v67","v68","v69","v70","v71","v72","v73","v74","v75","v76","v77","v78","v79","v80","v81","v82","v83","v84","v85","v86","v87","v88","v89","v90","v91","v92","v93","v94","v95","v96","v97","v98","v99","v100","v101","v102","v103","v104","v105","v106","v107","v108","v109","v110","v111"
#define VCLOB_QUANT "v4","v5","v6","v7","v8","v9","v10","v11","v12","v13","v14","v15","v16","v17","v18","v19","v20","v21","v22","v23","v24","v25","v26","v27","v28","v29","v30","v31","v32","v33","v34","v35","v36","v37","v38","v39","v40","v41","v42","v43","v44","v45","v46","v47"
#define SCLOB "s36","s37","s38","s39","s40","s41","s42","s43","s44","s45","s46","s47","s48","s49","s50","s51","s52","s53","s54","s55","s56","s57","s58","s59","s60","s61","s62","s63","s64","s65","s66","s67","s68","s69","s70","s71","s72","s73","s74","s75","s76","s77","s78","s79","s80","s81","s82","s83","s84","s85","s86","s87"
#define NITER 9
#define PACE 85
#define TFIRST 200
#define WBL2 0
#define G2C 1
#define RAW0 1
#define PFG 1

constexpr u32 WS_TICKET = 65536;
constexpr u32 WS_TMO    = 128;
constexpr u32 WS_EFLAG  = 8192;
constexpr u32 WS_G1     = 40960;
constexpr u32 WS_G2     = 45056;
constexpr u32 WS_XMETA  = 1u << 20;
constexpr u32 WS_XDATA  = 2u << 20;
constexpr u32 WS_XRAW3  = 26u << 20;
constexpr u32 WS_EX     = 34u << 20;
constexpr u32 WS_STAMP  = 52u << 20;
constexpr u32 WS_NEED   = 53u << 20;
constexpr u32 DEADLINE  = 40000000u;
constexpr u32 LDS_PUSH  = 32768, PUSH_STRIDE = 4096, LDS_SM = LDS_PUSH + 4 * PUSH_STRIDE;

struct Args {
    char* wsr[16];
    const float* x;
    const float* w;
    char* out;
    int rank;
};

__device__ __forceinline__ u32 rfl(u32 v) { return __builtin_amdgcn_readfirstlane(v); }
__device__ __forceinline__ u32x4 mksrd(const void* p, u32 bytes) {
    u64 a = (u64)p;
    u32x4 s;
    s[0] = (u32)a; s[1] = (u32)(a >> 32) & 0xffffu; s[2] = bytes; s[3] = 0x00020000u;
    return s;
}
__device__ __forceinline__ u32 ld_agent(const char* p) { return __hip_atomic_load((const u32*)p, __ATOMIC_RELAXED, __HIP_MEMORY_SCOPE_AGENT); }
__device__ __forceinline__ u64 ld_sys64(const char* p) { return __hip_atomic_load((const u64*)p, __ATOMIC_RELAXED, __HIP_MEMORY_SCOPE_SYSTEM); }
__device__ __forceinline__ bool expired(char* ws, u32 call, u64 t0) {
    if (ld_agent(ws + WS_TMO) == call || __builtin_amdgcn_s_memrealtime() - t0 > DEADLINE) {
        __hip_atomic_store((u32*)(ws + WS_TMO), call, __ATOMIC_RELAXED, __HIP_MEMORY_SCOPE_AGENT);
        return true;
    }
    return false;
}
template <int CTRL> __device__ __forceinline__ u32 dpp_max(u32 v) {
    u32 o = __builtin_amdgcn_update_dpp(v, v, CTRL, 0xf, 0xf, false);
    return v > o ? v : o;
}
__device__ __forceinline__ u32 wave_max(u32 v) {
    v = dpp_max<0xB1>(v);
    v = dpp_max<0x4E>(v);
    v = dpp_max<0x141>(v);
    v = dpp_max<0x140>(v);
    u32 a = __builtin_amdgcn_readlane(v, 0), b = __builtin_amdgcn_readlane(v, 16);
    u32 c = __builtin_amdgcn_readlane(v, 32), d = __builtin_amdgcn_readlane(v, 48);
    a = a > b ? a : b; c = c > d ? c : d;
    return a > c ? a : c;
}
__device__ __forceinline__ u32 cvt2(float a, float b) { u32 r; asm("v_cvt_pk_bf16_f32 %0, %1, %2" : "=v"(r) : "v"(a), "v"(b)); return r; }
__device__ __forceinline__ u32 pkmax(u32 a, u32 b) { u32 r; asm("v_pk_max_u16 %0, %1, %2" : "=v"(r) : "v"(a), "v"(b)); return r; }
__device__ __forceinline__ u32 pkmin(u32 a, u32 b) { u32 r; asm("v_pk_min_u16 %0, %1, %2" : "=v"(r) : "v"(a), "v"(b)); return r; }
__device__ __forceinline__ u32 pksub(u32 a, u32 b) { u32 r; asm("v_pk_sub_u16 %0, %1, %2" : "=v"(r) : "v"(a), "v"(b)); return r; }
__device__ __forceinline__ u32 below(u64 m) { return __builtin_amdgcn_mbcnt_hi((u32)(m >> 32), __builtin_amdgcn_mbcnt_lo((u32)m, 0)); }

__global__ __launch_bounds__(256, 1) void fused_gemm_a2a(Args a) {
    __shared__ __attribute__((aligned(1024))) u32 lds[LDS_SM / 4 + 128];
    u32* sm = lds + LDS_SM / 4;
    const u64 t0 = __builtin_amdgcn_s_memrealtime();
    const u32 b = blockIdx.x, xcd = b & 7, i = b >> 3, rank = a.rank & 7;
    const u32 tid = threadIdx.x, w = rfl(tid >> 6), l = tid & 63;
    char* ws = a.wsr[7];
#define STAMP_AT(n)
    const u32 sig = b >> 4, ps = b & 15, pkh = (ps >> 2) & 1, reg = (ps & 3) ^ (2 * pkh), blk = sig + 16 * w;
    const u32 prow = blk < 60 ? (ps & 3) : ((blk < 62 ? 2 : 0) + (reg & 1)) ^ (2 * pkh), pk = blk < 60 ? blk : 60 + 2 * (blk & 1) + (reg >> 1);
    const float* xp = a.x + (size_t)((ps >> 3) * 256 + prow * 64 + (l & 15)) * 4096 + pkh * 2048 + pk * 32 + (l >> 4) * 8;
    f32x4 r[4][2];
#pragma unroll
    for (int p = 0; p < 4; ++p) { r[p][0] = *(const f32x4*)(xp + p * 65536); r[p][1] = *(const f32x4*)(xp + p * 65536 + 4); }
    char* pw[8];
#pragma unroll
    for (int p = 0; p < 8; ++p) { pw[p] = a.wsr[p]; asm volatile("" : "+s"(pw[p])); }
    if (tid < 4) sm[20 + tid] = 0;
    const u32 pb = (rank * 4 + (ps >> 2)) * 64 + sig, pq = ps & 3;
    const u32 fo = WS_XMETA + pb * 512u + pq * 8u;
    u32x4 d[4]; u32 hi = 0;
    char* sc = (char*)lds + w * 4096;
#pragma unroll
    for (int p = 0; p < 4; ++p) {
        d[p][0] = cvt2(r[p][0][0], r[p][0][1]); d[p][1] = cvt2(r[p][0][2], r[p][0][3]);
        d[p][2] = cvt2(r[p][1][0], r[p][1][1]); d[p][3] = cvt2(r[p][1][2], r[p][1][3]);
        *(u32x4*)(sc + p * 1024 + l * 16) = d[p];
        const u32x4 g = d[p] & 0x7fff7fffu;
        hi = pkmax(hi, pkmax(pkmax(g[0], g[1]), pkmax(g[2], g[3])));
    }
    STAMP_AT(10);
    const u32 skip = sig < RAW0;
    if (skip && w == 0) {
        const u32 o0 = WS_XDATA + (pb * 4 + pq) * 3072u + l * 16u, o1 = WS_XRAW3 + (pb * 4 + pq) * 1024u + l * 16u;
#pragma unroll
        for (int p = 0; p < 8; ++p)
            asm volatile("global_store_dwordx4 %0, %2, %6 sc0 sc1\n\tglobal_store_dwordx4 %0, %3, %6 offset:1024 sc0 sc1\n\t"
                         "global_store_dwordx4 %0, %4, %6 offset:2048 sc0 sc1\n\tglobal_store_dwordx4 %1, %5, %6 sc0 sc1"
                         :: "v"(o0), "v"(o1), "v"(d[0]), "v"(d[1]), "v"(d[2]), "v"(d[3]), "s"(pw[p]) : "memory");
    }
    u32 tk = 0;
    if (w == 0) asm volatile("s_mov_b64 exec, 1\n\tglobal_atomic_add %0, %1, %2, %3 sc0\n\ts_mov_b64 exec, -1"
                             : "=&v"(tk) : "v"(WS_TICKET + b * 128u), "v"(1u), "s"(ws) : "memory");
    if (skip && w == 0) {
        char* fp = pw[0];
#pragma unroll
        for (int p = 1; p < 8; ++p) fp = l == p ? pw[p] : fp;
        fp += fo;
        asm volatile("s_waitcnt vmcnt(0)" : "+v"(tk), "+v"(fp) :: "memory");
        const u64 fv = (u64)0x80000000u << 32 | (rfl(tk) + 1);
        if (l < 8) asm volatile("global_store_dwordx2 %0, %1, off sc0 sc1" :: "v"(fp), "v"(fv) : "memory");
    } else {
        hi = pkmax(hi, hi >> 16) & 0xffffu;
        u32 top = wave_max(hi) >> 7;
#pragma unroll
        for (int k = 0; k < 3; ++k) if (top > 7 && __builtin_popcountll(__builtin_amdgcn_ballot_w64(hi >= top * 128u)) <= 6) --top;
        const u32 eb = (top > 7 ? top - 7 : 0) * 0x00800080u;
        u32x4 t[4]; u32 em = 0;
#pragma unroll
        for (int j = 0; j < 16; ++j) {
            t[j >> 2][j & 3] = pksub(d[j >> 2][j & 3], eb);
            em |= pkmin(t[j >> 2][j & 3] & 0x7c007c00u, 0x00010001u) << j;
        }
        const u32 nl = __builtin_popcount(em);
        const u64 b0 = __builtin_amdgcn_ballot_w64(nl & 1), b1 = __builtin_amdgcn_ballot_w64(nl & 2), b2 = __builtin_amdgcn_ballot_w64(nl & 4);
        const u32 n = __builtin_popcountll(b0) + 2 * __builtin_popcountll(b1) + 4 * __builtin_popcountll(b2);
        const bool plain = n > 63 || __builtin_amdgcn_ballot_w64(nl > 7) != 0;
        char* pa = (char*)lds + LDS_PUSH + w * PUSH_STRIDE;
        if (l < 4) {
            const u32 c = plain ? 64 : l < 2 ? 64 : l == 2 ? 48 + (n + 3) / 4 : 0;
            const u64 mk = c == 64 ? ~0ull : (1ull << c) - 1;
            u32x4 e; e[0] = (u32)mk; e[1] = (u32)(mk >> 32); e[2] = plain ? 0x80000000u : eb | n; e[3] = 0;
            *(u32x4*)(sm + 64 + w * 16 + l * 4) = e;
        }
        if (plain) {
#pragma unroll
            for (int p = 0; p < 4; ++p) *(u32x4*)(pa + p * 1024 + l * 16) = d[p];
        } else {
            u32x4 lo[2]; u32 h[8];
#pragma unroll
            for (int p = 0; p < 4; ++p) {
                h[2 * p] = __builtin_amdgcn_perm(t[p][1], t[p][0], 0x07050301u) & 0x83838383u;
                h[2 * p + 1] = __builtin_amdgcn_perm(t[p][3], t[p][2], 0x07050301u) & 0x83838383u;
                lo[p >> 1][(p & 1) * 2] = __builtin_amdgcn_perm(t[p][1], t[p][0], 0x06040200u);
                lo[p >> 1][(p & 1) * 2 + 1] = __builtin_amdgcn_perm(t[p][3], t[p][2], 0x06040200u);
            }
            *(u32x4*)(pa + l * 16) = lo[0];
            *(u32x4*)(pa + 1024 + l * 16) = lo[1];
            u32* np = (u32*)(pa + 2048 + l * 12);
            np[0] = h[0] | __builtin_amdgcn_alignbit(h[1], h[1], 29) | (h[6] & 0x03030303u) << 5;
            np[1] = h[2] | __builtin_amdgcn_alignbit(h[3], h[3], 29) | (h[7] & 0x03030303u) << 5;
            np[2] = h[4] | __builtin_amdgcn_alignbit(h[5], h[5], 29) | (h[6] & 0x80808080u) >> 2 | (h[7] & 0x80808080u) >> 1;
            u32* xl = (u32*)(pa + 2816) + below(b0) + 2 * below(b1) + 4 * below(b2);
            while (em) {
                const u32 k = __builtin_ctz(em), o = (k & 12) * 256 + (k & 3) * 4 + (k >> 4) * 2 + l * 16;
                em &= em - 1;
                *xl++ = (o + reg * 4096) << 16 | *(const unsigned short*)(sc + o);
            }
        }
    }
    if (!skip) asm volatile("s_waitcnt vmcnt(0)" : "+v"(tk) :: "memory");
    if (tid == 0) sm[16] = tk;
    __syncthreads();
    const u32 call = rfl(sm[16]) + 1;
    u32 xcc;
    asm volatile("s_getreg_b32 %0, hwreg(HW_REG_XCC_ID, 0, 4)" : "=s"(xcc));
    const u32 idv = (call << 4) | xcc;
    if (tid == 0) __hip_atomic_store((u32*)(ws + WS_EFLAG + b * 128u + 64u), idv, __ATOMIC_RELAXED, __HIP_MEMORY_SCOPE_AGENT);
    STAMP_AT(11);

    const u32 kh = (i >> 1) & 1, mt = (xcd & 3) * 4 + (i & 1) + ((i >> 2) & 1) * 2, nt = (xcd >> 2) * 4 + (i >> 3);
    u32x4 srdW = mksrd(a.w, 4096u * 65536u);
    u32x4 srdX = mksrd(ws, WS_NEED);
    const u32 cb = ((mt >> 1) * 4 + (mt & 1) * 2 + kh) * 64, cq = w ^ (2 * kh);
    u32 sW = kh * 2048u * 65536u, sX = WS_XDATA + (cb * 4 + cq) * 3072u, sM = WS_XMETA + cb * 512u + cq * 8u, cnt = NITER, cmp = call;
    const u32 vW0 = (l >> 4) * 8 * 65536u + (rank * 2048 + nt * 256 + w * 64 + (l & 15) * 4) * 4u, vL0 = l * 16u;
    const u32 vPT = sW + (l < 61 ? l + 3 : 63) * (32 * 65536u) + w * (8 * 65536u) + (rank * 2048 + nt * 256) * 4u;
    const u32 vPD = sW + (mt * 32 + (tid >> 3)) * 65536u + (rank * 2048 + nt * 256) * 4u + (tid & 7) * 128u;
    const u32 sec = ((mt * 4 + w) * 24 + (l < 24 ? l : 23)) * 2;
    const u32 vPL = sW + (sec >> 4) * 65536u + (rank * 2048 + nt * 256) * 4u + (sec & 15) * 64u;
    const u32 sOff = w == 3 ? WS_XRAW3 + (pb * 4 + pq) * 1024u : WS_XDATA + (pb * 4 + pq) * 3072u + w * 1024u;
    const u32 sInc = w == 3 ? 16 * 4096u : 16 * 12288u;
    const u32 sg = sig + 16 * skip, due = (u32)t0 + TFIRST + sg * PACE;
    char* exw = ws + WS_EX + b * 65536u + w * 16384u;
    char* exr = ws + WS_EX + (b ^ 16u) * 65536u + w * 16384u;
    u32 same; float m;
    f32x32 y0, y1, y2, y3;
    asm volatile("s_waitcnt lgkmcnt(0)\n\t" MAIN_ASM
        : [sW] "+s"(sW), [sX] "+s"(sX), [sM] "+s"(sM), [cnt] "+s"(cnt), [cmp] "+s"(cmp), [same] "=&s"(same), [m] "=&v"(m),
          "=&{v[112:143]}"(y0), "=&{v[144:175]}"(y1), "=&{v[176:207]}"(y2), "=&{v[208:239]}"(y3)
        : [srdW] "s"(srdW), [srdX] "s"(srdX), [sM0] "s"(w * 4096u), [ws] "s"(ws), [cq] "s"(cq), [ka] "s"(__builtin_amdgcn_kernarg_segment_ptr()),
          [exw] "s"(exw), [exr] "s"(exr), [sFP] "s"(WS_EFLAG + (b ^ 16u) * 128u + 64u), [sEF] "s"(WS_EFLAG + b * 128u + w * 4u), [sInc] "s"(sInc),
          [vOff] "v"(sOff + skip * sInc + vL0), [vPA] "v"(LDS_PUSH + skip * PUSH_STRIDE + w * 1024u + vL0), [sPM] "s"(LDS_SM + 256u + skip * 64u + w * 16u), [fi] "s"(16 * 512u),
          [call] "s"(call), [idv] "s"(idv), [due] "s"(due), [t0] "s"((u32)t0), [dl] "i"(DEADLINE),
          [vW0] "v"(vW0), [vL0] "v"(vL0), [vL12] "v"(l * 12u), [vL4] "v"(l * 4u), [fo] "v"(fo + skip * 16 * 512u), [sig] "s"(sg), [vPT] "v"(vPT), [vPD] "v"(vPD), [vPL] "v"(vPL)
        : "memory", "scc", "vcc", SCLOB, VCLOB_MAIN, ACLOB);
    STAMP_AT(5);
    u32 wm = wave_max(__builtin_bit_cast(u32, m));
    if (l == 0) sm[w] = wm;
    __syncthreads();
    if (w == 0) {
        u32 v = sm[l & 3];
        v = dpp_max<0xB1>(v); v = dpp_max<0x4E>(v);
        const u32 par = call & 1;
        bool bad = false;
        STAMP_AT(12);
        if (l == 0) __hip_atomic_store((u64*)(ws + WS_G1 + (par * 256 + xcd * 32 + i) * 8u), ((u64)call << 32) | v, __ATOMIC_RELAXED, __HIP_MEMORY_SCOPE_SYSTEM);
        if (i == 0) {
            u64 g;
            for (;;) {
                g = ld_sys64(ws + WS_G1 + (par * 256 + xcd * 32 + (l & 31)) * 8u);
                if (__all((u32)(g >> 32) == call)) break;
                if (expired(ws, call, t0)) { bad = true; break; }
                __builtin_amdgcn_s_sleep(1);
            }
            STAMP_AT(13);
            u32 gm = wave_max((u32)g);
            if (!bad && l < 8 * G2C) __hip_atomic_store((u64*)(a.wsr[l & 7] + WS_G2 + ((par * 8 + (l >> 3)) * 64 + rank * 8 + xcd) * 8u), ((u64)call << 32) | gm, __ATOMIC_RELAXED, __HIP_MEMORY_SCOPE_SYSTEM);
        }
    }
    asm volatile(RELU_ASM : "+{v[112:143]}"(y0), "+{v[144:175]}"(y1), "+{v[176:207]}"(y2), "+{v[208:239]}"(y3));
    if (w == 0) {
        const u32 par = call & 1;
        u64 g;
        for (;;) {
            g = ld_sys64(ws + WS_G2 + ((par * 8 + (xcd & (G2C - 1))) * 64 + l) * 8u);
            if (__all((u32)(g >> 32) == call)) break;
            if (expired(ws, call, t0)) break;
            __builtin_amdgcn_s_sleep(1);
        }
        STAMP_AT(14);
        u32 am = wave_max((u32)g);
        u32 tm = ld_agent(ws + WS_TMO);
        if (l == 0) { sm[8] = am; sm[9] = (tm == call); }
    }
    __syncthreads();
    STAMP_AT(6);
    const float amax = __builtin_bit_cast(float, sm[8]);
    const bool bad = sm[9] != 0;
    const float inv = amax > 0.f && !bad ? 127.0f / amax : 0.f, mg = bad ? 12582912.0f + 128.0f : 12582912.0f;
    char* p = a.out + (size_t)(mt * 256 + kh * 128) * 2048u + nt * 256u + w * 64u;
    asm volatile(QUANT_ASM
        : "+{v[112:143]}"(y0), "+{v[144:175]}"(y1), "+{v[176:207]}"(y2), "+{v[208:239]}"(y3)
        : [inv] "v"(inv), [mg] "v"(mg), [vo] "v"((l & 15) * 2048u + (l >> 4) * 16u), [outp] "s"(p)
        : "memory", "s36", VCLOB_QUANT);
}

extern "C" void kernel_launch(int rank, int nranks, ncclComm_t comm,
                              hipStream_t stream,
                              void* const* d_in, const int* in_sizes, int n_in,
                              void* d_out, int out_size,
                              const int* in_dtypes, int out_dtype,
                              void* const* d_out_all,
                              void* const* d_ws_all, size_t ws_bytes) {
    if (nranks != 8 || (unsigned)rank >= 8u || n_in < 2 || ws_bytes < WS_NEED ||
        in_sizes[0] != 512 * 4096 || in_sizes[1] != 4096 * 16384 || out_size != 4096 * 2048) {
        fprintf(stderr, "unexpected problem shape\n");
        return;
    }
    Args a;
    for (int p = 0; p < 16; ++p) a.wsr[p] = (char*)d_ws_all[(rank + 1 + p) & 7];
    a.x = (const float*)d_in[0];
    a.w = (const float*)d_in[1];
    a.out = (char*)d_out;
    a.rank = rank;
    fused_gemm_a2a<<<256, 256, 0, stream>>>(a);
}
```
